```python
import numpy as np

kernel_name = "gpu_gemm_rs_m4096_k4096_n4096_bf16_silu"


ABSMAX_RTOL = 0.008

COMPUTE_DTYPE = "bfloat16"
EPILOGUE = "silu"
IN_QUANT = None
OUT_QUANT = None
IN_DTYPES = {"x": "bfloat16", "w_mat": "bfloat16"}
WORLD_SIZE = 8

def _to_bf16(x):
    u = np.ascontiguousarray(x, dtype=np.float32).view(np.uint32)
    return ((u + 0x7FFF + ((u >> 16) & 1)) & 0xFFFF0000).view(np.float32)

def setup_inputs():

    x = _to_bf16(np.random.randn(4096, 4096).astype(np.float32))
    w_mat = _to_bf16((np.random.randn(4096, 4096) / np.sqrt(4096)).astype(np.float32))
    return {"x": x, "w_mat": w_mat}

def _epilogue(y):
    return (y / (1.0 + np.exp(-np.clip(y, -60.0, 60.0)))).astype(np.float32)

def reference(x, w_mat):
    y = (x @ w_mat).astype(np.float32)
    return _epilogue(y)
```

```cpp
#include <hip/hip_runtime.h>
#include <rccl/rccl.h>
#include <cstdio>

#define HIP_CHECK(expr) do { hipError_t _e = (expr); if (_e != hipSuccess) { \
    fprintf(stderr, "HIP error %s at %s:%d\n", hipGetErrorString(_e), __FILE__, __LINE__); \
    return; } } while (0)

namespace {

typedef unsigned u32;
typedef unsigned long long u64;
typedef unsigned u32x4 __attribute__((ext_vector_type(4)));
typedef unsigned u32x2 __attribute__((ext_vector_type(2)));
typedef unsigned short u16x2 __attribute__((ext_vector_type(2)));
typedef short bf16x8 __attribute__((ext_vector_type(8)));
typedef float f32x4 __attribute__((ext_vector_type(4)));
#define LAS __attribute__((address_space(3)))
#define GAS __attribute__((address_space(1)))
typedef GAS u32 gu32;
typedef GAS u64 gu64;
typedef __amdgpu_buffer_rsrc_t rsrc_t;

constexpr int NR = 8;
constexpr int NCOMM = 256;
constexpr int NGEMM = 256;
constexpr int NSEND = 128;
constexpr int NUNP = 512;
constexpr int NS = 4;
constexpr int NK = 64;
constexpr u64 DEADLINE = 50000000ull;

constexpr u32 OFF_CNT = 0;
constexpr u32 OFF_TMO = 8192;
constexpr u32 OFF_UDONE = 8192 + 256;
constexpr u32 OFF_DONE = 8192 + 1024;
constexpr u32 OFF_RDY = 16384;
constexpr u32 OFF_IFLAG = 65536;
constexpr u32 OFF_OFLAG = OFF_IFLAG + 8 * 256 * 64;
constexpr u32 OFF_UNP = 1u << 20;
constexpr u32 OFF_IPK = 17u << 20;
constexpr u32 OFF_IOV = 29u << 20;
constexpr u32 OFF_OPK = 37u << 20;
constexpr u32 OFF_OOV = 43u << 20;
constexpr u32 WS_USED = 47u << 20;
constexpr u32 WS_NEED = WS_USED;

struct Params {
    char* ws[NR];
    const char* x;
    const char* w;
    char* out;
    int rank;
    int pad;
};

struct Order { u32 e[256]; };
constexpr Order make_order() {
    Order o{};
    int n = 0;
    const int early[4] = {8, 5, 2, 1};
    for (int ks = 0; ks < 8; ++ks) {
        for (int t = 0; t < 16; ++t) o.e[n++] = (u32)(t << 3 | ks);
        for (int t = 0; t < 4; ++t) o.e[n++] = (u32)(128 | t << 3 | ks);
        for (int p = 1; p < 4; ++p)
            if (ks < early[p]) for (int t = 4 * p; t < 4 * p + 4; ++t) o.e[n++] = (u32)(128 | t << 3 | ks);
    }
    for (int p = 1; p < 4; ++p)
        for (int ks = early[p]; ks < 8; ++ks)
            for (int t = 4 * p; t < 4 * p + 4; ++t) o.e[n++] = (u32)(128 | t << 3 | ks);
    return o;
}
__constant__ const Order ORDER = make_order();

__device__ __forceinline__ u64 rt() { return __builtin_amdgcn_s_memrealtime(); }
__device__ __forceinline__ u32 uni(u32 v) { return __builtin_amdgcn_readfirstlane(v); }
__device__ __forceinline__ rsrc_t mkrs(const void* p, u32 bytes) { return __builtin_amdgcn_make_buffer_rsrc((void*)p, 0, bytes, 0x00020000); }
__device__ __forceinline__ u32x4 ld16(rsrc_t r, u32 voff, u32 soff) { return __builtin_amdgcn_raw_buffer_load_b128(r, voff, soff, 0); }
__device__ __forceinline__ u32x4 ld16_sys(rsrc_t r, u32 voff, u32 soff) { return __builtin_amdgcn_raw_buffer_load_b128(r, voff, soff, 17); }
__device__ __forceinline__ void st16_sys(u32x4 v, rsrc_t r, u32 voff, u32 soff) { __builtin_amdgcn_raw_buffer_store_b128(v, r, voff, soff, 17); }
__device__ __forceinline__ void st16_dev(u32x4 v, rsrc_t r, u32 voff, u32 soff) { __builtin_amdgcn_raw_buffer_store_b128(v, r, voff, soff, 16); }
#define FENCE() asm volatile("" ::: "memory")
#define WAITV(n) do { asm volatile("s_waitcnt vmcnt(" #n ")" ::: "memory"); __builtin_amdgcn_s_waitcnt(0xF70 | (n)); } while (0)
#define SB() __builtin_amdgcn_sched_barrier(0)

#define STAMPT(i) do {} while (0)

__device__ __forceinline__ u32 pkmax(u32 a, u32 b) {
    return __builtin_bit_cast(u32, __builtin_elementwise_max(__builtin_bit_cast(u16x2, a), __builtin_bit_cast(u16x2, b)));
}
__device__ __forceinline__ u32 wave_max(u32 v) {
    v = max(v, (u32)__builtin_amdgcn_update_dpp(0, (int)v, 0x111, 0xf, 0xf, true));
    v = max(v, (u32)__builtin_amdgcn_update_dpp(0, (int)v, 0x112, 0xf, 0xf, true));
    v = max(v, (u32)__builtin_amdgcn_update_dpp(0, (int)v, 0x114, 0xf, 0xf, true));
    v = max(v, (u32)__builtin_amdgcn_update_dpp(0, (int)v, 0x118, 0xf, 0xf, true));
    const u32 a = __builtin_amdgcn_readlane(v, 15), b = __builtin_amdgcn_readlane(v, 31);
    const u32 c = __builtin_amdgcn_readlane(v, 47), d = __builtin_amdgcn_readlane(v, 63);
    return max(max(a, b), max(c, d));
}

__device__ __forceinline__ void pack_hi(const u32 (&hi)[8], u32x4& nib, u32& base, u64& mask) {
    u32 e[8], ma = 0, mb = 0;
#pragma unroll
    for (int k = 0; k < 8; ++k) { e[k] = hi[k] & 0x7F7F7F7Fu; ma = pkmax(ma, e[k]); mb = pkmax(mb, e[k] << 8); }
    const u32 m2 = pkmax(ma, mb);
    const u32 top = wave_max(max(m2 >> 24, (m2 >> 8) & 0xFFu));
    base = top > 7u ? top - 7u : 0u;
    const u32 kk = 0x80808080u - base * 0x01010101u;
    u32 t[8], ok = 0x80808080u;
#pragma unroll
    for (int k = 0; k < 8; ++k) { const u32 d = e[k] + kk; ok &= d; t[k] = (hi[k] & 0x80808080u) | (d & 0x7F7F7F7Fu); }
    mask = __builtin_amdgcn_ballot_w64(ok != 0x80808080u);
#pragma unroll
    for (int k = 0; k < 4; ++k) nib[k] = t[2 * k] | __builtin_amdgcn_alignbit(t[2 * k + 1], t[2 * k + 1], 28);
}
__device__ __forceinline__ void unpack_hi(u32x4 nib, u32 base, u32 (&hi)[8]) {
    const u32 b4 = base * 0x01010101u;
#pragma unroll
    for (int k = 0; k < 4; ++k) {
        hi[2 * k] = (nib[k] & 0x87878787u) + b4;
        hi[2 * k + 1] = (__builtin_amdgcn_alignbit(nib[k], nib[k], 4) & 0x87878787u) + b4;
    }
}
__device__ __forceinline__ u64 flag_word(u32 call, u32 base, u64 mask, int which) {
    const u32 hi = (call << 4) | (which ? base >> 4 : base & 15u);
    const u32 lo = which ? (u32)(mask >> 32) : (u32)mask;
    return ((u64)hi << 32) | lo;
}
struct Blk { u32x4 l0, l1, nb, h0, h1; };
__device__ __forceinline__ void blk_load(Blk& b, rsrc_t rs, u32 voff, u32 soff, u32 ovoff, u64 mask) {
    b.l0 = ld16_sys(rs, voff, soff);
    b.l1 = ld16_sys(rs, voff + 1024, soff);
    b.nb = ld16_sys(rs, voff + 2048, soff);
    b.h0 = (u32x4){0, 0, 0, 0}; b.h1 = b.h0;
    if (mask) { b.h0 = ld16_sys(rs, voff, ovoff); b.h1 = ld16_sys(rs, voff + 1024, ovoff); }
}
__device__ __forceinline__ void blk_decode(const Blk& b, u32 base, u64 mask, int lane, u32 (&out)[16]) {
    u32 hi[8];
    unpack_hi(b.nb, base, hi);
    if (mask) {
        const bool esc = (mask >> lane) & 1;
#pragma unroll
        for (int k = 0; k < 4; ++k) { hi[k] = esc ? b.h0[k] : hi[k]; hi[4 + k] = esc ? b.h1[k] : hi[4 + k]; }
    }
#pragma unroll
    for (int k = 0; k < 8; ++k) {
        const u32 lo = k < 4 ? b.l0[k] : b.l1[k - 4];
        out[2 * k] = __builtin_amdgcn_perm(hi[k], lo, 0x05010400u);
        out[2 * k + 1] = __builtin_amdgcn_perm(hi[k], lo, 0x07030602u);
    }
}

__device__ __forceinline__ bool timed_out(gu32* tmo, u32 call, u64 t0) {
    if (uni(__hip_atomic_load(tmo, __ATOMIC_RELAXED, __HIP_MEMORY_SCOPE_AGENT)) == call) return true;
    if (rt() - t0 > DEADLINE) { __hip_atomic_store(tmo, call, __ATOMIC_RELAXED, __HIP_MEMORY_SCOPE_AGENT); return true; }
    return false;
}
template <int NW>
__device__ __forceinline__ bool wait_flags(const char* p, u32 call, gu32* tmo, int lane, u64& word) {
    const gu64* f = (const gu64*)p + (lane & (NW - 1));
    const u64 t0 = rt();
    for (u32 it = 1;; ++it) {
        word = __hip_atomic_load(f, __ATOMIC_RELAXED, __HIP_MEMORY_SCOPE_SYSTEM);
        if (__builtin_amdgcn_ballot_w64((u32)(word >> 36) < (call & 0xFFFFFFFu)) == 0) return true;
        if ((it & 63u) == 0 && timed_out(tmo, call, t0)) return false;
        __builtin_amdgcn_s_sleep(4);
    }
}
__device__ __forceinline__ void flag_decode(u64 word, int i, u32& base, u64& mask) {
    const u32 lo = (u32)word, hi = (u32)(word >> 32);
    const u32 m0 = __builtin_amdgcn_readlane(lo, 2 * i), m1 = __builtin_amdgcn_readlane(lo, 2 * i + 1);
    const u32 b0 = __builtin_amdgcn_readlane(hi, 2 * i), b1 = __builtin_amdgcn_readlane(hi, 2 * i + 1);
    base = (b0 & 15u) | ((b1 & 15u) << 4);
    mask = ((u64)m1 << 32) | m0;
}

__device__ __forceinline__ void unit_src(u32 op, u32 tile, u32 ks, u32 ab, u32 c, u32 half, int lane, u32& voff, u32& soff, u32& step) {
    const u32 sx = (512u * (tile >> 1) + 256u * ab + 128u * (tile & 1) + 64u * half) * 1024u + (256u * c + 32u * ks) * 2u;
    const u32 sw = (256u * c + 32u * ks + 4u * half) * 8192u + (2048u * ab + 128u * tile) * 2u;
    const u32 vx = (lane & 15) * 1024u + (lane >> 4) * 16u;
    const u32 vw = (lane >> 4) * 8u * 8192u + (lane & 15) * 16u;
    const u32 m = 0u - op;
    soff = uni((sw & m) | (sx & ~m));
    voff = (vw & m) | (vx & ~m);
    step = 16384u >> op;
}

__device__ __forceinline__ u32 cvt_pk_bf16(float lo, float hi) {
    typedef __bf16 bf2 __attribute__((ext_vector_type(2)));
    typedef float f2 __attribute__((ext_vector_type(2)));
    return __builtin_bit_cast(u32, __builtin_convertvector((f2){lo, hi}, bf2));
}
__device__ __forceinline__ float silu(float y) {
    const float c = __builtin_amdgcn_fmed3f(y, -60.0f, 60.0f);
    return y * __builtin_amdgcn_rcpf(1.0f + __builtin_amdgcn_exp2f(c * -1.44269504088896341f));
}

__global__ void __launch_bounds__(256, 2) fused(Params P) {
    __shared__ __attribute__((aligned(16))) char lds[NS * 16384];
    const int lane = threadIdx.x & 63;
    const int wid = uni(threadIdx.x >> 6);
    const u32 rank = P.rank;
    const u32 ra = rank >> 2, rb = (rank >> 1) & 1, rc = rank & 1;
    char* const my = P.ws[rank];
    gu32* const tmo = (gu32*)(my + OFF_TMO);
    gu32* const cnt = (gu32*)(my + OFF_CNT) + blockIdx.x * 4 + wid;
    const u32 cntv = __hip_atomic_load(cnt, __ATOMIC_RELAXED, __HIP_MEMORY_SCOPE_AGENT);
#define PUT_CALL() do { if (lane == 0) __hip_atomic_store(cnt, call, __ATOMIC_RELAXED, __HIP_MEMORY_SCOPE_AGENT); } while (0)
#define GET_CALL() const u32 call = uni(cntv) + 1u; PUT_CALL()
    STAMPT(0);
    const rsrc_t rmy = mkrs(my, WS_USED);
    const u32 v16 = lane * 16u;

    if (blockIdx.x < NCOMM) {
        const u32 cb = blockIdx.x;
        const rsrc_t rx = mkrs(P.x, 4096u * 512u * 2u), rw = mkrs(P.w, 512u * 4096u * 2u);
        if (wid == 0) {
            if (((cb >> 3) & 1) == 0) {
                const u32 sw = (cb & 7) | ((cb >> 4) << 3);
                const u32 half = sw & 1, c = (sw >> 1) & 1, ab = (sw >> 2) & 1;
                char* const dx0 = P.ws[ab * 4 + c]; char* const dx1 = P.ws[ab * 4 + 2 + c];
                char* const dw0 = P.ws[ab * 2 + c]; char* const dw1 = P.ws[4 + ab * 2 + c];
                u32x4 nx[4];
                u32 pos = sw >> 3;
                u32 ent = ORDER.e[pos];
                {
                    u32 voff, soff, step; unit_src(ent >> 7, (ent >> 3) & 15, ent & 7, ab, c, half, lane, voff, soff, step);
                    const rsrc_t rs = (ent >> 7) ? rw : rx;
#pragma unroll
                    for (int i = 0; i < 4; ++i) nx[i] = ld16(rs, voff, soff + i * step);
                }
                FENCE();
                const u32 call = uni(cntv) + 1u;
                char* pf1 = nullptr; u64 pw1 = 0;
                char* pf2 = nullptr; u64 pw2 = 0;
                constexpr int NIT = 256 / (NSEND / 8);
                for (int it = 0; it < NIT; ++it) {
                    WAITV(6);
                    const u32 op = ent >> 7;
                    u32 lo[8], hi[8];
                    if (op == 0) {
#pragma unroll
                        for (int k = 0; k < 8; ++k) {
                            const u32 a0 = nx[k >> 1][(k & 1) * 2], a1 = nx[k >> 1][(k & 1) * 2 + 1];
                            lo[k] = __builtin_amdgcn_perm(a1, a0, 0x06040200u); hi[k] = __builtin_amdgcn_perm(a1, a0, 0x07050301u);
                        }
                    } else {
#pragma unroll
                        for (int k = 0; k < 8; ++k) {
                            const u32 a0 = nx[(k & 1) * 2][k >> 1], a1 = nx[(k & 1) * 2 + 1][k >> 1];
                            lo[k] = __builtin_amdgcn_perm(a1, a0, 0x06020400u); hi[k] = __builtin_amdgcn_perm(a1, a0, 0x07030501u);
                        }
                    }
#pragma unroll
                    for (int k = 0; k < 8; ++k) asm volatile("" : "+v"(lo[k]), "+v"(hi[k]) :: "memory");
                    if (it > 1 && lane < 4) __hip_atomic_store((gu64*)pf2, pw2, __ATOMIC_RELAXED, __HIP_MEMORY_SCOPE_SYSTEM);
                    STAMPT(1 + it);
                    FENCE();
                    const u32 npos = it + 1 < NIT ? pos + NSEND / 8 : pos;
                    const u32 nent = ORDER.e[npos];
                    {
                        u32 voff, soff, step; unit_src(nent >> 7, (nent >> 3) & 15, nent & 7, ab, c, half, lane, voff, soff, step);
                        const rsrc_t rs = (nent >> 7) ? rw : rx;
#pragma unroll
                        for (int i = 0; i < 4; ++i) nx[i] = ld16(rs, voff, soff + i * step);
                    }
                    FENCE();
                    u32x4 nib; u32 base; u64 mask;
                    pack_hi(hi, nib, base, mask);
                    char* const d0 = op ? dw0 : dx0; char* const d1 = op ? dw1 : dx1;
                    const rsrc_t r0 = mkrs(d0, WS_USED), r1 = mkrs(d1, WS_USED);
                    const u32 blk = (rank * 256u + pos) * 2u + half;
                    if (mask) {
                        if ((mask >> lane) & 1) {
                            const u32x4 h0 = {hi[0], hi[1], hi[2], hi[3]}, h1 = {hi[4], hi[5], hi[6], hi[7]};
                            const u32 so = OFF_IOV + blk * 2048u;
                            st16_sys(h0, r0, v16, so); st16_sys(h1, r0, v16 + 1024, so);
                            st16_sys(h0, r1, v16, so); st16_sys(h1, r1, v16 + 1024, so);
                        }
                    }
                    FENCE();
                    {
                        const u32x4 l0 = {lo[0], lo[1], lo[2], lo[3]}, l1 = {lo[4], lo[5], lo[6], lo[7]};
                        const u32 so = OFF_IPK + blk * 3072u;
                        st16_sys(l0, r0, v16, so); st16_sys(l0, r1, v16, so);
                        st16_sys(l1, r0, v16 + 1024, so); st16_sys(l1, r1, v16 + 1024, so);
                        st16_sys(nib, r0, v16 + 2048, so); st16_sys(nib, r1, v16 + 2048, so);
                    }
                    FENCE();
                    pf2 = pf1; pw2 = pw1;
                    pf1 = ((lane & 2) ? d1 : d0) + OFF_IFLAG + (rank * 256u + pos) * 64u + half * 16u + (lane & 1) * 8u;
                    pw1 = flag_word(call, base, mask, lane & 1);
                    pos = npos; ent = nent;
                }
                WAITV(6);
                if (lane < 4) __hip_atomic_store((gu64*)pf2, pw2, __ATOMIC_RELAXED, __HIP_MEMORY_SCOPE_SYSTEM);
                WAITV(0);
                if (lane < 4) __hip_atomic_store((gu64*)pf1, pw1, __ATOMIC_RELAXED, __HIP_MEMORY_SCOPE_SYSTEM);
                PUT_CALL();
                STAMPT(20);
            } else if (cb == 8) {
                GET_CALL();
                const gu32* ud = (const gu32*)(my + OFF_UDONE);
                const u64 t0 = rt();
                bool ok = true;
                for (u32 it = 1; uni(__hip_atomic_load(ud, __ATOMIC_RELAXED, __HIP_MEMORY_SCOPE_AGENT)) < call * (u32)NUNP; ++it) {
                    if ((it & 15u) == 0 && timed_out(tmo, call, t0)) { ok = false; break; }
                    __builtin_amdgcn_s_sleep(32);
                }
                if (ok) {
                    if (lane < NR && lane != (int)rank) {
                        char* pw_ = P.ws[0];
#pragma unroll
                        for (int p = 1; p < NR; ++p) pw_ = lane == p ? P.ws[p] : pw_;
                        __hip_atomic_store((gu32*)(pw_ + OFF_DONE + rank * 128u), call, __ATOMIC_RELAXED, __HIP_MEMORY_SCOPE_SYSTEM);
                    }
                    const gu32* f = (const gu32*)(my + OFF_DONE + (lane & 7) * 128u);
                    for (u32 it = 1;; ++it) {
                        const u32 v = __hip_atomic_load(f, __ATOMIC_RELAXED, __HIP_MEMORY_SCOPE_SYSTEM);
                        if (__builtin_amdgcn_ballot_w64(v < call && (lane & 7) != (int)rank) == 0) break;
                        if ((it & 15u) == 0 && timed_out(tmo, call, t0)) { ok = false; break; }
                        __builtin_amdgcn_s_sleep(32);
                    }
                }
                if (!ok && lane == 0) __hip_atomic_store((gu32*)P.out, 0x7FC07FC0u, __ATOMIC_RELAXED, __HIP_MEMORY_SCOPE_AGENT);
                STAMPT(20);
            }
            return;
        }
        if (wid == 3) {
            const u32 op = cb >> 7, ks = (cb >> 4) & 7, tile = cb & 15;
            const rsrc_t rs = op ? rw : rx;
            u32x4 in[8];
#pragma unroll
            for (int h = 0; h < 2; ++h) {
                u32 voff, soff, step; unit_src(op, tile, ks, op ? rb : ra, rc, h, lane, voff, soff, step);
#pragma unroll
                for (int i = 0; i < 4; ++i) in[h * 4 + i] = ld16(rs, voff, soff + i * step);
            }
            FENCE();
            const u32 call = uni(cntv) + 1u;
            const u32 st = (op * 16u + tile) * 64u + ks;
            const u32 so = OFF_UNP + st * 8192u;
            if (op == 0) {
#pragma unroll
                for (int f = 0; f < 8; ++f) st16_dev(in[f], rmy, v16 + (f & 3) * 1024, so + (f >> 2) * 4096u);
            } else {
#pragma unroll
                for (int f = 0; f < 8; ++f) {
                    u32x4 v;
#pragma unroll
                    for (int p = 0; p < 4; ++p) v[p] = __builtin_amdgcn_perm(in[2 * p + 1][f >> 1], in[2 * p][f >> 1], (f & 1) ? 0x07060302u : 0x05040100u);
                    st16_dev(v, rmy, v16 + (f & 3) * 1024, so + (f >> 2) * 4096u);
                }
            }
            WAITV(0);
            if (lane == 0) __hip_atomic_store((gu32*)(my + OFF_RDY) + st, call, __ATOMIC_RELAXED, __HIP_MEMORY_SCOPE_AGENT);
            PUT_CALL();
            STAMPT(1);
            return;
        }
        GET_CALL();
        const u32 uw = cb * 2u + (wid - 1);
        bool ok = true;
        int si = 1;
        for (u32 j = uw; j < 7u * 256u; j += NUNP) {
            const u32 pos = j / 7u, sj = j - pos * 7u, src = (rank + 1u + sj) & 7u;
            const u32 ent = ORDER.e[pos], op = ent >> 7, tile = (ent >> 3) & 15, ks = ent & 7;
            u64 word;
            if (!wait_flags<4>(my + OFF_IFLAG + (src * 256u + pos) * 64u, call, tmo, lane, word)) { ok = false; break; }
            STAMPT(si); ++si;
            u32 b0, b1; u64 m0, m1;
            flag_decode(word, 0, b0, m0); flag_decode(word, 1, b1, m1);
            FENCE();
            const u32 blk = (src * 256u + pos) * 2u;
            Blk A, B;
            blk_load(A, rmy, v16, OFF_IPK + blk * 3072u, OFF_IOV + blk * 2048u, m0);
            blk_load(B, rmy, v16, OFF_IPK + blk * 3072u + 3072u, OFF_IOV + blk * 2048u + 2048u, m1);
            u32 oa[16], ob[16];
            blk_decode(A, b0, m0, lane, oa); blk_decode(B, b1, m1, lane, ob);
            const u32 st = (op * 16u + tile) * 64u + 8u + ks * 7u + sj;
            const u32 so = OFF_UNP + st * 8192u;
            if (op == 0) {
#pragma unroll
                for (int f = 0; f < 4; ++f) {
                    st16_dev((u32x4){oa[4 * f], oa[4 * f + 1], oa[4 * f + 2], oa[4 * f + 3]}, rmy, v16 + f * 1024, so);
                    st16_dev((u32x4){ob[4 * f], ob[4 * f + 1], ob[4 * f + 2], ob[4 * f + 3]}, rmy, v16 + f * 1024, so + 4096u);
                }
            } else {
#pragma unroll
                for (int f = 0; f < 8; ++f) {
                    const int q = 4 * (f >> 1) + (f & 1);
                    st16_dev((u32x4){oa[q], oa[q + 2], ob[q], ob[q + 2]}, rmy, v16 + (f & 3) * 1024, so + (f >> 2) * 4096u);
                }
            }
            WAITV(0);
            if (lane == 0) __hip_atomic_store((gu32*)(my + OFF_RDY) + st, call, __ATOMIC_RELAXED, __HIP_MEMORY_SCOPE_AGENT);
            STAMPT(si); ++si;
        }
        if (lane == 0) __hip_atomic_fetch_add((gu32*)(my + OFF_UDONE), 1u, __ATOMIC_RELAXED, __HIP_MEMORY_SCOPE_AGENT);
        {
            const u32 unit = uw;
            const u32 rq = unit & 15, cq = unit >> 4;
            const u32 row0 = rq * 32u, col0 = (cq & 1) * 2048u + (cq >> 1) * 128u;
            u64 word = 0;
            if (ok) ok = wait_flags<8>(my + OFF_OFLAG + unit * 64u, call, tmo, lane, word);
            STAMPT(16);
            if (uni(__hip_atomic_load(tmo, __ATOMIC_RELAXED, __HIP_MEMORY_SCOPE_AGENT)) == call) ok = false;
            const rsrc_t ro = mkrs(P.out, 512u * 4096u * 2u);
            const u32 ovoff = (lane >> 4) * 4u * 8192u + (lane & 15) * 16u;
            const u32 osoff = row0 * 8192u + col0 * 2u;
            if (ok) {
                u32 bs[4]; u64 mk[4];
#pragma unroll
                for (int i = 0; i < 4; ++i) flag_decode(word, i, bs[i], mk[i]);
                FENCE();
                Blk bl[4];
#pragma unroll
                for (int i = 0; i < 4; ++i) blk_load(bl[i], rmy, v16, OFF_OPK + (unit * 4u + i) * 3072u, OFF_OOV + (unit * 4u + i) * 2048u, mk[i]);
                u32 res[2][16];
#pragma unroll
                for (int wc = 0; wc < 2; ++wc) {
                    u32 p0[16], p1[16];
                    blk_decode(bl[wc], bs[wc], mk[wc], lane, p0);
                    blk_decode(bl[2 + wc], bs[2 + wc], mk[2 + wc], lane, p1);
#pragma unroll
                    for (int i = 0; i < 16; ++i) {
                        const float y0 = __builtin_bit_cast(float, p0[i] << 16) + __builtin_bit_cast(float, p1[i] << 16);
                        const float y1 = __builtin_bit_cast(float, p0[i] & 0xFFFF0000u) + __builtin_bit_cast(float, p1[i] & 0xFFFF0000u);
                        res[wc][i] = cvt_pk_bf16(silu(y0), silu(y1));
                    }
                }
#pragma unroll
                for (int r = 0; r < 8; ++r)
                    st16_dev((u32x4){res[0][2 * r], res[0][2 * r + 1], res[1][2 * r], res[1][2 * r + 1]}, ro, ovoff, osoff + (16u * (r >> 2) + (r & 3)) * 8192u);
            } else {
#pragma unroll
                for (int r = 0; r < 8; ++r)
                    st16_dev((u32x4){0x7FC07FC0u, 0x7FC07FC0u, 0x7FC07FC0u, 0x7FC07FC0u}, ro, ovoff, osoff + (16u * (r >> 2) + (r & 3)) * 8192u);
            }
            STAMPT(17);
        }
        return;
    }

    GET_CALL();
    const u32 gb = blockIdx.x - NCOMM;
    const u32 xcd = gb & 7, idx = gb >> 3;
    const u32 rt_ = (xcd >> 1) * 4 + (idx & 3), ct = (idx >> 2) * 2 + (xcd & 1);
    const rsrc_t rs = mkrs(my + OFF_UNP + ((wid >> 1) ? (16u + ct) : rt_) * (NK * 8192u) + (wid & 1) * 4096u, NK * 8192u);
    char* const lw = lds + wid * 4096;
#define ISSUE1(ks, slot, i) __builtin_amdgcn_raw_ptr_buffer_load_lds(rs, (LAS void*)(lw + (slot) * 16384), 16, v16, (ks) * 8192, (i) * 1024, 16)
    const gu32* const fx = (const gu32*)(my + OFF_RDY) + rt_ * 64u + lane;
    const gu32* const fw = (const gu32*)(my + OFF_RDY) + (16u + ct) * 64u + lane;
    int avail = 0;
    bool bad = false;
    auto need = [&](int n) {
        if (wid == 0 && avail < n) {
            const u64 t0 = rt();
            for (u32 it = 1;; ++it) {
                const u32 a = __hip_atomic_load(fx, __ATOMIC_RELAXED, __HIP_MEMORY_SCOPE_AGENT);
                const u32 b = __hip_atomic_load(fw, __ATOMIC_RELAXED, __HIP_MEMORY_SCOPE_AGENT);
                const u64 miss = __builtin_amdgcn_ballot_w64(min(a, b) < call);
                avail = miss ? __builtin_ctzll(miss) : 64;
                if (avail >= n) break;
                if ((it & 63u) == 0 && timed_out(tmo, call, t0)) { avail = 64; break; }
                __builtin_amdgcn_s_sleep(2);
            }
        }
    };
    const int wr = wid >> 1, wc = wid & 1;
    const char* const la = lds + wr * 4096 + lane * 16;
    const char* const lb = lds + 8192 + wc * 4096 + lane * 16;
    f32x4 acc[4][4];
#pragma unroll
    for (int m = 0; m < 4; ++m)
#pragma unroll
        for (int n = 0; n < 4; ++n) acc[m][n] = (f32x4){0.f, 0.f, 0.f, 0.f};
    bf16x8 A0[4], B0[4], A1[4], B1[4];
#define RDA(A, i, slot) A[i] = *(const bf16x8*)(la + (slot) * 16384 + (i) * 1024)
#define RDB(B, i, slot) B[i] = *(const bf16x8*)(lb + (slot) * 16384 + (i) * 1024)
#define M1(A, B, m, n) acc[m][n] = __builtin_amdgcn_mfma_f32_16x16x32_bf16(A[m], B[n], acc[m][n], 0, 0, 0)
    need(NS);
    FENCE(); __builtin_amdgcn_s_barrier(); FENCE();
    STAMPT(1);
#pragma unroll
    for (int s = 0; s < NS; ++s) { ISSUE1(s, s, 0); ISSUE1(s, s, 1); ISSUE1(s, s, 2); ISSUE1(s, s, 3); }
    asm volatile("s_waitcnt vmcnt(%0)" :: "n"(4 * (NS - 1)) : "memory");
    __builtin_amdgcn_s_barrier(); FENCE();
#pragma unroll
    for (int i = 0; i < 4; ++i) { RDA(A0, i, 0); RDB(B0, i, 0); }
#define STEP(Ac, Bc, An, Bn, s_, srd, swr) do { \
        const int ks_ = (s_) + NS < NK ? (s_) + NS : NK - 1; \
        need(ks_ + 1); \
        __builtin_amdgcn_s_waitcnt(0xC07F); \
        asm volatile("s_waitcnt vmcnt(%0)" :: "n"(4 * (NS - 2)) : "memory"); \
        __builtin_amdgcn_s_barrier(); FENCE(); SB(); \
        M1(Ac, Bc, 0, 0); SB(); RDA(An, 0, srd); SB(); M1(Ac, Bc, 0, 1); SB(); RDB(Bn, 0, srd); SB(); \
        M1(Ac, Bc, 0, 2); SB(); RDB(Bn, 1, srd); SB(); M1(Ac, Bc, 0, 3); SB(); ISSUE1(ks_, swr, 0); SB(); \
        M1(Ac, Bc, 1, 0); SB(); RDB(Bn, 2, srd); SB(); M1(Ac, Bc, 1, 1); SB(); RDB(Bn, 3, srd); SB(); \
        M1(Ac, Bc, 1, 2); SB(); RDA(An, 1, srd); SB(); M1(Ac, Bc, 1, 3); SB(); ISSUE1(ks_, swr, 1); SB(); \
        M1(Ac, Bc, 2, 0); SB(); RDA(An, 2, srd); SB(); M1(Ac, Bc, 2, 1); SB(); RDA(An, 3, srd); SB(); \
        M1(Ac, Bc, 2, 2); SB(); M1(Ac, Bc, 2, 3); SB(); ISSUE1(ks_, swr, 2); SB(); \
        M1(Ac, Bc, 3, 0); SB(); M1(Ac, Bc, 3, 1); SB(); ISSUE1(ks_, swr, 3); SB(); \
        M1(Ac, Bc, 3, 2); SB(); M1(Ac, Bc, 3, 3); SB(); \
    } while (0)
    static_assert(NS == 4, "the loop below is unrolled over the four ring slots");
    for (int s = 0; s < NK; s += 4) {
        STEP(A0, B0, A1, B1, s, 1, 0);
        STEP(A1, B1, A0, B0, s + 1, 2, 1);
        STEP(A0, B0, A1, B1, s + 2, 3, 2);
        STEP(A1, B1, A0, B0, s + 3, 0, 3);
    }
    WAITV(0);
    bad = uni(__hip_atomic_load(tmo, __ATOMIC_RELAXED, __HIP_MEMORY_SCOPE_AGENT)) == call;
    {
        const u32 owner = rt_ >> 1;
        char* dst = P.ws[0];
#pragma unroll
        for (int p = 1; p < NR; ++p) dst = owner == (u32)p ? P.ws[p] : dst;
        const rsrc_t rd = mkrs(dst, WS_USED);
        u64 word = 0;
        const u32 unit0 = ((ct * 2u + rb) * 4u + (ra * 2u + (rt_ & 1))) * 4u + wr * 2u;
#pragma unroll
        for (int h = 0; h < 2; ++h) {
            u32 lo[8], hi[8];
#pragma unroll
            for (int k = 0; k < 8; ++k) {
                const int m = 2 * h + (k >> 2), j = k & 3;
                u32 a0 = cvt_pk_bf16(acc[m][0][j], acc[m][1][j]), a1 = cvt_pk_bf16(acc[m][2][j], acc[m][3][j]);
                if (bad) { a0 = 0x7FC07FC0u; a1 = 0x7FC07FC0u; }
                lo[k] = __builtin_amdgcn_perm(a1, a0, 0x06040200u); hi[k] = __builtin_amdgcn_perm(a1, a0, 0x07050301u);
            }
            u32x4 nib; u32 base; u64 mask;
            pack_hi(hi, nib, base, mask);
            const u32 blk = (unit0 + h) * 4u + rc * 2u + wc;
            if (mask) {
                if ((mask >> lane) & 1) {
                    st16_sys((u32x4){hi[0], hi[1], hi[2], hi[3]}, rd, v16, OFF_OOV + blk * 2048u);
                    st16_sys((u32x4){hi[4], hi[5], hi[6], hi[7]}, rd, v16 + 1024, OFF_OOV + blk * 2048u);
                }
            }
            st16_sys((u32x4){lo[0], lo[1], lo[2], lo[3]}, rd, v16, OFF_OPK + blk * 3072u);
            st16_sys((u32x4){lo[4], lo[5], lo[6], lo[7]}, rd, v16 + 1024, OFF_OPK + blk * 3072u);
            st16_sys(nib, rd, v16 + 2048, OFF_OPK + blk * 3072u);
            const u64 wd = flag_word(call, base, mask, lane & 1);
            if (h == 0 || (lane & 2)) word = wd;
        }
        STAMPT(12);
        WAITV(0);
        if (lane < 4)
            __hip_atomic_store((gu64*)(dst + OFF_OFLAG + (unit0 + (lane >> 1)) * 64u + (rc * 2u + wc) * 16u + (lane & 1) * 8u), word,
                               __ATOMIC_RELAXED, __HIP_MEMORY_SCOPE_SYSTEM);
        STAMPT(13);
    }
}

}

extern "C" void kernel_launch(int rank, int nranks, ncclComm_t comm,
                              hipStream_t stream,
                              void* const* d_in, const int* in_sizes, int n_in,
                              void* d_out, int out_size,
                              const int* in_dtypes, int out_dtype,
                              void* const* d_out_all,
                              void* const* d_ws_all, size_t ws_bytes) {
    if (nranks != NR || (unsigned)rank >= (unsigned)NR || n_in != 2 || ws_bytes < WS_NEED ||
        in_sizes[0] != 4096 * 512 || in_sizes[1] != 512 * 4096 || out_size != 512 * 4096) {
        fprintf(stderr, "kernel_launch: unexpected problem shape\n");
        return;
    }
    Params P;
    for (int p = 0; p < NR; ++p) P.ws[p] = (char*)d_ws_all[p];
    P.x = (const char*)d_in[0];
    P.w = (const char*)d_in[1];
    P.out = (char*)d_out;
    P.rank = rank;
    P.pad = 0;
    hipLaunchKernelGGL(fused, dim3(NCOMM + NGEMM), dim3(256), 0, stream, P);
    HIP_CHECK(hipGetLastError());
}
```

```cpp
#include <hip/hip_runtime.h>
#include <rccl/rccl.h>
#include <cstdio>

#define HIP_CHECK(expr) do { hipError_t _e = (expr); if (_e != hipSuccess) { \
    fprintf(stderr, "HIP error %s at %s:%d\n", hipGetErrorString(_e), __FILE__, __LINE__); \
    return; } } while (0)

namespace {

typedef unsigned u32;
typedef unsigned long long u64;
typedef unsigned u32x4 __attribute__((ext_vector_type(4)));
typedef unsigned u32x2 __attribute__((ext_vector_type(2)));
typedef unsigned short u16x2 __attribute__((ext_vector_type(2)));
typedef short bf16x8 __attribute__((ext_vector_type(8)));
typedef float f32x4 __attribute__((ext_vector_type(4)));
#define LAS __attribute__((address_space(3)))
#define GAS __attribute__((address_space(1)))
typedef GAS u32 gu32;
typedef GAS u64 gu64;
typedef __amdgpu_buffer_rsrc_t rsrc_t;

constexpr int NR = 8;
constexpr int NCOMM = 256;
constexpr int NGEMM = 256;
#define NPW 24
#define PWPC 3
#define NRS 28
#define RWPC 4
#define CPRIO 3
#define LSLEEP 4
#define PSTAG 24
#define RFAST 12
#define RSLOW 25
#define REARLY 1
#define RBASE 40
constexpr int NK = 64;
constexpr u64 DEADLINE = 50000000ull;

constexpr u32 OFF_CNT = 0;
constexpr u32 OFF_TMO = 16384;
constexpr u32 OFF_TKI = 16384 + 1024;
constexpr u32 OFF_TKO = 16384 + 2048;
constexpr u32 OFF_TOK = 196608;
constexpr u32 OFF_RDY = 32768;
constexpr u32 OFF_IHD = 65536;
constexpr u32 IHD_PAR = 32768;
constexpr u32 OFF_OHD = 131072;
constexpr u32 OFF_UNP = 1u << 20;
constexpr u32 OFF_IPK = 17u << 20;
constexpr u32 IPK_PAR = 6u << 20;
constexpr u32 OFF_IOV = 29u << 20;
constexpr u32 IOV_PAR = 4u << 20;
constexpr u32 OFF_OPK = 37u << 20;
constexpr u32 OFF_OOV = 43u << 20;
constexpr u32 WS_USED = 47u << 20;
constexpr u32 WS_NEED = WS_USED;

struct Params {
    char* ws[NR];
    const char* x;
    const char* w;
    char* out;
    int rank;
    int pad;
};

struct Order { u32 e[256]; };
constexpr Order make_order() {
    constexpr int SLOW0 = 16 - 4 * ((NPW + 8 * PWPC - 1) / (8 * PWPC));
    int key[17][8] = {};
    for (int ct = 0; ct < 16; ++ct) {
        int t = 8 * ct - RBASE - (ct < REARLY ? 8 * ct : 8 * (REARLY - 1) * (16 - ct) / (17 - REARLY));
        for (int ks = 7; ks >= 0; --ks) { t -= (ct >= SLOW0 && t <= 0) ? RSLOW : RFAST; key[ct][ks] = t; }
    }
    for (int ks = 0; ks < 8; ++ks) { int m = key[0][ks]; for (int ct = 1; ct < 16; ++ct) m = key[ct][ks] < m ? key[ct][ks] : m; key[16][ks] = m; }
    Order o{};
    int n = 0, next[17] = {};
    while (n < 256) {
        int best = -1;
        for (int i = 16; i >= 0; --i) if (next[i] < 8 && (best < 0 || key[i][next[i]] < key[best][next[best]])) best = i;
        if (best == 16) for (int t = 0; t < 16; ++t) o.e[n++] = (u32)(t << 3 | next[16]);
        else o.e[n++] = (u32)(128 | best << 3 | next[best]);
        ++next[best];
    }
    return o;
}
__constant__ const Order ORDER = make_order();

__device__ __forceinline__ u64 rt() { return __builtin_amdgcn_s_memrealtime(); }
__device__ __forceinline__ u32 uni(u32 v) { return __builtin_amdgcn_readfirstlane(v); }
__device__ __forceinline__ u32 rdl(u32 v, u32 l) { return __builtin_amdgcn_readlane(v, l); }
__device__ __forceinline__ rsrc_t mkrs(const void* p, u32 bytes) { return __builtin_amdgcn_make_buffer_rsrc((void*)p, 0, bytes, 0x00020000); }
__device__ __forceinline__ u32x4 ld16(rsrc_t r, u32 voff, u32 soff) { return __builtin_amdgcn_raw_buffer_load_b128(r, voff, soff, 0); }
__device__ __forceinline__ u32x2 ld8(rsrc_t r, u32 voff, u32 soff) { return __builtin_amdgcn_raw_buffer_load_b64(r, voff, soff, 0); }
__device__ __forceinline__ u32x4 ld16_sys(rsrc_t r, u32 voff, u32 soff) { return __builtin_amdgcn_raw_buffer_load_b128(r, voff, soff, 17); }
__device__ __forceinline__ u32x2 ld8_sys(rsrc_t r, u32 voff, u32 soff) { return __builtin_amdgcn_raw_buffer_load_b64(r, voff, soff, 17); }
__device__ __forceinline__ void st16_sys(u32x4 v, rsrc_t r, u32 voff, u32 soff) { __builtin_amdgcn_raw_buffer_store_b128(v, r, voff + soff, 0, 17); }
__device__ __forceinline__ void st16_dev(u32x4 v, rsrc_t r, u32 voff, u32 soff) { __builtin_amdgcn_raw_buffer_store_b128(v, r, voff + soff, 0, 16); }
__device__ __forceinline__ void st4_dev(u32 v, rsrc_t r, u32 voff, u32 soff) { __builtin_amdgcn_raw_buffer_store_b32(v, r, voff, soff, 16); }
__device__ __forceinline__ void st8_sys(u32x2 v, rsrc_t r, u32 voff, u32 soff) { __builtin_amdgcn_raw_buffer_store_b64(v, r, voff, soff, 17); }
#define FENCE() asm volatile("" ::: "memory")
#define WAITV(n) do { asm volatile("s_waitcnt vmcnt(" #n ")" ::: "memory"); __builtin_amdgcn_s_waitcnt(0xF70 | (n)); } while (0)
#define WAITL0() do { asm volatile("s_waitcnt lgkmcnt(0)" ::: "memory"); __builtin_amdgcn_s_waitcnt(0xC07F); } while (0)
#define SB() __builtin_amdgcn_sched_barrier(0)
#define BAR() do { FENCE(); __builtin_amdgcn_s_barrier(); FENCE(); } while (0)

#define STAMPT(i) do {} while (0)
#define LOGDECL() do {} while (0)
#define LOGU(u) do {} while (0)
#define LOGOUT() do {} while (0)

__device__ __forceinline__ u32 pkmax(u32 a, u32 b) {
    return __builtin_bit_cast(u32, __builtin_elementwise_max(__builtin_bit_cast(u16x2, a), __builtin_bit_cast(u16x2, b)));
}
__device__ __forceinline__ u32 wave_max(u32 v) {
    v = max(v, (u32)__builtin_amdgcn_update_dpp(0, (int)v, 0x111, 0xf, 0xf, true));
    v = max(v, (u32)__builtin_amdgcn_update_dpp(0, (int)v, 0x112, 0xf, 0xf, true));
    v = max(v, (u32)__builtin_amdgcn_update_dpp(0, (int)v, 0x114, 0xf, 0xf, true));
    v = max(v, (u32)__builtin_amdgcn_update_dpp(0, (int)v, 0x118, 0xf, 0xf, true));
    return max(max(rdl(v, 15), rdl(v, 31)), max(rdl(v, 47), rdl(v, 63)));
}

struct Hdr { u32 base, w0, w1; };
__device__ __forceinline__ void pack_hi(const u32 (&hi)[8], int lane, u32x4& nib, Hdr& h) {
    u32 e[8], ma = 0, mb = 0;
#pragma unroll
    for (int k = 0; k < 8; ++k) { e[k] = hi[k] & 0x7F7F7F7Fu; ma = pkmax(ma, e[k]); mb = pkmax(mb, e[k] << 8); }
    const u32 m2 = pkmax(ma, mb);
    const u32 top = wave_max(max(m2 >> 24, (m2 >> 8) & 0xFFu));
    h.base = top > 7u ? top - 7u : 0u;
    const u32 kk = 0x80808080u - h.base * 0x01010101u;
    u32 t[8], ok = 0x80808080u;
#pragma unroll
    for (int k = 0; k < 8; ++k) { e[k] += kk; ok &= e[k]; t[k] = (hi[k] & 0x80808080u) | (e[k] & 0x07070707u); }
#pragma unroll
    for (int k = 0; k < 4; ++k) nib[k] = t[2 * k] | __builtin_amdgcn_alignbit(t[2 * k + 1], t[2 * k + 1], 28);
    h.w0 = 0; h.w1 = 0;
    u64 lanes = __builtin_amdgcn_ballot_w64(ok != 0x80808080u);
    if (lanes) {
        u32 bits = 0;
#pragma unroll
        for (int k = 0; k < 8; ++k) bits |= (~e[k] & 0x80808080u) >> (7 - k);
        u32 ex[3] = {0, 0, 0}, cnt = 0;
#pragma unroll
        for (int i = 0; i < 3; ++i) {
            if (lanes) {
                const u32 L = __builtin_ctzll(lanes), m = rdl(bits, L), idx = __builtin_ctz(m), k = idx & 7, b = idx >> 3;
                u32 v = rdl(hi[0], L);
#pragma unroll
                for (int q = 1; q < 8; ++q) { const u32 vq = rdl(hi[q], L); v = k == (u32)q ? vq : v; }
                ex[i] = ((v >> (8 * b)) & 0xFFu) << 11 | (k >> 2) << 10 | (b >> 1) << 9 | L << 3 | (k & 3) << 1 | (b & 1);
                if ((u32)lane == L) bits &= bits - 1;
                if ((m & (m - 1)) == 0) lanes &= lanes - 1;
                cnt = i + 1;
            }
        }
        if (lanes) cnt = 4;
        h.w0 = ex[0] | ex[1] << 19;
        h.w1 = ex[2] | (ex[1] >> 13) << 19 | cnt << 25;
    }
}
__device__ __forceinline__ u32x2 hdr_word(const Hdr& h, u32 call, int which) {
    return (u32x2){which ? h.w1 : h.w0, (call << 4) | (which ? h.base >> 4 : h.base & 15u)};
}
__device__ __forceinline__ u32 hdr_exc(u32 w0, u32 w1, u32 i) {
    const u32 e0 = w0 & 0x7FFFFu, e1 = (w0 >> 19) | ((w1 >> 19) & 63u) << 13, e2 = w1 & 0x7FFFFu;
    const u32 m0 = 0u - (u32)(i == 0), m1 = 0u - (u32)(i == 1);
    return (e0 & m0) | (e1 & m1) | (e2 & ~(m0 | m1));
}

__device__ __forceinline__ bool timed_out(gu32* tmo, u32 call, u64 t0) {
    if (uni(__hip_atomic_load(tmo, __ATOMIC_RELAXED, __HIP_MEMORY_SCOPE_AGENT)) == call) return true;
    if (rt() - t0 > DEADLINE) { __hip_atomic_store(tmo, call, __ATOMIC_RELAXED, __HIP_MEMORY_SCOPE_AGENT); return true; }
    return false;
}
__device__ __forceinline__ bool hdr_valid(u64 word, u32 call) { return __builtin_amdgcn_ballot_w64((u32)(word >> 36) < (call & 0xFFFFFFFu)) == 0; }
__device__ __forceinline__ bool hdr_wait(const gu64* f, u64& word, u32 call, gu32* tmo) {
    const u64 t0 = rt();
    for (u32 it = 1;; ++it) {
        word = __hip_atomic_load(f, __ATOMIC_RELAXED, __HIP_MEMORY_SCOPE_SYSTEM);
        if (hdr_valid(word, call)) return true;
        if ((it & 63u) == 0 && timed_out(tmo, call, t0)) return false;
        __builtin_amdgcn_s_sleep(1);
    }
}
__device__ __forceinline__ void unpack_hi(u32x4 nb, u64 word, int i, rsrc_t rs, u32 ov, int lane, u32 (&hi)[8]) {
    const u32 w1 = rdl((u32)word, 2 * i + 1);
    const u32 b4 = ((rdl((u32)(word >> 32), 2 * i) & 15u) | (rdl((u32)(word >> 32), 2 * i + 1) & 15u) << 4) * 0x01010101u;
#pragma unroll
    for (int k = 0; k < 4; ++k) {
        hi[2 * k] = (nb[k] & 0x87878787u) + b4;
        hi[2 * k + 1] = (__builtin_amdgcn_alignbit(nb[k], nb[k], 4) & 0x87878787u) + b4;
    }
    const u32 cn = w1 >> 25;
    if (cn >= 4) {
        const u32x4 h0 = ld16_sys(rs, lane * 16u, ov), h1 = ld16_sys(rs, lane * 16u + 1024u, ov);
        WAITV(0);
#pragma unroll
        for (int k = 0; k < 4; ++k) { hi[k] = h0[k]; hi[4 + k] = h1[k]; }
    } else {
        const u32 w0 = rdl((u32)word, 2 * i);
        for (u32 j = 0; j < cn; ++j) {
            const u32 ex = hdr_exc(w0, w1, j);
            const u32 sh = (((ex >> 9) & 1) * 2u + (ex & 1)) * 8u, kk = ((ex >> 10) & 1) * 4u + ((ex >> 1) & 3);
            const u32 m = (u32)lane == ((ex >> 3) & 63u) ? 0xFFu << sh : 0u, v = (ex >> 11) << sh;
#pragma unroll
            for (int k = 0; k < 8; ++k) { const u32 mk = kk == (u32)k ? m : 0u; hi[k] = (hi[k] & ~mk) | (v & mk); }
        }
    }
}
__device__ __forceinline__ void store_frags(const u32 (&hi)[8], u32x4 l0, u32x4 l1, rsrc_t rs, u32 voff) {
#pragma unroll
    for (int f = 0; f < 4; ++f) {
        u32x4 v;
#pragma unroll
        for (int p = 0; p < 4; ++p) v[p] = __builtin_amdgcn_perm(hi[4 * (f >> 1) + p], (f >> 1) ? l1[p] : l0[p], (f & 1) ? 0x07030602u : 0x05010400u);
        st16_dev(v, rs, voff + f * 1024u, 0);
    }
}

__device__ __forceinline__ void unit_load(u32 (&in)[16], rsrc_t rx, rsrc_t rw, u32 ent, u32 ab, u32 c, u32 half, int lane) {
    const u32 tile = (ent >> 3) & 15, ks = ent & 7;
    if (ent >> 7) {
        const u32 so = uni((256u * c + 32u * ks) * 8192u + (2048u * ab + 128u * tile + 64u * half) * 2u);
        const u32 vo = (lane >> 4) * 8u * 8192u + (lane & 7) * 16u + (lane & 8);
#pragma unroll
        for (int r = 0; r < 8; ++r) { const u32x2 v = ld8(rw, vo, so + r * 8192u); in[2 * r] = v[0]; in[2 * r + 1] = v[1]; }
    } else {
        const u32 so = uni((512u * (tile >> 1) + 256u * ab + 128u * (tile & 1) + 64u * half) * 1024u + (256u * c + 32u * ks) * 2u);
        const u32 vo = (lane & 15) * 1024u + (lane >> 4) * 16u;
#pragma unroll
        for (int m = 0; m < 4; ++m) { const u32x4 v = ld16(rx, vo, so + m * 16384u); in[4 * m] = v[0]; in[4 * m + 1] = v[1]; in[4 * m + 2] = v[2]; in[4 * m + 3] = v[3]; }
    }
}
__device__ __forceinline__ void unit_split(const u32 (&in)[16], u32 op, u32 (&lo)[8], u32 (&hi)[8]) {
    if (op) {
#pragma unroll
        for (int k = 0; k < 8; ++k) {
            const u32 a0 = in[4 * (k & 3) + (k >> 2)], a1 = in[4 * (k & 3) + 2 + (k >> 2)];
            lo[k] = __builtin_amdgcn_perm(a1, a0, 0x06020400u); hi[k] = __builtin_amdgcn_perm(a1, a0, 0x07030501u);
        }
    } else {
#pragma unroll
        for (int k = 0; k < 8; ++k) {
            const u32 a0 = in[8 * (k >> 2) + (k & 3)], a1 = in[8 * (k >> 2) + 4 + (k & 3)];
            lo[k] = __builtin_amdgcn_perm(a1, a0, 0x06040200u); hi[k] = __builtin_amdgcn_perm(a1, a0, 0x07050301u);
        }
    }
}

__device__ __forceinline__ u32 cvt_pk_bf16(float lo, float hi) {
    typedef __bf16 bf2 __attribute__((ext_vector_type(2)));
    typedef float f2 __attribute__((ext_vector_type(2)));
    return __builtin_bit_cast(u32, __builtin_convertvector((f2){lo, hi}, bf2));
}
__device__ __forceinline__ float silu(float y) {
    const float c = __builtin_amdgcn_fmed3f(y, -60.0f, 60.0f);
    return y * __builtin_amdgcn_rcpf(1.0f + __builtin_amdgcn_exp2f(c * -1.44269504088896341f));
}

__global__ void __launch_bounds__(512, 4) fused(Params P) {
    constexpr int NS = 5;
    __shared__ __attribute__((aligned(16))) char lds[NS * 16384];
    const int lane = threadIdx.x & 63;
    const int wid = uni(threadIdx.x >> 6);
    const u32 rank = P.rank;
    const u32 ra = rank >> 2, rb = (rank >> 1) & 1, rc = rank & 1;
    char* const my = P.ws[rank];
    gu32* const tmo = (gu32*)(my + OFF_TMO);
    gu32* const cnt = (gu32*)(my + OFF_CNT) + blockIdx.x * 8 + wid;
    const u32 cntv = __hip_atomic_load(cnt, __ATOMIC_RELAXED, __HIP_MEMORY_SCOPE_AGENT);
#define PUT_CALL() do { if (lane == 0) __hip_atomic_store(cnt, call, __ATOMIC_RELAXED, __HIP_MEMORY_SCOPE_AGENT); } while (0)
    STAMPT(0);
    const rsrc_t rmy = mkrs(my, WS_USED);
    const u32 v16 = lane * 16u;
    const u32 hvoff = lane < 2 ? lane * 8u : 0x80000000u;
    const u32 fvoff = lane == 0 ? 0u : 0x80000000u;

    if (blockIdx.x < NCOMM) {
        const u32 cb = blockIdx.x;
        const u32 call = uni(cntv) + 1u;
        const u32 src = (cb >> 3) & 7;
        const bool puller = wid < 4;
        const u32 wl = wid & 3;
        const u32 wi = ((puller ? (3u - (cb >> 6)) * PWPC : (cb >> 5) * RWPC) + wl) * 8u + (cb & 7);
        if (puller ? (wl >= PWPC || wi >= NPW || src == rank) : (wl >= RWPC || wi >= NRS)) return;
        PUT_CALL();
        __builtin_amdgcn_s_setprio(CPRIO);
        bool ok = true;
        LOGDECL();
        if (puller) {
            char* const sw = P.ws[src];
            const rsrc_t rs = mkrs(sw, WS_USED);
            const u32 par = call & 1, sj = 8u + ((src - rank - 1u) & 7u);
            gu32* const tk = (gu32*)(my + OFF_TKI + src * 128u);
            const gu64* const hb = (const gu64*)(sw + OFF_IHD + par * IHD_PAR) + (lane & 1);
#define HB1 (hb + b1 * 2u)
#define IBLK(u_, e_) (((u_) >> 1) * 8u + (((e_) >> 7) ? rb : ra) * 4u + rc * 2u + ((u_) & 1))
            u32 u0 = wi, u1 = wi + NPW;
            u32 e0 = ORDER.e[u0 >> 1], e1 = ORDER.e[u1 >> 1];
            u32 b0 = IBLK(u0, e0), b1 = IBLK(u1, e1);
            u64 h0, h1;
            ok = hdr_wait(hb + b0 * 2u, h0, call, tmo);
            STAMPT(2);
            u32x4 l0, l1, nb; u32 tkt = 0;
#define IISSUE() do { FENCE(); const u32 so_ = OFF_IPK + par * IPK_PAR + b0 * 3072u; \
                l0 = ld16_sys(rs, v16, so_); l1 = ld16_sys(rs, v16 + 1024, so_); nb = ld16_sys(rs, v16 + 2048, so_); \
                h1 = __hip_atomic_load(HB1, __ATOMIC_RELAXED, __HIP_MEMORY_SCOPE_SYSTEM); \
                tkt = __builtin_amdgcn_raw_ptr_buffer_atomic_add_i32(1, rmy, fvoff, OFF_TKI + src * 128u, 0); FENCE(); } while (0)
            if (ok) {
                IISSUE();
                WAITV(0);
                for (;;) {
                    u32 hi[8];
                    unpack_hi(nb, h0, 0, rs, OFF_IOV + par * IOV_PAR + b0 * 2048u, lane, hi);
                    const u32 kpos = sj + (e0 & 7) * 7u;
                    store_frags(hi, l0, l1, rmy, v16 + OFF_UNP + (((e0 >> 3) * 64u + kpos) * 2u + (u0 & 1)) * 4096u);
                    const u32 ro = OFF_RDY + (((e0 >> 3) * 2u + (u0 & 1)) * 64u + kpos) * 4u;
                    const u32 su = u0; (void)su;
                    const u32 u2 = uni(tkt) - (call - 1u) * 512u + 2u * NPW;
                    bool pub = false;
                    if (u1 >= 512u || !hdr_valid(h1, call)) {
                        WAITV(0);
                        st4_dev(call, rmy, fvoff, ro);
                        pub = true;
                        LOGU(src * 512u + su);
                        if (u1 >= 512u) break;
                        if (!hdr_wait(hb + b1 * 2u, h1, call, tmo)) { ok = false; break; }
                    }
                    if (u2 >= 512u) st4_dev(call, rmy, fvoff, OFF_TOK + (src * 64u + wi) * 64u);
                    u0 = u1; e0 = e1; b0 = b1; h0 = h1;
                    u1 = u2; e1 = ORDER.e[(u1 >> 1) & 255]; b1 = IBLK(u1 & 511u, e1);
                    IISSUE();
                    WAITV(5);
                    st4_dev(call, rmy, fvoff, ro);
                    if (!pub) LOGU(src * 512u + su);
                    (void)pub;
                    WAITV(1);
                }
            }
            (void)tk;
            STAMPT(3);
        } else {
            const u32 st = src & 3;
            char* const s0 = P.ws[st * 2u]; char* const s1 = P.ws[st * 2u + 1u];
            const rsrc_t r0 = mkrs(s0, WS_USED), r1 = mkrs(s1, WS_USED);
            const rsrc_t ro = mkrs(P.out, 512u * 4096u * 2u);
            const gu64* const hb = (const gu64*)(((lane & 2) ? s1 : s0) + OFF_OHD + rank * 4096u) + (lane & 1);
            const u32 ovoff = ((lane >> 4) * 4u + ((lane >> 3) & 1)) * 8192u + (lane & 7) * 16u;
            u32 v0 = wi, v1 = v0 + NRS;
            u64 h0 = 0, h1 = 0;
            {
                const u32 ts = st * 2u + ((wi >> 3) & 1);
                const gu32* tok = (const gu32*)(my + OFF_TOK + ((ts == rank ? ts ^ 1u : ts) * 64u + wi % NPW) * 64u);
                const u64 t0 = rt();
                for (u32 it = 1;; ++it) {
                    const u32 tv = __hip_atomic_load(tok, __ATOMIC_RELAXED, __HIP_MEMORY_SCOPE_AGENT);
                    if (uni(tv) >= call) break;
                    if ((it & 63u) == 0 && timed_out(tmo, call, t0)) { ok = false; break; }
                    __builtin_amdgcn_s_sleep(4);
                }
            }
            STAMPT(2);
            u32x4 na0, na1, nan_, nc0, nc1, ncn; u32 tkt, tm;
#define OISSUE(v_, vn_) do { FENCE(); const u32 so_ = OFF_OPK + (rank * 256u + (v_)) * 3072u; \
                nan_ = ld16_sys(r0, v16 + 2048, so_); ncn = ld16_sys(r1, v16 + 2048, so_); \
                na0 = ld16_sys(r0, v16, so_); nc0 = ld16_sys(r1, v16, so_); \
                na1 = ld16_sys(r0, v16 + 1024, so_); nc1 = ld16_sys(r1, v16 + 1024, so_); \
                h1 = __hip_atomic_load(hb + ((vn_) & 255u) * 2u, __ATOMIC_RELAXED, __HIP_MEMORY_SCOPE_SYSTEM); \
                tkt = __builtin_amdgcn_raw_ptr_buffer_atomic_add_i32(1, rmy, fvoff, OFF_TKO + st * 128u, 0); \
                tm = __hip_atomic_load(tmo, __ATOMIC_RELAXED, __HIP_MEMORY_SCOPE_AGENT); FENCE(); } while (0)
            if (ok) ok = hdr_wait(hb + v0 * 2u, h0, call, tmo);
            OISSUE(v0, v1);
            WAITV(0);
            for (;;) {
                if (!ok) h0 = 0;
                const u32 ov = OFF_OOV + (rank * 256u + v0) * 2048u;
                u32 ha[8], hc[8], res[16];
                unpack_hi(nan_, h0, 0, r0, ov, lane, ha);
                unpack_hi(ncn, h0, 1, r1, ov, lane, hc);
                u32x4 a0 = na0, a1 = na1, c0 = nc0, c1 = nc1;
                asm volatile("" : "+v"(a0), "+v"(a1), "+v"(c0), "+v"(c1));
                const bool bad = !ok || uni(tm) == call;
                const u32 v2 = uni(tkt) - (call - 1u) * 256u + 2u * NRS;
                const u64 hn = h1;
                const bool early = v1 < 256u && hdr_valid(hn, call);
                if (early) OISSUE(v1, v2);
#pragma unroll
                for (int k = 0; k < 8; ++k) {
                    float y[4];
#pragma unroll
                    for (int e = 0; e < 4; ++e) {
                        const u32 sel = 0x04000c0cu + e * 0x01010000u;
                        y[e] = silu(__builtin_bit_cast(float, __builtin_amdgcn_perm(ha[k], k < 4 ? a0[k & 3] : a1[k & 3], sel)) +
                                    __builtin_bit_cast(float, __builtin_amdgcn_perm(hc[k], k < 4 ? c0[k & 3] : c1[k & 3], sel)));
                    }
                    res[2 * k] = cvt_pk_bf16(y[0], y[1]); res[2 * k + 1] = cvt_pk_bf16(y[2], y[3]);
                }
                if (bad) {
#pragma unroll
                    for (int i = 0; i < 16; ++i) res[i] = 0x7FC07FC0u;
                }
                const u32 row0 = (st >> 1) * 256u + ((v0 >> 3) & 1) * 128u + ((v0 >> 2) & 1) * 64u + (v0 & 1) * 32u;
                const u32 col0 = (st & 1) * 2048u + (v0 >> 4) * 128u + ((v0 >> 1) & 1) * 64u;
#pragma unroll
                for (int k = 0; k < 8; k += 2) {
                    u32x4 o;
#pragma unroll
                    for (int d = 0; d < 2; ++d) {
                        const u32 x = res[2 * k + d], y = res[2 * k + 2 + d];
                        o[d] = __builtin_amdgcn_update_dpp(x, y, 0x128, 0xf, 0xc, false);
                        o[2 + d] = __builtin_amdgcn_update_dpp(y, x, 0x128, 0xf, 0x3, false);
                    }
                    st16_dev(o, ro, ovoff, (row0 + 16u * (k >> 2) + (k & 3)) * 8192u + col0 * 2u);
                }
                LOGU(4096u + st * 256u + v0);
                if (v1 >= 256u) break;
                h0 = hn;
                if (!early) {
                    if (ok) ok = hdr_wait(hb + v1 * 2u, h0, call, tmo);
                    OISSUE(v1, v2);
                    WAITV(0);
                } else {
                    WAITV(4);
                }
                v0 = v1; v1 = v2;
            }
        }
        STAMPT(4);
        LOGOUT();
        return;
    }

    const u32 gb = blockIdx.x - NCOMM;
    u32 call;
        {
            const rsrc_t rx = mkrs(P.x, 4096u * 512u * 2u), rw = mkrs(P.w, 512u * 4096u * 2u);
            const u32 blk = wid * 256u + gb, sub = blk & 7, ent = ORDER.e[blk >> 3], op = ent >> 7;
            for (int i = 0; i < wid; ++i) __builtin_amdgcn_s_sleep(PSTAG);
            u32 in[16], lo[8], hi[8];
            unit_load(in, rx, rw, ent, sub >> 2, (sub >> 1) & 1, sub & 1, lane);
            FENCE();
            call = uni(cntv) + 1u;
            const u32 par = call & 1;
            unit_split(in, op, lo, hi);
            const u32x4 l0 = {lo[0], lo[1], lo[2], lo[3]}, l1 = {lo[4], lo[5], lo[6], lo[7]};
            const u32 so = OFF_IPK + par * IPK_PAR + blk * 3072u;
            st16_sys(l0, rmy, v16, so); st16_sys(l1, rmy, v16 + 1024, so);
            FENCE();
            u32x4 nib; Hdr h;
            pack_hi(hi, lane, nib, h);
            st16_sys(nib, rmy, v16 + 2048, so);
            if (h.w1 >> 27) {
                const u32 ov = OFF_IOV + par * IOV_PAR + blk * 2048u;
                st16_sys((u32x4){hi[0], hi[1], hi[2], hi[3]}, rmy, v16, ov); st16_sys((u32x4){hi[4], hi[5], hi[6], hi[7]}, rmy, v16 + 1024, ov);
            }
            const bool own = (sub >> 1) == (op ? rb : ra) * 2u + rc;
            if (own) store_frags(hi, l0, l1, rmy, v16 + OFF_UNP + (((ent >> 3) * 64u + (ent & 7)) * 2u + (sub & 1)) * 4096u);
            WAITV(0);
            st8_sys(hdr_word(h, call, lane & 1), rmy, hvoff, OFF_IHD + par * IHD_PAR + blk * 16u);
            if (own) st4_dev(call, rmy, fvoff, OFF_RDY + (((ent >> 3) * 2u + (sub & 1)) * 64u + (ent & 7)) * 4u);
            PUT_CALL();
            STAMPT(14);
        }
    const u32 xcd = gb & 7, idx = gb >> 3;
    const u32 rt_ = (xcd >> 1) * 4 + (idx & 3), ct = (idx >> 2) * 2 + (xcd & 1);

    if (wid >= 4) {
        const u32 l = wid - 4;
        const u32 tl = (l >> 1) ? 16u + ct : rt_;
        const rsrc_t rs = mkrs(my + OFF_UNP + tl * (NK * 8192u) + (l & 1) * 4096u, NK * 8192u);
        const gu32* const fl = (const gu32*)(my + OFF_RDY) + (tl * 2u + (l & 1)) * 64u + lane;
        int avail = 0;
        auto issue = [&](int s, int slot) {
            const int s1 = s < NK ? s : NK - 1;
            if (avail <= s1) {
                const u64 t0 = rt();
                for (u32 it = 1;; ++it) {
                    const u64 miss = __builtin_amdgcn_ballot_w64(__hip_atomic_load(fl, __ATOMIC_RELAXED, __HIP_MEMORY_SCOPE_AGENT) < call);
                    avail = miss ? __builtin_ctzll(miss) : 64;
                    if (avail > s1) break;
                    if ((it & 63u) == 0 && timed_out(tmo, call, t0)) { avail = 64; break; }
                    __builtin_amdgcn_s_sleep(LSLEEP);
                }
                FENCE();
            }
            LAS void* const d = (LAS void*)((LAS char*)lds + slot * 16384 + l * 4096);
            __builtin_amdgcn_raw_ptr_buffer_load_lds(rs, d, 16, v16, s1 * 8192, 0, 16);
            __builtin_amdgcn_raw_ptr_buffer_load_lds(rs, d, 16, v16, s1 * 8192, 1024, 16);
            __builtin_amdgcn_raw_ptr_buffer_load_lds(rs, d, 16, v16, s1 * 8192, 2048, 16);
            __builtin_amdgcn_raw_ptr_buffer_load_lds(rs, d, 16, v16, s1 * 8192, 3072, 16);
        };
        issue(0, 0); issue(1, 1); issue(2, 2); issue(3, 3);
        asm volatile("s_waitcnt vmcnt(12)" ::: "memory"); BAR();
        STAMPT(1);
        asm volatile("s_waitcnt vmcnt(8)" ::: "memory"); BAR();
        int slot = NS - 1;
        for (int s = 0; s < NK; ++s) {
            issue(s + 4, slot);
            slot = slot + 1 == NS ? 0 : slot + 1;
            asm volatile("s_waitcnt vmcnt(8)" ::: "memory"); BAR();
        }
        asm volatile("s_waitcnt vmcnt(0)" ::: "memory");
        return;
    }

    const int wr = wid >> 1, wc = wid & 1;
    const char* const la = lds + wr * 4096 + lane * 16;
    const char* const lb = lds + 8192 + wc * 4096 + lane * 16;
    f32x4 acc[4][4];
#pragma unroll
    for (int m = 0; m < 4; ++m)
#pragma unroll
        for (int n = 0; n < 4; ++n) acc[m][n] = (f32x4){0.f, 0.f, 0.f, 0.f};
    bf16x8 A[4], B0[4], B1[4];
#define RDA(i) A[i] = *(const bf16x8*)(la + so + (i) * 1024)
#define RDB(B, i) B[i] = *(const bf16x8*)(lb + so + (i) * 1024)
#define M1(B, m, n) acc[m][n] = __builtin_amdgcn_mfma_f32_16x16x32_bf16(A[m], B[n], acc[m][n], 0, 0, 0)
    int so = 0;
    BAR();
#pragma unroll
    for (int i = 0; i < 4; ++i) { RDA(i); RDB(B0, i); }
    BAR();
#define STEP(Bc, Bn) do { so = so + 16384 == NS * 16384 ? 0 : so + 16384; SB(); \
        M1(Bc, 0, 0); SB(); RDB(Bn, 0); SB(); M1(Bc, 0, 1); SB(); RDB(Bn, 1); SB(); M1(Bc, 0, 2); SB(); M1(Bc, 0, 3); SB(); RDA(0); SB(); \
        M1(Bc, 1, 0); SB(); RDB(Bn, 2); SB(); M1(Bc, 1, 1); SB(); RDB(Bn, 3); SB(); M1(Bc, 1, 2); SB(); M1(Bc, 1, 3); SB(); RDA(1); SB(); \
        M1(Bc, 2, 0); SB(); M1(Bc, 2, 1); SB(); M1(Bc, 2, 2); SB(); M1(Bc, 2, 3); SB(); RDA(2); SB(); \
        M1(Bc, 3, 0); SB(); M1(Bc, 3, 1); SB(); M1(Bc, 3, 2); SB(); M1(Bc, 3, 3); SB(); RDA(3); SB(); \
        BAR(); \
    } while (0)
    for (int s = 0; s < NK; s += 2) {
        STEP(B0, B1);
        STEP(B1, B0);
    }
    const u32 tm = __hip_atomic_load(tmo, __ATOMIC_RELAXED, __HIP_MEMORY_SCOPE_AGENT);
    {
        const u32 blk0 = (rt_ >> 1) * 256u + ((ct * 2u + (rt_ & 1)) * 4u + wid) * 2u;
        u32x2 word[2];
#pragma unroll
        for (int h = 0; h < 2; ++h) {
            u32 lo[8], hi[8];
#pragma unroll
            for (int k = 0; k < 8; ++k) {
                const int m = 2 * h + (k >> 2), j = k & 3;
                const u32 a0 = cvt_pk_bf16(acc[m][0][j], acc[m][1][j]), a1 = cvt_pk_bf16(acc[m][2][j], acc[m][3][j]);
                lo[k] = __builtin_amdgcn_perm(a1, a0, 0x06040200u); hi[k] = __builtin_amdgcn_perm(a1, a0, 0x07050301u);
            }
            const u32 so_ = OFF_OPK + (blk0 + h) * 3072u;
            st16_sys((u32x4){lo[0], lo[1], lo[2], lo[3]}, rmy, v16, so_);
            st16_sys((u32x4){lo[4], lo[5], lo[6], lo[7]}, rmy, v16 + 1024, so_);
            u32x4 nib; Hdr hd;
            pack_hi(hi, lane, nib, hd);
            st16_sys(nib, rmy, v16 + 2048, so_);
            if (hd.w1 >> 27) {
                st16_sys((u32x4){hi[0], hi[1], hi[2], hi[3]}, rmy, v16, OFF_OOV + (blk0 + h) * 2048u);
                st16_sys((u32x4){hi[4], hi[5], hi[6], hi[7]}, rmy, v16 + 1024, OFF_OOV + (blk0 + h) * 2048u);
            }
            word[h] = hdr_word(hd, call, lane & 1);
        }
        STAMPT(12);
        WAITV(0);
        if (uni(tm) != call) st8_sys((lane & 2) ? word[1] : word[0], rmy, lane < 4 ? lane * 8u : 0x80000000u, OFF_OHD + blk0 * 16u);
        STAMPT(13);
    }
}

}

extern "C" void kernel_launch(int rank, int nranks, ncclComm_t comm,
                              hipStream_t stream,
                              void* const* d_in, const int* in_sizes, int n_in,
                              void* d_out, int out_size,
                              const int* in_dtypes, int out_dtype,
                              void* const* d_out_all,
                              void* const* d_ws_all, size_t ws_bytes) {
    if (nranks != NR || (unsigned)rank >= (unsigned)NR || n_in != 2 || ws_bytes < WS_NEED ||
        in_sizes[0] != 4096 * 512 || in_sizes[1] != 512 * 4096 || out_size != 512 * 4096) {
        fprintf(stderr, "kernel_launch: unexpected problem shape\n");
        return;
    }
    Params P;
    for (int p = 0; p < NR; ++p) P.ws[p] = (char*)d_ws_all[p];
    P.x = (const char*)d_in[0];
    P.w = (const char*)d_in[1];
    P.out = (char*)d_out;
    P.rank = rank;
    P.pad = 0;
    hipLaunchKernelGGL(fused, dim3(NCOMM + NGEMM), dim3(512), 0, stream, P);
    HIP_CHECK(hipGetLastError());
}
```

```cpp
#include <hip/hip_runtime.h>
#include <rccl/rccl.h>
#include <cstdio>

#define HIP_CHECK(expr) do { hipError_t _e = (expr); if (_e != hipSuccess) { \
    fprintf(stderr, "HIP error %s at %s:%d\n", hipGetErrorString(_e), __FILE__, __LINE__); \
    return; } } while (0)

namespace {

typedef unsigned u32;
typedef unsigned long long u64;
typedef unsigned u32x4 __attribute__((ext_vector_type(4)));
typedef unsigned u32x3 __attribute__((ext_vector_type(3)));
typedef unsigned u32x2 __attribute__((ext_vector_type(2)));
typedef short i16x2 __attribute__((ext_vector_type(2)));
typedef unsigned short u16x2 __attribute__((ext_vector_type(2)));
typedef short bf16x8 __attribute__((ext_vector_type(8)));
typedef float f32x4 __attribute__((ext_vector_type(4)));
#define LAS __attribute__((address_space(3)))
#define GAS __attribute__((address_space(1)))
typedef GAS u32 gu32;
typedef GAS u64 gu64;
typedef __amdgpu_buffer_rsrc_t rsrc_t;

constexpr int NR = 8;
constexpr int NCOMM = 256;
constexpr int NGEMM = 256;
#define NPW 24
#define PWPC 3
#define NRS 48
#define RWPC 3
#define CPRIO 3
#define NPUSH 48
#define PUSHD 44
#define RFIRST NPW
#define RDELAY 8
#define LPRIO 2
#define LAHEAD 10
#define LWIN 12
#define LWINB 6
#define LSLEEP 4
#define PSTAG 24
#define PSTEP 1
#define TOKE NPW
#define RFAST 11
#define RSLOW 25
#define REARLY 3
#define RBASE 40
constexpr u32 NU = 480;
constexpr u64 DEADLINE = 50000000ull;

constexpr u32 BLK = 2944;
constexpr u32 OFF_CNT = 0;
constexpr u32 OFF_TMO = 16384;
constexpr u32 OFF_TKI = 16384 + 1024;
constexpr u32 OFF_TKO = 16384 + 2048;
constexpr u32 OFF_PCN = 16384 + 4096;
constexpr u32 OFF_IFL = 24576;
constexpr u32 OFF_TOK = 65536;
constexpr u32 OFF_IFA = 262144;
constexpr u32 OFF_OFL = 196608;
constexpr u32 OFF_DFL = 229376;
constexpr u32 ID_XD = 2048, ID_WD = 2112, NID = 2624;
constexpr u32 OFF_RDY = 1u << 20;
constexpr u32 OFF_IMG = 2u << 20;
constexpr u32 OFF_IPK = 23u << 20;
constexpr u32 IPK_PAR = 6u << 20;
constexpr u32 OFF_IOV = 35u << 20;
constexpr u32 IOV_PAR = 4u << 20;
constexpr u32 OFF_OPK = 43u << 20;
constexpr u32 OFF_OOV = 49u << 20;
constexpr u32 OFF_DIA = 53u << 20;
constexpr u32 OFF_PSH = 61u << 20;
constexpr u32 PSH_PAR = 3u << 19;
constexpr u32 WS_USED = 64u << 20;
static_assert(OFF_RDY + NID * 256u <= OFF_IMG && OFF_IMG + NID * 8192u <= OFF_IPK && 2048u * BLK <= IPK_PAR, "layout");
constexpr u32 WS_NEED = WS_USED;

struct Params {
    const char* x;
    const char* w;
    char* my;
    char* out;
    int rank;
    int pad;
    char* ws[NR];
};

struct Order { u32 e[256]; };
constexpr Order make_order() {
    constexpr int SLOW0 = 16 - 4 * ((NPW + 8 * PWPC - 1) / (8 * PWPC));
    int key[17][8] = {};
    for (int ct = 0; ct < 16; ++ct) {
        int t = 8 * ct - RBASE - (ct < REARLY ? 8 * ct : 8 * (REARLY - 1) * (16 - ct) / (17 - REARLY));
        for (int ks = 7; ks >= 0; --ks) { t -= (ct >= SLOW0 && t <= 0) ? RSLOW : RFAST; key[ct][ks] = t; }
    }
    for (int ks = 0; ks < 8; ++ks) { int m = key[0][ks]; for (int ct = 1; ct < 16; ++ct) m = key[ct][ks] < m ? key[ct][ks] : m; key[16][ks] = m; }
    Order o{};
    int n = 0, next[17] = {};
    while (n < 240) {
        int best = -1;
        for (int i = 16; i >= 0; --i) if (next[i] < 8 && (best < 0 || key[i][next[i]] < key[best][next[best]])) best = i;
        if (best == 16) for (int t = 0; t < 14; ++t) o.e[n++] = (u32)(t << 3 | next[16]);
        else o.e[n++] = (u32)(128 | best << 3 | next[best]);
        ++next[best];
    }
    for (int ks = 0; ks < 8; ++ks) { o.e[n++] = (u32)(14 << 3 | ks); o.e[n++] = (u32)(15 << 3 | ks); }
    return o;
}
__constant__ const Order ORDER = make_order();
__device__ __forceinline__ u32 entry_fix(u32 e, u32 src) {
    const u32 own = e - 112u + src * 16u, oth = e + (e >= src * 16u ? 16u : 0u), x = e >= 112u ? own : oth;
    return e >= 128u ? e : x;
}
__device__ __forceinline__ u32 entry(u32 u, u32 src) { return entry_fix(ORDER.e[(u >> 1) & 255u], src); }

__device__ __forceinline__ u64 rt() { return __builtin_amdgcn_s_memrealtime(); }
__device__ __forceinline__ u32 uni(u32 v) { return __builtin_amdgcn_readfirstlane(v); }
__device__ __forceinline__ u32 rdl(u32 v, u32 l) { return __builtin_amdgcn_readlane(v, l); }
__device__ __forceinline__ rsrc_t mkrs(const void* p, u32 bytes) { return __builtin_amdgcn_make_buffer_rsrc((void*)p, 0, bytes, 0x00020000); }
__device__ __forceinline__ u32x4 ld16(rsrc_t r, u32 voff, u32 soff) { return __builtin_amdgcn_raw_buffer_load_b128(r, voff, soff, 0); }
__device__ __forceinline__ u32x2 ld8(rsrc_t r, u32 voff, u32 soff) { return __builtin_amdgcn_raw_buffer_load_b64(r, voff, soff, 0); }
__device__ __forceinline__ u32x4 ld16_sys(rsrc_t r, u32 voff, u32 soff) { return __builtin_amdgcn_raw_buffer_load_b128(r, voff, soff, 17); }
__device__ __forceinline__ u32x2 ld8_sys(rsrc_t r, u32 voff, u32 soff) { return __builtin_amdgcn_raw_buffer_load_b64(r, voff, soff, 17); }
__device__ __forceinline__ void st16_sys(u32x4 v, rsrc_t r, u32 voff, u32 soff) { __builtin_amdgcn_raw_buffer_store_b128(v, r, voff + soff, 0, 17); }
__device__ __forceinline__ void st16_dev(u32x4 v, rsrc_t r, u32 voff, u32 soff) { __builtin_amdgcn_raw_buffer_store_b128(v, r, voff + soff, 0, 16); }
__device__ __forceinline__ void st4_dev(u32 v, rsrc_t r, u32 voff, u32 soff) { __builtin_amdgcn_raw_buffer_store_b32(v, r, voff, soff, 16); }
__device__ __forceinline__ void st8_sys(u32x2 v, rsrc_t r, u32 voff, u32 soff) { __builtin_amdgcn_raw_buffer_store_b64(v, r, voff, soff, 17); }
#define FENCE() asm volatile("" ::: "memory")
#define WAITV(n) do { asm volatile("s_waitcnt vmcnt(" #n ")" ::: "memory"); __builtin_amdgcn_s_waitcnt(0xF70 | (n)); } while (0)
#define WAITL0() do { asm volatile("s_waitcnt lgkmcnt(0)" ::: "memory"); __builtin_amdgcn_s_waitcnt(0xC07F); } while (0)
#define SB() __builtin_amdgcn_sched_barrier(0)
#define BAR() do { FENCE(); __builtin_amdgcn_s_barrier(); FENCE(); } while (0)

#define STAMPT(i) do {} while (0)
#define LOGDECL() do {} while (0)
#define LOGU(u) do {} while (0)
#define LOGOUT() do {} while (0)

__device__ __forceinline__ u32 pkmax(u32 a, u32 b) {
    return __builtin_bit_cast(u32, __builtin_elementwise_max(__builtin_bit_cast(u16x2, a), __builtin_bit_cast(u16x2, b)));
}
__device__ __forceinline__ u32 pkmaxi(u32 a, u32 b) {
    return __builtin_bit_cast(u32, __builtin_elementwise_max(__builtin_bit_cast(i16x2, a), __builtin_bit_cast(i16x2, b)));
}
struct Pk { u32x3 cs; u32 tail; bool raw; };
__device__ __forceinline__ u32 bfi(u32 m, u32 a, u32 b) { return (a & m) | (b & ~m); }
#define DPP0(v, ctrl, rm) (u32)__builtin_amdgcn_update_dpp(0, (int)(v), ctrl, rm, 0xf, true)
__device__ __forceinline__ void pack_hi(const u32 (&lo)[8], const u32 (&hi)[8], u32 m, int lane, LAS char* sc, Pk& p) {
    m = max(m, DPP0(m, 0x111, 0xf)); m = max(m, DPP0(m, 0x112, 0xf)); m = max(m, DPP0(m, 0x114, 0xf)); m = max(m, DPP0(m, 0x118, 0xf));
    const u32 T = min(min(rdl(m, 15), rdl(m, 31)), min(rdl(m, 47), rdl(m, 63)));
    const u32 L = max(T, 7u) - 7u, s = L & 1u, base = (L + s) >> 1;
    const u32 k4 = ((128u - base) & 127u) * 0x01010101u, s4 = s * 0x01010101u;
    u32 c7f = 0x7F7F7F7Fu, c7c = 0x7C7C7C7Cu;
    asm volatile("" : "+s"(c7f), "+s"(c7c));
    u32 x[8], cc[2], sg = 0, bits = 0;
#pragma unroll
    for (int k = 0; k < 8; ++k) {
        const u32 d = (hi[k] & c7f) + ((lo[k] >> 7) & s4) + k4;
        x[k] = d & c7c;
        cc[k >> 2] = (k & 3) ? bfi(((1u << (2 * (k & 3))) - 1u) * 0x01010101u, cc[k >> 2], d << (2 * (k & 3))) : d;
        const u32 low = (0xFFu >> k) * 0x01010101u;
        sg = k ? bfi(low, hi[k] >> k, sg) : hi[k];
        bits = k ? bfi(low, (x[k] + c7c) >> k, bits) : x[k] + c7c;
    }
    p.cs = (u32x3){cc[0], cc[1], sg};
    *(LAS u32x4*)(sc + lane * 16) = (u32x4){x[0], x[1], x[2], x[3]};
    *(LAS u32x4*)(sc + 1024 + lane * 16) = (u32x4){x[4], x[5], x[6], x[7]};
    const u32 cnt = __builtin_popcount(bits);
    u32 sum = cnt;
    sum += DPP0(sum, 0x111, 0xf); sum += DPP0(sum, 0x112, 0xf); sum += DPP0(sum, 0x114, 0xf); sum += DPP0(sum, 0x118, 0xf);
    sum += DPP0(sum, 0x142, 0xa); sum += DPP0(sum, 0x143, 0xc);
    const u32 n = rdl(sum, 63);
    p.raw = n > 62u;
    u32 ent = 0;
    if (!p.raw) {
        LAS unsigned short* q = (LAS unsigned short*)(sc + 2048) + (sum - cnt);
        const u32 l5 = (u32)lane << 5;
#pragma unroll
        for (int j = 0; j < 3; ++j)
            if (bits) { q[j] = (unsigned short)(l5 | __builtin_ctz(bits)); bits &= bits - 1u; }
        q += 3;
        while (bits) { *q++ = (unsigned short)(l5 | __builtin_ctz(bits)); bits &= bits - 1u; }
        FENCE();
        const u32 e = ((LAS unsigned short*)(sc + 2048))[lane];
        const u32 pos = ((~e & 4u) << 8) | ((e >> 1) & 0x3F0u) | ((~e & 3u) << 2) | ((e >> 3) & 3u);
        ent = pos | (u32)*(LAS unsigned char*)(sc + pos) << 9;
    }
    const u32 hd = lane == 63 ? (base - s) | s << 7 : min(n, 63u);
    p.tail = lane >= 62 ? hd : ent;
    FENCE();
}
__device__ __forceinline__ void store_blk_cs(const Pk& p, rsrc_t rs, u32 so, int lane) {
    __builtin_amdgcn_raw_buffer_store_b96(p.cs, rs, so + 2048u + lane * 12u, 0, 17);
    __builtin_amdgcn_raw_buffer_store_b16((unsigned short)p.tail, rs, so + 2816u + lane * 2u, 0, 17);
}
__device__ __forceinline__ void store_blk_raw(const Pk& p, const u32 (&hi)[8], rsrc_t rs, u32 ov, int lane) {
    if (p.raw) {
        st16_sys((u32x4){hi[0], hi[1], hi[2], hi[3]}, rs, lane * 16u, ov);
        st16_sys((u32x4){hi[4], hi[5], hi[6], hi[7]}, rs, lane * 16u + 1024u, ov);
    }
}
struct Blk { u32x4 l0, l1; u32x3 cs; u32 tail; };
__device__ __forceinline__ void load_blk(Blk& b, rsrc_t rs, u32 so, int lane) {
    b.tail = __builtin_amdgcn_raw_buffer_load_b16(rs, 2816u + lane * 2u, so, 17);
    b.cs = __builtin_amdgcn_raw_buffer_load_b96(rs, 2048u + lane * 12u, so, 17);
    b.l0 = ld16_sys(rs, lane * 16u, so);
    b.l1 = ld16_sys(rs, lane * 16u + 1024u, so);
}
__device__ __forceinline__ void unpack_hi(const Blk& b, rsrc_t rs, u32 ov, int lane, LAS char* img, u32 (&hi)[8]) {
    const u32 cn = rdl(b.tail, 62), hb = rdl(b.tail, 63);
    if (cn > 62u) {
        const u32x4 h0 = ld16_sys(rs, lane * 16u, ov), h1 = ld16_sys(rs, lane * 16u + 1024u, ov);
        WAITV(0);
#pragma unroll
        for (int k = 0; k < 4; ++k) { hi[k] = h0[k]; hi[4 + k] = h1[k]; }
    } else {
        LAS unsigned char* const q = (LAS unsigned char*)img + (b.tail & 0x7FFu);
        if ((u32)lane < cn) *q = (unsigned char)((b.tail >> 9) & 0x7Cu);
        FENCE();
        const u32x4 d0 = *(LAS u32x4*)(img + lane * 16), d1 = *(LAS u32x4*)(img + 1024 + lane * 16);
        FENCE();
        if ((u32)lane < cn) *q = 0;
        FENCE();
        const u32 b4 = (hb & 127u) * 0x01010101u, s4 = ((hb >> 7) & 1u) * 0x01010101u;
#pragma unroll
        for (int k = 0; k < 8; ++k) {
            const u32 lo = k < 4 ? b.l0[k & 3] : b.l1[k & 3];
            const u32 u = bfi(0x03030303u, b.cs[k >> 2] >> (2 * (k & 3)), k < 4 ? d0[k & 3] : d1[k & 3]) + (~(lo >> 7) & s4) + b4;
            hi[k] = bfi(0x7F7F7F7Fu, u, b.cs[2] << k);
        }
    }
}

__device__ __forceinline__ bool timed_out(gu32* tmo, u32 call, u64 t0) {
    if (uni(__hip_atomic_load(tmo, __ATOMIC_RELAXED, __HIP_MEMORY_SCOPE_AGENT)) == call) return true;
    if (rt() - t0 > DEADLINE) { __hip_atomic_store(tmo, call, __ATOMIC_RELAXED, __HIP_MEMORY_SCOPE_AGENT); return true; }
    return false;
}
__device__ __forceinline__ void store_frags(const u32 (&hi)[8], u32x4 l0, u32x4 l1, rsrc_t rs, u32 voff) {
#pragma unroll
    for (int f = 0; f < 4; ++f) {
        u32x4 v;
#pragma unroll
        for (int p = 0; p < 4; ++p) v[p] = __builtin_amdgcn_perm(hi[4 * (f >> 1) + p], (f >> 1) ? l1[p] : l0[p], (f & 1) ? 0x07030602u : 0x05010400u);
        st16_dev(v, rs, voff + f * 1024u, 0);
    }
}

__device__ __forceinline__ void unit_load(u32 (&in)[16], rsrc_t rx, rsrc_t rw, u32 ent, u32 ab, u32 c, u32 half, int lane) {
    const u32 tile = (ent >> 3) & 15, ks = ent & 7;
    if (ent >> 7) {
        const u32 so = uni((256u * c + 32u * ks) * 8192u + (2048u * ab + 128u * tile + 64u * half) * 2u);
        const u32 vo = (lane >> 4) * 8u * 8192u + (lane & 7) * 16u + (lane & 8);
#pragma unroll
        for (int r = 0; r < 8; ++r) { const u32x2 v = ld8(rw, vo, so + r * 8192u); in[2 * r] = v[0]; in[2 * r + 1] = v[1]; }
    } else {
        const u32 so = uni((512u * (tile >> 1) + 256u * ab + 128u * (tile & 1) + 64u * half) * 1024u + (256u * c + 32u * ks) * 2u);
        const u32 vo = (lane & 15) * 1024u + (lane >> 4) * 16u;
#pragma unroll
        for (int m = 0; m < 4; ++m) { const u32x4 v = ld16(rx, vo, so + m * 16384u); in[4 * m] = v[0]; in[4 * m + 1] = v[1]; in[4 * m + 2] = v[2]; in[4 * m + 3] = v[3]; }
    }
}
__device__ __forceinline__ void unit_split(const u32 (&in)[16], u32 op, u32 (&lo)[8], u32 (&hi)[8]) {
    if (op) {
#pragma unroll
        for (int k = 0; k < 8; ++k) {
            const u32 a0 = in[4 * (k & 3) + (k >> 2)], a1 = in[4 * (k & 3) + 2 + (k >> 2)];
            lo[k] = __builtin_amdgcn_perm(a1, a0, 0x06020400u); hi[k] = __builtin_amdgcn_perm(a1, a0, 0x07030501u);
        }
    } else {
#pragma unroll
        for (int k = 0; k < 8; ++k) {
            const u32 a0 = in[8 * (k >> 2) + (k & 3)], a1 = in[8 * (k >> 2) + 4 + (k & 3)];
            lo[k] = __builtin_amdgcn_perm(a1, a0, 0x06040200u); hi[k] = __builtin_amdgcn_perm(a1, a0, 0x07050301u);
        }
    }
}

__device__ __forceinline__ u32 cvt_pk_bf16(float lo, float hi) {
    typedef __bf16 bf2 __attribute__((ext_vector_type(2)));
    typedef float f2 __attribute__((ext_vector_type(2)));
    return __builtin_bit_cast(u32, __builtin_convertvector((f2){lo, hi}, bf2));
}
typedef float f32x2 __attribute__((ext_vector_type(2)));
__device__ __forceinline__ u32 silu2(f32x2 y) {
    constexpr float LOG2E = 1.44269504088896341f;
    const f32x2 t = y * -LOG2E;
    f32x2 e;
    e.x = __builtin_amdgcn_exp2f(__builtin_amdgcn_fmed3f(t.x, -60.0f * LOG2E, 60.0f * LOG2E));
    e.y = __builtin_amdgcn_exp2f(__builtin_amdgcn_fmed3f(t.y, -60.0f * LOG2E, 60.0f * LOG2E));
    e += 1.0f;
    e.x = __builtin_amdgcn_rcpf(e.x); e.y = __builtin_amdgcn_rcpf(e.y);
    y *= e;
    return cvt_pk_bf16(y.x, y.y);
}

__global__ void __launch_bounds__(512, 4) fused(Params P) {
    constexpr int NS = 5;
    __shared__ __attribute__((aligned(16))) char lds[NS * 16384];
    const int wid = uni(threadIdx.x >> 6);
#define LANE_VARS() int tz_ = threadIdx.x; asm volatile("" : "+v"(tz_)); const int lane = tz_ & 63; const u32 v16 = lane * 16u; \
    const u32 fvoff = lane == 0 ? 0u : 0x80000000u;   (void)v16; (void)fvoff
    const u32 pc = blockIdx.x & 3, grp = (blockIdx.x >> 2) & 63, bg = (grp - NPUSH / 2) & 63u,
        u = wid >= 2 ? 126u + bg * 6u + wid : grp < NPUSH / 2 ? wid * (NPUSH / 2) + grp : NPUSH / 2 + grp + wid * (64u - NPUSH / 2);
    const u32 ent0 = ORDER.e[u >> 1];
    asm volatile("" :: "s"(ent0), "s"(P.x), "s"(P.w), "s"(P.my), "s"(P.rank));
    const u32 rank = P.rank;
    const u32 ra = rank >> 2, rb = (rank >> 1) & 1, rc = rank & 1;
    char* const my = P.my;
    gu32* const tmo = (gu32*)(my + OFF_TMO);
    gu32* const cnt = (gu32*)(my + OFF_CNT) + blockIdx.x * 8 + wid;
#define PUT_CALL() do { if (lane == 0) __hip_atomic_store(cnt, call, __ATOMIC_RELAXED, __HIP_MEMORY_SCOPE_AGENT); } while (0)
    STAMPT(0);
    const rsrc_t rmy = mkrs(my, WS_USED);

    if (blockIdx.x < NCOMM) {
        const u32 cb = blockIdx.x;
        const u32 src = (cb >> 3) & 7;
        const bool shot = cb - 64u < 128u && wid >= 4;
        const bool puller = cb >= 128u || shot;
        const u32 wi = (shot ? ((cb >> 6) - 1u) * 4u + (wid - 4) : (puller ? (3u - (cb >> 6)) * PWPC : (cb >> 5) * RWPC) + wid) * 8u + (cb & 7);
        if (shot ? (wi >= NPUSH || src == rank) : puller ? (wid >= PWPC || wi >= NPW || src == rank) : (wid >= RWPC || wi >= NRS)) return;
        LANE_VARS();
        const u32 call = uni(__hip_atomic_load(cnt, __ATOMIC_RELAXED, __HIP_MEMORY_SCOPE_AGENT)) + 1u;
        PUT_CALL();
        __builtin_amdgcn_s_setprio(CPRIO);
        bool ok = true;
        LOGDECL();
        LAS char* const img = (LAS char*)lds + wid * 4096;
#pragma unroll
        for (int i = 0; i < 4; ++i) *(LAS u32x4*)(img + i * 1024 + v16) = (u32x4){0, 0, 0, 0};
        FENCE();
        if (puller) {
            const rsrc_t rs = mkrs(P.ws[src], WS_USED);
            const u32 par = call & 1, sj = 8u + ((src - rank - 1u) & 7u);
            const gu32* const fla = (const gu32*)(my + OFF_IFA + src * 16384u);
            const gu32* const flb = (const gu32*)(my + OFF_IFL + src * 1024u + (lane & 7) * 128u);
            u64 ma = 0, mb = shot ? ~0ull : 0ull;
            auto ready = [&](u32 u) { return u >= 64u && (((u < 128u ? ma >> (u - 64u) : mb >> ((u - 128u) / 48u))) & 1) != 0; };
            auto wait = [&](u32 u) {
                const u64 t0 = rt();
                for (u32 it = 1;; ++it) {
                    if (u < 128u) {
                        if (uni(__hip_atomic_load(fla + u * 32u, __ATOMIC_RELAXED, __HIP_MEMORY_SCOPE_SYSTEM)) >= call) return true;
                    } else {
                        mb = __builtin_amdgcn_ballot_w64(__hip_atomic_load(flb, __ATOMIC_RELAXED, __HIP_MEMORY_SCOPE_SYSTEM) >= call);
                        if (ready(u)) return true;
                    }
                    if ((it & 63u) == 0 && timed_out(tmo, call, t0)) return false;
                    __builtin_amdgcn_s_sleep(1);
                }
            };
            static_assert(NPUSH + NPW >= 64 && NPUSH <= 64, "tickets start at unit 64 or later");
            u32 u0 = shot ? wi : NPUSH + wi, e0 = entry(u0, src);
            u32 b0 = (((e0 >> 7) ? rb : ra) * 2u + rc) * 512u + u0;
            const rsrc_t rd = mkrs(shot ? my + OFF_PSH + par * PSH_PAR + (src * 64u + u0) * BLK - b0 * BLK : P.ws[src] + OFF_IPK + par * IPK_PAR, 2048u * BLK);
            const u32 tvoff = shot ? 0x80000000u : fvoff;
            ok = wait(u0);
            STAMPT(2);
            Blk N; u32 tkt = 0, fa = 0, fb = 0;
#define IISSUE(b_) do { FENCE(); \
                if (!shot && lane + 64u > ((b_) & 511u) && lane + 64u >= NPUSH + NPW) fa = __hip_atomic_load(fla + (lane + 64u) * 32u, __ATOMIC_RELAXED, __HIP_MEMORY_SCOPE_SYSTEM); \
                if (~mb) fb = __hip_atomic_load(flb, __ATOMIC_RELAXED, __HIP_MEMORY_SCOPE_SYSTEM); \
                FENCE(); tkt = __builtin_amdgcn_raw_ptr_buffer_atomic_add_i32(1, rmy, tvoff, OFF_TKI + src * 128u, 0); \
                load_blk(N, rd, (b_) * BLK, lane); FENCE(); } while (0)
            if (ok) {
                IISSUE(b0);
                WAITV(0);
                bool tok = shot;
                u32 rp = 0x80000000u;
                for (;;) {
                    const u32 u1 = shot ? NU : uni(tkt) - (call - 1u) * (NU - NPUSH) + NPUSH + NPW;
                    const u32 e1 = entry(u1, src);
                    const u32 b1 = (((e1 >> 7) ? rb : ra) * 2u + rc) * 512u + (u1 & 511u);
                    ma |= __builtin_amdgcn_ballot_w64(fa >= call); mb |= __builtin_amdgcn_ballot_w64(fb >= call);
                    Blk B = N;
                    asm volatile("" : "+v"(B.l0), "+v"(B.l1), "+v"(B.cs), "+v"(B.tail));
                    if (!tok && u1 + TOKE >= NU) { st4_dev(call, rmy, fvoff, OFF_TOK + (src * 64u + wi) * 64u); tok = true; }
                    const bool early = u1 < NU && ready(u1);
                    if (early) { IISSUE(b1); WAITV(5); } else { WAITV(0); }
                    st4_dev(call, rmy, fvoff | rp, 0);
                    u32 hi[8];
                    unpack_hi(B, rs, OFF_IOV + par * IOV_PAR + b0 * 2048u, lane, img, hi);
                    const u32 ih = ((e0 >> 3) * 64u + sj + (e0 & 7) * 7u) * 2u + (u0 & 1);
                    store_frags(hi, B.l0, B.l1, rmy, v16 + OFF_IMG + ih * 4096u);
                    rp = OFF_RDY + ih * 128u;
                    LOGU(src * 512u + u0);
                    if (early) {
                        WAITV(5);
                    } else {
                        WAITV(0);
                        st4_dev(call, rmy, fvoff | rp, 0);
                        rp = 0x80000000u;
                        if (u1 >= NU) break;
                        if (!wait(u1)) { ok = false; break; }
                        IISSUE(b1);
                        WAITV(0);
                    }
                    u0 = u1; e0 = e1; b0 = b1;
                }
            }
            STAMPT(3);
        } else {
            const u32 st = src & 3;
            const rsrc_t r0 = mkrs(P.ws[st * 2u], WS_USED), r1 = mkrs(P.ws[st * 2u + 1u], WS_USED);
            const rsrc_t ro = mkrs(P.out, 512u * 4096u * 2u);
            const gu32* const fl = (const gu32*)(my + ((lane & 2) ? OFF_DFL + st * 4096u : OFF_OFL + st * 8192u + (lane & 1) * 4096u));
            const u32 ovoff = ((lane >> 4) * 4u + ((lane >> 3) & 1)) * 8192u + (lane & 7) * 16u;
            auto wait = [&](u32 v) {
                const u64 t0 = rt();
                for (u32 it = 1;; ++it) {
                    if (__builtin_amdgcn_ballot_w64(__hip_atomic_load(fl + (v >> 3) * 32u, __ATOMIC_RELAXED, __HIP_MEMORY_SCOPE_SYSTEM) < call) == 0) return true;
                    if ((it & 63u) == 0 && timed_out(tmo, call, t0)) return false;
                    __builtin_amdgcn_s_sleep(1);
                }
            };
            u32 v0 = wi;
            {
                const u32 ts = st * 2u + ((wi >> 3) & 1);
                const gu32* tok = (const gu32*)(my + OFF_TOK + ((ts == rank ? ts ^ 1u : ts) * 64u + wi % NPW) * 64u);
                const u64 t0 = rt();
                for (u32 it = 1;; ++it) {
                    const u32 tv = __hip_atomic_load(tok, __ATOMIC_RELAXED, __HIP_MEMORY_SCOPE_AGENT);
                    if (uni(tv) >= call) break;
                    if ((it & 63u) == 0 && timed_out(tmo, call, t0)) { ok = false; break; }
                    __builtin_amdgcn_s_sleep(4);
                }
            }
            STAMPT(2);
            if (wi >= RFIRST) for (int i = 0; i < RDELAY; ++i) __builtin_amdgcn_s_sleep(8);
            bool rdy = false;
            for (;;) {
                if (ok && !rdy) ok = wait(v0);
                FENCE();
                Blk A, C;
                const u32 tkt = __builtin_amdgcn_raw_ptr_buffer_atomic_add_i32(1, rmy, fvoff, OFF_TKO + st * 128u, 0);
                const u32 tm = __hip_atomic_load(tmo, __ATOMIC_RELAXED, __HIP_MEMORY_SCOPE_AGENT);
                u32x4 dg[8];
#pragma unroll
                for (int k = 0; k < 8; ++k) dg[k] = __builtin_amdgcn_raw_buffer_load_b128(rmy, v16 + (k & 3) * 1024u, OFF_DIA + (st * 256u + v0) * 8192u + (k >> 2) * 4096u, 16);
                FENCE();
                load_blk(A, r0, OFF_OPK + (rank * 256u + v0) * BLK, lane);
                load_blk(C, r1, OFF_OPK + (rank * 256u + v0) * BLK, lane);
                FENCE();
                const bool bad = !ok || uni(tm) == call;
                if (bad) { A.tail = 0; C.tail = 0; }
                const u32 v1 = uni(tkt) - (call - 1u) * 256u + NRS;
                const u32 ov = OFF_OOV + (rank * 256u + v0) * 2048u;
                u32 ha[8], hc[8], res[16];
                unpack_hi(A, r0, ov, lane, img, ha);
                unpack_hi(C, r1, ov, lane, img + 2048, hc);
#pragma unroll
                for (int k = 0; k < 8; ++k) asm volatile("" : "+v"(ha[k]), "+v"(hc[k]));
                FENCE();
                const u32 fv = __hip_atomic_load(fl + ((v1 >> 3) & 31u) * 32u, __ATOMIC_RELAXED, __HIP_MEMORY_SCOPE_SYSTEM);
                FENCE();
                const u32x4 a0 = A.l0, a1 = A.l1, c0 = C.l0, c1 = C.l1;
#pragma unroll
                for (int k = 0; k < 8; ++k) {
                    float ya[4], yc[4];
#pragma unroll
                    for (int e = 0; e < 4; ++e) {
                        const u32 sel = 0x04000c0cu + e * 0x01010000u;
                        ya[e] = __builtin_bit_cast(float, __builtin_amdgcn_perm(ha[k], k < 4 ? a0[k & 3] : a1[k & 3], sel));
                        yc[e] = __builtin_bit_cast(float, __builtin_amdgcn_perm(hc[k], k < 4 ? c0[k & 3] : c1[k & 3], sel));
                    }
                    const f32x4 yd = __builtin_bit_cast(f32x4, dg[k]);
                    res[2 * k] = silu2((f32x2){ya[0], ya[1]} + (f32x2){yc[0], yc[1]} + (f32x2){yd[0], yd[1]});
                    res[2 * k + 1] = silu2((f32x2){ya[2], ya[3]} + (f32x2){yc[2], yc[3]} + (f32x2){yd[2], yd[3]});
                }
                if (bad) {
#pragma unroll
                    for (int i = 0; i < 16; ++i) res[i] = 0x7FC07FC0u;
                }
                const u32 row0 = (st >> 1) * 256u + ((v0 >> 3) & 1) * 128u + ((v0 >> 2) & 1) * 64u + (v0 & 1) * 32u;
                const u32 col0 = (st & 1) * 2048u + (v0 >> 4) * 128u + ((v0 >> 1) & 1) * 64u;
#pragma unroll
                for (int k = 0; k < 8; k += 2) {
                    u32x4 o;
#pragma unroll
                    for (int d = 0; d < 2; ++d) {
                        const u32 x = res[2 * k + d], y = res[2 * k + 2 + d];
                        o[d] = __builtin_amdgcn_update_dpp(x, y, 0x128, 0xf, 0xc, false);
                        o[2 + d] = __builtin_amdgcn_update_dpp(y, x, 0x128, 0xf, 0x3, false);
                    }
                    st16_dev(o, ro, ovoff, (row0 + 16u * (k >> 2) + (k & 3)) * 8192u + col0 * 2u);
                }
                LOGU(4096u + st * 256u + v0);
                if (v1 >= 256u) break;
                rdy = __builtin_amdgcn_ballot_w64(fv < call) == 0;
                v0 = v1;
            }
        }
        STAMPT(4);
        LOGOUT();
        return;
    }

    const u32 gb = blockIdx.x - NCOMM;
    u32 call;
    {
        LANE_VARS();
        const rsrc_t rx = mkrs(P.x, 4096u * 512u * 2u), rw = mkrs(P.w, 512u * 4096u * 2u);
        const u32 ent = entry_fix(ent0, rank), op = ent >> 7;
        const bool push = u < NPUSH;
        if (wid > 1) { __builtin_amdgcn_s_sleep(PSTAG); for (u32 i = 0; i < bg; ++i) __builtin_amdgcn_s_sleep(PSTEP); }
        STAMPT(10);
        u32 in[16], lo[8], hi[8];
        unit_load(in, rx, rw, ent, pc >> 1, pc & 1, u & 1, lane);
        FENCE();
        call = uni(__hip_atomic_load(cnt, __ATOMIC_RELAXED, __HIP_MEMORY_SCOPE_AGENT)) + 1u;
        const u32 par = call & 1;
        const u32 g0 = (op ? (pc & 2) : (pc & 2) * 2u) + (pc & 1), g1 = g0 + (op ? 4u : 2u);
        const u32 po = OFF_PSH + par * PSH_PAR + (rank * 64u + u) * BLK;
        const rsrc_t d0 = push ? mkrs(P.ws[g0 & 7] + po, g0 == rank ? 0u : BLK) : mkrs(my + OFF_IPK + par * IPK_PAR + (pc * 512u + u) * BLK, BLK);
        const rsrc_t d1 = mkrs(P.ws[g1 & 7] + po, g1 == rank ? 0u : BLK);
        unit_split(in, op, lo, hi);
        const u32x4 l0 = {lo[0], lo[1], lo[2], lo[3]}, l1 = {lo[4], lo[5], lo[6], lo[7]};
        if (push && wid) { asm volatile("" :: "v"(l0), "v"(l1)); __builtin_amdgcn_s_sleep(PUSHD); }
        STAMPT(11);
        st16_sys(l0, d0, v16, 0); st16_sys(l1, d0, v16 + 1024, 0);
        if (push) { st16_sys(l0, d1, v16, 0); st16_sys(l1, d1, v16 + 1024, 0); }
        FENCE();
        u32 mu = in[0], mi = in[1];
#pragma unroll
        for (int i = 2; i < 16; i += 2) { mu = pkmax(mu, in[i]); mi = pkmaxi(mi, in[i + 1]); }
        mu = pkmax(mu & 0x7FFF7FFFu, mi & 0x7FFF7FFFu);
        Pk pk;
        pack_hi(lo, hi, max(mu >> 16, mu & 0xFFFFu) >> 7, lane, (LAS char*)lds + wid * 4096, pk);
        STAMPT(15);
        store_blk_cs(pk, d0, 0, lane);
        if (push) store_blk_cs(pk, d1, 0, lane);
        store_blk_raw(pk, hi, rmy, OFF_IOV + par * IOV_PAR + (pc * 512u + u) * 2048u, lane);
        const bool own = op || u >= NU || pc == ra * 2u + rc;
        const u32 tile = (ent >> 3) & 15u, kd = (pc & 1) * 8u + (ent & 7);
        const u32 ih = (op ? ID_WD + ((pc >> 1) * 16u + tile) * 16u + kd : u >= NU ? ID_XD + ((pc & 2) + (tile & 1)) * 16u + kd : tile * 64u + (ent & 7)) * 2u + (u & 1);
        if (own) store_frags(hi, l0, l1, rmy, v16 + OFF_IMG + ih * 4096u);
        const u32 dr = (lane & 3) * 2u + (pc & 1);
        u64 q0 = (u64)P.ws[pc & 1], q1 = (u64)P.ws[2 + (pc & 1)], q2 = (u64)P.ws[4 + (pc & 1)], q3 = (u64)P.ws[6 + (pc & 1)];
        asm volatile("" : "+s"(q0), "+s"(q1), "+s"(q2), "+s"(q3));
        char* const dw = (char*)((lane & 2) ? ((lane & 1) ? q3 : q2) : ((lane & 1) ? q1 : q0));
        const u32 da = (lane >> 1) & 1, db = lane & 1, ab = pc >> 1;
        WAITV(0);
        if (wid < 2 && lane < 4 && dr != rank && (op ? db : da) == ab)
            __hip_atomic_store((gu32*)(dw + OFF_IFA + rank * 16384u + u * 128u), call, __ATOMIC_RELAXED, __HIP_MEMORY_SCOPE_SYSTEM);
        if (own) st4_dev(call, rmy, fvoff, OFF_RDY + ih * 128u);
        PUT_CALL();
        STAMPT(14);
        BAR();
        if (wid == 2) {
            const u32 t = __builtin_amdgcn_raw_ptr_buffer_atomic_add_i32(1, rmy, fvoff, OFF_PCN + (bg >> 3) * 128u, 0);
            if (uni(t) == call * 32u - 1u) {
                u64 r0 = (u64)P.ws[(pc & 1) ^ 1], r1 = (u64)P.ws[2 + ((pc & 1) ^ 1)], r2 = (u64)P.ws[4 + ((pc & 1) ^ 1)], r3 = (u64)P.ws[6 + ((pc & 1) ^ 1)];
                asm volatile("" : "+s"(r0), "+s"(r1), "+s"(r2), "+s"(r3));
                char* const dv = (lane & 4) ? (char*)((lane & 2) ? ((lane & 1) ? r3 : r2) : ((lane & 1) ? r1 : r0)) : dw;
                if (lane < 8) __hip_atomic_store((gu32*)(dv + OFF_IFL + rank * 1024u + (bg >> 3) * 128u), call, __ATOMIC_RELAXED, __HIP_MEMORY_SCOPE_SYSTEM);
            }
        }
    }
    const u32 xcd = gb & 7, idx = gb >> 3;
    const u32 rt_ = (xcd >> 1) * 4 + (idx & 3), ct = (idx >> 2) * 2 + (xcd & 1);

    const int pass0 = gb - 64u < 128u ? 0 : 1;
    const u32 drt = idx & 3, dct = xcd * 4u + ((idx >> 2) & 3);
    const u32 owner = rt_ >> 1;
    u32 pv[16];

    if (wid >= 4) {
        LANE_VARS();
        const u32 l = wid - 4;
        const rsrc_t rs = mkrs(my + OFF_IMG + (l & 1) * 4096u, NID * 8192u - 4096u);
        const rsrc_t rf = mkrs(my + OFF_RDY + (l & 1) * 128u, NID * 256u - 128u);
        __builtin_amdgcn_s_setprio(LPRIO);
#define LSTAMP(s_)
        for (int pass = pass0; pass < 2; ++pass) {
            const int N = pass ? 56 : 16;
            u32 idv;
            {
                const u32 hole = (owner - rank - 1u) & 7u;
                const u32 j = lane - 8u, ks = j / 6u, r = j - ks * 6u;
                const u32 kp = hole == 7u ? min(lane + 8u, 63u) : lane < 8 ? lane : lane < 56 ? 8u + ks * 7u + r + (r >= hole) : 8u + hole;
                const u32 cube = (l >> 1) ? (kp < 8u ? ID_WD + (rb * 16u + ct) * 16u + rc * 8u + kp : (16u + ct) * 64u + kp) : rt_ * 64u + kp;
                idv = pass ? cube : ((l >> 1) ? ID_WD + dct * 16u : ID_XD + drt * 16u) + (lane & 15);
            }
            int avail = 0;
            auto window = [&](u32 n) { return (u32)(lane - avail) < n ? idv * 256u : 0x80000000u; };
            auto seen = [&](u32 f) { if (avail < 64) { const u64 miss = __builtin_amdgcn_ballot_w64(f < call) >> avail << avail; avail = miss ? (int)__builtin_ctzll(miss) : 64; } };
            auto need = [&](int s1) {
                if (avail <= s1) {
                    const u64 t0 = rt();
                    for (u32 it = 1;; ++it) {
                        FENCE();
                        seen(__builtin_amdgcn_raw_buffer_load_b32(rf, window(LWINB), 0, 16));
                        if (avail > s1) break;
                        if ((it & 63u) == 0 && timed_out(tmo, call, t0)) { avail = 64; break; }
                        __builtin_amdgcn_s_sleep(LSLEEP);
                    }
                    FENCE();
                }
            };
            int slot = 0;
            u32 so = rdl(idv, 0) * 8192u;
            auto issue = [&](int s1) {
                need(s1);
                LAS void* const d = (LAS void*)((LAS char*)lds + slot * 16384 + l * 4096);
                __builtin_amdgcn_raw_ptr_buffer_load_lds(rs, d, 16, v16, so, 0, 16);
                __builtin_amdgcn_raw_ptr_buffer_load_lds(rs, d, 16, v16, so, 1024, 16);
                __builtin_amdgcn_raw_ptr_buffer_load_lds(rs, d, 16, v16, so, 2048, 16);
                __builtin_amdgcn_raw_ptr_buffer_load_lds(rs, d, 16, v16, so, 3072, 16);
                FENCE(); SB();
                slot = slot + 1 == NS ? 0 : slot + 1;
                so = rdl(idv, s1 + 1) * 8192u;
                asm volatile("" : "+s"(so), "+s"(slot));
            };
            for (int s = 0; s < 5; ++s) issue(s);
            asm volatile("s_waitcnt vmcnt(16)" ::: "memory"); BAR();
            asm volatile("s_waitcnt vmcnt(12)" ::: "memory"); BAR();
            u32 f = 0;
            int s = 5;
            for (; s < N - 3; s += 4) {
                issue(s);
                seen(f);
                asm volatile("buffer_load_dword %0, %1, %2, 0 offen sc1" : "=v"(f) : "v"(avail < s + LAHEAD ? window(LWIN) : 0x80000000u), "s"(rf) : "memory");
                LSTAMP(s - 6);
                asm volatile("s_waitcnt vmcnt(13)" ::: "memory"); BAR();
                issue(s + 1); LSTAMP(s - 5);
                asm volatile("s_waitcnt vmcnt(13)" ::: "memory"); BAR();
                issue(s + 2); LSTAMP(s - 4);
                asm volatile("s_waitcnt vmcnt(13)" ::: "memory"); BAR();
                issue(s + 3); LSTAMP(s - 3);
                asm volatile("s_waitcnt vmcnt(12)" : "+v"(f) :: "memory"); BAR();
            }
            seen(f);
            for (; s < N; ++s) { issue(s); LSTAMP(s - 6); asm volatile("s_waitcnt vmcnt(12)" ::: "memory"); BAR(); }
            asm volatile("s_waitcnt vmcnt(8)" ::: "memory"); BAR();
            asm volatile("s_waitcnt vmcnt(4)" ::: "memory"); BAR();
            asm volatile("s_waitcnt vmcnt(0)" ::: "memory"); BAR();
            if (pass && l == 2) need(56);
            BAR(); BAR();
            LSTAMP(55);
        }
        __builtin_amdgcn_s_setprio(0);
        BAR();
#pragma unroll
        for (int i = 0; i < 4; ++i) {
            const u32x4 v = *(LAS u32x4*)((LAS char*)lds + 32768 + l * 4096 + i * 1024 + v16);
            pv[4 * i] = v[0]; pv[4 * i + 1] = v[1]; pv[4 * i + 2] = v[2]; pv[4 * i + 3] = v[3];
        }
    } else {
        LANE_VARS();
        const int la = (wid >> 1) * 4096, lb = 8192 + (wid & 1) * 4096;
        f32x4 acc[4][4];
        bf16x8 A0, A1, C0[2], C1[2], B0[4], B1[4];
#define RD(d, p, i) d = *(const LAS bf16x8*)((LAS char*)lds + v16 + s##p + (i) * 1024)
#define M1(a, B, m, n) acc[m][n] = __builtin_amdgcn_mfma_f32_16x16x32_bf16(a, B[n], acc[m][n], 0, 0, 0)
#define STEP(Cc, Bc, Cn, Bn) do { so = so + 16384 == NS * 16384 ? 0 : so + 16384; sla = so + la; slb = so + lb; asm volatile("" : "+s"(sla), "+s"(slb)); SB(); \
            M1(A0, Bc, 0, 0); SB(); RD(Bn[0], lb, 0); SB(); M1(A0, Bc, 0, 1); SB(); RD(Bn[1], lb, 1); SB(); M1(A0, Bc, 0, 2); SB(); RD(Bn[2], lb, 2); SB(); M1(A0, Bc, 0, 3); SB(); RD(A0, la, 0); SB(); \
            M1(A1, Bc, 1, 0); SB(); RD(Bn[3], lb, 3); SB(); M1(A1, Bc, 1, 1); SB(); RD(Cn[0], la, 2); SB(); M1(A1, Bc, 1, 2); SB(); RD(Cn[1], la, 3); SB(); M1(A1, Bc, 1, 3); SB(); RD(A1, la, 1); SB(); \
            M1(Cc[0], Bc, 2, 0); SB(); M1(Cc[0], Bc, 2, 1); SB(); M1(Cc[0], Bc, 2, 2); SB(); M1(Cc[0], Bc, 2, 3); SB(); \
            M1(Cc[1], Bc, 3, 0); SB(); M1(Cc[1], Bc, 3, 1); SB(); M1(Cc[1], Bc, 3, 2); SB(); M1(Cc[1], Bc, 3, 3); SB(); \
            WAITL0(); BAR(); \
        } while (0)
        for (int pass = pass0;; ++pass) {
#pragma unroll
            for (int m = 0; m < 4; ++m)
#pragma unroll
                for (int n = 0; n < 4; ++n) acc[m][n] = (f32x4){0.f, 0.f, 0.f, 0.f};
            int so = 0, sla = la, slb = lb;
            BAR();
            if (pass > pass0 && wid == 0 && v16 == 0 && __hip_atomic_load(tmo, __ATOMIC_RELAXED, __HIP_MEMORY_SCOPE_AGENT) != call)
                __hip_atomic_store((gu32*)(my + OFF_DFL + (((drt & 2) + (dct >> 4)) * 32u + (dct & 15u) * 2u + (drt & 1)) * 128u), call, __ATOMIC_RELAXED, __HIP_MEMORY_SCOPE_AGENT);
            RD(A0, la, 0); RD(A1, la, 1); RD(C0[0], la, 2); RD(C0[1], la, 3);
#pragma unroll
            for (int i = 0; i < 4; ++i) RD(B0[i], lb, i);
            WAITL0(); BAR();
            for (int s = pass ? 56 : 16; s > 0; s -= 2) {
                STEP(C0, B0, C1, B1);
                STEP(C1, B1, C0, B0);
            }
            if (pass) break;
            u32 vo = v16; asm volatile("" : "+v"(vo));
            vo += OFF_DIA + (((drt & 2) + (dct >> 4)) * 256u + (((dct & 15u) * 2u + (drt & 1)) * 4u + wid) * 2u) * 8192u;
#pragma unroll
            for (int m = 0; m < 4; ++m)
#pragma unroll
                for (int j = 0; j < 4; ++j)
                    st16_dev(__builtin_bit_cast(u32x4, (f32x4){acc[m][0][j], acc[m][1][j], acc[m][2][j], acc[m][3][j]}), rmy, vo + m * 4096u + j * 1024u, 0);
            WAITV(0);
        }
        STAMPT(9);
#pragma unroll
        for (int i = 0; i < 4; ++i) {
            u32x4 v;
#pragma unroll
            for (int e = 0; e < 4; ++e) { const int k = 2 * i + (e >> 1), m = 2 + (k >> 2), j = k & 3; v[e] = cvt_pk_bf16(acc[m][2 * (e & 1)][j], acc[m][2 * (e & 1) + 1][j]); }
            *(LAS u32x4*)((LAS char*)lds + 32768 + wid * 4096 + i * 1024 + v16) = v;
        }
        WAITL0(); BAR();
#pragma unroll
        for (int k = 0; k < 8; ++k) { pv[2 * k] = cvt_pk_bf16(acc[k >> 2][0][k & 3], acc[k >> 2][1][k & 3]); pv[2 * k + 1] = cvt_pk_bf16(acc[k >> 2][2][k & 3], acc[k >> 2][3][k & 3]); }
    }
    {
        int tz = threadIdx.x; asm volatile("" : "+v"(tz));
        const int ln = tz & 63;
        const u32 blk = owner * 256u + ((ct * 2u + (rt_ & 1)) * 4u + (wid & 3)) * 2u + (wid >> 2);
        u32 lo[8], hi[8], mu = 0, mi = 0x80008000u;
#pragma unroll
        for (int k = 0; k < 8; ++k) {
            lo[k] = __builtin_amdgcn_perm(pv[2 * k + 1], pv[2 * k], 0x06040200u); hi[k] = __builtin_amdgcn_perm(pv[2 * k + 1], pv[2 * k], 0x07050301u);
            mu = pkmax(mu, pv[2 * k]); mi = pkmaxi(mi, pv[2 * k + 1]);
        }
        mu = pkmax(mu & 0x7FFF7FFFu, mi & 0x7FFF7FFFu);
        const u32 so_ = OFF_OPK + blk * BLK;
        st16_sys((u32x4){lo[0], lo[1], lo[2], lo[3]}, rmy, ln * 16u, so_);
        st16_sys((u32x4){lo[4], lo[5], lo[6], lo[7]}, rmy, ln * 16u + 1024u, so_);
        Pk pk;
        pack_hi(lo, hi, max(mu >> 16, mu & 0xFFFFu) >> 7, ln, (LAS char*)lds + wid * 4096, pk);
        store_blk_cs(pk, rmy, so_, ln);
        store_blk_raw(pk, hi, rmy, OFF_OOV + blk * 2048u, ln);
        STAMPT(12);
        FENCE();
        const u32 tm = __hip_atomic_load(tmo, __ATOMIC_RELAXED, __HIP_MEMORY_SCOPE_AGENT);
        WAITV(0);
        BAR();
        if (tz == 0 && tm != call)
            __hip_atomic_store((gu32*)(P.ws[owner & 7] + OFF_OFL + rank * 4096u + (ct * 2u + (rt_ & 1)) * 128u), call, __ATOMIC_RELAXED, __HIP_MEMORY_SCOPE_SYSTEM);
        STAMPT(13);
    }
}

}

extern "C" void kernel_launch(int rank, int nranks, ncclComm_t comm,
                              hipStream_t stream,
                              void* const* d_in, const int* in_sizes, int n_in,
                              void* d_out, int out_size,
                              const int* in_dtypes, int out_dtype,
                              void* const* d_out_all,
                              void* const* d_ws_all, size_t ws_bytes) {
    if (nranks != NR || (unsigned)rank >= (unsigned)NR || n_in != 2 || ws_bytes < WS_NEED ||
        in_sizes[0] != 4096 * 512 || in_sizes[1] != 512 * 4096 || out_size != 512 * 4096) {
        fprintf(stderr, "kernel_launch: unexpected problem shape\n");
        return;
    }
    Params P;
    for (int p = 0; p < NR; ++p) P.ws[p] = (char*)d_ws_all[p];
    P.x = (const char*)d_in[0];
    P.w = (const char*)d_in[1];
    P.out = (char*)d_out;
    P.my = P.ws[rank];
    P.rank = rank;
    P.pad = 0;
    hipLaunchKernelGGL(fused, dim3(NCOMM + NGEMM), dim3(512), 0, stream, P);
    HIP_CHECK(hipGetLastError());
}
```

```cpp
#include <hip/hip_runtime.h>
#include <rccl/rccl.h>
#include <cstdio>

#define HIP_CHECK(expr) do { hipError_t _e = (expr); if (_e != hipSuccess) { \
    fprintf(stderr, "HIP error %s at %s:%d\n", hipGetErrorString(_e), __FILE__, __LINE__); \
    return; } } while (0)

namespace {

typedef unsigned u32;
typedef unsigned long long u64;
typedef unsigned u32x4 __attribute__((ext_vector_type(4)));
typedef unsigned u32x3 __attribute__((ext_vector_type(3)));
typedef unsigned u32x2 __attribute__((ext_vector_type(2)));
typedef short i16x2 __attribute__((ext_vector_type(2)));
typedef unsigned short u16x2 __attribute__((ext_vector_type(2)));
typedef short bf16x8 __attribute__((ext_vector_type(8)));
typedef float f32x4 __attribute__((ext_vector_type(4)));
#define LAS __attribute__((address_space(3)))
#define GAS __attribute__((address_space(1)))
typedef GAS u32 gu32;
typedef GAS u64 gu64;
typedef __amdgpu_buffer_rsrc_t rsrc_t;

constexpr int NR = 8;
constexpr int NCOMM = 256;
constexpr int NGEMM = 256;
#define NPW 22
#define PWPC 3
#define NRS 44
#define RWPC 3
#define CPRIO 3
#define NPUSH 48
#define PUSHD 36
#define RFIRST NPW
#define RDELAY 8
#define LPRIO 2
#define LAHEAD 10
#define LWIN 12
#define LWINB 6
#define LSLEEP 4
#define PSTAG 24
#define PSTAG2 8
#define PSTEP 1
#define TOKE NPW
#define RFAST 10
#define RSLOW 18
#define REARLY 3
#define RBASE 40
constexpr u32 NU = 480;
constexpr u64 DEADLINE = 50000000ull;

constexpr u32 BLK = 2944;
constexpr u32 OFF_CNT = 0;
constexpr u32 OFF_TMO = 16384;
constexpr u32 OFF_TKI = 16384 + 1024;
constexpr u32 OFF_TKO = 16384 + 2048;
constexpr u32 OFF_PCN = 16384 + 4096;
constexpr u32 OFF_IFL = 24576;
constexpr u32 OFF_TOK = 65536;
constexpr u32 OFF_IFA = 262144;
constexpr u32 OFF_OFL = 196608;
constexpr u32 OFF_DFL = 229376;
constexpr u32 ID_XD = 2048, ID_WD = 2112, NID = 2624;
constexpr u32 OFF_RDY = 1u << 20;
constexpr u32 OFF_IMG = 2u << 20;
constexpr u32 OFF_IPK = 23u << 20;
constexpr u32 IPK_PAR = 6u << 20;
constexpr u32 OFF_IOV = 35u << 20;
constexpr u32 IOV_PAR = 4u << 20;
constexpr u32 OFF_OPK = 43u << 20;
constexpr u32 OFF_OOV = 49u << 20;
constexpr u32 OFF_DIA = 53u << 20;
constexpr u32 OFF_PSH = 61u << 20;
constexpr u32 PSH_PAR = 3u << 19;
constexpr u32 WS_USED = 64u << 20;
static_assert(OFF_RDY + NID * 256u <= OFF_IMG && OFF_IMG + NID * 8192u <= OFF_IPK && 2048u * BLK <= IPK_PAR, "layout");
constexpr u32 WS_NEED = WS_USED;

struct Params {
    const char* x;
    const char* w;
    char* my;
    char* out;
    int rank;
    int pad;
    char* ws[NR];
};

struct Order { u32 e[256]; };
constexpr Order make_order() {
    constexpr int SLOW0 = 16 - 4 * ((NPW + 8 * PWPC - 1) / (8 * PWPC));
    int key[17][8] = {};
    for (int ct = 0; ct < 16; ++ct) {
        int t = 8 * ct - RBASE - (ct < REARLY ? 8 * ct : 8 * (REARLY - 1) * (16 - ct) / (17 - REARLY));
        for (int ks = 7; ks >= 0; --ks) { t -= (ct >= SLOW0 && t <= 0) ? RSLOW : RFAST; key[ct][ks] = t; }
    }
    for (int ks = 0; ks < 8; ++ks) { int m = key[0][ks]; for (int ct = 1; ct < 16; ++ct) m = key[ct][ks] < m ? key[ct][ks] : m; key[16][ks] = m; }
    Order o{};
    int n = 0, next[17] = {};
    while (n < 240) {
        int best = -1;
        for (int i = 16; i >= 0; --i) if (next[i] < 8 && (best < 0 || key[i][next[i]] < key[best][next[best]])) best = i;
        if (best == 16) for (int t = 0; t < 14; ++t) o.e[n++] = (u32)(t << 3 | next[16]);
        else o.e[n++] = (u32)(128 | best << 3 | next[best]);
        ++next[best];
    }
    for (int ks = 0; ks < 8; ++ks) { o.e[n++] = (u32)(14 << 3 | ks); o.e[n++] = (u32)(15 << 3 | ks); }
    return o;
}
__constant__ const Order ORDER = make_order();
__device__ __forceinline__ u32 entry_fix(u32 e, u32 src) {
    const u32 own = e - 112u + src * 16u, oth = e + (e >= src * 16u ? 16u : 0u), x = e >= 112u ? own : oth;
    return e >= 128u ? e : x;
}
__device__ __forceinline__ u32 entry(u32 u, u32 src) { return entry_fix(ORDER.e[(u >> 1) & 255u], src); }

__device__ __forceinline__ u64 rt() { return __builtin_amdgcn_s_memrealtime(); }
__device__ __forceinline__ u32 uni(u32 v) { return __builtin_amdgcn_readfirstlane(v); }
__device__ __forceinline__ u32 rdl(u32 v, u32 l) { return __builtin_amdgcn_readlane(v, l); }
__device__ __forceinline__ rsrc_t mkrs(const void* p, u32 bytes) { return __builtin_amdgcn_make_buffer_rsrc((void*)p, 0, bytes, 0x00020000); }
__device__ __forceinline__ u32x4 ld16(rsrc_t r, u32 voff, u32 soff) { return __builtin_amdgcn_raw_buffer_load_b128(r, voff, soff, 0); }
__device__ __forceinline__ u32x2 ld8(rsrc_t r, u32 voff, u32 soff) { return __builtin_amdgcn_raw_buffer_load_b64(r, voff, soff, 0); }
__device__ __forceinline__ u32x4 ld16_sys(rsrc_t r, u32 voff, u32 soff) { return __builtin_amdgcn_raw_buffer_load_b128(r, voff, soff, 17); }
__device__ __forceinline__ u32x2 ld8_sys(rsrc_t r, u32 voff, u32 soff) { return __builtin_amdgcn_raw_buffer_load_b64(r, voff, soff, 17); }
__device__ __forceinline__ void st16_sys(u32x4 v, rsrc_t r, u32 voff, u32 soff) { __builtin_amdgcn_raw_buffer_store_b128(v, r, voff + soff, 0, 17); }
__device__ __forceinline__ void st16_dev(u32x4 v, rsrc_t r, u32 voff, u32 soff) { __builtin_amdgcn_raw_buffer_store_b128(v, r, voff + soff, 0, 16); }
__device__ __forceinline__ void st4_dev(u32 v, rsrc_t r, u32 voff, u32 soff) { __builtin_amdgcn_raw_buffer_store_b32(v, r, voff, soff, 16); }
__device__ __forceinline__ void st8_sys(u32x2 v, rsrc_t r, u32 voff, u32 soff) { __builtin_amdgcn_raw_buffer_store_b64(v, r, voff, soff, 17); }
#define FENCE() asm volatile("" ::: "memory")
#define WAITV(n) do { asm volatile("s_waitcnt vmcnt(" #n ")" ::: "memory"); __builtin_amdgcn_s_waitcnt(0xF70 | (n)); } while (0)
#define WAITL0() do { asm volatile("s_waitcnt lgkmcnt(0)" ::: "memory"); __builtin_amdgcn_s_waitcnt(0xC07F); } while (0)
#define SB() __builtin_amdgcn_sched_barrier(0)
#define BAR() do { FENCE(); __builtin_amdgcn_s_barrier(); FENCE(); } while (0)

#define STAMPT(i) do {} while (0)
#define LOGDECL() do {} while (0)
#define LOGU(u) do {} while (0)
#define LOGOUT() do {} while (0)

__device__ __forceinline__ u32 pkmax(u32 a, u32 b) {
    return __builtin_bit_cast(u32, __builtin_elementwise_max(__builtin_bit_cast(u16x2, a), __builtin_bit_cast(u16x2, b)));
}
__device__ __forceinline__ u32 pkmaxi(u32 a, u32 b) {
    return __builtin_bit_cast(u32, __builtin_elementwise_max(__builtin_bit_cast(i16x2, a), __builtin_bit_cast(i16x2, b)));
}
struct Pk { u32x3 cs; u32 tail; bool raw; };
__device__ __forceinline__ u32 bfi(u32 m, u32 a, u32 b) { return (a & m) | (b & ~m); }
#define DPP0(v, ctrl, rm) (u32)__builtin_amdgcn_update_dpp(0, (int)(v), ctrl, rm, 0xf, true)
__device__ __forceinline__ void pack_hi(const u32 (&lo)[8], const u32 (&hi)[8], u32 m, int lane, LAS char* sc, Pk& p) {
    m = max(m, DPP0(m, 0x111, 0xf)); m = max(m, DPP0(m, 0x112, 0xf)); m = max(m, DPP0(m, 0x114, 0xf)); m = max(m, DPP0(m, 0x118, 0xf));
    const u32 T = min(min(rdl(m, 15), rdl(m, 31)), min(rdl(m, 47), rdl(m, 63)));
    const u32 L = max(T, 7u) - 7u, s = L & 1u, base = (L + s) >> 1;
    const u32 k4 = ((128u - base) & 127u) * 0x01010101u, s4 = s * 0x01010101u;
    u32 c7f = 0x7F7F7F7Fu, c7c = 0x7C7C7C7Cu;
    asm volatile("" : "+s"(c7f), "+s"(c7c));
    u32 x[8], cc[2], sg = 0, bits = 0;
#pragma unroll
    for (int k = 0; k < 8; ++k) {
        const u32 d = (hi[k] & c7f) + ((lo[k] >> 7) & s4) + k4;
        x[k] = d & c7c;
        cc[k >> 2] = (k & 3) ? bfi(((1u << (2 * (k & 3))) - 1u) * 0x01010101u, cc[k >> 2], d << (2 * (k & 3))) : d;
        const u32 low = (0xFFu >> k) * 0x01010101u;
        sg = k ? bfi(low, hi[k] >> k, sg) : hi[k];
        bits = k ? bfi(low, (x[k] + c7c) >> k, bits) : x[k] + c7c;
    }
    p.cs = (u32x3){cc[0], cc[1], sg};
    *(LAS u32x4*)(sc + lane * 16) = (u32x4){x[0], x[1], x[2], x[3]};
    *(LAS u32x4*)(sc + 1024 + lane * 16) = (u32x4){x[4], x[5], x[6], x[7]};
    const u32 cnt = __builtin_popcount(bits);
    u32 sum = cnt;
    sum += DPP0(sum, 0x111, 0xf); sum += DPP0(sum, 0x112, 0xf); sum += DPP0(sum, 0x114, 0xf); sum += DPP0(sum, 0x118, 0xf);
    sum += DPP0(sum, 0x142, 0xa); sum += DPP0(sum, 0x143, 0xc);
    const u32 n = rdl(sum, 63);
    p.raw = n > 62u;
    u32 ent = 0;
    if (!p.raw) {
        LAS unsigned short* q = (LAS unsigned short*)(sc + 2048) + (sum - cnt);
        const u32 l5 = (u32)lane << 5;
#pragma unroll
        for (int j = 0; j < 3; ++j)
            if (bits) { q[j] = (unsigned short)(l5 | __builtin_ctz(bits)); bits &= bits - 1u; }
        q += 3;
        while (bits) { *q++ = (unsigned short)(l5 | __builtin_ctz(bits)); bits &= bits - 1u; }
        FENCE();
        const u32 e = ((LAS unsigned short*)(sc + 2048))[lane];
        const u32 pos = ((~e & 4u) << 8) | ((e >> 1) & 0x3F0u) | ((~e & 3u) << 2) | ((e >> 3) & 3u);
        ent = pos | (u32)*(LAS unsigned char*)(sc + pos) << 9;
    }
    const u32 hd = lane == 63 ? (base - s) | s << 7 : min(n, 63u);
    p.tail = lane >= 62 ? hd : ent;
    FENCE();
}
__device__ __forceinline__ void store_blk_cs(const Pk& p, rsrc_t rs, u32 so, int lane) {
    __builtin_amdgcn_raw_buffer_store_b96(p.cs, rs, so + 2048u + lane * 12u, 0, 17);
    __builtin_amdgcn_raw_buffer_store_b16((unsigned short)p.tail, rs, so + 2816u + lane * 2u, 0, 17);
}
__device__ __forceinline__ void store_blk_raw(const Pk& p, const u32 (&hi)[8], rsrc_t rs, u32 ov, int lane) {
    if (p.raw) {
        st16_sys((u32x4){hi[0], hi[1], hi[2], hi[3]}, rs, lane * 16u, ov);
        st16_sys((u32x4){hi[4], hi[5], hi[6], hi[7]}, rs, lane * 16u + 1024u, ov);
    }
}
struct Blk { u32x4 l0, l1; u32x3 cs; u32 tail; };
__device__ __forceinline__ void load_blk(Blk& b, rsrc_t rs, u32 so, int lane) {
    b.tail = __builtin_amdgcn_raw_buffer_load_b16(rs, 2816u + lane * 2u, so, 17);
    b.cs = __builtin_amdgcn_raw_buffer_load_b96(rs, 2048u + lane * 12u, so, 17);
    b.l0 = ld16_sys(rs, lane * 16u, so);
    b.l1 = ld16_sys(rs, lane * 16u + 1024u, so);
}
__device__ __forceinline__ void unpack_hi(const Blk& b, rsrc_t rs, u32 ov, int lane, LAS char* img, u32 (&hi)[8]) {
    const u32 cn = rdl(b.tail, 62), hb = rdl(b.tail, 63);
    if (cn > 62u) {
        const u32x4 h0 = ld16_sys(rs, lane * 16u, ov), h1 = ld16_sys(rs, lane * 16u + 1024u, ov);
        WAITV(0);
#pragma unroll
        for (int k = 0; k < 4; ++k) { hi[k] = h0[k]; hi[4 + k] = h1[k]; }
    } else {
        LAS unsigned char* const q = (LAS unsigned char*)img + (b.tail & 0x7FFu);
        if ((u32)lane < cn) *q = (unsigned char)((b.tail >> 9) & 0x7Cu);
        FENCE();
        const u32x4 d0 = *(LAS u32x4*)(img + lane * 16), d1 = *(LAS u32x4*)(img + 1024 + lane * 16);
        FENCE();
        if ((u32)lane < cn) *q = 0;
        FENCE();
        const u32 b4 = (hb & 127u) * 0x01010101u, s4 = ((hb >> 7) & 1u) * 0x01010101u;
#pragma unroll
        for (int k = 0; k < 8; ++k) {
            const u32 lo = k < 4 ? b.l0[k & 3] : b.l1[k & 3];
            const u32 u = bfi(0x03030303u, b.cs[k >> 2] >> (2 * (k & 3)), k < 4 ? d0[k & 3] : d1[k & 3]) + (~(lo >> 7) & s4) + b4;
            hi[k] = bfi(0x7F7F7F7Fu, u, b.cs[2] << k);
        }
    }
}

__device__ __forceinline__ bool timed_out(gu32* tmo, u32 call, u64 t0) {
    if (uni(__hip_atomic_load(tmo, __ATOMIC_RELAXED, __HIP_MEMORY_SCOPE_AGENT)) == call) return true;
    if (rt() - t0 > DEADLINE) { __hip_atomic_store(tmo, call, __ATOMIC_RELAXED, __HIP_MEMORY_SCOPE_AGENT); return true; }
    return false;
}
__device__ __forceinline__ void store_frags(const u32 (&hi)[8], u32x4 l0, u32x4 l1, rsrc_t rs, u32 voff) {
#pragma unroll
    for (int f = 0; f < 4; ++f) {
        u32x4 v;
#pragma unroll
        for (int p = 0; p < 4; ++p) v[p] = __builtin_amdgcn_perm(hi[4 * (f >> 1) + p], (f >> 1) ? l1[p] : l0[p], (f & 1) ? 0x07030602u : 0x05010400u);
        st16_dev(v, rs, voff + f * 1024u, 0);
    }
}

__device__ __forceinline__ void unit_load(u32 (&in)[16], rsrc_t rx, rsrc_t rw, u32 ent, u32 ab, u32 c, u32 half, int lane) {
    const u32 tile = (ent >> 3) & 15, ks = ent & 7;
    if (ent >> 7) {
        const u32 so = uni((256u * c + 32u * ks) * 8192u + (2048u * ab + 128u * tile + 64u * half) * 2u);
        const u32 vo = (lane >> 4) * 8u * 8192u + (lane & 7) * 16u + (lane & 8);
#pragma unroll
        for (int r = 0; r < 8; ++r) { const u32x2 v = ld8(rw, vo, so + r * 8192u); in[2 * r] = v[0]; in[2 * r + 1] = v[1]; }
    } else {
        const u32 so = uni((512u * (tile >> 1) + 256u * ab + 128u * (tile & 1) + 64u * half) * 1024u + (256u * c + 32u * ks) * 2u);
        const u32 vo = (lane & 15) * 1024u + (lane >> 4) * 16u;
#pragma unroll
        for (int m = 0; m < 4; ++m) { const u32x4 v = ld16(rx, vo, so + m * 16384u); in[4 * m] = v[0]; in[4 * m + 1] = v[1]; in[4 * m + 2] = v[2]; in[4 * m + 3] = v[3]; }
    }
}
__device__ __forceinline__ void unit_split(const u32 (&in)[16], u32 op, u32 (&lo)[8], u32 (&hi)[8]) {
    if (op) {
#pragma unroll
        for (int k = 0; k < 8; ++k) {
            const u32 a0 = in[4 * (k & 3) + (k >> 2)], a1 = in[4 * (k & 3) + 2 + (k >> 2)];
            lo[k] = __builtin_amdgcn_perm(a1, a0, 0x06020400u); hi[k] = __builtin_amdgcn_perm(a1, a0, 0x07030501u);
        }
    } else {
#pragma unroll
        for (int k = 0; k < 8; ++k) {
            const u32 a0 = in[8 * (k >> 2) + (k & 3)], a1 = in[8 * (k >> 2) + 4 + (k & 3)];
            lo[k] = __builtin_amdgcn_perm(a1, a0, 0x06040200u); hi[k] = __builtin_amdgcn_perm(a1, a0, 0x07050301u);
        }
    }
}

__device__ __forceinline__ u32 cvt_pk_bf16(float lo, float hi) {
    typedef __bf16 bf2 __attribute__((ext_vector_type(2)));
    typedef float f2 __attribute__((ext_vector_type(2)));
    return __builtin_bit_cast(u32, __builtin_convertvector((f2){lo, hi}, bf2));
}
typedef float f32x2 __attribute__((ext_vector_type(2)));
__device__ __forceinline__ u32 silu2(f32x2 y) {
    constexpr float LOG2E = 1.44269504088896341f;
    const f32x2 t = y * -LOG2E;
    f32x2 e;
    e.x = __builtin_amdgcn_exp2f(__builtin_amdgcn_fmed3f(t.x, -60.0f * LOG2E, 60.0f * LOG2E));
    e.y = __builtin_amdgcn_exp2f(__builtin_amdgcn_fmed3f(t.y, -60.0f * LOG2E, 60.0f * LOG2E));
    e += 1.0f;
    e.x = __builtin_amdgcn_rcpf(e.x); e.y = __builtin_amdgcn_rcpf(e.y);
    y *= e;
    return cvt_pk_bf16(y.x, y.y);
}

#define CODE_LEN 25600u
#define KSYM "_ZN12_GLOBAL__N_15fusedENS_6ParamsE"
__global__ void __launch_bounds__(512, 4) fused(Params P) {
    constexpr int NS = 5;
    __shared__ __attribute__((aligned(16))) char lds[NS * 16384];
    const int wid = uni(threadIdx.x >> 6);
#define LANE_VARS() int tz_ = threadIdx.x; asm volatile("" : "+v"(tz_)); const int lane = tz_ & 63; const u32 v16 = lane * 16u; \
    const u32 fvoff = lane == 0 ? 0u : 0x80000000u;   (void)v16; (void)fvoff
    constexpr u32 G0 = NPUSH / 2, UA = 144u - G0, UB = UA + 128u, UC = UB + (48u - G0) * 5u;
    static_assert(G0 % 8 == 0 && G0 <= 32 && UA - 64u <= NPUSH + NPW, "whole flag groups; one look covers the ticketed early units");
    const u32 pc = blockIdx.x & 3, grp = (blockIdx.x >> 2) & 63, bg = grp >= 48u ? grp - 48u : grp >= G0 ? grp - G0 + 16u : grp + 64u - G0;
    const bool early = wid < (grp < G0 ? 2 : grp < 48u ? 3 : 0);
    const u32 u = grp < G0 ? (early ? wid * G0 + grp : UC + grp * 6u + (wid - 2))
                : grp < 48u ? (early ? G0 + grp + wid * (48u - G0) : UB + (grp - G0) * 5u + (wid - 3)) : UA + bg * 8u + wid;
    const u32 ent0 = ORDER.e[u >> 1];
    asm volatile("" :: "s"(ent0), "s"(P.x), "s"(P.w), "s"(P.my), "s"(P.rank));
    const u32 rank = P.rank;
    const u32 ra = rank >> 2, rb = (rank >> 1) & 1, rc = rank & 1;
    char* const my = P.my;
    gu32* const tmo = (gu32*)(my + OFF_TMO);
    gu32* const cnt = (gu32*)(my + OFF_CNT) + blockIdx.x * 8 + wid;
#define PUT_CALL() do { if (lane == 0) __hip_atomic_store(cnt, call, __ATOMIC_RELAXED, __HIP_MEMORY_SCOPE_AGENT); } while (0)
    STAMPT(0);
    const rsrc_t rmy = mkrs(my, WS_USED);

    if (blockIdx.x < NCOMM) {
        const u32 cb = blockIdx.x;
        const u32 src = (cb >> 3) & 7;
        const bool shot = cb - 64u < 128u && wid >= 4;
        const bool puller = cb >= 128u || shot;
        const u32 wi = (shot ? ((cb >> 6) - 1u) * 4u + (wid - 4) : (puller ? (3u - (cb >> 6)) * PWPC : (cb >> 5) * RWPC) + wid) * 8u + (cb & 7);
        if (cb - 64u < 64u && wid == 0) {
            u64 pc0; u32 off;
            asm volatile("s_getpc_b64 %0\n.Lcw%=:\n\ts_mov_b32 %1, .Lcw%=-" KSYM : "=s"(pc0), "=s"(off));
            pc0 -= off;
            const u32 len = CODE_LEN;
            const u32 w_ = __builtin_amdgcn_raw_buffer_load_b32(mkrs((const void*)pc0, len), src * 8192u + (threadIdx.x & 63) * 128u, 0, 0);
            asm volatile("" :: "v"(w_));
            return;
        }
        if (shot ? (wi >= NPUSH || src == rank) : puller ? (wid >= PWPC || wi >= NPW || src == rank) : (wid >= RWPC || wi >= NRS)) return;
        LANE_VARS();
        const u32 call = uni(__hip_atomic_load(cnt, __ATOMIC_RELAXED, __HIP_MEMORY_SCOPE_AGENT)) + 1u;
        PUT_CALL();
        __builtin_amdgcn_s_setprio(CPRIO);
        bool ok = true;
        LOGDECL();
        LAS char* const img = (LAS char*)lds + wid * 4096;
#pragma unroll
        for (int i = 0; i < 4; ++i) *(LAS u32x4*)(img + i * 1024 + v16) = (u32x4){0, 0, 0, 0};
        FENCE();
        if (puller) {
            const rsrc_t rs = mkrs(P.ws[src], WS_USED);
            const u32 par = call & 1, sj = 8u + ((src - rank - 1u) & 7u);
            const gu32* const fla = (const gu32*)(my + OFF_IFA + src * 16384u);
            const gu32* const flb = (const gu32*)(my + OFF_IFL + src * 1024u + (lane & 7) * 128u);
            u64 ma = 0, mb = shot ? ~0ull : 0ull;
            auto ready = [&](u32 u) {
                const u32 sg = u < UB ? (u - UA) >> 6 : u < UC ? 2u + (u - UB) / 40u : (64u - G0) / 8u + (u - UC) / 48u;
                return ((u < UA ? ma >> (u + 64u - UA) : mb >> sg) & 1) != 0;
            };
            auto look = [&](u32 u) {
                if (u < UA) return uni(__hip_atomic_load(fla + u * 32u, __ATOMIC_RELAXED, __HIP_MEMORY_SCOPE_SYSTEM)) >= call;
                mb = __builtin_amdgcn_ballot_w64(__hip_atomic_load(flb, __ATOMIC_RELAXED, __HIP_MEMORY_SCOPE_SYSTEM) >= call);
                return ready(u);
            };
            auto wait = [&](u32 u) {
                const u64 t0 = rt();
                for (u32 it = 1;; ++it) {
                    if (look(u)) return true;
                    if ((it & 63u) == 0 && timed_out(tmo, call, t0)) return false;
                    __builtin_amdgcn_s_sleep(1);
                }
            };
            u32 u0 = shot ? wi : NPUSH + wi, e0 = entry(u0, src);
            u32 b0 = (((e0 >> 7) ? rb : ra) * 2u + rc) * 512u + u0;
            const rsrc_t rd = mkrs(shot ? my + OFF_PSH + par * PSH_PAR + (src * 64u + u0) * BLK - b0 * BLK : P.ws[src] + OFF_IPK + par * IPK_PAR, 2048u * BLK);
            const u32 tvoff = shot ? 0x80000000u : fvoff;
            ok = wait(u0);
            STAMPT(2);
            Blk N; u32 tkt = 0, fa = 0, fb = 0;
#define IISSUE(N_, b_) do { FENCE(); tkt = __builtin_amdgcn_raw_ptr_buffer_atomic_add_i32(1, rmy, tvoff, OFF_TKI + src * 128u, 0); \
                load_blk(N_, rd, (b_) * BLK, lane); FENCE(); } while (0)
#define ILOOK(b_) do { fb = __builtin_amdgcn_raw_buffer_load_b32(rmy, OFF_IFL + src * 1024u + (lane & 7) * 128u, 0, 17); \
                fa = __builtin_amdgcn_raw_buffer_load_b32(rmy, lane + (UA - 64u) > max((b_) & 511u, NPUSH + NPW - 1u) ? \
                        OFF_IFA + src * 16384u + (lane + (UA - 64u)) * 128u : 0x80000000u, 0, 17); FENCE(); } while (0)
#define INEXT() do { u1 = shot ? NU : uni(tkt) - (call - 1u) * (NU - NPUSH) + NPUSH + NPW; e1 = entry(u1, src); \
                b1 = (((e1 >> 7) ? rb : ra) * 2u + rc) * 512u + (u1 & 511u); } while (0)
#define IUNPACK(B_) do { u32 hi_[8]; unpack_hi(B_, rs, OFF_IOV + par * IOV_PAR + b0 * 2048u, lane, img, hi_); \
                const u32 ih_ = ((e0 >> 3) * 64u + sj + (e0 & 7) * 7u) * 2u + (u0 & 1); \
                store_frags(hi_, B_.l0, B_.l1, rmy, v16 + OFF_IMG + ih_ * 4096u); rp = OFF_RDY + ih_ * 128u; LOGU(src * 512u + u0); } while (0)
            if (ok) {
                IISSUE(N, b0);
                if (!shot) ILOOK(b0);
                WAITV(0);
                STAMPT(5);
                u32 rp = 0x80000000u;
                u32 u1, e1, b1;
                INEXT();
                for (;;) {
                    ma |= __builtin_amdgcn_ballot_w64(fa >= call); mb |= __builtin_amdgcn_ballot_w64(fb >= call);
                    if (u1 >= NU || (u1 >= UA && !~mb)) break;
                    Blk B = N;
                    asm volatile("" : "+v"(B.l0), "+v"(B.l1), "+v"(B.cs), "+v"(B.tail));
                    const bool ahead = ready(u1) || look(u1);
                    if (ahead) { IISSUE(N, b1); ILOOK(b1); WAITV(7); } else { WAITV(0); }
                    st4_dev(call, rmy, fvoff | rp, 0);
                    IUNPACK(B);
                    if (ahead) {
                        WAITV(5);
                    } else {
                        WAITV(0);
                        st4_dev(call, rmy, fvoff | rp, 0);
                        rp = 0x80000000u;
                        if (!wait(u1)) { ok = false; break; }
                        IISSUE(N, b1); ILOOK(b1);
                        WAITV(0);
                    }
                    u0 = u1; e0 = e1; b0 = b1;
                    INEXT();
                }
                if (ok) {
                    Blk M;
                    bool tok = shot;
#define ISTEP(C_, N_) { if (!tok && u1 + TOKE >= NU) { st4_dev(call, rmy, fvoff, OFF_TOK + (src * 64u + wi) * 64u); tok = true; } \
                        if (u1 >= NU) { WAITV(0); st4_dev(call, rmy, fvoff | rp, 0); IUNPACK(C_); break; } \
                        IISSUE(N_, b1); \
                        WAITV(5);             \
                        st4_dev(call, rmy, fvoff | rp, 0); \
                        IUNPACK(C_); \
                        u0 = u1; e0 = e1; b0 = b1; \
                        WAITV(9); INEXT(); asm volatile("" : "+s"(u1), "+s"(e1), "+s"(b1)); \
                        WAITV(5); }
                    for (;;) { ISTEP(N, M) ISTEP(M, N) }
                    WAITV(0);
                    st4_dev(call, rmy, fvoff | rp, 0);
                }
            }
            STAMPT(3);
        } else {
            const u32 st = src & 3;
            const rsrc_t r0 = mkrs(P.ws[st * 2u], WS_USED), r1 = mkrs(P.ws[st * 2u + 1u], WS_USED);
            const rsrc_t ro = mkrs(P.out, 512u * 4096u * 2u);
            const gu32* const fl = (const gu32*)(my + ((lane & 2) ? OFF_DFL + st * 4096u : OFF_OFL + st * 8192u + (lane & 1) * 4096u));
            const u32 ovoff = ((lane >> 4) * 4u + ((lane >> 3) & 1)) * 8192u + (lane & 7) * 16u;
            auto wait = [&](u32 v) {
                const u64 t0 = rt();
                for (u32 it = 1;; ++it) {
                    if (__builtin_amdgcn_ballot_w64(__hip_atomic_load(fl + (v >> 3) * 32u, __ATOMIC_RELAXED, __HIP_MEMORY_SCOPE_SYSTEM) < call) == 0) return true;
                    if ((it & 63u) == 0 && timed_out(tmo, call, t0)) return false;
                    __builtin_amdgcn_s_sleep(1);
                }
            };
            u32 v0 = wi;
            {
                const u32 ts = st * 2u + ((wi >> 3) & 1);
                const gu32* tok = (const gu32*)(my + OFF_TOK + ((ts == rank ? ts ^ 1u : ts) * 64u + wi % NPW) * 64u);
                const u64 t0 = rt();
                for (u32 it = 1;; ++it) {
                    const u32 tv = __hip_atomic_load(tok, __ATOMIC_RELAXED, __HIP_MEMORY_SCOPE_AGENT);
                    if (uni(tv) >= call) break;
                    if ((it & 63u) == 0 && timed_out(tmo, call, t0)) { ok = false; break; }
                    __builtin_amdgcn_s_sleep(4);
                }
            }
            STAMPT(2);
            if (wi >= RFIRST) for (int i = 0; i < RDELAY; ++i) __builtin_amdgcn_s_sleep(8);
            bool rdy = false;
            for (;;) {
                if (ok && !rdy) ok = wait(v0);
                FENCE();
                Blk A, C;
                load_blk(A, r0, OFF_OPK + (rank * 256u + v0) * BLK, lane);
                load_blk(C, r1, OFF_OPK + (rank * 256u + v0) * BLK, lane);
                FENCE();
                const u32 tkt = __builtin_amdgcn_raw_ptr_buffer_atomic_add_i32(1, rmy, fvoff, OFF_TKO + st * 128u, 0);
                const u32 tm = __hip_atomic_load(tmo, __ATOMIC_RELAXED, __HIP_MEMORY_SCOPE_AGENT);
                u32x4 dg[8];
#pragma unroll
                for (int k = 0; k < 8; ++k) dg[k] = __builtin_amdgcn_raw_buffer_load_b128(rmy, v16 + (k & 3) * 1024u, OFF_DIA + (st * 256u + v0) * 8192u + (k >> 2) * 4096u, 16);
                FENCE();
                const bool bad = !ok || uni(tm) == call;
                if (bad) { A.tail = 0; C.tail = 0; }
                const u32 v1 = uni(tkt) - (call - 1u) * 256u + NRS;
                const u32 ov = OFF_OOV + (rank * 256u + v0) * 2048u;
                u32 ha[8], hc[8], res[16];
                unpack_hi(A, r0, ov, lane, img, ha);
                unpack_hi(C, r1, ov, lane, img + 2048, hc);
#pragma unroll
                for (int k = 0; k < 8; ++k) asm volatile("" : "+v"(ha[k]), "+v"(hc[k]));
                FENCE();
                const u32 fv = __hip_atomic_load(fl + ((v1 >> 3) & 31u) * 32u, __ATOMIC_RELAXED, __HIP_MEMORY_SCOPE_SYSTEM);
                FENCE();
                const u32x4 a0 = A.l0, a1 = A.l1, c0 = C.l0, c1 = C.l1;
#pragma unroll
                for (int k = 0; k < 8; ++k) {
                    float ya[4], yc[4];
#pragma unroll
                    for (int e = 0; e < 4; ++e) {
                        const u32 sel = 0x04000c0cu + e * 0x01010000u;
                        ya[e] = __builtin_bit_cast(float, __builtin_amdgcn_perm(ha[k], k < 4 ? a0[k & 3] : a1[k & 3], sel));
                        yc[e] = __builtin_bit_cast(float, __builtin_amdgcn_perm(hc[k], k < 4 ? c0[k & 3] : c1[k & 3], sel));
                    }
                    const f32x4 yd = __builtin_bit_cast(f32x4, dg[k]);
                    res[2 * k] = silu2((f32x2){ya[0], ya[1]} + (f32x2){yc[0], yc[1]} + (f32x2){yd[0], yd[1]});
                    res[2 * k + 1] = silu2((f32x2){ya[2], ya[3]} + (f32x2){yc[2], yc[3]} + (f32x2){yd[2], yd[3]});
                }
                if (bad) {
#pragma unroll
                    for (int i = 0; i < 16; ++i) res[i] = 0x7FC07FC0u;
                }
                const u32 row0 = (st >> 1) * 256u + ((v0 >> 3) & 1) * 128u + ((v0 >> 2) & 1) * 64u + (v0 & 1) * 32u;
                const u32 col0 = (st & 1) * 2048u + (v0 >> 4) * 128u + ((v0 >> 1) & 1) * 64u;
#pragma unroll
                for (int k = 0; k < 8; k += 2) {
                    u32x4 o;
#pragma unroll
                    for (int d = 0; d < 2; ++d) {
                        const u32 x = res[2 * k + d], y = res[2 * k + 2 + d];
                        o[d] = __builtin_amdgcn_update_dpp(x, y, 0x128, 0xf, 0xc, false);
                        o[2 + d] = __builtin_amdgcn_update_dpp(y, x, 0x128, 0xf, 0x3, false);
                    }
                    st16_dev(o, ro, ovoff, (row0 + 16u * (k >> 2) + (k & 3)) * 8192u + col0 * 2u);
                }
                LOGU(4096u + st * 256u + v0);
                if (v1 >= 256u) break;
                rdy = __builtin_amdgcn_ballot_w64(fv < call) == 0;
                v0 = v1;
            }
        }
        STAMPT(4);
        LOGOUT();
        return;
    }

    const u32 gb = blockIdx.x - NCOMM;
    u32 call;
    {
        LANE_VARS();
        const rsrc_t rx = mkrs(P.x, 4096u * 512u * 2u), rw = mkrs(P.w, 512u * 4096u * 2u);
        const u32 ent = entry_fix(ent0, rank), op = ent >> 7;
        const bool push = u < NPUSH;
        if (grp >= 48u) __builtin_amdgcn_s_sleep(PSTAG2);
        else if (!early) { __builtin_amdgcn_s_sleep(PSTAG); for (u32 i = 16; i < bg; ++i) __builtin_amdgcn_s_sleep(PSTEP); }
        STAMPT(10);
        u32 in[16], lo[8], hi[8];
        unit_load(in, rx, rw, ent, pc >> 1, pc & 1, u & 1, lane);
        FENCE();
        call = uni(__hip_atomic_load(cnt, __ATOMIC_RELAXED, __HIP_MEMORY_SCOPE_AGENT)) + 1u;
        const u32 par = call & 1;
        const u32 g0 = (op ? (pc & 2) : (pc & 2) * 2u) + (pc & 1), g1 = g0 + (op ? 4u : 2u);
        const u32 po = OFF_PSH + par * PSH_PAR + (rank * 64u + u) * BLK;
        const rsrc_t d0 = push ? mkrs(P.ws[g0 & 7] + po, g0 == rank ? 0u : BLK) : mkrs(my + OFF_IPK + par * IPK_PAR + (pc * 512u + u) * BLK, BLK);
        const rsrc_t d1 = mkrs(P.ws[g1 & 7] + po, g1 == rank ? 0u : BLK);
        unit_split(in, op, lo, hi);
        const u32x4 l0 = {lo[0], lo[1], lo[2], lo[3]}, l1 = {lo[4], lo[5], lo[6], lo[7]};
        if (push && wid) { asm volatile("" :: "v"(l0), "v"(l1)); __builtin_amdgcn_s_sleep(PUSHD); }
        STAMPT(11);
        st16_sys(l0, d0, v16, 0); st16_sys(l1, d0, v16 + 1024, 0);
        if (push) { st16_sys(l0, d1, v16, 0); st16_sys(l1, d1, v16 + 1024, 0); }
        FENCE();
        u32 mu = in[0], mi = in[1];
#pragma unroll
        for (int i = 2; i < 16; i += 2) { mu = pkmax(mu, in[i]); mi = pkmaxi(mi, in[i + 1]); }
        mu = pkmax(mu & 0x7FFF7FFFu, mi & 0x7FFF7FFFu);
        Pk pk;
        pack_hi(lo, hi, max(mu >> 16, mu & 0xFFFFu) >> 7, lane, (LAS char*)lds + wid * 4096, pk);
        STAMPT(15);
        store_blk_cs(pk, d0, 0, lane);
        if (push) store_blk_cs(pk, d1, 0, lane);
        store_blk_raw(pk, hi, rmy, OFF_IOV + par * IOV_PAR + (pc * 512u + u) * 2048u, lane);
        const bool own = op || u >= NU || pc == ra * 2u + rc;
        const u32 tile = (ent >> 3) & 15u, kd = (pc & 1) * 8u + (ent & 7);
        const u32 ih = (op ? ID_WD + ((pc >> 1) * 16u + tile) * 16u + kd : u >= NU ? ID_XD + ((pc & 2) + (tile & 1)) * 16u + kd : tile * 64u + (ent & 7)) * 2u + (u & 1);
        if (own) store_frags(hi, l0, l1, rmy, v16 + OFF_IMG + ih * 4096u);
        const u32 dr = (lane & 3) * 2u + (pc & 1);
        u64 q0 = (u64)P.ws[pc & 1], q1 = (u64)P.ws[2 + (pc & 1)], q2 = (u64)P.ws[4 + (pc & 1)], q3 = (u64)P.ws[6 + (pc & 1)];
        asm volatile("" : "+s"(q0), "+s"(q1), "+s"(q2), "+s"(q3));
        char* const dw = (char*)((lane & 2) ? ((lane & 1) ? q3 : q2) : ((lane & 1) ? q1 : q0));
        const u32 da = (lane >> 1) & 1, db = lane & 1, ab = pc >> 1;
        WAITV(0);
        if (early && lane < 4 && dr != rank && (op ? db : da) == ab)
            __hip_atomic_store((gu32*)(dw + OFF_IFA + rank * 16384u + u * 128u), call, __ATOMIC_RELAXED, __HIP_MEMORY_SCOPE_SYSTEM);
        if (own) st4_dev(call, rmy, fvoff, OFF_RDY + ih * 128u);
        PUT_CALL();
        STAMPT(14);
        BAR();
        if (wid == 3) {
            const u32 t = __builtin_amdgcn_raw_ptr_buffer_atomic_add_i32(1, rmy, fvoff, OFF_PCN + (bg >> 3) * 128u, 0);
            if (uni(t) == call * 32u - 1u) {
                u64 r0 = (u64)P.ws[(pc & 1) ^ 1], r1 = (u64)P.ws[2 + ((pc & 1) ^ 1)], r2 = (u64)P.ws[4 + ((pc & 1) ^ 1)], r3 = (u64)P.ws[6 + ((pc & 1) ^ 1)];
                asm volatile("" : "+s"(r0), "+s"(r1), "+s"(r2), "+s"(r3));
                char* const dv = (lane & 4) ? (char*)((lane & 2) ? ((lane & 1) ? r3 : r2) : ((lane & 1) ? r1 : r0)) : dw;
                if (lane < 8) __hip_atomic_store((gu32*)(dv + OFF_IFL + rank * 1024u + (bg >> 3) * 128u), call, __ATOMIC_RELAXED, __HIP_MEMORY_SCOPE_SYSTEM);
            }
        }
    }
    const u32 xcd = gb & 7, idx = gb >> 3;
    const u32 rt_ = (xcd >> 1) * 4 + (idx & 3), ct = (idx >> 2) * 2 + (xcd & 1);

    const int pass0 = gb - 64u < 128u ? 0 : 1;
    const u32 drt = idx & 3, dct = xcd * 4u + ((idx >> 2) & 3);
    const u32 owner = rt_ >> 1;
    u32 pv[16];

    if (wid >= 4) {
        LANE_VARS();
        const u32 l = wid - 4;
        const rsrc_t rs = mkrs(my + OFF_IMG + (l & 1) * 4096u, NID * 8192u - 4096u);
        const rsrc_t rf = mkrs(my + OFF_RDY + (l & 1) * 128u, NID * 256u - 128u);
        __builtin_amdgcn_s_setprio(LPRIO);
#define LSTAMP(s_)
        for (int pass = pass0; pass < 2; ++pass) {
            const int N = pass ? 56 : 16;
            u32 idv;
            {
                const u32 hole = (owner - rank - 1u) & 7u;
                const u32 j = lane - 8u, ks = j / 6u, r = j - ks * 6u;
                const u32 kp = hole == 7u ? min(lane + 8u, 63u) : lane < 8 ? lane : lane < 56 ? 8u + ks * 7u + r + (r >= hole) : 8u + hole;
                const u32 cube = (l >> 1) ? (kp < 8u ? ID_WD + (rb * 16u + ct) * 16u + rc * 8u + kp : (16u + ct) * 64u + kp) : rt_ * 64u + kp;
                idv = pass ? cube : ((l >> 1) ? ID_WD + dct * 16u : ID_XD + drt * 16u) + (lane & 15);
            }
            int avail = 0;
            auto window = [&](u32 n) { return (u32)(lane - avail) < n ? idv * 256u : 0x80000000u; };
            auto seen = [&](u32 f) { if (avail < 64) { const u64 miss = __builtin_amdgcn_ballot_w64(f < call) >> avail << avail; avail = miss ? (int)__builtin_ctzll(miss) : 64; } };
            auto need = [&](int s1) {
                if (__builtin_expect(avail <= s1, 0)) {
                    const u64 t0 = rt();
                    for (u32 it = 1;; ++it) {
                        FENCE();
                        seen(__builtin_amdgcn_raw_buffer_load_b32(rf, window(LWINB), 0, 16));
                        if (avail > s1) break;
                        if ((it & 63u) == 0 && timed_out(tmo, call, t0)) { avail = 64; break; }
                        __builtin_amdgcn_s_sleep(LSLEEP);
                    }
                    WAITL0();
                }
            };
            const u32 sov = idv * 8192u, dl0 = l * 4096u;
            u32 so = rdl(sov, 0), dl = dl0, f = 0, fo = 0x80000000u;
            auto issue = [&](int s1, bool look) {
                need(s1);
                LAS void* const d = (LAS void*)((LAS char*)lds + dl);
                __builtin_amdgcn_raw_ptr_buffer_load_lds(rs, d, 16, v16, so, 0, 16);
                SB();
                u32 son = rdl(sov, s1 + 1), dln = dl + 16384u;
                dln = dln == dl0 + NS * 16384u ? dl0 : dln;
                asm volatile("" : "+s"(son), "+s"(dln));
                SB();
                __builtin_amdgcn_raw_ptr_buffer_load_lds(rs, d, 16, v16, so, 1024, 16);
                SB();
                if (look) { const u64 miss = __builtin_amdgcn_ballot_w64(f < call && lane >= avail); avail = miss ? (int)__builtin_ctzll(miss) : 64; asm volatile("" : "+s"(avail)); }
                SB();
                __builtin_amdgcn_raw_ptr_buffer_load_lds(rs, d, 16, v16, so, 2048, 16);
                SB();
                if (look) { fo = window(avail < s1 + LAHEAD ? LWIN : 0); asm volatile("" : "+v"(fo)); }
                SB();
                __builtin_amdgcn_raw_ptr_buffer_load_lds(rs, d, 16, v16, so, 3072, 16);
                FENCE(); SB();
                so = son; dl = dln;
            };
            for (int s = 0; s < 5; ++s) issue(s, false);
            asm volatile("s_waitcnt vmcnt(16)" ::: "memory"); BAR();
            asm volatile("s_waitcnt vmcnt(12)" ::: "memory"); BAR();
            int s = 5;
            for (; s < N - 3; s += 4) {
                issue(s, true);
                asm volatile("buffer_load_dword %0, %1, %2, 0 offen sc1" : "=v"(f) : "v"(fo), "s"(rf) : "memory");
                LSTAMP(s - 6);
                asm volatile("s_waitcnt vmcnt(13)" ::: "memory"); BAR();
                issue(s + 1, false);
                asm volatile("s_waitcnt vmcnt(13)" ::: "memory"); BAR();
                issue(s + 2, false);
                asm volatile("s_waitcnt vmcnt(13)" ::: "memory"); BAR();
                issue(s + 3, false);
                asm volatile("s_waitcnt vmcnt(12)" : "+v"(f) :: "memory"); BAR();
            }
            seen(f);
            for (; s < N; ++s) { issue(s, false); LSTAMP(s - 6); asm volatile("s_waitcnt vmcnt(12)" ::: "memory"); BAR(); }
            asm volatile("s_waitcnt vmcnt(8)" ::: "memory"); BAR();
            asm volatile("s_waitcnt vmcnt(4)" ::: "memory"); BAR();
            asm volatile("s_waitcnt vmcnt(0)" ::: "memory"); BAR();
            if (pass && l == 2) need(56);
            BAR(); BAR();
            LSTAMP(55);
        }
        __builtin_amdgcn_s_setprio(0);
        BAR();
#pragma unroll
        for (int i = 0; i < 4; ++i) {
            const u32x4 v = *(LAS u32x4*)((LAS char*)lds + 32768 + l * 4096 + i * 1024 + v16);
            pv[4 * i] = v[0]; pv[4 * i + 1] = v[1]; pv[4 * i + 2] = v[2]; pv[4 * i + 3] = v[3];
        }
    } else {
        const u32 la = (wid >> 1) * 4096, lb = 8192 + (wid & 1) * 4096;
        u32 pla = threadIdx.x * 16u, plb;
#define V16() (pla & 1023u)
        f32x4 acc[4][4];
        bf16x8 A0, A1, C0[2], C1[2], B0[4], B1[4];
#define RD(d, q, i) d = *(const LAS bf16x8*)((LAS char*)lds + p##q + (i) * 1024)
#define M1(a, B, m, n) acc[m][n] = __builtin_amdgcn_mfma_f32_16x16x32_bf16(a, B[n], acc[m][n], 0, 0, 0)
#define NEXT() do { const u32 d_ = so == (NS - 1) * 16384u ? 0u - so : 16384u; so += d_; pla += d_; plb += d_; asm volatile("" : "+v"(pla), "+v"(plb)); } while (0)
#define STEP(Cc, Bc, Cn, Bn) do { SB(); \
            M1(A0, Bc, 0, 0); SB(); RD(Bn[0], lb, 0); SB(); M1(A0, Bc, 0, 1); SB(); RD(Bn[1], lb, 1); SB(); M1(A0, Bc, 0, 2); SB(); RD(Bn[2], lb, 2); SB(); M1(A0, Bc, 0, 3); SB(); RD(A0, la, 0); SB(); \
            M1(A1, Bc, 1, 0); SB(); RD(Bn[3], lb, 3); SB(); M1(A1, Bc, 1, 1); SB(); RD(Cn[0], la, 2); SB(); M1(A1, Bc, 1, 2); SB(); RD(Cn[1], la, 3); SB(); M1(A1, Bc, 1, 3); SB(); RD(A1, la, 1); SB(); \
            M1(Cc[0], Bc, 2, 0); SB(); M1(Cc[0], Bc, 2, 1); SB(); NEXT(); SB(); M1(Cc[0], Bc, 2, 2); SB(); M1(Cc[0], Bc, 2, 3); SB(); \
            M1(Cc[1], Bc, 3, 0); SB(); M1(Cc[1], Bc, 3, 1); SB(); M1(Cc[1], Bc, 3, 2); SB(); WAITL0(); SB(); M1(Cc[1], Bc, 3, 3); SB(); \
            BAR(); \
        } while (0)
        for (int pass = pass0;; ++pass) {
#pragma unroll
            for (int m = 0; m < 4; ++m)
#pragma unroll
                for (int n = 0; n < 4; ++n) acc[m][n] = (f32x4){0.f, 0.f, 0.f, 0.f};
            u32 so = 0;
            asm volatile("" : "+v"(pla));
            plb = V16() + lb; pla = V16() + la;
            BAR();
            if (pass > pass0 && wid == 0 && pla == 0 && __hip_atomic_load(tmo, __ATOMIC_RELAXED, __HIP_MEMORY_SCOPE_AGENT) != call)
                __hip_atomic_store((gu32*)(my + OFF_DFL + (((drt & 2) + (dct >> 4)) * 32u + (dct & 15u) * 2u + (drt & 1)) * 128u), call, __ATOMIC_RELAXED, __HIP_MEMORY_SCOPE_AGENT);
            RD(A0, la, 0); RD(A1, la, 1); RD(C0[0], la, 2); RD(C0[1], la, 3);
#pragma unroll
            for (int i = 0; i < 4; ++i) RD(B0[i], lb, i);
            SB(); NEXT();
            WAITL0(); BAR();
            for (int s = pass ? 56 : 16; s > 0; s -= 2) {
                STEP(C0, B0, C1, B1);
                STEP(C1, B1, C0, B0);
            }
            if (pass) break;
            const u32 vo = V16() + OFF_DIA + (((drt & 2) + (dct >> 4)) * 256u + (((dct & 15u) * 2u + (drt & 1)) * 4u + wid) * 2u) * 8192u;
#pragma unroll
            for (int m = 0; m < 4; ++m)
#pragma unroll
                for (int j = 0; j < 4; ++j)
                    st16_dev(__builtin_bit_cast(u32x4, (f32x4){acc[m][0][j], acc[m][1][j], acc[m][2][j], acc[m][3][j]}), rmy, vo + m * 4096u + j * 1024u, 0);
            WAITV(0);
        }
        STAMPT(9);
#pragma unroll
        for (int i = 0; i < 4; ++i) {
            u32x4 v;
#pragma unroll
            for (int e = 0; e < 4; ++e) { const int k = 2 * i + (e >> 1), m = 2 + (k >> 2), j = k & 3; v[e] = cvt_pk_bf16(acc[m][2 * (e & 1)][j], acc[m][2 * (e & 1) + 1][j]); }
            *(LAS u32x4*)((LAS char*)lds + 32768 + wid * 4096 + i * 1024 + V16()) = v;
        }
        WAITL0(); BAR();
#pragma unroll
        for (int k = 0; k < 8; ++k) { pv[2 * k] = cvt_pk_bf16(acc[k >> 2][0][k & 3], acc[k >> 2][1][k & 3]); pv[2 * k + 1] = cvt_pk_bf16(acc[k >> 2][2][k & 3], acc[k >> 2][3][k & 3]); }
    }
    {
        int tz = threadIdx.x; asm volatile("" : "+v"(tz));
        const int ln = tz & 63;
        const u32 blk = owner * 256u + ((ct * 2u + (rt_ & 1)) * 4u + (wid & 3)) * 2u + (wid >> 2);
        u32 lo[8], hi[8], mu = 0, mi = 0x80008000u;
#pragma unroll
        for (int k = 0; k < 8; ++k) {
            lo[k] = __builtin_amdgcn_perm(pv[2 * k + 1], pv[2 * k], 0x06040200u); hi[k] = __builtin_amdgcn_perm(pv[2 * k + 1], pv[2 * k], 0x07050301u);
            mu = pkmax(mu, pv[2 * k]); mi = pkmaxi(mi, pv[2 * k + 1]);
        }
        mu = pkmax(mu & 0x7FFF7FFFu, mi & 0x7FFF7FFFu);
        const u32 so_ = OFF_OPK + blk * BLK;
        st16_sys((u32x4){lo[0], lo[1], lo[2], lo[3]}, rmy, ln * 16u, so_);
        st16_sys((u32x4){lo[4], lo[5], lo[6], lo[7]}, rmy, ln * 16u + 1024u, so_);
        Pk pk;
        pack_hi(lo, hi, max(mu >> 16, mu & 0xFFFFu) >> 7, ln, (LAS char*)lds + wid * 4096, pk);
        store_blk_cs(pk, rmy, so_, ln);
        store_blk_raw(pk, hi, rmy, OFF_OOV + blk * 2048u, ln);
        STAMPT(12);
        FENCE();
        const u32 tm = __hip_atomic_load(tmo, __ATOMIC_RELAXED, __HIP_MEMORY_SCOPE_AGENT);
        WAITV(0);
        BAR();
        if (tz == 0 && tm != call)
            __hip_atomic_store((gu32*)(P.ws[owner & 7] + OFF_OFL + rank * 4096u + (ct * 2u + (rt_ & 1)) * 128u), call, __ATOMIC_RELAXED, __HIP_MEMORY_SCOPE_SYSTEM);
        STAMPT(13);
    }
}

}

extern "C" void kernel_launch(int rank, int nranks, ncclComm_t comm,
                              hipStream_t stream,
                              void* const* d_in, const int* in_sizes, int n_in,
                              void* d_out, int out_size,
                              const int* in_dtypes, int out_dtype,
                              void* const* d_out_all,
                              void* const* d_ws_all, size_t ws_bytes) {
    if (nranks != NR || (unsigned)rank >= (unsigned)NR || n_in != 2 || ws_bytes < WS_NEED ||
        in_sizes[0] != 4096 * 512 || in_sizes[1] != 512 * 4096 || out_size != 512 * 4096) {
        fprintf(stderr, "kernel_launch: unexpected problem shape\n");
        return;
    }
    Params P;
    for (int p = 0; p < NR; ++p) P.ws[p] = (char*)d_ws_all[p];
    P.x = (const char*)d_in[0];
    P.w = (const char*)d_in[1];
    P.out = (char*)d_out;
    P.my = P.ws[rank];
    P.rank = rank;
    P.pad = 0;
    hipLaunchKernelGGL(fused, dim3(NCOMM + NGEMM), dim3(512), 0, stream, P);
    HIP_CHECK(hipGetLastError());
}
```

```cpp
#include <hip/hip_runtime.h>
#include <rccl/rccl.h>
#include <cstdio>

#define HIP_CHECK(expr) do { hipError_t _e = (expr); if (_e != hipSuccess) { \
    fprintf(stderr, "HIP error %s at %s:%d\n", hipGetErrorString(_e), __FILE__, __LINE__); \
    return; } } while (0)

namespace {

typedef unsigned u32;
typedef unsigned long long u64;
typedef unsigned u32x4 __attribute__((ext_vector_type(4)));
typedef unsigned u32x3 __attribute__((ext_vector_type(3)));
typedef unsigned u32x2 __attribute__((ext_vector_type(2)));
typedef short i16x2 __attribute__((ext_vector_type(2)));
typedef unsigned short u16x2 __attribute__((ext_vector_type(2)));
typedef short bf16x8 __attribute__((ext_vector_type(8)));
typedef float f32x4 __attribute__((ext_vector_type(4)));
#define LAS __attribute__((address_space(3)))
#define GAS __attribute__((address_space(1)))
typedef GAS u32 gu32;
typedef GAS u64 gu64;
typedef __amdgpu_buffer_rsrc_t rsrc_t;

constexpr int NR = 8;
constexpr int NCOMM = 256;
constexpr int NGEMM = 256;
#define NPW 21
#define HAND 170u
#define GATEK 20
#define NRS 44
#define RWPC 3
#define CPRIO 3
#define NPUSH 48
#define PUSHD 25
#define RFIRST NPW
#define RDELAY 6
#define LPRIO 2
#define LAHEAD 10
#define LWIN 8
#define TSLEEP 4
#define GSLEEP 24
#define HSLEEP 2
#define SSLEEP 1
#define PSLEEP 1
#define RSLEEP 1
#define LWIN1 LWINB
#define LWINB 4
#define LSLEEP 12
#define PSTAG 24
#define PSTAG2 0
#define PSTEP 1
#define TOKE 24
#define RFAST 10
#define RSLOW 14
#define REARLY 3
#define RBASE 40
constexpr u32 NU = 480;
constexpr u64 DEADLINE = 50000000ull;

constexpr u32 BLK = 2944;
constexpr u32 OFF_CNT = 0;
constexpr u32 OFF_TMO = 16384;
constexpr u32 OFF_TKI = 16384 + 1024;
constexpr u32 OFF_TKO = 16384 + 2048;
constexpr u32 OFF_PCN = 16384 + 4096;
constexpr u32 OFF_IFL = 24576;
constexpr u32 OFF_TOK = 65536;
constexpr u32 OFF_IFA = 262144;
constexpr u32 OFF_OFL = 196608;
constexpr u32 OFF_DFL = 229376;
constexpr u32 OFF_HND = 98304;
constexpr u32 ID_WR = 1024, ID_XD = 1792, ID_WD = 1856, NID = 2368;
constexpr u32 KIB = 1024;
constexpr u32 OFF_RDY = 1024 * KIB;
constexpr u32 OFF_IMG = 2048 * KIB;
constexpr u32 OFF_IPK = 25088 * KIB;
constexpr u32 IPK_PAR = 6144 * KIB;
constexpr u32 OFF_IOV = 37376 * KIB;
constexpr u32 IOV_PAR = 4096 * KIB;
constexpr u32 OFF_OPK = 45568 * KIB;
constexpr u32 OFF_OOV = 51456 * KIB;
constexpr u32 OFF_DIA = 55552 * KIB;
constexpr u32 OFF_PSH = 63744 * KIB;
constexpr u32 PSH_PAR = 8u * NPUSH * BLK;
constexpr u32 WS_USED = OFF_PSH + 2u * PSH_PAR;
static_assert(WS_USED <= 65u * 1024u * KIB && OFF_OPK + 2048u * BLK <= OFF_OOV, "layout");
static_assert(OFF_RDY + NID * 256u <= OFF_IMG && OFF_IMG + NID * 8192u <= OFF_IPK && 2048u * BLK <= IPK_PAR, "layout");
constexpr u32 WS_NEED = WS_USED;

struct Params {
    const char* x;
    const char* w;
    char* my;
    char* out;
    int rank;
    int pad;
    char* ws[NR];
};

struct Order { u32 e[256]; };
constexpr Order make_order() {
    constexpr int SLOW0 = 12;
    int key[17][8] = {};
    for (int ct = 0; ct < 16; ++ct) {
        int t = 8 * ct - RBASE - (ct < REARLY ? 8 * ct : 8 * (REARLY - 1) * (16 - ct) / (17 - REARLY));
        for (int ks = 7; ks >= 0; --ks) { t -= (ct >= SLOW0 && t <= 0) ? RSLOW : RFAST; key[ct][ks] = t; }
    }
    for (int ks = 0; ks < 8; ++ks) { int m = key[0][ks]; for (int ct = 1; ct < 16; ++ct) m = key[ct][ks] < m ? key[ct][ks] : m; key[16][ks] = m; }
    Order o{};
    int n = 0, next[17] = {};
    while (n < 240) {
        int best = -1;
        for (int i = 16; i >= 0; --i) if (next[i] < 8 && (best < 0 || key[i][next[i]] < key[best][next[best]])) best = i;
        if (best == 16) for (int t = 0; t < 14; ++t) o.e[n++] = (u32)(t << 3 | next[16]);
        else o.e[n++] = (u32)(128 | best << 3 | next[best]);
        ++next[best];
    }
    for (int ks = 0; ks < 8; ++ks) { o.e[n++] = (u32)(14 << 3 | ks); o.e[n++] = (u32)(15 << 3 | ks); }
    return o;
}
__constant__ const Order ORDER = make_order();
constexpr u32 G0 = NPUSH / 2, UA = 144u - G0, UB = UA + 128u, UC = UB + (48u - G0) * 5u;
static_assert(G0 % 8 == 0 && G0 <= 32 && UA - 64u <= NPUSH + NPW, "whole flag groups; one look covers the ticketed early units");
struct PackTab { u32 e[512][2]; };
constexpr PackTab make_pack() {
    const Order o = make_order();
    PackTab t{};
    for (u32 g = 0; g < 64; ++g)
        for (u32 w = 0; w < 8; ++w) {
            const u32 bg = g >= 48u ? g - 48u : g >= G0 ? g - G0 + 16u : g + 64u - G0;
            const bool early = w < (g < G0 ? 2u : g < 48u ? 3u : 0u);
            const u32 u = g < G0 ? (early ? w * G0 + g : UC + g * 6u + (w - 2))
                        : g < 48u ? (early ? G0 + g + w * (48u - G0) : UB + (g - G0) * 5u + (w - 3)) : UA + bg * 8u + w;
            const u32 e = o.e[u >> 1], tile = (e >> 3) & 15u, ks = e & 7u, half = u & 1u;
            const u32 delay = early ? 0u : g >= 48u ? PSTAG2 : PSTAG + (bg - 16u) * PSTEP;
            t.e[g * 8 + w][0] = e | u << 8 | (bg >> 3) << 17 | delay << 20;
            t.e[g * 8 + w][1] = e >= 128u ? 32u * ks * 8192u + (128u * tile + 64u * half) * 2u
                                          : ((tile < 14u ? 512u * (tile >> 1) : 0u) + 128u * (tile & 1u) + 64u * half) * 1024u + 64u * ks;
        }
    return t;
}
__constant__ const PackTab PACK = make_pack();
__device__ __forceinline__ u32 entry_fix(u32 e, u32 src) {
    const u32 own = e - 112u + src * 16u, oth = e + (e >= src * 16u ? 16u : 0u), x = e >= 112u ? own : oth;
    return e >= 128u ? e : x;
}
__device__ __forceinline__ u32 entry(u32 u, u32 src) { return entry_fix(ORDER.e[(u >> 1) & 255u], src); }

__device__ __forceinline__ u64 rt() { return __builtin_amdgcn_s_memrealtime(); }
__device__ __forceinline__ u32 uni(u32 v) { return __builtin_amdgcn_readfirstlane(v); }
__device__ __forceinline__ u32 rdl(u32 v, u32 l) { return __builtin_amdgcn_readlane(v, l); }
__device__ __forceinline__ rsrc_t mkrs(const void* p, u32 bytes) { return __builtin_amdgcn_make_buffer_rsrc((void*)p, 0, bytes, 0x00020000); }
__device__ __forceinline__ u32x4 ld16(rsrc_t r, u32 voff, u32 soff) { return __builtin_amdgcn_raw_buffer_load_b128(r, voff, soff, 0); }
__device__ __forceinline__ u32x2 ld8(rsrc_t r, u32 voff, u32 soff) { return __builtin_amdgcn_raw_buffer_load_b64(r, voff, soff, 0); }
__device__ __forceinline__ u32x4 ld16_sys(rsrc_t r, u32 voff, u32 soff) { return __builtin_amdgcn_raw_buffer_load_b128(r, voff, soff, 17); }
__device__ __forceinline__ u32x2 ld8_sys(rsrc_t r, u32 voff, u32 soff) { return __builtin_amdgcn_raw_buffer_load_b64(r, voff, soff, 17); }
__device__ __forceinline__ void st16_sys(u32x4 v, rsrc_t r, u32 voff, u32 soff) { __builtin_amdgcn_raw_buffer_store_b128(v, r, voff + soff, 0, 17); }
__device__ __forceinline__ void st16_dev(u32x4 v, rsrc_t r, u32 voff, u32 soff) { __builtin_amdgcn_raw_buffer_store_b128(v, r, voff + soff, 0, 16); }
__device__ __forceinline__ void st4_dev(u32 v, rsrc_t r, u32 voff, u32 soff) { __builtin_amdgcn_raw_buffer_store_b32(v, r, voff, soff, 16); }
__device__ __forceinline__ void st8_sys(u32x2 v, rsrc_t r, u32 voff, u32 soff) { __builtin_amdgcn_raw_buffer_store_b64(v, r, voff, soff, 17); }
#define FENCE() asm volatile("" ::: "memory")
#define WAITV(n) do { asm volatile("s_waitcnt vmcnt(" #n ")" ::: "memory"); __builtin_amdgcn_s_waitcnt(0xF70 | (n)); } while (0)
#define WAITL0() do { asm volatile("s_waitcnt lgkmcnt(0)" ::: "memory"); __builtin_amdgcn_s_waitcnt(0xC07F); } while (0)
#define SB() __builtin_amdgcn_sched_barrier(0)
#define BAR() do { FENCE(); __builtin_amdgcn_s_barrier(); FENCE(); } while (0)

#define STAMPT(i) do {} while (0)
#define LOGDECL() do {} while (0)
#define LOGU(u) do {} while (0)
#define LOGOUT() do {} while (0)

__device__ __forceinline__ u32 pkmax(u32 a, u32 b) {
    return __builtin_bit_cast(u32, __builtin_elementwise_max(__builtin_bit_cast(u16x2, a), __builtin_bit_cast(u16x2, b)));
}
__device__ __forceinline__ u32 pkmaxi(u32 a, u32 b) {
    return __builtin_bit_cast(u32, __builtin_elementwise_max(__builtin_bit_cast(i16x2, a), __builtin_bit_cast(i16x2, b)));
}
struct Pk { u32x3 cs; u32 tail; bool raw; };
__device__ __forceinline__ u32 bfi(u32 m, u32 a, u32 b) { return __builtin_amdgcn_bitop3_b32(m, a, b, 0xCA); }
#define DPP0(v, ctrl, rm) (u32)__builtin_amdgcn_update_dpp(0, (int)(v), ctrl, rm, 0xf, true)
__device__ __forceinline__ void pack_hi(const u32 (&lo)[8], const u32 (&hi)[8], u32 m, int lane, LAS char* sc, Pk& p) {
    m = max(m, DPP0(m, 0x111, 0xf)); m = max(m, DPP0(m, 0x112, 0xf)); m = max(m, DPP0(m, 0x114, 0xf)); m = max(m, DPP0(m, 0x118, 0xf));
    u32 T0 = min(rdl(m, 15), rdl(m, 31)), T1 = min(rdl(m, 47), rdl(m, 63));
    asm volatile("" : "+s"(T0), "+s"(T1));
    u32 T = max(min(T0, T1), 7u);
    asm volatile("" : "+s"(T));
    const u32 L = T - 7u, s = L & 1u, base = (L + s) >> 1;
    const u32 k4 = (128u - base) * 0x01010101u,
              s4 = s * 0x01010101u;
    u32 c7f = 0x7F7F7F7Fu, c7c = 0x7C7C7C7Cu;
    asm volatile("" : "+s"(c7f), "+s"(c7c));
    u32 d[8], x[8], bits = 0;
#pragma unroll
    for (int k = 7; k >= 0; --k) {
        d[k] = (hi[k] & c7f) + ((lo[k] >> 7) & s4) + k4;
        x[k] = d[k] & c7c;
        bits = k < 7 ? bfi(c7f, bits >> 1, x[k] + c7c) : x[k] + c7c;
    }
    *(LAS u32x4*)(sc + lane * 16) = (u32x4){x[0], x[1], x[2], x[3]};
    *(LAS u32x4*)(sc + 1024 + lane * 16) = (u32x4){x[4], x[5], x[6], x[7]};
    const u32 cnt = __builtin_popcount(bits);
    u32 sum = cnt;
    sum += DPP0(sum, 0x111, 0xf); sum += DPP0(sum, 0x112, 0xf); sum += DPP0(sum, 0x114, 0xf); sum += DPP0(sum, 0x118, 0xf);
    sum += DPP0(sum, 0x142, 0xa); sum += DPP0(sum, 0x143, 0xc);
    asm volatile("" : "+v"(sum));
    const u32 n = rdl(sum, 63);
    p.raw = n > 62u;
    {
        LAS unsigned short* const list = (LAS unsigned short*)(sc + 2048);
        const u32 l5 = (u32)lane << 5;
        LAS unsigned short* q = list + (sum - cnt);
        if (p.raw) bits = 0;
#pragma unroll
        for (int j = 0; j < 3; ++j) {
            u32 lb; asm("v_ffbl_b32 %0, %1" : "=v"(lb) : "v"(bits));
            (bits ? q : list + 64)[j] = (unsigned short)(l5 | lb); bits &= bits - 1u;
        }
        q += 3;
        while (bits) { *q++ = (unsigned short)(l5 | __builtin_ctz(bits)); bits &= bits - 1u; }
    }
    FENCE();
    const u32 e = ((LAS unsigned short*)(sc + 2048))[lane];
    SB();
    u32 cc[2];
#pragma unroll
    for (int k = 7; k >= 0; --k) cc[k >> 2] = (k & 3) < 3 ? bfi(0x03030303u, d[k], cc[k >> 2] << 2) : d[k];
    SB();
    const u32 pos = ((~e & 4u) << 8) | ((e >> 1) & 0x3F0u) | ((~e & 3u) << 2) | ((e >> 3) & 3u);
    const u32 val = *(LAS unsigned char*)(sc + pos);
    SB();
    u32 sg = hi[7];
#pragma unroll
    for (int k = 6; k >= 0; --k) sg = bfi(c7f, sg >> 1, hi[k]);
    p.cs = (u32x3){cc[0], cc[1], sg};
    const u32 hd = lane == 63 ? (base - s) | s << 7 : min(n, 63u);
    SB();
    p.tail = lane >= 62 ? hd : pos | val << 9;
    FENCE();
}
__device__ __forceinline__ void store_blk_cs(const Pk& p, rsrc_t rs, u32 so, int lane) {
    __builtin_amdgcn_raw_buffer_store_b96(p.cs, rs, so + 2048u + lane * 12u, 0, 17);
    __builtin_amdgcn_raw_buffer_store_b16((unsigned short)p.tail, rs, so + 2816u + lane * 2u, 0, 17);
}
__device__ __forceinline__ void store_blk_raw(const Pk& p, const u32 (&hi)[8], rsrc_t rs, u32 ov, int lane) {
    if (p.raw) {
        st16_sys((u32x4){hi[0], hi[1], hi[2], hi[3]}, rs, lane * 16u, ov);
        st16_sys((u32x4){hi[4], hi[5], hi[6], hi[7]}, rs, lane * 16u + 1024u, ov);
    }
}
struct Blk { u32x4 l0, l1; u32x3 cs; u32 tail; };
__device__ __forceinline__ void load_blk(Blk& b, rsrc_t rs, u32 so, int lane) {
    b.tail = __builtin_amdgcn_raw_buffer_load_b16(rs, 2816u + lane * 2u, so, 17);
    b.cs = __builtin_amdgcn_raw_buffer_load_b96(rs, 2048u + lane * 12u, so, 17);
    b.l0 = ld16_sys(rs, lane * 16u, so);
    b.l1 = ld16_sys(rs, lane * 16u + 1024u, so);
}
__device__ __forceinline__ void unpack_hi(const Blk& b, rsrc_t rs, u32 ov, int lane, LAS char* img, u32 (&hi)[8]) {
    const u32 cn = rdl(b.tail, 62), hb = rdl(b.tail, 63);
    if (cn > 62u) {
        const u32x4 h0 = ld16_sys(rs, lane * 16u, ov), h1 = ld16_sys(rs, lane * 16u + 1024u, ov);
        WAITV(0);
#pragma unroll
        for (int k = 0; k < 4; ++k) { hi[k] = h0[k]; hi[4 + k] = h1[k]; }
    } else {
        LAS unsigned char* const q = (LAS unsigned char*)img + (b.tail & 0x7FFu);
        if ((u32)lane < cn) *q = (unsigned char)((b.tail >> 9) & 0x7Cu);
        FENCE();
        const u32x4 d0 = *(LAS u32x4*)(img + lane * 16), d1 = *(LAS u32x4*)(img + 1024 + lane * 16);
        FENCE();
        if ((u32)lane < cn) *q = 0;
        FENCE();
        const u32 b4 = (hb & 127u) * 0x01010101u, s4 = ((hb >> 7) & 1u) * 0x01010101u;
#pragma unroll
        for (int k = 0; k < 8; ++k) {
            const u32 lo = k < 4 ? b.l0[k & 3] : b.l1[k & 3];
            const u32 u = bfi(0x03030303u, b.cs[k >> 2] >> (2 * (k & 3)), k < 4 ? d0[k & 3] : d1[k & 3]) + (~(lo >> 7) & s4) + b4;
            hi[k] = bfi(0x7F7F7F7Fu, u, b.cs[2] << k);
        }
    }
}

__device__ __forceinline__ bool timed_out(gu32* tmo, u32 call, u64 t0) {
    if (uni(__hip_atomic_load(tmo, __ATOMIC_RELAXED, __HIP_MEMORY_SCOPE_AGENT)) == call) return true;
    if (rt() - t0 > DEADLINE) { __hip_atomic_store(tmo, call, __ATOMIC_RELAXED, __HIP_MEMORY_SCOPE_AGENT); return true; }
    return false;
}
__device__ __forceinline__ void store_frags(const u32 (&hi)[8], u32x4 l0, u32x4 l1, rsrc_t rs, u32 voff) {
#pragma unroll
    for (int f = 0; f < 4; ++f) {
        u32x4 v;
#pragma unroll
        for (int p = 0; p < 4; ++p) v[p] = __builtin_amdgcn_perm(hi[4 * (f >> 1) + p], (f >> 1) ? l1[p] : l0[p], (f & 1) ? 0x07030602u : 0x05010400u);
        st16_dev(v, rs, voff + f * 1024u, 0);
    }
}

__device__ __forceinline__ void unit_load(u32 (&in)[16], const char* x, const char* w, u32 op, u32 so, int lane) {
    if (op) {
        const rsrc_t rw = mkrs(w, 512u * 4096u * 2u);
        const u32 vo = (lane >> 4) * 8u * 8192u + (lane & 7) * 16u + (lane & 8);
#pragma unroll
        for (int r = 0; r < 8; ++r) { const u32x2 v = ld8(rw, vo, so + r * 8192u); in[2 * r] = v[0]; in[2 * r + 1] = v[1]; }
    } else {
        const rsrc_t rx = mkrs(x, 4096u * 512u * 2u);
        const u32 vo = (lane & 15) * 1024u + (lane >> 4) * 16u;
#pragma unroll
        for (int m = 0; m < 4; ++m) { const u32x4 v = ld16(rx, vo, so + m * 16384u); in[4 * m] = v[0]; in[4 * m + 1] = v[1]; in[4 * m + 2] = v[2]; in[4 * m + 3] = v[3]; }
    }
}
__device__ __forceinline__ void unit_split(const u32 (&in)[16], u32 op, u32 (&lo)[8], u32 (&hi)[8]) {
    if (op) {
#pragma unroll
        for (int k = 0; k < 8; ++k) {
            const u32 a0 = in[4 * (k & 3) + (k >> 2)], a1 = in[4 * (k & 3) + 2 + (k >> 2)];
            lo[k] = __builtin_amdgcn_perm(a1, a0, 0x06020400u); hi[k] = __builtin_amdgcn_perm(a1, a0, 0x07030501u);
        }
    } else {
#pragma unroll
        for (int k = 0; k < 8; ++k) {
            const u32 a0 = in[8 * (k >> 2) + (k & 3)], a1 = in[8 * (k >> 2) + 4 + (k & 3)];
            lo[k] = __builtin_amdgcn_perm(a1, a0, 0x06040200u); hi[k] = __builtin_amdgcn_perm(a1, a0, 0x07050301u);
        }
    }
}

__device__ __forceinline__ u32 cvt_pk_bf16(float lo, float hi) {
    typedef __bf16 bf2 __attribute__((ext_vector_type(2)));
    typedef float f2 __attribute__((ext_vector_type(2)));
    return __builtin_bit_cast(u32, __builtin_convertvector((f2){lo, hi}, bf2));
}
typedef float f32x2 __attribute__((ext_vector_type(2)));
__device__ __forceinline__ u32 silu2(f32x2 y) {
    constexpr float LOG2E = 1.44269504088896341f;
    const f32x2 t = y * -LOG2E;
    f32x2 e;
    e.x = __builtin_amdgcn_exp2f(__builtin_amdgcn_fmed3f(t.x, -60.0f * LOG2E, 60.0f * LOG2E));
    e.y = __builtin_amdgcn_exp2f(__builtin_amdgcn_fmed3f(t.y, -60.0f * LOG2E, 60.0f * LOG2E));
    e += 1.0f;
    e.x = __builtin_amdgcn_rcpf(e.x); e.y = __builtin_amdgcn_rcpf(e.y);
    y *= e;
    return cvt_pk_bf16(y.x, y.y);
}

#define BID (blockIdx.x ^ 256u)
__global__ void __launch_bounds__(512, 4) fused(Params P) {
    constexpr int NS = 5;
    __shared__ __attribute__((aligned(16))) char lds[NS * 16384];
    const int wid = uni(threadIdx.x >> 6);
#define LANE_VARS() int tz_ = threadIdx.x; asm volatile("" : "+v"(tz_)); const int lane = tz_ & 63; const u32 v16 = lane * 16u; \
    const u32 fvoff = lane == 0 ? 0u : 0x80000000u;   (void)v16; (void)fvoff
    const u32 pk0 = PACK.e[((blockIdx.x >> 2) & 63u) * 8u + wid][0], pk1 = PACK.e[((blockIdx.x >> 2) & 63u) * 8u + wid][1];
    asm volatile("" :: "s"(pk0), "s"(pk1), "s"(P.x), "s"(P.w), "s"(P.my), "s"(P.rank));
    const u32 rank = P.rank;
    const u32 ra = rank >> 2, rb = (rank >> 1) & 1, rc = rank & 1;
    char* const my = P.my;
    gu32* const tmo = (gu32*)(my + OFF_TMO);
    gu32* const cnt = (gu32*)(my + OFF_CNT) + BID * 8 + wid;
#define PUT_CALL() do { if (lane == 0) __hip_atomic_store(cnt, call, __ATOMIC_RELAXED, __HIP_MEMORY_SCOPE_AGENT); } while (0)
    STAMPT(0);
    const rsrc_t rmy = mkrs(my, WS_USED);

    if (BID < NCOMM) {
        const u32 cb = BID;
        const bool shot = cb - 64u < 128u && wid >= 4;
        const bool puller = cb >= 128u || shot, second = cb < 192u && !shot;
        const u32 src = (cb >> 3) & 7;
        const u32 wi = shot ? (((cb >> 6) - 1u) * 4u + (wid - 4)) * 8u + (cb & 7)
                     : !puller ? ((cb >> 5) * RWPC + wid) * 8u + (cb & 7)
                     : wid * 8u + ((cb + src) & 7);
        if (shot ? (wi >= NPUSH || (src ^ rank) < 2u) : puller ? (wi >= NPW || (src ^ rank) < 2u) : (wid >= RWPC || wi >= NRS)) return;
        LANE_VARS();
        const u32 call = uni(__hip_atomic_load(cnt, __ATOMIC_RELAXED, __HIP_MEMORY_SCOPE_AGENT)) + 1u;
        PUT_CALL();
        __builtin_amdgcn_s_setprio(CPRIO);
        bool ok = true;
        LOGDECL();
        LAS char* const img = (LAS char*)lds + wid * 4096;
#pragma unroll
        for (int i = 0; i < 4; ++i) *(LAS u32x4*)(img + i * 1024 + v16) = (u32x4){0, 0, 0, 0};
        FENCE();
        if (puller) {
            const rsrc_t rs = mkrs(P.ws[src], WS_USED);
            const u32 par = call & 1, sj = 14u + (src ^ rank);
            const gu32* const fla = (const gu32*)(my + OFF_IFA + src * 16384u);
            const gu32* const flb = (const gu32*)(my + OFF_IFL + src * 1024u + (lane & 7) * 128u);
            u64 ma = 0, mb = shot ? ~0ull : 0ull;
            auto ready = [&](u32 u) {
                const u32 sg = u < UB ? (u - UA) >> 6 : u < UC ? 2u + (u - UB) / 40u : (64u - G0) / 8u + (u - UC) / 48u;
                return ((u < UA ? ma >> (u + 64u - UA) : mb >> sg) & 1) != 0;
            };
            auto look = [&](u32 u) {
                if (u < UA) return uni(__hip_atomic_load(fla + u * 32u, __ATOMIC_RELAXED, __HIP_MEMORY_SCOPE_SYSTEM)) >= call;
                mb = __builtin_amdgcn_ballot_w64(__hip_atomic_load(flb, __ATOMIC_RELAXED, __HIP_MEMORY_SCOPE_SYSTEM) >= call);
                return ready(u);
            };
            auto wait = [&](u32 u) {
                const u64 t0 = rt();
                for (u32 it = 1;; ++it) {
                    if (look(u)) return true;
                    if ((it & 63u) == 0 && timed_out(tmo, call, t0)) return false;
                    if (shot) __builtin_amdgcn_s_sleep(SSLEEP); else __builtin_amdgcn_s_sleep(PSLEEP);
                }
            };
            u32 u0 = shot ? wi : NPUSH + wi;
            const u32 ho = OFF_HND + (src * 32u + wi) * 128u, tbase = NPUSH + NPW - (call - 1u) * (NU - NPUSH);
            if (second) {
                u32 hv;
                const u64 t0 = rt();
                for (u32 it = 1;; ++it) {
                    hv = uni(__hip_atomic_load((const gu32*)(my + ho), __ATOMIC_RELAXED, __HIP_MEMORY_SCOPE_AGENT));
                    if ((hv & 0x7FFFFFFFu) >= call) break;
                    if ((it & 63u) == 0 && timed_out(tmo, call, t0)) { ok = false; break; }
                    if (~mb) mb = __builtin_amdgcn_ballot_w64(__hip_atomic_load(flb, __ATOMIC_RELAXED, __HIP_MEMORY_SCOPE_SYSTEM) >= call);
                    __builtin_amdgcn_s_sleep(HSLEEP);
                }
                u0 = !ok || (hv >> 31) ? NU : uni(__builtin_amdgcn_raw_ptr_buffer_atomic_add_i32(1, rmy, fvoff, OFF_TKI + src * 128u, 0)) + tbase;
                if (u0 >= NU) {
                    st4_dev(call, rmy, fvoff, OFF_TOK + (src * 64u + wi) * 64u);
                    return;
                }
            }
            u32 e0 = entry(u0, src);
            u32 b0 = (((e0 >> 7) ? rb : ra) * 2u + rc) * 512u + u0;
            const rsrc_t rd = mkrs(shot ? my + OFF_PSH + par * PSH_PAR + (src * NPUSH + u0) * BLK - b0 * BLK : P.ws[src] + OFF_IPK + par * IPK_PAR, 2048u * BLK);
            u32 tvoff = shot ? 0x80000000u : fvoff;
            bool last = shot;
            if (ok && !(second && ready(u0))) ok = wait(u0);
            STAMPT(2);
            Blk N; u32 tkt = 0, fa = 0, fb = 0;
#define IISSUE(N_, b_) do { FENCE(); tkt = __builtin_amdgcn_raw_ptr_buffer_atomic_add_i32(1, rmy, tvoff, OFF_TKI + src * 128u, 0); \
                load_blk(N_, rd, (b_) * BLK, lane); FENCE(); } while (0)
#define ILOOK(b_) do { fb = __builtin_amdgcn_raw_buffer_load_b32(rmy, OFF_IFL + src * 1024u + (lane & 7) * 128u, 0, 17); \
                fa = __builtin_amdgcn_raw_buffer_load_b32(rmy, lane + (UA - 64u) > max((b_) & 511u, NPUSH + NPW - 1u) ? \
                        OFF_IFA + src * 16384u + (lane + (UA - 64u)) * 128u : 0x80000000u, 0, 17); FENCE(); } while (0)
#define INEXT() do { u1 = last ? NU : uni(tkt) + tbase; e1 = entry(u1, src); \
                b1 = (((e1 >> 7) ? rb : ra) * 2u + rc) * 512u + (u1 & 511u); \
                if (!second && !last && u1 >= HAND) { st4_dev(u1 < NU ? call : call | 0x80000000u, rmy, fvoff, ho); tvoff = 0x80000000u; last = true; } } while (0)
#define IUNPACK(B_) do { u32 hi_[8]; unpack_hi(B_, rs, OFF_IOV + par * IOV_PAR + b0 * 2048u, lane, img, hi_); \
                const u32 ih_ = ((e0 >= 128u ? ID_WR - 16u + ((e0 >> 3) & 15u) * 48u : (e0 >> 3) * 64u) + sj + (e0 & 7) * 6u) * 2u + (u0 & 1); \
                store_frags(hi_, B_.l0, B_.l1, rmy, v16 + OFF_IMG + ih_ * 4096u); rp = OFF_RDY + ih_ * 128u; LOGU(src * 512u + u0); } while (0)
            if (ok) {
                IISSUE(N, b0);
                if (!shot) ILOOK(b0);
                WAITV(0);
                STAMPT(5);
                u32 rp = 0x80000000u;
                u32 u1, e1, b1;
                INEXT();
                bool go = false;
                for (;;) {
                    if (!go) {
                        ma |= __builtin_amdgcn_ballot_w64(fa >= call); mb |= __builtin_amdgcn_ballot_w64(fb >= call);
                        if (u1 >= NU || (u1 >= UA && !~mb)) break;
                        go = ready(u1) || look(u1);
                    }
                    Blk B = N;
                    asm volatile("" : "+v"(B.l0), "+v"(B.l1), "+v"(B.cs), "+v"(B.tail));
                    if (go) {
                        IISSUE(N, b1);
                        ma |= __builtin_amdgcn_ballot_w64(fa >= call); mb |= __builtin_amdgcn_ballot_w64(fb >= call);
                        ILOOK(b1); WAITV(7);
                    } else {
                        WAITV(0);
                    }
                    st4_dev(call, rmy, fvoff | rp, 0);
                    IUNPACK(B);
                    if (!go) {
                        WAITV(0);
                        st4_dev(call, rmy, fvoff | rp, 0);
                        rp = 0x80000000u;
                        if (!wait(u1)) { ok = false; break; }
                        IISSUE(N, b1); ILOOK(b1);
                        WAITV(0);
                    }
                    u0 = u1; e0 = e1; b0 = b1;
                    WAITV(11); INEXT();
                    go = u1 < NU && (u1 < UA || ~mb) && ready(u1);
                    { u32 g_ = go; asm volatile("" : "+s"(u1), "+s"(e1), "+s"(b1), "+s"(g_)); go = g_; }
                    WAITV(5);
                }
                if (ok) {
                    Blk M;
                    bool tok = !second;
#define ISTEP(C_, N_) { if (!tok && u1 + TOKE >= NU) { st4_dev(call, rmy, fvoff, OFF_TOK + (src * 64u + wi) * 64u); tok = true; } \
                        if (u1 >= NU) { WAITV(0); st4_dev(call, rmy, fvoff | rp, 0); IUNPACK(C_); break; } \
                        IISSUE(N_, b1); \
                        WAITV(5);             \
                        st4_dev(call, rmy, fvoff | rp, 0); \
                        IUNPACK(C_); \
                        u0 = u1; e0 = e1; b0 = b1; \
                        WAITV(9); INEXT(); asm volatile("" : "+s"(u1), "+s"(e1), "+s"(b1)); \
                        WAITV(5); }
                    for (;;) { ISTEP(N, M) ISTEP(M, N) }
                    WAITV(0);
                    st4_dev(call, rmy, fvoff | rp, 0);
                }
            }
            STAMPT(3);
        } else {
            const u32 st = src & 3;
            const rsrc_t r0 = mkrs(P.ws[st * 2u], WS_USED), r1 = mkrs(P.ws[st * 2u + 1u], WS_USED);
            const rsrc_t ro = mkrs(P.out, 512u * 4096u * 2u);
            const gu32* const fl = (const gu32*)(my + ((lane & 2) ? OFF_DFL + st * 4096u : OFF_OFL + st * 8192u + (lane & 1) * 4096u));
            const u32 ovoff = ((lane >> 4) * 4u + ((lane >> 3) & 1)) * 8192u + (lane & 7) * 16u;
            auto wait = [&](u32 v) {
                const u64 t0 = rt();
                for (u32 it = 1;; ++it) {
                    if (__builtin_amdgcn_ballot_w64(__hip_atomic_load(fl + (v >> 3) * 32u, __ATOMIC_RELAXED, __HIP_MEMORY_SCOPE_SYSTEM) < call) == 0) return true;
                    if ((it & 63u) == 0 && timed_out(tmo, call, t0)) return false;
                    __builtin_amdgcn_s_sleep(RSLEEP);
                }
            };
            u32 v0 = wi;
            {
                const u32 ts = st * 2u + ((wi >> 3) & 1);
                const gu32* tok = (const gu32*)(my + OFF_TOK + (((ts ^ rank) < 2u ? ts ^ 2u : ts) * 64u + wi % NPW) * 64u);
                const u64 t0 = rt();
                for (u32 it = 1;; ++it) {
                    const u32 tv = __hip_atomic_load(tok, __ATOMIC_RELAXED, __HIP_MEMORY_SCOPE_AGENT);
                    if (uni(tv) >= call) break;
                    if ((it & 63u) == 0 && timed_out(tmo, call, t0)) { ok = false; break; }
                    __builtin_amdgcn_s_sleep(TSLEEP);
                }
            }
            STAMPT(2);
            if (wi >= RFIRST) for (int i = 0; i < RDELAY; ++i) __builtin_amdgcn_s_sleep(8);
            bool rdy = false;
            for (;;) {
                if (ok && !rdy) ok = wait(v0);
                FENCE();
                Blk A, C;
                load_blk(A, r0, OFF_OPK + (rank * 256u + v0) * BLK, lane);
                load_blk(C, r1, OFF_OPK + (rank * 256u + v0) * BLK, lane);
                FENCE();
                const u32 tkt = __builtin_amdgcn_raw_ptr_buffer_atomic_add_i32(1, rmy, fvoff, OFF_TKO + st * 128u, 0);
                const u32 tm = __hip_atomic_load(tmo, __ATOMIC_RELAXED, __HIP_MEMORY_SCOPE_AGENT);
                u32x4 dg[8];
#pragma unroll
                for (int k = 0; k < 8; ++k) dg[k] = __builtin_amdgcn_raw_buffer_load_b128(rmy, v16 + (k & 3) * 1024u, OFF_DIA + (st * 256u + v0) * 8192u + (k >> 2) * 4096u, 16);
                FENCE();
                const bool bad = !ok || uni(tm) == call;
                if (bad) { A.tail = 0; C.tail = 0; }
                const u32 v1 = uni(tkt) - (call - 1u) * 256u + NRS;
                const u32 ov = OFF_OOV + (rank * 256u + v0) * 2048u;
                u32 ha[8], hc[8], res[16];
                unpack_hi(A, r0, ov, lane, img, ha);
                unpack_hi(C, r1, ov, lane, img + 2048, hc);
#pragma unroll
                for (int k = 0; k < 8; ++k) asm volatile("" : "+v"(ha[k]), "+v"(hc[k]));
                FENCE();
                const u32 fv = __hip_atomic_load(fl + ((v1 >> 3) & 31u) * 32u, __ATOMIC_RELAXED, __HIP_MEMORY_SCOPE_SYSTEM);
                FENCE();
                const u32x4 a0 = A.l0, a1 = A.l1, c0 = C.l0, c1 = C.l1;
#pragma unroll
                for (int k = 0; k < 8; ++k) {
                    float ya[4], yc[4];
#pragma unroll
                    for (int e = 0; e < 4; ++e) {
                        const u32 sel = 0x04000c0cu + e * 0x01010000u;
                        ya[e] = __builtin_bit_cast(float, __builtin_amdgcn_perm(ha[k], k < 4 ? a0[k & 3] : a1[k & 3], sel));
                        yc[e] = __builtin_bit_cast(float, __builtin_amdgcn_perm(hc[k], k < 4 ? c0[k & 3] : c1[k & 3], sel));
                    }
                    const f32x4 yd = __builtin_bit_cast(f32x4, dg[k]);
                    res[2 * k] = silu2((f32x2){ya[0], ya[1]} + (f32x2){yc[0], yc[1]} + (f32x2){yd[0], yd[1]});
                    res[2 * k + 1] = silu2((f32x2){ya[2], ya[3]} + (f32x2){yc[2], yc[3]} + (f32x2){yd[2], yd[3]});
                }
                if (bad) {
#pragma unroll
                    for (int i = 0; i < 16; ++i) res[i] = 0x7FC07FC0u;
                }
                const u32 row0 = (st >> 1) * 256u + ((v0 >> 3) & 1) * 128u + ((v0 >> 2) & 1) * 64u + (v0 & 1) * 32u;
                const u32 col0 = (st & 1) * 2048u + (v0 >> 4) * 128u + ((v0 >> 1) & 1) * 64u;
#pragma unroll
                for (int k = 0; k < 8; k += 2) {
                    u32x4 o;
#pragma unroll
                    for (int d = 0; d < 2; ++d) {
                        const u32 x = res[2 * k + d], y = res[2 * k + 2 + d];
                        o[d] = __builtin_amdgcn_update_dpp(x, y, 0x128, 0xf, 0xc, false);
                        o[2 + d] = __builtin_amdgcn_update_dpp(y, x, 0x128, 0xf, 0x3, false);
                    }
                    st16_dev(o, ro, ovoff, (row0 + 16u * (k >> 2) + (k & 3)) * 8192u + col0 * 2u);
                }
                LOGU(4096u + st * 256u + v0);
                if (v1 >= 256u) break;
                rdy = __builtin_amdgcn_ballot_w64(fv < call) == 0;
                v0 = v1;
            }
        }
        STAMPT(4);
        LOGOUT();
        return;
    }

    const u32 gb = BID - NCOMM;
    u32 call;
    {
        LANE_VARS();
        const u32 pc = gb & 3, ent0 = pk0 & 255u, op = ent0 >> 7;
        for (u32 i = pk0 >> 20; i; --i) __builtin_amdgcn_s_sleep(1);
        STAMPT(10);
        u32 in[16], lo[8], hi[8];
        unit_load(in, P.x, P.w, op, pk1 + (op ? (pc >> 1) * 4096u + (pc & 1) * 2097152u
                  : (pc >> 1) * 262144u + (pc & 1) * 512u + (ent0 >= 112u ? rank * 524288u : ent0 >= rank * 16u ? 524288u : 0u)), lane);
        SB();
        const u32 u = (pk0 >> 8) & 511u, sg = (pk0 >> 17) & 7u, ent = entry_fix(ent0, rank);
        const bool push = u < NPUSH, early = u < UA;
        FENCE();
        const u32 cntv = __hip_atomic_load(cnt, __ATOMIC_RELAXED, __HIP_MEMORY_SCOPE_AGENT);
        FENCE();
        const u32 g0 = (op ? (pc & 2) : (pc & 2) * 2u) + (pc & 1), g1 = g0 + (op ? 4u : 2u);
        const u32 po = OFF_PSH + (rank * NPUSH + min(u, NPUSH - 1u)) * BLK;
        u64 w0 = (u64)P.ws[g0 & 7], w1 = (u64)P.ws[g1 & 7];
        asm volatile("" : "+s"(w0), "+s"(w1));
        u64 a0 = push ? w0 + po : (u64)my + OFF_IPK + (pc * 512u + u) * BLK, a1 = w1 + po;
        gu32* const df = (gu32*)((lane ? w1 : w0) + OFF_IFA + rank * 16384u + u * 128u);
        const bool dof = early && lane < 2 && ((lane ? g1 : g0) ^ rank) >= 2u;
        const u32 tile = (ent >> 3) & 15u, kd = (pc & 1) * 8u + (ent & 7);
        u32 ih = (op ? ID_WD + ((pc >> 1) * 16u + tile) * 16u + kd : u >= NU ? ID_XD + ((pc & 2) + (tile & 1)) * 16u + kd : tile * 64u + kd) * 2u + (u & 1);
        if (!(op || u >= NU || (pc >> 1) == ra)) ih = 0xFFFFFFFFu;
        asm volatile("" : "+s"(ih) : "v"(df), "s"(__builtin_amdgcn_ballot_w64(dof)));
        u32 n0 = push && (g0 ^ rank) < 2u ? 0u : BLK, n1 = !push || (g1 ^ rank) < 2u ? 0u : BLK, ps = push ? PSH_PAR : IPK_PAR, hold = push ? (u32)wid : 0u;
        asm volatile("" : "+s"(a0), "+s"(a1), "+s"(n0), "+s"(n1), "+s"(ps), "+s"(hold));
        SB();
        call = uni(cntv) + 1u;
        const u32 par = call & 1;
        const rsrc_t d0 = mkrs((const char*)a0 + (par ? ps : 0u), n0), d1 = mkrs((const char*)a1 + (par ? ps : 0u), n1);
        unit_split(in, op, lo, hi);
        const u32x4 l0 = {lo[0], lo[1], lo[2], lo[3]}, l1 = {lo[4], lo[5], lo[6], lo[7]};
        if (hold) { asm volatile("" :: "v"(l0), "v"(l1)); __builtin_amdgcn_s_sleep(PUSHD); }
        STAMPT(11);
        st16_sys(l0, d0, v16, 0); st16_sys(l1, d0, v16 + 1024, 0);
        st16_sys(l0, d1, v16, 0); st16_sys(l1, d1, v16 + 1024, 0);
        FENCE(); SB();
        u32 mu = in[0], mi = in[1];
#pragma unroll
        for (int i = 2; i < 16; i += 2) { mu = pkmax(mu, in[i]); mi = pkmaxi(mi, in[i + 1]); }
        mu = pkmax(mu & 0x7FFF7FFFu, mi & 0x7FFF7FFFu);
        Pk pk;
        pack_hi(lo, hi, max(mu >> 16, mu & 0xFFFFu) >> 7, lane, (LAS char*)lds + wid * 4096, pk);
        STAMPT(15);
        store_blk_cs(pk, d0, 0, lane);
        store_blk_cs(pk, d1, 0, lane);
        store_blk_raw(pk, hi, rmy, OFF_IOV + par * IOV_PAR + (pc * 512u + u) * 2048u, lane);
        if (early) {
            WAITV(0);
            if (dof) __hip_atomic_store(df, call, __ATOMIC_RELAXED, __HIP_MEMORY_SCOPE_SYSTEM);
            STAMPT(14);
        }
        if (ih != 0xFFFFFFFFu) store_frags(hi, l0, l1, rmy, v16 + OFF_IMG + ih * 4096u);
        WAITV(0);
        if (ih != 0xFFFFFFFFu) st4_dev(call, rmy, fvoff, OFF_RDY + ih * 128u);
        PUT_CALL();
        if (!early) STAMPT(14);
        BAR();
        if (wid == 3) {
            const u32 t = __builtin_amdgcn_raw_ptr_buffer_atomic_add_i32(1, rmy, fvoff, OFF_PCN + sg * 128u, 0);
            if (uni(t) == call * 32u - 1u) {
                u64 q[NR];
#pragma unroll
                for (int i = 0; i < NR; ++i) { q[i] = (u64)P.ws[i]; asm volatile("" : "+s"(q[i])); }
                u64 dv = q[0];
#pragma unroll
                for (int i = 1; i < NR; ++i) dv = lane == i ? q[i] : dv;
                if (lane < 8) __hip_atomic_store((gu32*)(dv + OFF_IFL + rank * 1024u + sg * 128u), call, __ATOMIC_RELAXED, __HIP_MEMORY_SCOPE_SYSTEM);
            }
        }
    }
    const u32 xcd = gb & 7, idx = gb >> 3;
    const u32 rt_ = (xcd >> 1) * 4 + (idx & 3), ct = (((idx >> 2) * 2) ^ ((idx >> 2) & 4u)) + (xcd & 1);

    const u32 owner = rt_ >> 1;
    const bool noown = owner == rank;
    const int pass0 = (gb & 64u) ? 0 : 1;
    const u32 drt = idx & 3, dct = xcd * 4u + ((idx >> 2) & 1) + ((idx >> 4) & 1) * 2u;
    const int N1 = noown ? 48 : (owner ^ rank) == 1u ? 64 : 56;
    u32 pv[16];
    const u32 blk = owner * 256u + ((ct * 2u + (rt_ & 1)) * 4u + (wid & 3)) * 2u + (wid >> 2);
    u64 eblk = (u64)(my + OFF_OPK + blk * BLK), eflag = (u64)(P.ws[owner & 7] + OFF_OFL + rank * 4096u + (ct * 2u + (rt_ & 1)) * 128u);
    asm volatile("" : "+s"(eblk), "+s"(eflag));

    if (wid >= 4) {
        LANE_VARS();
        const u32 l = wid - 4;
        const rsrc_t rs = mkrs(my + OFF_IMG + (l & 1) * 4096u, NID * 8192u - 4096u);
        const rsrc_t rf = mkrs(my + OFF_RDY + (l & 1) * 128u, NID * 256u - 128u);
        __builtin_amdgcn_s_setprio(LPRIO);
#define LSTAMP(s_)
        for (int pass = pass0; pass < 2; ++pass) {
            const int N = pass ? N1 : 16;
            u32 idv;
            {
                const u32 hole = (owner ^ rank) - 2u;
                const u32 i = noown ? lane + 16u : lane, j = i - 16u, ks = j / 5u, r = j - ks * 5u;
                const u32 kp = min(i < 16u || hole > 5u ? i : 16u + ks * 6u + r + (r >= hole), 63u);
                const u32 cube = (l >> 1) ? (kp < 16u ? ID_WD + (rb * 16u + ct) * 16u : ID_WR - 16u + ct * 48u) + kp : rt_ * 64u + kp;
                const u32 first = (l >> 1) ? ID_WD + dct * 16u : ID_XD + drt * 16u;
                idv = pass ? cube : first + (lane & 15);
            }
            int avail = 0;
            auto window = [&](u32 n) { return (u32)(lane - avail) < n ? idv * 256u : 0x80000000u; };
            auto seen = [&](u32 f) { if (avail < 64) { const u64 miss = __builtin_amdgcn_ballot_w64(f < call) >> avail << avail; avail = miss ? (int)__builtin_ctzll(miss) : 64; } };
            auto need = [&](int s1) {
                if (__builtin_expect(avail <= s1, 0)) {
                    const u64 t0 = rt();
                    for (u32 it = 1;; ++it) {
                        FENCE();
                        seen(__builtin_amdgcn_raw_buffer_load_b32(rf, window(it == 1 ? LWIN1 : LWINB), 0, 16));
                        if (avail > s1) break;
                        if ((it & 63u) == 0 && timed_out(tmo, call, t0)) { avail = 64; break; }
                        __builtin_amdgcn_s_sleep(LSLEEP);
                    }
                    WAITL0();
                }
            };
            const u32 sov = idv * 8192u, dl0 = l * 4096u;
            u32 so = rdl(sov, 0), dl = dl0, f = 0, fo = 0x80000000u;
            auto issue = [&](int s1, bool look) {
                need(s1);
                LAS void* const d = (LAS void*)((LAS char*)lds + dl);
                __builtin_amdgcn_raw_ptr_buffer_load_lds(rs, d, 16, v16, so, 0, 16);
                SB();
                u32 son = rdl(sov, (s1 + 1) & 63), dln = dl + 16384u;
                dln = dln == dl0 + NS * 16384u ? dl0 : dln;
                asm volatile("" : "+s"(son), "+s"(dln));
                SB();
                __builtin_amdgcn_raw_ptr_buffer_load_lds(rs, d, 16, v16, so, 1024, 16);
                SB();
                if (look) { const u64 miss = __builtin_amdgcn_ballot_w64(f < call && lane >= avail); avail = miss ? (int)__builtin_ctzll(miss) : 64; asm volatile("" : "+s"(avail)); }
                SB();
                __builtin_amdgcn_raw_ptr_buffer_load_lds(rs, d, 16, v16, so, 2048, 16);
                SB();
                if (look) { fo = window(avail < s1 + LAHEAD ? LWIN : 0); asm volatile("" : "+v"(fo)); }
                SB();
                __builtin_amdgcn_raw_ptr_buffer_load_lds(rs, d, 16, v16, so, 3072, 16);
                FENCE(); SB();
                so = son; dl = dln;
            };
            if (pass && l == 2) {
                const u64 t0 = rt();
                for (u32 it = 1; uni(__hip_atomic_load((const gu32*)(my + OFF_IFL + owner * 1024u), __ATOMIC_RELAXED, __HIP_MEMORY_SCOPE_SYSTEM)) < call; ++it) {
                    if ((it & 63u) == 0 && timed_out(tmo, call, t0)) break;
                    __builtin_amdgcn_s_sleep(LSLEEP);
                }
            }
            if (!pass && gb >= 192u && ((idx ^ rank) & 6u) != 0) {
                const gu32* const h = (const gu32*)(my + OFF_HND + ((idx & 7u) * 32u + ((gb + idx) & 7u)) * 128u);
                const u64 t0 = rt();
                for (u32 it = 1; (uni(__hip_atomic_load(h, __ATOMIC_RELAXED, __HIP_MEMORY_SCOPE_AGENT)) & 0x7FFFFFFFu) < call; ++it) {
                    if ((it & 15u) == 0 && timed_out(tmo, call, t0)) break;
                    __builtin_amdgcn_s_sleep(GSLEEP);
                }
                __builtin_amdgcn_s_sleep(GATEK);
            }
            for (int s = 0; s < 5; ++s) issue(s, false);
            asm volatile("s_waitcnt vmcnt(16)" ::: "memory"); BAR();
            asm volatile("s_waitcnt vmcnt(12)" ::: "memory"); BAR();
            int s = 5;
            for (; s < N - 3; s += 4) {
                issue(s, true);
                asm volatile("buffer_load_dword %0, %1, %2, 0 offen sc1" : "=v"(f) : "v"(fo), "s"(rf) : "memory");
                LSTAMP(s - 6);
                asm volatile("s_waitcnt vmcnt(13)" ::: "memory"); BAR();
                issue(s + 1, false);
                asm volatile("s_waitcnt vmcnt(13)" ::: "memory"); BAR();
                issue(s + 2, false);
                asm volatile("s_waitcnt vmcnt(13)" ::: "memory"); BAR();
                issue(s + 3, false);
                asm volatile("s_waitcnt vmcnt(12)" : "+v"(f) :: "memory"); BAR();
            }
            seen(f);
            for (; s < N; ++s) { issue(s, false); LSTAMP(s - 6); asm volatile("s_waitcnt vmcnt(12)" ::: "memory"); BAR(); }
            asm volatile("s_waitcnt vmcnt(8)" ::: "memory"); BAR();
            asm volatile("s_waitcnt vmcnt(4)" ::: "memory"); BAR();
            asm volatile("s_waitcnt vmcnt(0)" ::: "memory"); BAR();
            BAR(); BAR();
            if (!pass) BAR();
            LSTAMP(55);
        }
        __builtin_amdgcn_s_setprio(0);
        BAR();
#pragma unroll
        for (int i = 0; i < 4; ++i) {
            const u32x4 v = *(LAS u32x4*)((LAS char*)lds + 32768 + l * 4096 + i * 1024 + v16);
            pv[4 * i] = v[0]; pv[4 * i + 1] = v[1]; pv[4 * i + 2] = v[2]; pv[4 * i + 3] = v[3];
        }
    } else {
        const u32 la = (wid >> 1) * 4096, lb = 8192 + (wid & 1) * 4096;
        u32 pla = threadIdx.x * 16u, plb;
#define V16() (pla & 1023u)
        f32x4 acc[4][4];
        bf16x8 A0, A1, C0[2], C1[2], B0[4], B1[4];
#define RD(d, q, i) d = *(const LAS bf16x8*)((LAS char*)lds + p##q + (i) * 1024)
#define M1(a, B, m, n) acc[m][n] = __builtin_amdgcn_mfma_f32_16x16x32_bf16(a, B[n], acc[m][n], 0, 0, 0)
#define NEXT() do { const u32 d_ = so == (NS - 1) * 16384u ? 0u - so : 16384u; so += d_; pla += d_; plb += d_; asm volatile("" : "+v"(pla), "+v"(plb)); } while (0)
#define STEP(Cc, Bc, Cn, Bn) do { SB(); \
            M1(A0, Bc, 0, 0); SB(); RD(Bn[0], lb, 0); SB(); M1(A0, Bc, 0, 1); SB(); RD(Bn[1], lb, 1); SB(); M1(A0, Bc, 0, 2); SB(); RD(Bn[2], lb, 2); SB(); M1(A0, Bc, 0, 3); SB(); RD(A0, la, 0); SB(); \
            M1(A1, Bc, 1, 0); SB(); RD(Bn[3], lb, 3); SB(); M1(A1, Bc, 1, 1); SB(); RD(Cn[0], la, 2); SB(); M1(A1, Bc, 1, 2); SB(); RD(Cn[1], la, 3); SB(); M1(A1, Bc, 1, 3); SB(); RD(A1, la, 1); SB(); \
            M1(Cc[0], Bc, 2, 0); SB(); M1(Cc[0], Bc, 2, 1); SB(); NEXT(); SB(); M1(Cc[0], Bc, 2, 2); SB(); M1(Cc[0], Bc, 2, 3); SB(); \
            M1(Cc[1], Bc, 3, 0); SB(); M1(Cc[1], Bc, 3, 1); SB(); M1(Cc[1], Bc, 3, 2); SB(); WAITL0(); SB(); M1(Cc[1], Bc, 3, 3); SB(); \
            BAR(); \
        } while (0)
        for (int pass = pass0;; ++pass) {
#pragma unroll
            for (int m = 0; m < 4; ++m)
#pragma unroll
                for (int n = 0; n < 4; ++n) acc[m][n] = (f32x4){0.f, 0.f, 0.f, 0.f};
            u32 so = 0;
            asm volatile("" : "+v"(pla));
            plb = V16() + lb; pla = V16() + la;
            BAR();
            RD(A0, la, 0); RD(A1, la, 1); RD(C0[0], la, 2); RD(C0[1], la, 3);
#pragma unroll
            for (int i = 0; i < 4; ++i) RD(B0[i], lb, i);
            SB(); NEXT();
            WAITL0(); BAR();
            for (int s = pass ? N1 : 16; s > 0; s -= 2) {
                STEP(C0, B0, C1, B1);
                STEP(C1, B1, C0, B0);
            }
            if (pass) break;
            {
                const u32 vo = V16() + OFF_DIA + (((drt & 2) + (dct >> 4)) * 256u + (((dct & 15u) * 2u + (drt & 1)) * 4u + wid) * 2u) * 8192u;
#pragma unroll
                for (int m = 0; m < 4; ++m)
#pragma unroll
                    for (int j = 0; j < 4; ++j)
                        st16_dev(__builtin_bit_cast(u32x4, (f32x4){acc[m][0][j], acc[m][1][j], acc[m][2][j], acc[m][3][j]}), rmy, vo + m * 4096u + j * 1024u, 0);
            }
            WAITV(0);
            BAR();
            if (wid == 0 && V16() == 0 && __hip_atomic_load(tmo, __ATOMIC_RELAXED, __HIP_MEMORY_SCOPE_AGENT) != call)
                __hip_atomic_store((gu32*)(my + OFF_DFL + (((drt & 2) + (dct >> 4)) * 32u + (dct & 15u) * 2u + (drt & 1)) * 128u), call, __ATOMIC_RELAXED, __HIP_MEMORY_SCOPE_AGENT);
        }
        STAMPT(9);
#pragma unroll
        for (int i = 0; i < 4; ++i) {
            u32x4 v;
#pragma unroll
            for (int e = 0; e < 4; ++e) { const int k = 2 * i + (e >> 1), m = 2 + (k >> 2), j = k & 3; v[e] = cvt_pk_bf16(acc[m][2 * (e & 1)][j], acc[m][2 * (e & 1) + 1][j]); }
            *(LAS u32x4*)((LAS char*)lds + 32768 + wid * 4096 + i * 1024 + V16()) = v;
        }
        WAITL0(); BAR();
#pragma unroll
        for (int k = 0; k < 8; ++k) { pv[2 * k] = cvt_pk_bf16(acc[k >> 2][0][k & 3], acc[k >> 2][1][k & 3]); pv[2 * k + 1] = cvt_pk_bf16(acc[k >> 2][2][k & 3], acc[k >> 2][3][k & 3]); }
    }
    {
        int tz = threadIdx.x; asm volatile("" : "+v"(tz));
        const int ln = tz & 63;
        const rsrc_t re = mkrs((const void*)eblk, BLK);
        u32 lo[8], hi[8], mu = 0, mi = 0x80008000u;
#pragma unroll
        for (int k = 0; k < 8; ++k) {
            lo[k] = __builtin_amdgcn_perm(pv[2 * k + 1], pv[2 * k], 0x06040200u); hi[k] = __builtin_amdgcn_perm(pv[2 * k + 1], pv[2 * k], 0x07050301u);
            mu = pkmax(mu, pv[2 * k]); mi = pkmaxi(mi, pv[2 * k + 1]);
        }
        mu = pkmax(mu & 0x7FFF7FFFu, mi & 0x7FFF7FFFu);
        st16_sys((u32x4){lo[0], lo[1], lo[2], lo[3]}, re, ln * 16u, 0);
        st16_sys((u32x4){lo[4], lo[5], lo[6], lo[7]}, re, ln * 16u + 1024u, 0);
        Pk pk;
        pack_hi(lo, hi, max(mu >> 16, mu & 0xFFFFu) >> 7, ln, (LAS char*)lds + wid * 4096, pk);
        store_blk_cs(pk, re, 0, ln);
        store_blk_raw(pk, hi, rmy, OFF_OOV + blk * 2048u, ln);
        STAMPT(12);
        FENCE();
        const u32 tm = __hip_atomic_load(tmo, __ATOMIC_RELAXED, __HIP_MEMORY_SCOPE_AGENT);
        WAITV(0);
        BAR();
        if (tz == 0 && tm != call)
            __hip_atomic_store((gu32*)eflag, call, __ATOMIC_RELAXED, __HIP_MEMORY_SCOPE_SYSTEM);
        STAMPT(13);
    }
}

}

extern "C" void kernel_launch(int rank, int nranks, ncclComm_t comm,
                              hipStream_t stream,
                              void* const* d_in, const int* in_sizes, int n_in,
                              void* d_out, int out_size,
                              const int* in_dtypes, int out_dtype,
                              void* const* d_out_all,
                              void* const* d_ws_all, size_t ws_bytes) {
    if (nranks != NR || (unsigned)rank >= (unsigned)NR || n_in != 2 || ws_bytes < WS_NEED ||
        in_sizes[0] != 4096 * 512 || in_sizes[1] != 512 * 4096 || out_size != 512 * 4096) {
        fprintf(stderr, "kernel_launch: unexpected problem shape\n");
        return;
    }
    Params P;
    for (int p = 0; p < NR; ++p) P.ws[p] = (char*)d_ws_all[p];
    P.x = (const char*)d_in[0];
    P.w = (const char*)d_in[1];
    P.out = (char*)d_out;
    P.my = P.ws[rank];
    P.rank = rank;
    P.pad = 0;
    hipLaunchKernelGGL(fused, dim3(NCOMM + NGEMM), dim3(512), 0, stream, P);
    HIP_CHECK(hipGetLastError());
}
```

```cpp
#include <hip/hip_runtime.h>
#include <rccl/rccl.h>
#include <cstdio>

#define HIP_CHECK(expr) do { hipError_t _e = (expr); if (_e != hipSuccess) { \
    fprintf(stderr, "HIP error %s at %s:%d\n", hipGetErrorString(_e), __FILE__, __LINE__); \
    return; } } while (0)

namespace {

typedef unsigned u32;
typedef unsigned long long u64;
typedef unsigned u32x4 __attribute__((ext_vector_type(4)));
typedef unsigned u32x3 __attribute__((ext_vector_type(3)));
typedef unsigned u32x2 __attribute__((ext_vector_type(2)));
typedef short i16x2 __attribute__((ext_vector_type(2)));
typedef unsigned short u16x2 __attribute__((ext_vector_type(2)));
typedef short bf16x8 __attribute__((ext_vector_type(8)));
typedef float f32x4 __attribute__((ext_vector_type(4)));
#define LAS __attribute__((address_space(3)))
#define GAS __attribute__((address_space(1)))
typedef GAS u32 gu32;
typedef GAS u64 gu64;
typedef __amdgpu_buffer_rsrc_t rsrc_t;

constexpr int NR = 8;
constexpr int NCOMM = 256;
constexpr int NGEMM = 256;
constexpr u32 NPW = 21;
constexpr u32 HAND = 170;
constexpr u32 TOKE = 24;
constexpr u32 NRS = 42, RWPC = 3;
constexpr int RDELAY = 5;
constexpr u32 NPUSH = 48;
constexpr u32 PUSH0 = 16, PUSHD = 20;
constexpr u32 PSTAG = 24, PSTEP = 1;
constexpr int RFAST = 10, RSLOW = 14;
constexpr int RBASE = 40, REARLY = 3;
constexpr u32 LWIN = 8, LWINB = 4;
constexpr int LAHEAD = 10;
constexpr u32 NU = 480;
constexpr u64 DEADLINE = 50000000ull;

constexpr u32 BLK = 2944;
constexpr u32 OFF_CNT = 0;
constexpr u32 OFF_TMO = 16384;
constexpr u32 OFF_TKI = 16384 + 1024;
constexpr u32 OFF_TKO = 16384 + 2048;
constexpr u32 OFF_PCN = 16384 + 4096;
constexpr u32 OFF_IFL = 24576;
constexpr u32 OFF_TOK = 65536;
constexpr u32 OFF_IFA = 262144;
constexpr u32 OFF_DUM = 393216;
constexpr u32 OFF_OFL = 196608;
constexpr u32 OFF_DFL = 229376;
constexpr u32 OFF_HND = 98304;
constexpr u32 ID_WR = 1024, ID_XD = 1792, ID_WD = 1856, NID = 2368;
constexpr u32 KIB = 1024;
constexpr u32 OFF_RDY = 1024 * KIB;
constexpr u32 OFF_IMG = 2048 * KIB;
constexpr u32 OFF_IPK = 25088 * KIB;
constexpr u32 IPK_PAR = 6144 * KIB;
constexpr u32 OFF_IOV = 37376 * KIB;
constexpr u32 IOV_PAR = 4096 * KIB;
constexpr u32 OFF_OPK = 45568 * KIB;
constexpr u32 OFF_OOV = 51456 * KIB;
constexpr u32 OFF_DIA = 55552 * KIB;
constexpr u32 OFF_PSH = 63744 * KIB;
constexpr u32 PSH_PAR = 8u * NPUSH * BLK;
constexpr u32 WS_USED = OFF_PSH + 2u * PSH_PAR;
static_assert(WS_USED <= 65u * 1024u * KIB && OFF_OPK + 2048u * BLK <= OFF_OOV, "layout");
static_assert(OFF_RDY + NID * 256u <= OFF_IMG && OFF_IMG + NID * 8192u <= OFF_IPK && 2048u * BLK <= IPK_PAR, "layout");

struct Params {
    const char* x;
    const char* w;
    char* my;
    char* out;
    int rank;
    int pad;
    char* ws[NR];
};

struct Order { u32 e[256]; };
constexpr Order make_order() {
    constexpr int SLOW0 = 12;
    int key[17][8] = {};
    for (int ct = 0; ct < 16; ++ct) {
        int t = 8 * ct - RBASE - (ct < REARLY ? 8 * ct : 8 * (REARLY - 1) * (16 - ct) / (17 - REARLY));
        for (int ks = 7; ks >= 0; --ks) { t -= (ct >= SLOW0 && t <= 0) ? RSLOW : RFAST; key[ct][ks] = t; }
    }
    for (int ks = 0; ks < 8; ++ks) { int m = key[0][ks]; for (int ct = 1; ct < 16; ++ct) m = key[ct][ks] < m ? key[ct][ks] : m; key[16][ks] = m; }
    Order o{};
    int n = 0, next[17] = {};
    while (n < 240) {
        int best = -1;
        for (int i = 16; i >= 0; --i) if (next[i] < 8 && (best < 0 || key[i][next[i]] < key[best][next[best]])) best = i;
        if (best == 16) for (int t = 0; t < 14; ++t) o.e[n++] = (u32)(t << 3 | next[16]);
        else o.e[n++] = (u32)(128 | best << 3 | next[best]);
        ++next[best];
    }
    for (int ks = 0; ks < 8; ++ks) { o.e[n++] = (u32)(14 << 3 | ks); o.e[n++] = (u32)(15 << 3 | ks); }
    return o;
}
__constant__ const Order ORDER = make_order();
constexpr u32 G0 = 24, UA = 144u - G0, UB = UA + 128u, UC = UB + (48u - G0) * 5u;
static_assert(NPUSH >= G0 && NPUSH <= 2 * G0 && UA - 64u <= NPUSH + NPW, "whole flag groups; one look covers the ticketed early units");
struct PackTab { u32 e[512][2]; };
constexpr PackTab make_pack() {
    const Order o = make_order();
    PackTab t{};
    for (u32 g = 0; g < 64; ++g)
        for (u32 w = 0; w < 8; ++w) {
            const u32 bg = g >= 48u ? g - 48u : g >= G0 ? g - G0 + 16u : g + 64u - G0;
            const bool early = w < (g < G0 ? 2u : g < 48u ? 3u : 0u);
            const u32 u = g < G0 ? (early ? w * G0 + g + (w * G0 + g < NPUSH ? 0u : 3u * (48u - G0)) : UC + g * 6u + (w - 2))
                        : g < 48u ? (early ? NPUSH - G0 + g + w * (48u - G0) : UB + (g - G0) * 5u + (w - 3)) : UA + bg * 8u + w;
            const u32 e = o.e[u >> 1], tile = (e >> 3) & 15u, ks = e & 7u, half = u & 1u;
            const u32 delay = early || g >= 48u ? 0u : PSTAG + (bg - 16u) * PSTEP;
            t.e[g * 8 + w][0] = e | u << 8 | (bg >> 3) << 17 | delay << 20;
            t.e[g * 8 + w][1] = e >= 128u ? 32u * ks * 8192u + (128u * tile + 64u * half) * 2u
                                          : ((tile < 14u ? 512u * (tile >> 1) : 0u) + 128u * (tile & 1u) + 64u * half) * 1024u + 64u * ks;
        }
    return t;
}
__constant__ const PackTab PACK = make_pack();
__device__ __forceinline__ u32 entry_fix(u32 e, u32 src) {
    const u32 own = e - 112u + src * 16u, oth = e + (e >= src * 16u ? 16u : 0u), x = e >= 112u ? own : oth;
    return e >= 128u ? e : x;
}
__device__ __forceinline__ u32 entry(u32 u, u32 src) { return entry_fix(ORDER.e[(u >> 1) & 255u], src); }

__device__ __forceinline__ u64 rt() { return __builtin_amdgcn_s_memrealtime(); }
__device__ __forceinline__ u32 uni(u32 v) { return __builtin_amdgcn_readfirstlane(v); }
__device__ __forceinline__ u32 rdl(u32 v, u32 l) { return __builtin_amdgcn_readlane(v, l); }
__device__ __forceinline__ rsrc_t mkrs(const void* p, u32 bytes) { return __builtin_amdgcn_make_buffer_rsrc((void*)p, 0, bytes, 0x00020000); }
__device__ __forceinline__ u32x4 ld16(rsrc_t r, u32 voff, u32 soff) { return __builtin_amdgcn_raw_buffer_load_b128(r, voff, soff, 0); }
__device__ __forceinline__ u32x2 ld8(rsrc_t r, u32 voff, u32 soff) { return __builtin_amdgcn_raw_buffer_load_b64(r, voff, soff, 0); }
__device__ __forceinline__ u32x4 ld16_sys(rsrc_t r, u32 voff, u32 soff) { return __builtin_amdgcn_raw_buffer_load_b128(r, voff, soff, 17); }
__device__ __forceinline__ void st16_sys(u32x4 v, rsrc_t r, u32 voff, u32 soff) { __builtin_amdgcn_raw_buffer_store_b128(v, r, voff + soff, 0, 17); }
__device__ __forceinline__ void st16_dev(u32x4 v, rsrc_t r, u32 voff, u32 soff) { __builtin_amdgcn_raw_buffer_store_b128(v, r, voff + soff, 0, 16); }
__device__ __forceinline__ void st4_dev(u32 v, rsrc_t r, u32 voff, u32 soff) { __builtin_amdgcn_raw_buffer_store_b32(v, r, voff, soff, 16); }
#define FENCE() asm volatile("" ::: "memory")
#define WAITV(n) do { asm volatile("s_waitcnt vmcnt(" #n ")" ::: "memory"); __builtin_amdgcn_s_waitcnt(0xF70 | (n)); } while (0)
#define WAITL0() do { asm volatile("s_waitcnt lgkmcnt(0)" ::: "memory"); __builtin_amdgcn_s_waitcnt(0xC07F); } while (0)
#define SB() __builtin_amdgcn_sched_barrier(0)
#define BAR() do { FENCE(); __builtin_amdgcn_s_barrier(); FENCE(); } while (0)


__device__ __forceinline__ u32 pkmax(u32 a, u32 b) {
    return __builtin_bit_cast(u32, __builtin_elementwise_max(__builtin_bit_cast(u16x2, a), __builtin_bit_cast(u16x2, b)));
}
__device__ __forceinline__ u32 pkmaxi(u32 a, u32 b) {
    return __builtin_bit_cast(u32, __builtin_elementwise_max(__builtin_bit_cast(i16x2, a), __builtin_bit_cast(i16x2, b)));
}
struct Pk { u32x3 cs; u32 tail; bool raw; };
__device__ __forceinline__ u32 bfi(u32 m, u32 a, u32 b) { return __builtin_amdgcn_bitop3_b32(m, a, b, 0xCA); }
#define DPP0(v, ctrl, rm) (u32)__builtin_amdgcn_update_dpp(0, (int)(v), ctrl, rm, 0xf, true)
__device__ __forceinline__ void pack_hi(const u32 (&lo)[8], const u32 (&hi)[8], u32 m, int lane, LAS char* sc, Pk& p) {
    m = max(m, DPP0(m, 0x111, 0xf)); m = max(m, DPP0(m, 0x112, 0xf)); m = max(m, DPP0(m, 0x114, 0xf)); m = max(m, DPP0(m, 0x118, 0xf));
    u32 T0 = min(rdl(m, 15), rdl(m, 31)), T1 = min(rdl(m, 47), rdl(m, 63));
    asm volatile("" : "+s"(T0), "+s"(T1));
    u32 T = max(min(T0, T1), 7u);
    asm volatile("" : "+s"(T));
    const u32 L = T - 7u, s = L & 1u, base = (L + s) >> 1;
    const u32 k4 = (128u - base) * 0x01010101u,
              s4 = s * 0x01010101u;
    u32 c7f = 0x7F7F7F7Fu, c7c = 0x7C7C7C7Cu;
    asm volatile("" : "+s"(c7f), "+s"(c7c));
    u32 d[8], x[8], bits = 0;
#pragma unroll
    for (int k = 7; k >= 0; --k) {
        d[k] = (hi[k] & c7f) + ((lo[k] >> 7) & s4) + k4;
        x[k] = d[k] & c7c;
        bits = k < 7 ? bfi(c7f, bits >> 1, x[k] + c7c) : x[k] + c7c;
    }
    *(LAS u32x4*)(sc + lane * 16) = (u32x4){x[0], x[1], x[2], x[3]};
    *(LAS u32x4*)(sc + 1024 + lane * 16) = (u32x4){x[4], x[5], x[6], x[7]};
    const u32 cnt = __builtin_popcount(bits);
    u32 sum = cnt;
    sum += DPP0(sum, 0x111, 0xf); sum += DPP0(sum, 0x112, 0xf); sum += DPP0(sum, 0x114, 0xf); sum += DPP0(sum, 0x118, 0xf);
    sum += DPP0(sum, 0x142, 0xa); sum += DPP0(sum, 0x143, 0xc);
    asm volatile("" : "+v"(sum));
    const u32 n = rdl(sum, 63);
    p.raw = n > 62u;
    {
        LAS unsigned short* const list = (LAS unsigned short*)(sc + 2048);
        const u32 l5 = (u32)lane << 5;
        LAS unsigned short* q = list + (sum - cnt);
        if (p.raw) bits = 0;
#pragma unroll
        for (int j = 0; j < 3; ++j) {
            u32 lb; asm("v_ffbl_b32 %0, %1" : "=v"(lb) : "v"(bits));
            (bits ? q : list + 64)[j] = (unsigned short)(l5 | lb); bits &= bits - 1u;
        }
        q += 3;
        while (bits) { *q++ = (unsigned short)(l5 | __builtin_ctz(bits)); bits &= bits - 1u; }
    }
    FENCE();
    const u32 e = ((LAS unsigned short*)(sc + 2048))[lane];
    SB();
    u32 cc[2];
#pragma unroll
    for (int k = 7; k >= 0; --k) cc[k >> 2] = (k & 3) < 3 ? bfi(0x03030303u, d[k], cc[k >> 2] << 2) : d[k];
    SB();
    const u32 pos = ((~e & 4u) << 8) | ((e >> 1) & 0x3F0u) | ((~e & 3u) << 2) | ((e >> 3) & 3u);
    const u32 val = *(LAS unsigned char*)(sc + pos);
    SB();
    u32 sg = hi[7];
#pragma unroll
    for (int k = 6; k >= 0; --k) sg = bfi(c7f, sg >> 1, hi[k]);
    p.cs = (u32x3){cc[0], cc[1], sg};
    const u32 hd = lane == 63 ? (base - s) | s << 7 : min(n, 63u);
    SB();
    p.tail = lane >= 62 ? hd : pos | val << 9;
    FENCE();
}
__device__ __forceinline__ void store_blk_cs(const Pk& p, rsrc_t rs, u32 so, int lane) {
    __builtin_amdgcn_raw_buffer_store_b96(p.cs, rs, so + 2048u + lane * 12u, 0, 17);
    __builtin_amdgcn_raw_buffer_store_b16((unsigned short)p.tail, rs, so + 2816u + lane * 2u, 0, 17);
}
__device__ __forceinline__ void store_blk_raw(const Pk& p, const u32 (&hi)[8], rsrc_t rs, u32 ov, int lane) {
    if (p.raw) {
        st16_sys((u32x4){hi[0], hi[1], hi[2], hi[3]}, rs, lane * 16u, ov);
        st16_sys((u32x4){hi[4], hi[5], hi[6], hi[7]}, rs, lane * 16u + 1024u, ov);
    }
}
struct Blk { u32x4 l0, l1; u32x3 cs; u32 tail; };
__device__ __forceinline__ void load_blk(Blk& b, rsrc_t rs, u32 so, int lane) {
    b.tail = __builtin_amdgcn_raw_buffer_load_b16(rs, 2816u + lane * 2u, so, 17);
    b.cs = __builtin_amdgcn_raw_buffer_load_b96(rs, 2048u + lane * 12u, so, 17);
    b.l0 = ld16_sys(rs, lane * 16u, so);
    b.l1 = ld16_sys(rs, lane * 16u + 1024u, so);
}
__device__ __forceinline__ void unpack_exc(u32 tail, int lane, LAS char* img, u32x4& d0, u32x4& d1) {
    const u32 cn = rdl(tail, 62);
    const bool mine = (u32)lane < (cn > 62u ? 0u : cn);
    LAS unsigned char* const q = (LAS unsigned char*)img + (tail & 0x7FFu);
    if (mine) *q = (unsigned char)((tail >> 9) & 0x7Cu);
    FENCE();
    d0 = *(LAS u32x4*)(img + lane * 16); d1 = *(LAS u32x4*)(img + 1024 + lane * 16);
    FENCE();
    if (mine) *q = 0;
    FENCE();
}
__device__ __forceinline__ void unpack_hi(const Blk& b, u32x4 d0, u32x4 d1, rsrc_t rs, u32 ov, int lane, u32 (&hi)[8]) {
    const u32 hb = rdl(b.tail, 63);
    const u32 b4 = (hb & 127u) * 0x01010101u, s4 = ((hb >> 7) & 1u) * 0x01010101u;
#pragma unroll
    for (int k = 0; k < 8; ++k) {
        const u32 lo = k < 4 ? b.l0[k & 3] : b.l1[k & 3];
        const u32 u = bfi(0x03030303u, b.cs[k >> 2] >> (2 * (k & 3)), k < 4 ? d0[k & 3] : d1[k & 3]) + (~(lo >> 7) & s4) + b4;
        hi[k] = bfi(0x7F7F7F7Fu, u, b.cs[2] << k);
    }
    if (__builtin_expect(rdl(b.tail, 62) > 62u, 0)) {
        const u32x4 h0 = ld16_sys(rs, lane * 16u, ov), h1 = ld16_sys(rs, lane * 16u + 1024u, ov);
        WAITV(0);
#pragma unroll
        for (int k = 0; k < 4; ++k) { hi[k] = h0[k]; hi[4 + k] = h1[k]; }
    }
}

__device__ __forceinline__ bool timed_out(gu32* tmo, u32 call, u64 t0) {
    if (uni(__hip_atomic_load(tmo, __ATOMIC_RELAXED, __HIP_MEMORY_SCOPE_AGENT)) == call) return true;
    if (rt() - t0 > DEADLINE) { __hip_atomic_store(tmo, call, __ATOMIC_RELAXED, __HIP_MEMORY_SCOPE_AGENT); return true; }
    return false;
}
__device__ __forceinline__ void store_frags(const u32 (&hi)[8], u32x4 l0, u32x4 l1, rsrc_t rs, u32 voff) {
#pragma unroll
    for (int f = 0; f < 4; ++f) {
        u32x4 v;
#pragma unroll
        for (int p = 0; p < 4; ++p) v[p] = __builtin_amdgcn_perm(hi[4 * (f >> 1) + p], (f >> 1) ? l1[p] : l0[p], (f & 1) ? 0x07030602u : 0x05010400u);
        st16_dev(v, rs, voff + f * 1024u, 0);
    }
}

__device__ __forceinline__ void unit_load(u32 (&in)[16], const char* x, const char* w, u32 op, u32 so, int lane) {
    if (op) {
        const rsrc_t rw = mkrs(w, 512u * 4096u * 2u);
        const u32 vo = (lane >> 4) * 8u * 8192u + (lane & 7) * 16u + (lane & 8);
#pragma unroll
        for (int r = 0; r < 8; ++r) { const u32x2 v = ld8(rw, vo, so + r * 8192u); in[2 * r] = v[0]; in[2 * r + 1] = v[1]; }
    } else {
        const rsrc_t rx = mkrs(x, 4096u * 512u * 2u);
        const u32 vo = (lane & 15) * 1024u + (lane >> 4) * 16u;
#pragma unroll
        for (int m = 0; m < 4; ++m) { const u32x4 v = ld16(rx, vo, so + m * 16384u); in[4 * m] = v[0]; in[4 * m + 1] = v[1]; in[4 * m + 2] = v[2]; in[4 * m + 3] = v[3]; }
    }
}
__device__ __forceinline__ void unit_split(const u32 (&in)[16], u32 op, u32 (&lo)[8], u32 (&hi)[8]) {
    if (op) {
#pragma unroll
        for (int k = 0; k < 8; ++k) {
            const u32 a0 = in[4 * (k & 3) + (k >> 2)], a1 = in[4 * (k & 3) + 2 + (k >> 2)];
            lo[k] = __builtin_amdgcn_perm(a1, a0, 0x06020400u); hi[k] = __builtin_amdgcn_perm(a1, a0, 0x07030501u);
        }
    } else {
#pragma unroll
        for (int k = 0; k < 8; ++k) {
            const u32 a0 = in[8 * (k >> 2) + (k & 3)], a1 = in[8 * (k >> 2) + 4 + (k & 3)];
            lo[k] = __builtin_amdgcn_perm(a1, a0, 0x06040200u); hi[k] = __builtin_amdgcn_perm(a1, a0, 0x07050301u);
        }
    }
}

__device__ __forceinline__ u32 cvt_pk_bf16(float lo, float hi) {
    typedef __bf16 bf2 __attribute__((ext_vector_type(2)));
    typedef float f2 __attribute__((ext_vector_type(2)));
    return __builtin_bit_cast(u32, __builtin_convertvector((f2){lo, hi}, bf2));
}
typedef float f32x2 __attribute__((ext_vector_type(2)));
__device__ __forceinline__ float bf_top(u32 hi, u32 lo, int e) { return __builtin_bit_cast(float, __builtin_amdgcn_perm(hi, lo, 0x04000c0cu + e * 0x01010000u)); }
__device__ __forceinline__ u32 silu2(f32x2 y) {
    constexpr float LOG2E = 1.44269504088896341f;
    const f32x2 t = y * -LOG2E;
    f32x2 e;
    e.x = __builtin_amdgcn_exp2f(__builtin_amdgcn_fmed3f(t.x, -60.0f * LOG2E, 60.0f * LOG2E));
    e.y = __builtin_amdgcn_exp2f(__builtin_amdgcn_fmed3f(t.y, -60.0f * LOG2E, 60.0f * LOG2E));
    e += 1.0f;
    e.x = __builtin_amdgcn_rcpf(e.x); e.y = __builtin_amdgcn_rcpf(e.y);
    y *= e;
    return cvt_pk_bf16(y.x, y.y);
}

#define BID (blockIdx.x ^ 256u)
__global__ void __launch_bounds__(512, 4) fused(Params P) {
    constexpr int NS = 5;
    __shared__ __attribute__((aligned(16))) char lds[NS * 16384];
    const int wid = uni(threadIdx.x >> 6);
#define LANE_VARS() int tz_ = threadIdx.x; asm volatile("" : "+v"(tz_)); const int lane = tz_ & 63; const u32 v16 = lane * 16u; \
    const u32 fvoff = lane == 0 ? 0u : 0x80000000u;   (void)v16; (void)fvoff
    const u32 pk0 = PACK.e[((blockIdx.x >> 2) & 63u) * 8u + wid][0], pk1 = PACK.e[((blockIdx.x >> 2) & 63u) * 8u + wid][1];
    asm volatile("" :: "s"(pk0), "s"(pk1), "s"(P.x), "s"(P.w), "s"(P.my), "s"(P.rank));
    const u32 rank = P.rank;
    const u32 ra = rank >> 2, rb = (rank >> 1) & 1, rc = rank & 1;
    char* const my = P.my;
    gu32* const tmo = (gu32*)(my + OFF_TMO);
    gu32* const cnt = (gu32*)(my + OFF_CNT) + BID * 8 + wid;
#define PUT_CALL() do { if (lane == 0) __hip_atomic_store(cnt, call, __ATOMIC_RELAXED, __HIP_MEMORY_SCOPE_AGENT); } while (0)
    const rsrc_t rmy = mkrs(my, WS_USED);

    if (BID < NCOMM) {
        const u32 cb = BID;
        const bool shot = cb - 64u < 128u && wid >= 4;
        const bool puller = cb >= 128u || shot, second = cb < 192u && !shot;
        const u32 src = (cb >> 3) & 7;
        const u32 wi = shot ? (((cb >> 6) - 1u) * 4u + (wid - 4)) * 8u + (cb & 7)
                     : !puller ? ((cb >> 5) * RWPC + wid) * 8u + (cb & 7)
                     : wid * 8u + ((cb + src) & 7);
        if (shot ? (wi >= NPUSH || (src ^ rank) < 2u) : puller ? (wi >= NPW || (src ^ rank) < 2u) : (wid >= RWPC || wi >= NRS)) return;
        LANE_VARS();
        const u32 call = uni(__hip_atomic_load(cnt, __ATOMIC_RELAXED, __HIP_MEMORY_SCOPE_AGENT)) + 1u;
        PUT_CALL();
        __builtin_amdgcn_s_setprio(3);
        bool ok = true;
        LAS char* const img = (LAS char*)lds + wid * 4096;
#pragma unroll
        for (int i = 0; i < 4; ++i) *(LAS u32x4*)(img + i * 1024 + v16) = (u32x4){0, 0, 0, 0};
        FENCE();
        if (puller) {
            const rsrc_t rs = mkrs(P.ws[src], WS_USED);
            const u32 par = call & 1, sj = 14u + (src ^ rank);
            const gu32* const fla = (const gu32*)(my + OFF_IFA + src * 16384u);
            const gu32* const flb = (const gu32*)(my + OFF_IFL + src * 1024u + (lane & 7) * 128u);
            u64 ma = 0, mb = shot ? ~0ull : 0ull;
            auto ready = [&](u32 u) {
                const u32 sg = u < UB ? (u - UA) >> 6 : u < UC ? 2u + (u - UB) / 40u : (64u - G0) / 8u + (u - UC) / 48u;
                return ((u < UA ? ma >> (u + 64u - UA) : mb >> sg) & 1) != 0;
            };
            auto look = [&](u32 u) {
                if (u < UA) return uni(__hip_atomic_load(fla + u * 32u, __ATOMIC_RELAXED, __HIP_MEMORY_SCOPE_SYSTEM)) >= call;
                mb = __builtin_amdgcn_ballot_w64(__hip_atomic_load(flb, __ATOMIC_RELAXED, __HIP_MEMORY_SCOPE_SYSTEM) >= call);
                return ready(u);
            };
            auto wait = [&](u32 u) {
                const u64 t0 = rt();
                for (u32 it = 1;; ++it) {
                    if (look(u)) return true;
                    if ((it & 63u) == 0 && timed_out(tmo, call, t0)) return false;
                    __builtin_amdgcn_s_sleep(1);
                }
            };
            u32 u0 = shot ? wi : NPUSH + wi;
            const u32 ho = OFF_HND + (src * 32u + wi) * 128u, tbase = NPUSH + NPW - (call - 1u) * (NU - NPUSH);
            if (second) {
                u32 hv;
                const u64 t0 = rt();
                for (u32 it = 1;; ++it) {
                    hv = uni(__hip_atomic_load((const gu32*)(my + ho), __ATOMIC_RELAXED, __HIP_MEMORY_SCOPE_AGENT));
                    if ((hv & 0x7FFFFFFFu) >= call) break;
                    if ((it & 63u) == 0 && timed_out(tmo, call, t0)) { ok = false; break; }
                    if (~mb) mb = __builtin_amdgcn_ballot_w64(__hip_atomic_load(flb, __ATOMIC_RELAXED, __HIP_MEMORY_SCOPE_SYSTEM) >= call);
                    __builtin_amdgcn_s_sleep(2);
                }
                u0 = !ok || (hv >> 31) ? NU : uni(__builtin_amdgcn_raw_ptr_buffer_atomic_add_i32(1, rmy, fvoff, OFF_TKI + src * 128u, 0)) + tbase;
                if (u0 >= NU) {
                    st4_dev(call, rmy, fvoff, OFF_TOK + (src * 64u + wi) * 64u);
                    return;
                }
            }
            u32 e0 = entry(u0, src);
            u32 b0 = (((e0 >> 7) ? rb : ra) * 2u + rc) * 512u + u0;
            const rsrc_t rd = mkrs(shot ? my + OFF_PSH + par * PSH_PAR + (src * NPUSH + u0) * BLK - b0 * BLK : P.ws[src] + OFF_IPK + par * IPK_PAR, 2048u * BLK);
            u32 tinc = shot ? 0u : 1u;
            const u32 dummy = OFF_DUM + (cb * 8u + wid) * 64u, tko = shot ? dummy : OFF_TKI + src * 128u;
            bool last = shot;
            if (ok && !(second && ready(u0))) ok = wait(u0);
            Blk N; u32 tkt = 0, fa = 0, fb = 0;
#define IISSUE(N_, b_) do { FENCE(); tkt = __builtin_amdgcn_raw_ptr_buffer_atomic_add_i32(tinc, rmy, fvoff, tko, 0); \
                load_blk(N_, rd, (b_) * BLK, lane); FENCE(); } while (0)
#define ILOOK(b_) do { fb = __builtin_amdgcn_raw_buffer_load_b32(rmy, OFF_IFL + src * 1024u + (lane & 7) * 128u, 0, 17); \
                fa = __builtin_amdgcn_raw_buffer_load_b32(rmy, lane + (UA - 64u) > max((b_) & 511u, NPUSH + NPW - 1u) ? \
                        OFF_IFA + src * 16384u + (lane + (UA - 64u)) * 128u : fvoff | (OFF_IFA + src * 16384u + (UA - 64u) * 128u), 0, 17); FENCE(); } while (0)
#define INEXT() do { u1 = last ? NU : uni(tkt) + tbase; e1 = entry(u1, src); \
                b1 = (((e1 >> 7) ? rb : ra) * 2u + rc) * 512u + (u1 & 511u); \
                if (!second && !last && u1 >= HAND) { st4_dev(u1 < NU ? call : call | 0x80000000u, rmy, fvoff, ho); tinc = 0; last = true; } } while (0)
#define IUNPACK(B_) do { u32 hi_[8]; u32x4 d0_, d1_; unpack_exc(B_.tail, lane, img, d0_, d1_); \
                unpack_hi(B_, d0_, d1_, rs, OFF_IOV + par * IOV_PAR + b0 * 2048u, lane, hi_); \
                const u32 ih_ = ((e0 >= 128u ? ID_WR - 16u + ((e0 >> 3) & 15u) * 48u : (e0 >> 3) * 64u) + sj + (e0 & 7) * 6u) * 2u + (u0 & 1); \
                store_frags(hi_, B_.l0, B_.l1, rmy, v16 + OFF_IMG + ih_ * 4096u); rp = OFF_RDY + ih_ * 128u; } while (0)
            if (ok) {
                IISSUE(N, b0);
                if (!shot) ILOOK(b0);
                WAITV(0);
                u32 rp = dummy;
                u32 u1, e1, b1;
                INEXT();
                bool go = false;
                for (;;) {
                    if (!go) {
                        ma |= __builtin_amdgcn_ballot_w64(fa >= call); mb |= __builtin_amdgcn_ballot_w64(fb >= call);
                        if (u1 >= NU || (u1 >= UA && !~mb)) break;
                        go = ready(u1) || look(u1);
                    }
                    Blk B = N;
                    asm volatile("" : "+v"(B.l0), "+v"(B.l1), "+v"(B.cs), "+v"(B.tail));
                    if (go) {
                        IISSUE(N, b1);
                        ma |= __builtin_amdgcn_ballot_w64(fa >= call); mb |= __builtin_amdgcn_ballot_w64(fb >= call);
                        ILOOK(b1); WAITV(7);
                    } else {
                        WAITV(0);
                    }
                    st4_dev(call, rmy, fvoff | rp, 0);
                    IUNPACK(B);
                    if (!go) {
                        WAITV(0);
                        st4_dev(call, rmy, fvoff | rp, 0);
                        rp = dummy;
                        if (!wait(u1)) { ok = false; break; }
                        IISSUE(N, b1); ILOOK(b1);
                        WAITV(0);
                    }
                    u0 = u1; e0 = e1; b0 = b1;
                    WAITV(11); INEXT();
                    go = u1 < NU && (u1 < UA || ~mb) && ready(u1);
                    { u32 g_ = go; asm volatile("" : "+s"(u1), "+s"(e1), "+s"(b1), "+s"(g_)); go = g_; }
                    WAITV(5);
                }
                if (ok) {
                    Blk M;
                    bool tok = !second;
#define ISTEP(C_, N_) { if (!tok && u1 + TOKE >= NU) { st4_dev(call, rmy, fvoff, OFF_TOK + (src * 64u + wi) * 64u); tok = true; } \
                        if (u1 >= NU) { WAITV(0); st4_dev(call, rmy, fvoff | rp, 0); IUNPACK(C_); break; } \
                        IISSUE(N_, b1); \
                        WAITV(5);             \
                        st4_dev(call, rmy, fvoff | rp, 0); \
                        IUNPACK(C_); \
                        u0 = u1; e0 = e1; b0 = b1; \
                        WAITV(9); INEXT(); asm volatile("" : "+s"(u1), "+s"(e1), "+s"(b1)); \
                        WAITV(5); }
                    for (;;) { ISTEP(N, M) ISTEP(M, N) }
                    WAITV(0);
                    st4_dev(call, rmy, fvoff | rp, 0);
                }
            }
        } else {
            const u32 st = src & 3;
            const rsrc_t r0 = mkrs(P.ws[st * 2u], WS_USED), r1 = mkrs(P.ws[st * 2u + 1u], WS_USED);
            const rsrc_t ro = mkrs(P.out, 512u * 4096u * 2u);
            const gu32* const fl = (const gu32*)(my + ((lane & 2) ? OFF_DFL + st * 4096u : OFF_OFL + st * 8192u + (lane & 1) * 4096u));
            const u32 ovoff = ((lane >> 4) * 4u + ((lane >> 3) & 1)) * 8192u + (lane & 7) * 16u + (st >> 1) * 2097152u + (st & 1) * 4096u;
            auto wait = [&](u32 v) {
                const u64 t0 = rt();
                for (u32 it = 1;; ++it) {
                    if (__builtin_amdgcn_ballot_w64(__hip_atomic_load(fl + (v >> 3) * 32u, __ATOMIC_RELAXED, __HIP_MEMORY_SCOPE_SYSTEM) < call) == 0) return true;
                    if ((it & 63u) == 0 && timed_out(tmo, call, t0)) return false;
                    __builtin_amdgcn_s_sleep(1);
                }
            };
            u32 v0 = wi;
            {
                const u32 ts = st * 2u + ((wi >> 3) & 1);
                const gu32* tok = (const gu32*)(my + OFF_TOK + (((ts ^ rank) < 2u ? ts ^ 2u : ts) * 64u + wi % NPW) * 64u);
                const u64 t0 = rt();
                for (u32 it = 1;; ++it) {
                    const u32 tv = __hip_atomic_load(tok, __ATOMIC_RELAXED, __HIP_MEMORY_SCOPE_AGENT);
                    if (uni(tv) >= call) break;
                    if ((it & 63u) == 0 && timed_out(tmo, call, t0)) { ok = false; break; }
                    __builtin_amdgcn_s_sleep(4);
                }
            }
            if (wi >= NPW) for (int i = 0; i < RDELAY; ++i) __builtin_amdgcn_s_sleep(8);
            bool rdy = false;
            for (;;) {
                if (ok && !rdy) ok = wait(v0);
                FENCE();
                Blk A, C;
                {
                    const u32 so = OFF_OPK + (rank * 256u + v0) * BLK;
                    A.tail = __builtin_amdgcn_raw_buffer_load_b16(r0, 2816u + lane * 2u, so, 17);
                    C.tail = __builtin_amdgcn_raw_buffer_load_b16(r1, 2816u + lane * 2u, so, 17);
                    FENCE();
                    A.cs = __builtin_amdgcn_raw_buffer_load_b96(r0, 2048u + lane * 12u, so, 17);
                    C.cs = __builtin_amdgcn_raw_buffer_load_b96(r1, 2048u + lane * 12u, so, 17);
                    A.l0 = ld16_sys(r0, v16, so); A.l1 = ld16_sys(r0, v16 + 1024u, so);
                    C.l0 = ld16_sys(r1, v16, so); C.l1 = ld16_sys(r1, v16 + 1024u, so);
                }
                FENCE();
                const u32 tkt = __builtin_amdgcn_raw_ptr_buffer_atomic_add_i32(1, rmy, fvoff, OFF_TKO + st * 128u, 0);
                const u32 tm = __hip_atomic_load(tmo, __ATOMIC_RELAXED, __HIP_MEMORY_SCOPE_AGENT);
                u32x4 dg[8];
#pragma unroll
                for (int k = 0; k < 8; ++k) dg[k] = __builtin_amdgcn_raw_buffer_load_b128(rmy, v16 + (k & 3) * 1024u, OFF_DIA + (st * 256u + v0) * 8192u + (k >> 2) * 4096u, 16);
                FENCE();
                u32 oo = ovoff + (((v0 >> 2) & 3) * 64u + (v0 & 1) * 32u) * 8192u + ((v0 >> 4) * 128u + ((v0 >> 1) & 1) * 64u) * 2u;
                u32 ov = OFF_OOV + (rank * 256u + v0) * 2048u;
                asm volatile("" : "+v"(oo)); asm volatile("" : "+s"(ov));
                SB();
                if (!ok) { A.tail = 0; C.tail = 0; }
                u32 ha[8], hc[8];
                u32x4 da0, da1, dc0, dc1;
                unpack_exc(A.tail, lane, img, da0, da1);
                unpack_exc(C.tail, lane, img + 2048, dc0, dc1);
                unpack_hi(A, da0, da1, r0, ov, lane, ha);
                unpack_hi(C, dc0, dc1, r1, ov, lane, hc);
#pragma unroll
                for (int k = 0; k < 8; ++k) asm volatile("" : "+v"(ha[k]), "+v"(hc[k]));
                const u32 v1 = uni(tkt) - (call - 1u) * 256u + NRS;
                FENCE();
                const u32 fv = __hip_atomic_load(fl + ((v1 >> 3) & 31u) * 32u, __ATOMIC_RELAXED, __HIP_MEMORY_SCOPE_SYSTEM);
                FENCE();
                if (!ok || uni(tm) == call) {
#pragma unroll
                    for (int k = 0; k < 8; ++k) { ha[k] = 0x7F7F7F7Fu; A.l0[k & 3] = 0xC0C0C0C0u; A.l1[k & 3] = 0xC0C0C0C0u; }
                }
#pragma unroll
                for (int k = 0; k < 8; k += 2) {
                    u32 r[4];
#pragma unroll
                    for (int j = 0; j < 2; ++j) {
                        const u32 la_ = k < 4 ? A.l0[(k + j) & 3] : A.l1[(k + j) & 3], lc_ = k < 4 ? C.l0[(k + j) & 3] : C.l1[(k + j) & 3];
                        const f32x4 yd = __builtin_bit_cast(f32x4, dg[k + j]);
#pragma unroll
                        for (int e = 0; e < 2; ++e)
                            r[2 * j + e] = silu2((f32x2){bf_top(ha[k + j], la_, 2 * e), bf_top(ha[k + j], la_, 2 * e + 1)} +
                                                 (f32x2){bf_top(hc[k + j], lc_, 2 * e), bf_top(hc[k + j], lc_, 2 * e + 1)} + (f32x2){yd[2 * e], yd[2 * e + 1]});
                    }
                    u32x4 o;
#pragma unroll
                    for (int d = 0; d < 2; ++d) {
                        o[d] = __builtin_amdgcn_update_dpp(r[d], r[2 + d], 0x128, 0xf, 0xc, false);
                        o[2 + d] = __builtin_amdgcn_update_dpp(r[2 + d], r[d], 0x128, 0xf, 0x3, false);
                    }
                    st16_dev(o, ro, oo + (16u * (k >> 2) + (k & 3)) * 8192u, 0);
                }
                if (v1 >= 256u) break;
                rdy = __builtin_amdgcn_ballot_w64(fv < call) == 0;
                v0 = v1;
            }
        }
        return;
    }

    const u32 gb = BID - NCOMM;
    u32 call;
    {
        LANE_VARS();
        const u32 pc = gb & 3, ent0 = pk0 & 255u, op = ent0 >> 7;
        for (u32 i = pk0 >> 20; i; --i) __builtin_amdgcn_s_sleep(1);
        u32 in[16], lo[8], hi[8];
        unit_load(in, P.x, P.w, op, pk1 + (op ? (pc >> 1) * 4096u + (pc & 1) * 2097152u
                  : (pc >> 1) * 262144u + (pc & 1) * 512u + (ent0 >= 112u ? rank * 524288u : ent0 >= rank * 16u ? 524288u : 0u)), lane);
        SB();
        const u32 u = (pk0 >> 8) & 511u, sg = (pk0 >> 17) & 7u, ent = entry_fix(ent0, rank);
        const bool push = u < NPUSH, early = u < UA;
        FENCE();
        const u32 cntv = __hip_atomic_load(cnt, __ATOMIC_RELAXED, __HIP_MEMORY_SCOPE_AGENT);
        FENCE();
        const u32 g0 = (op ? (pc & 2) : (pc & 2) * 2u) + (pc & 1), g1 = g0 + (op ? 4u : 2u);
        const u32 po = OFF_PSH + (rank * NPUSH + min(u, NPUSH - 1u)) * BLK;
        u64 w0 = (u64)P.ws[g0 & 7], w1 = (u64)P.ws[g1 & 7];
        asm volatile("" : "+s"(w0), "+s"(w1));
        u64 a0 = push ? w0 + po : (u64)my + OFF_IPK + (pc * 512u + u) * BLK, a1 = w1 + po;
        gu32* const df = (gu32*)((lane ? w1 : w0) + OFF_IFA + rank * 16384u + u * 128u);
        const bool dof = early && lane < 2 && ((lane ? g1 : g0) ^ rank) >= 2u;
        const u32 tile = (ent >> 3) & 15u, kd = (pc & 1) * 8u + (ent & 7);
        u32 ih = (op ? ID_WD + ((pc >> 1) * 16u + tile) * 16u + kd : u >= NU ? ID_XD + ((pc & 2) + (tile & 1)) * 16u + kd : tile * 64u + kd) * 2u + (u & 1);
        if (!(op || u >= NU || (pc >> 1) == ra)) ih = 0xFFFFFFFFu;
        asm volatile("" : "+s"(ih) : "v"(df), "s"(__builtin_amdgcn_ballot_w64(dof)));
        u32 n0 = push && (g0 ^ rank) < 2u ? 0u : BLK, n1 = !push || (g1 ^ rank) < 2u ? 0u : BLK, ps = push ? PSH_PAR : IPK_PAR, hold = push && u > PUSH0 ? ((u - PUSH0) * PUSHD) >> 4 : 0u;
        asm volatile("" : "+s"(a0), "+s"(a1), "+s"(n0), "+s"(n1), "+s"(ps), "+s"(hold));
        SB();
        call = uni(cntv) + 1u;
        const u32 par = call & 1;
        const rsrc_t d0 = mkrs((const char*)a0 + (par ? ps : 0u), n0), d1 = mkrs((const char*)a1 + (par ? ps : 0u), n1);
        unit_split(in, op, lo, hi);
        const u32x4 l0 = {lo[0], lo[1], lo[2], lo[3]}, l1 = {lo[4], lo[5], lo[6], lo[7]};
        if (hold) { asm volatile("" :: "v"(l0), "v"(l1)); for (; hold; --hold) __builtin_amdgcn_s_sleep(1); }
        st16_sys(l0, d0, v16, 0); st16_sys(l1, d0, v16 + 1024, 0);
        st16_sys(l0, d1, v16, 0); st16_sys(l1, d1, v16 + 1024, 0);
        FENCE(); SB();
        u32 mu = in[0], mi = in[1];
#pragma unroll
        for (int i = 2; i < 16; i += 2) { mu = pkmax(mu, in[i]); mi = pkmaxi(mi, in[i + 1]); }
        mu = pkmax(mu & 0x7FFF7FFFu, mi & 0x7FFF7FFFu);
        Pk pk;
        pack_hi(lo, hi, max(mu >> 16, mu & 0xFFFFu) >> 7, lane, (LAS char*)lds + wid * 4096, pk);
        store_blk_cs(pk, d0, 0, lane);
        store_blk_cs(pk, d1, 0, lane);
        store_blk_raw(pk, hi, rmy, OFF_IOV + par * IOV_PAR + (pc * 512u + u) * 2048u, lane);
        if (early) {
            WAITV(0);
            if (dof) __hip_atomic_store(df, call, __ATOMIC_RELAXED, __HIP_MEMORY_SCOPE_SYSTEM);
        }
        if (ih != 0xFFFFFFFFu) store_frags(hi, l0, l1, rmy, v16 + OFF_IMG + ih * 4096u);
        WAITV(0);
        if (ih != 0xFFFFFFFFu) st4_dev(call, rmy, fvoff, OFF_RDY + ih * 128u);
        PUT_CALL();
        BAR();
        if (wid == 3) {
            const u32 t = __builtin_amdgcn_raw_ptr_buffer_atomic_add_i32(1, rmy, fvoff, OFF_PCN + sg * 128u, 0);
            if (uni(t) == call * 32u - 1u) {
                u64 q[NR];
#pragma unroll
                for (int i = 0; i < NR; ++i) { q[i] = (u64)P.ws[i]; asm volatile("" : "+s"(q[i])); }
                u64 dv = q[0];
#pragma unroll
                for (int i = 1; i < NR; ++i) dv = lane == i ? q[i] : dv;
                if (lane < 8) __hip_atomic_store((gu32*)(dv + OFF_IFL + rank * 1024u + sg * 128u), call, __ATOMIC_RELAXED, __HIP_MEMORY_SCOPE_SYSTEM);
            }
        }
    }
    const u32 xcd = gb & 7, idx = gb >> 3;
    const u32 rt_ = (xcd >> 1) * 4 + (idx & 3), ct = (((idx >> 2) * 2) ^ ((idx >> 2) & 4u)) + (xcd & 1);

    const u32 owner = rt_ >> 1;
    const bool noown = owner == rank;
    const int pass0 = (gb & 64u) ? 0 : 1;
    const u32 drt = idx & 3, dct = xcd * 4u + ((idx >> 2) & 1) + ((idx >> 4) & 1) * 2u;
    const int N1 = noown ? 48 : (owner ^ rank) == 1u ? 64 : 56;
    u32 pv[16];
    const u32 blk = owner * 256u + ((ct * 2u + (rt_ & 1)) * 4u + (wid & 3)) * 2u + (wid >> 2);
    u64 eblk = (u64)(my + OFF_OPK + blk * BLK), eflag = (u64)(P.ws[owner & 7] + OFF_OFL + rank * 4096u + (ct * 2u + (rt_ & 1)) * 128u);
    asm volatile("" : "+s"(eblk), "+s"(eflag));

    if (wid >= 4) {
        LANE_VARS();
        const u32 l = wid - 4;
        const rsrc_t rs = mkrs(my + OFF_IMG + (l & 1) * 4096u, NID * 8192u - 4096u);
        const rsrc_t rf = mkrs(my + OFF_RDY + (l & 1) * 128u, NID * 256u - 128u);
        __builtin_amdgcn_s_setprio(2);
        for (int pass = pass0; pass < 2; ++pass) {
            const int N = pass ? N1 : 16;
            u32 idv;
            {
                const u32 hole = (owner ^ rank) - 2u;
                const u32 i = noown ? lane + 16u : lane, j = i - 16u, ks = j / 5u, r = j - ks * 5u;
                const u32 kp = min(i < 16u || hole > 5u ? i : 16u + ks * 6u + r + (r >= hole), 63u);
                const u32 cube = (l >> 1) ? (kp < 16u ? ID_WD + (rb * 16u + ct) * 16u : ID_WR - 16u + ct * 48u) + kp : rt_ * 64u + kp;
                const u32 first = (l >> 1) ? ID_WD + dct * 16u : ID_XD + drt * 16u;
                idv = pass ? cube : first + (lane & 15);
            }
            int avail = 0;
            auto window = [&](u32 n) { return (u32)(lane - avail) < n ? idv * 256u : 0x80000000u; };
            auto seen = [&](u32 f) { if (avail < 64) { const u64 miss = __builtin_amdgcn_ballot_w64(f < call) >> avail << avail; avail = miss ? (int)__builtin_ctzll(miss) : 64; } };
            auto need = [&](int s1) {
                if (__builtin_expect(avail <= s1, 0)) {
                    const u64 t0 = rt();
                    for (u32 it = 1;; ++it) {
                        FENCE();
                        seen(__builtin_amdgcn_raw_buffer_load_b32(rf, window(LWINB), 0, 16));
                        if (avail > s1) break;
                        if ((it & 63u) == 0 && timed_out(tmo, call, t0)) { avail = 64; break; }
                        __builtin_amdgcn_s_sleep(12);
                    }
                    WAITL0();
                }
            };
            const u32 sov = idv * 8192u, dl0 = l * 4096u;
            u32 so = rdl(sov, 0), dl = dl0, f = 0, fo = 0x80000000u;
            auto issue = [&](int s1, bool look) {
                need(s1);
                LAS void* const d = (LAS void*)((LAS char*)lds + dl);
                __builtin_amdgcn_raw_ptr_buffer_load_lds(rs, d, 16, v16, so, 0, 17);
                SB();
                u32 son = rdl(sov, (s1 + 1) & 63), dln = dl + 16384u;
                dln = dln == dl0 + NS * 16384u ? dl0 : dln;
                asm volatile("" : "+s"(son), "+s"(dln));
                SB();
                __builtin_amdgcn_raw_ptr_buffer_load_lds(rs, d, 16, v16, so, 1024, 17);
                SB();
                if (look) { const u64 miss = __builtin_amdgcn_ballot_w64(f < call && lane >= avail); avail = miss ? (int)__builtin_ctzll(miss) : 64; asm volatile("" : "+s"(avail)); }
                SB();
                __builtin_amdgcn_raw_ptr_buffer_load_lds(rs, d, 16, v16, so, 2048, 17);
                SB();
                if (look) { fo = window(avail < s1 + LAHEAD ? LWIN : 0); asm volatile("" : "+v"(fo)); }
                SB();
                __builtin_amdgcn_raw_ptr_buffer_load_lds(rs, d, 16, v16, so, 3072, 17);
                FENCE(); SB();
                so = son; dl = dln;
            };
            if (pass && l == 2) {
                const u64 t0 = rt();
                for (u32 it = 1; uni(__hip_atomic_load((const gu32*)(my + OFF_IFL + owner * 1024u), __ATOMIC_RELAXED, __HIP_MEMORY_SCOPE_SYSTEM)) < call; ++it) {
                    if ((it & 63u) == 0 && timed_out(tmo, call, t0)) break;
                    __builtin_amdgcn_s_sleep(12);
                }
            }
            if (!pass && gb >= 192u && ((idx ^ rank) & 6u) != 0) {
                const gu32* const h = (const gu32*)(my + OFF_HND + ((idx & 7u) * 32u + ((gb + idx) & 7u)) * 128u);
                const u64 t0 = rt();
                for (u32 it = 1; (uni(__hip_atomic_load(h, __ATOMIC_RELAXED, __HIP_MEMORY_SCOPE_AGENT)) & 0x7FFFFFFFu) < call; ++it) {
                    if ((it & 15u) == 0 && timed_out(tmo, call, t0)) break;
                    __builtin_amdgcn_s_sleep(24);
                }
                __builtin_amdgcn_s_sleep(20);
            }
            for (int s = 0; s < 5; ++s) issue(s, false);
            asm volatile("s_waitcnt vmcnt(16)" ::: "memory"); BAR();
            asm volatile("s_waitcnt vmcnt(12)" ::: "memory"); BAR();
            int s = 5;
            for (; s < N - 3; s += 4) {
                issue(s, true);
                asm volatile("buffer_load_dword %0, %1, %2, 0 offen sc1" : "=v"(f) : "v"(fo), "s"(rf) : "memory");
                asm volatile("s_waitcnt vmcnt(12)" ::: "memory"); BAR();
                issue(s + 1, false);
                asm volatile("s_waitcnt vmcnt(12)" ::: "memory"); BAR();
                issue(s + 2, false);
                asm volatile("s_waitcnt vmcnt(12)" ::: "memory"); BAR();
                issue(s + 3, false);
                asm volatile("s_waitcnt vmcnt(12)" : "+v"(f) :: "memory"); BAR();
            }
            seen(f);
            for (; s < N; ++s) { issue(s, false); asm volatile("s_waitcnt vmcnt(12)" ::: "memory"); BAR(); }
            asm volatile("s_waitcnt vmcnt(8)" ::: "memory"); BAR();
            asm volatile("s_waitcnt vmcnt(4)" ::: "memory"); BAR();
            asm volatile("s_waitcnt vmcnt(0)" ::: "memory"); BAR();
            BAR(); BAR();
            if (!pass) BAR();
        }
        __builtin_amdgcn_s_setprio(0);
        BAR();
#pragma unroll
        for (int i = 0; i < 4; ++i) {
            const u32x4 v = *(LAS u32x4*)((LAS char*)lds + 32768 + l * 4096 + i * 1024 + v16);
            pv[4 * i] = v[0]; pv[4 * i + 1] = v[1]; pv[4 * i + 2] = v[2]; pv[4 * i + 3] = v[3];
        }
    } else {
        const u32 la = (wid >> 1) * 4096, lb = 8192 + (wid & 1) * 4096;
        u32 pla = threadIdx.x * 16u, plb;
#define V16() (pla & 1023u)
        f32x4 acc[4][4];
        bf16x8 A0, A1, C0[2], C1[2], B0[4], B1[4];
#define RD(d, q, i) d = *(const LAS bf16x8*)((LAS char*)lds + p##q + (i) * 1024)
#define M1(a, B, m, n) acc[m][n] = __builtin_amdgcn_mfma_f32_16x16x32_bf16(a, B[n], acc[m][n], 0, 0, 0)
#define NEXT() do { const u32 d_ = so == (NS - 1) * 16384u ? 0u - so : 16384u; so += d_; pla += d_; plb += d_; asm volatile("" : "+v"(pla), "+v"(plb)); } while (0)
#define STEP(Cc, Bc, Cn, Bn) do { SB(); \
            M1(A0, Bc, 0, 0); SB(); RD(Bn[0], lb, 0); SB(); M1(A0, Bc, 0, 1); SB(); RD(Bn[1], lb, 1); SB(); M1(A0, Bc, 0, 2); SB(); RD(Bn[2], lb, 2); SB(); M1(A0, Bc, 0, 3); SB(); RD(A0, la, 0); SB(); \
            M1(A1, Bc, 1, 0); SB(); RD(Bn[3], lb, 3); SB(); M1(A1, Bc, 1, 1); SB(); RD(Cn[0], la, 2); SB(); M1(A1, Bc, 1, 2); SB(); RD(Cn[1], la, 3); SB(); M1(A1, Bc, 1, 3); SB(); RD(A1, la, 1); SB(); \
            M1(Cc[0], Bc, 2, 0); SB(); M1(Cc[0], Bc, 2, 1); SB(); NEXT(); SB(); M1(Cc[0], Bc, 2, 2); SB(); M1(Cc[0], Bc, 2, 3); SB(); \
            M1(Cc[1], Bc, 3, 0); SB(); M1(Cc[1], Bc, 3, 1); SB(); M1(Cc[1], Bc, 3, 2); SB(); WAITL0(); SB(); M1(Cc[1], Bc, 3, 3); SB(); \
            BAR(); \
        } while (0)
        for (int pass = pass0;; ++pass) {
#pragma unroll
            for (int m = 0; m < 4; ++m)
#pragma unroll
                for (int n = 0; n < 4; ++n) acc[m][n] = (f32x4){0.f, 0.f, 0.f, 0.f};
            u32 so = 0;
            asm volatile("" : "+v"(pla));
            plb = V16() + lb; pla = V16() + la;
            BAR();
            RD(A0, la, 0); RD(A1, la, 1); RD(C0[0], la, 2); RD(C0[1], la, 3);
#pragma unroll
            for (int i = 0; i < 4; ++i) RD(B0[i], lb, i);
            SB(); NEXT();
            WAITL0(); BAR();
            for (int s = pass ? N1 : 16; s > 0; s -= 2) {
                STEP(C0, B0, C1, B1);
                STEP(C1, B1, C0, B0);
            }
            if (pass) break;
            {
                const u32 vo = V16() + OFF_DIA + (((drt & 2) + (dct >> 4)) * 256u + (((dct & 15u) * 2u + (drt & 1)) * 4u + wid) * 2u) * 8192u;
#pragma unroll
                for (int m = 0; m < 4; ++m)
#pragma unroll
                    for (int j = 0; j < 4; ++j)
                        st16_dev(__builtin_bit_cast(u32x4, (f32x4){acc[m][0][j], acc[m][1][j], acc[m][2][j], acc[m][3][j]}), rmy, vo + m * 4096u + j * 1024u, 0);
            }
            WAITV(0);
            BAR();
            if (wid == 0 && V16() == 0 && __hip_atomic_load(tmo, __ATOMIC_RELAXED, __HIP_MEMORY_SCOPE_AGENT) != call)
                __hip_atomic_store((gu32*)(my + OFF_DFL + (((drt & 2) + (dct >> 4)) * 32u + (dct & 15u) * 2u + (drt & 1)) * 128u), call, __ATOMIC_RELAXED, __HIP_MEMORY_SCOPE_AGENT);
        }
#pragma unroll
        for (int i = 0; i < 4; ++i) {
            u32x4 v;
#pragma unroll
            for (int e = 0; e < 4; ++e) { const int k = 2 * i + (e >> 1), m = 2 + (k >> 2), j = k & 3; v[e] = cvt_pk_bf16(acc[m][2 * (e & 1)][j], acc[m][2 * (e & 1) + 1][j]); }
            *(LAS u32x4*)((LAS char*)lds + 32768 + wid * 4096 + i * 1024 + V16()) = v;
        }
        WAITL0(); BAR();
#pragma unroll
        for (int k = 0; k < 8; ++k) { pv[2 * k] = cvt_pk_bf16(acc[k >> 2][0][k & 3], acc[k >> 2][1][k & 3]); pv[2 * k + 1] = cvt_pk_bf16(acc[k >> 2][2][k & 3], acc[k >> 2][3][k & 3]); }
    }
    {
        int tz = threadIdx.x; asm volatile("" : "+v"(tz));
        const int ln = tz & 63;
        const rsrc_t re = mkrs((const void*)eblk, BLK);
        u32 lo[8], hi[8], mu = 0, mi = 0x80008000u;
#pragma unroll
        for (int k = 0; k < 8; ++k) {
            lo[k] = __builtin_amdgcn_perm(pv[2 * k + 1], pv[2 * k], 0x06040200u); hi[k] = __builtin_amdgcn_perm(pv[2 * k + 1], pv[2 * k], 0x07050301u);
            mu = pkmax(mu, pv[2 * k]); mi = pkmaxi(mi, pv[2 * k + 1]);
        }
        mu = pkmax(mu & 0x7FFF7FFFu, mi & 0x7FFF7FFFu);
        st16_sys((u32x4){lo[0], lo[1], lo[2], lo[3]}, re, ln * 16u, 0);
        st16_sys((u32x4){lo[4], lo[5], lo[6], lo[7]}, re, ln * 16u + 1024u, 0);
        Pk pk;
        pack_hi(lo, hi, max(mu >> 16, mu & 0xFFFFu) >> 7, ln, (LAS char*)lds + wid * 4096, pk);
        store_blk_cs(pk, re, 0, ln);
        store_blk_raw(pk, hi, rmy, OFF_OOV + blk * 2048u, ln);
        FENCE();
        const u32 tm = __hip_atomic_load(tmo, __ATOMIC_RELAXED, __HIP_MEMORY_SCOPE_AGENT);
        WAITV(0);
        BAR();
        if (tz == 0 && tm != call)
            __hip_atomic_store((gu32*)eflag, call, __ATOMIC_RELAXED, __HIP_MEMORY_SCOPE_SYSTEM);
    }
}

}

extern "C" void kernel_launch(int rank, int nranks, ncclComm_t comm,
                              hipStream_t stream,
                              void* const* d_in, const int* in_sizes, int n_in,
                              void* d_out, int out_size,
                              const int* in_dtypes, int out_dtype,
                              void* const* d_out_all,
                              void* const* d_ws_all, size_t ws_bytes) {
    if (nranks != NR || (unsigned)rank >= (unsigned)NR || n_in != 2 || ws_bytes < WS_USED ||
        in_sizes[0] != 4096 * 512 || in_sizes[1] != 512 * 4096 || out_size != 512 * 4096) {
        fprintf(stderr, "kernel_launch: unexpected problem shape\n");
        return;
    }
    Params P;
    for (int p = 0; p < NR; ++p) P.ws[p] = (char*)d_ws_all[p];
    P.x = (const char*)d_in[0];
    P.w = (const char*)d_in[1];
    P.out = (char*)d_out;
    P.my = P.ws[rank];
    P.rank = rank;
    P.pad = 0;
    hipLaunchKernelGGL(fused, dim3(NCOMM + NGEMM), dim3(512), 0, stream, P);
    HIP_CHECK(hipGetLastError());
}
```

```cpp
#include <hip/hip_runtime.h>
#include <rccl/rccl.h>
#include <cstdio>

#define HIP_CHECK(expr) do { hipError_t _e = (expr); if (_e != hipSuccess) { \
    fprintf(stderr, "HIP error %s at %s:%d\n", hipGetErrorString(_e), __FILE__, __LINE__); \
    return; } } while (0)

namespace {

typedef unsigned u32;
typedef unsigned long long u64;
typedef unsigned u32x4 __attribute__((ext_vector_type(4)));
typedef unsigned u32x3 __attribute__((ext_vector_type(3)));
typedef unsigned u32x2 __attribute__((ext_vector_type(2)));
typedef short i16x2 __attribute__((ext_vector_type(2)));
typedef unsigned short u16x2 __attribute__((ext_vector_type(2)));
typedef short bf16x8 __attribute__((ext_vector_type(8)));
typedef float f32x4 __attribute__((ext_vector_type(4)));
#define LAS __attribute__((address_space(3)))
#define GAS __attribute__((address_space(1)))
typedef GAS u32 gu32;
typedef GAS u64 gu64;
typedef __amdgpu_buffer_rsrc_t rsrc_t;

constexpr int NR = 8;
constexpr int NCOMM = 256;
constexpr int NGEMM = 256;
constexpr u32 NPW = 21;
constexpr u32 HAND = 170;
constexpr u32 TOKE = 24;
constexpr u32 NRS = 42, RWPC = 3;
constexpr int RDELAY = 5;
constexpr u32 NPUSH = 48;
constexpr u32 PUSH0 = 16, PUSHD = 20;
constexpr u32 PSTAG = 24, PSTEP = 1;
constexpr int RFAST = 10, RSLOW = 14;
constexpr int RBASE = 40, REARLY = 3;
constexpr u32 LWIN = 8, LWINB = 4;
constexpr int LAHEAD = 10;
constexpr u32 NU = 480;
constexpr u64 DEADLINE = 50000000ull;

constexpr u32 BLK = 2944;
constexpr u32 OFF_CNT = 0;
constexpr u32 OFF_TMO = 16384;
constexpr u32 OFF_TKI = 16384 + 1024;
constexpr u32 OFF_TKO = 16384 + 2048;
constexpr u32 OFF_PCN = 16384 + 4096;
constexpr u32 OFF_IFL = 24576;
constexpr u32 OFF_TOK = 65536;
constexpr u32 OFF_IFA = 262144;
constexpr u32 OFF_DUM = 393216;
constexpr u32 OFF_OFL = 196608;
constexpr u32 OFF_DFL = 229376;
constexpr u32 OFF_HND = 98304;
constexpr u32 ID_WR = 1024, ID_XD = 1792, ID_WD = 1856, NID = 2368;
constexpr u32 KIB = 1024;
constexpr u32 OFF_RDY = 1024 * KIB;
constexpr u32 OFF_IMG = 2048 * KIB;
constexpr u32 OFF_IPK = 25088 * KIB;
constexpr u32 IPK_PAR = 6144 * KIB;
constexpr u32 OFF_IOV = 37376 * KIB;
constexpr u32 IOV_PAR = 4096 * KIB;
constexpr u32 OFF_OPK = 45568 * KIB;
constexpr u32 OFF_OOV = 51456 * KIB;
constexpr u32 OFF_DIA = 55552 * KIB;
constexpr u32 OFF_PSH = 63744 * KIB;
constexpr u32 PSH_PAR = 8u * NPUSH * BLK;
constexpr u32 WS_USED = OFF_PSH + 2u * PSH_PAR;
static_assert(WS_USED <= 65u * 1024u * KIB && OFF_OPK + 2048u * BLK <= OFF_OOV, "layout");
static_assert(OFF_RDY + NID * 256u <= OFF_IMG && OFF_IMG + NID * 8192u <= OFF_IPK && 2048u * BLK <= IPK_PAR, "layout");

struct Params {
    const char* x;
    const char* w;
    char* my;
    char* out;
    int rank;
    int pad;
    char* ws[NR];
};

struct Order { u32 e[256]; };
constexpr Order make_order() {
    constexpr int SLOW0 = 12;
    int key[17][8] = {};
    for (int ct = 0; ct < 16; ++ct) {
        int t = 8 * ct - RBASE - (ct < REARLY ? 8 * ct : 8 * (REARLY - 1) * (16 - ct) / (17 - REARLY));
        for (int ks = 7; ks >= 0; --ks) { t -= (ct >= SLOW0 && t <= 0) ? RSLOW : RFAST; key[ct][ks] = t; }
    }
    for (int ks = 0; ks < 8; ++ks) { int m = key[0][ks]; for (int ct = 1; ct < 16; ++ct) m = key[ct][ks] < m ? key[ct][ks] : m; key[16][ks] = m; }
    Order o{};
    int n = 0, next[17] = {};
    while (n < 240) {
        int best = -1;
        for (int i = 16; i >= 0; --i) if (next[i] < 8 && (best < 0 || key[i][next[i]] < key[best][next[best]])) best = i;
        if (best == 16) for (int t = 0; t < 14; ++t) o.e[n++] = (u32)(t << 3 | next[16]);
        else o.e[n++] = (u32)(128 | best << 3 | next[best]);
        ++next[best];
    }
    for (int ks = 0; ks < 8; ++ks) { o.e[n++] = (u32)(14 << 3 | ks); o.e[n++] = (u32)(15 << 3 | ks); }
    return o;
}
__constant__ const Order ORDER = make_order();
constexpr u32 G0 = 24, UA = 144u - G0, UB = UA + 128u, UC = UB + (48u - G0) * 5u;
static_assert(NPUSH >= G0 && NPUSH <= 2 * G0 && UA - 64u <= NPUSH + NPW, "whole flag groups; one look covers the ticketed early units");
struct PackTab { u32 e[512][2]; };
constexpr PackTab make_pack() {
    const Order o = make_order();
    PackTab t{};
    for (u32 g = 0; g < 64; ++g)
        for (u32 w = 0; w < 8; ++w) {
            const u32 bg = g >= 48u ? g - 48u : g >= G0 ? g - G0 + 16u : g + 64u - G0;
            const bool early = w < (g < G0 ? 2u : g < 48u ? 3u : 0u);
            const u32 u = g < G0 ? (early ? w * G0 + g + (w * G0 + g < NPUSH ? 0u : 3u * (48u - G0)) : UC + g * 6u + (w - 2))
                        : g < 48u ? (early ? NPUSH - G0 + g + w * (48u - G0) : UB + (g - G0) * 5u + (w - 3)) : UA + bg * 8u + w;
            const u32 e = o.e[u >> 1], tile = (e >> 3) & 15u, ks = e & 7u, half = u & 1u;
            const u32 delay = early || g >= 48u ? 0u : PSTAG + (bg - 16u) * PSTEP;
            t.e[g * 8 + w][0] = e | u << 8 | (bg >> 3) << 17 | delay << 20;
            t.e[g * 8 + w][1] = e >= 128u ? 32u * ks * 8192u + (128u * tile + 64u * half) * 2u
                                          : ((tile < 14u ? 512u * (tile >> 1) : 0u) + 128u * (tile & 1u) + 64u * half) * 1024u + 64u * ks;
        }
    return t;
}
__constant__ const PackTab PACK = make_pack();
__device__ __forceinline__ u32 entry_fix(u32 e, u32 src) {
    const u32 own = e - 112u + src * 16u, oth = e + (e >= src * 16u ? 16u : 0u), x = e >= 112u ? own : oth;
    return e >= 128u ? e : x;
}
__device__ __forceinline__ u32 entry(u32 u, u32 src) { return entry_fix(ORDER.e[(u >> 1) & 255u], src); }

__device__ __forceinline__ u64 rt() { return __builtin_amdgcn_s_memrealtime(); }
__device__ __forceinline__ u32 uni(u32 v) { return __builtin_amdgcn_readfirstlane(v); }
__device__ __forceinline__ u32 rdl(u32 v, u32 l) { return __builtin_amdgcn_readlane(v, l); }
__device__ __forceinline__ rsrc_t mkrs(const void* p, u32 bytes) { return __builtin_amdgcn_make_buffer_rsrc((void*)p, 0, bytes, 0x00020000); }
__device__ __forceinline__ u32x4 ld16(rsrc_t r, u32 voff, u32 soff) { return __builtin_amdgcn_raw_buffer_load_b128(r, voff, soff, 0); }
__device__ __forceinline__ u32x2 ld8(rsrc_t r, u32 voff, u32 soff) { return __builtin_amdgcn_raw_buffer_load_b64(r, voff, soff, 0); }
__device__ __forceinline__ u32x4 ld16_sys(rsrc_t r, u32 voff, u32 soff) { return __builtin_amdgcn_raw_buffer_load_b128(r, voff, soff, 17); }
__device__ __forceinline__ void st16_sys(u32x4 v, rsrc_t r, u32 voff, u32 soff) { __builtin_amdgcn_raw_buffer_store_b128(v, r, voff + soff, 0, 17); }
__device__ __forceinline__ void st16_dev(u32x4 v, rsrc_t r, u32 voff, u32 soff) { __builtin_amdgcn_raw_buffer_store_b128(v, r, voff + soff, 0, 16); }
__device__ __forceinline__ void st4_dev(u32 v, rsrc_t r, u32 voff, u32 soff) { __builtin_amdgcn_raw_buffer_store_b32(v, r, voff, soff, 16); }
#define FENCE() asm volatile("" ::: "memory")
#define WAITV(n) do { asm volatile("s_waitcnt vmcnt(" #n ")" ::: "memory"); __builtin_amdgcn_s_waitcnt(0xF70 | (n)); } while (0)
#define WAITL0() do { asm volatile("s_waitcnt lgkmcnt(0)" ::: "memory"); __builtin_amdgcn_s_waitcnt(0xC07F); } while (0)
#define SB() __builtin_amdgcn_sched_barrier(0)
#define BAR() do { FENCE(); __builtin_amdgcn_s_barrier(); FENCE(); } while (0)


__device__ __forceinline__ u32 pkmax(u32 a, u32 b) {
    return __builtin_bit_cast(u32, __builtin_elementwise_max(__builtin_bit_cast(u16x2, a), __builtin_bit_cast(u16x2, b)));
}
__device__ __forceinline__ u32 pkmaxi(u32 a, u32 b) {
    return __builtin_bit_cast(u32, __builtin_elementwise_max(__builtin_bit_cast(i16x2, a), __builtin_bit_cast(i16x2, b)));
}
struct Pk { u32x3 cs; u32 tail; bool raw; };
__device__ __forceinline__ u32 bfi(u32 m, u32 a, u32 b) { return __builtin_amdgcn_bitop3_b32(m, a, b, 0xCA); }
#define DPP0(v, ctrl, rm) (u32)__builtin_amdgcn_update_dpp(0, (int)(v), ctrl, rm, 0xf, true)
__device__ __forceinline__ void pack_hi(const u32 (&lo)[8], const u32 (&hi)[8], u32 m, int lane, LAS char* sc, Pk& p) {
    m = max(m, DPP0(m, 0x111, 0xf)); m = max(m, DPP0(m, 0x112, 0xf)); m = max(m, DPP0(m, 0x114, 0xf)); m = max(m, DPP0(m, 0x118, 0xf));
    u32 T0 = min(rdl(m, 15), rdl(m, 31)), T1 = min(rdl(m, 47), rdl(m, 63));
    asm volatile("" : "+s"(T0), "+s"(T1));
    u32 T = max(min(T0, T1), 7u);
    asm volatile("" : "+s"(T));
    const u32 L = T - 7u, s = L & 1u, base = (L + s) >> 1;
    const u32 k4 = (128u - base) * 0x01010101u,
              s4 = s * 0x01010101u;
    u32 c7f = 0x7F7F7F7Fu, c7c = 0x7C7C7C7Cu;
    asm volatile("" : "+s"(c7f), "+s"(c7c));
    u32 d[8], x[8], bits = 0;
#pragma unroll
    for (int k = 7; k >= 0; --k) {
        d[k] = (hi[k] & c7f) + ((lo[k] >> 7) & s4) + k4;
        x[k] = d[k] & c7c;
        bits = k < 7 ? bfi(c7f, bits >> 1, x[k] + c7c) : x[k] + c7c;
    }
    *(LAS u32x4*)(sc + lane * 16) = (u32x4){x[0], x[1], x[2], x[3]};
    *(LAS u32x4*)(sc + 1024 + lane * 16) = (u32x4){x[4], x[5], x[6], x[7]};
    const u32 cnt = __builtin_popcount(bits);
    u32 sum = cnt;
    sum += DPP0(sum, 0x111, 0xf); sum += DPP0(sum, 0x112, 0xf); sum += DPP0(sum, 0x114, 0xf); sum += DPP0(sum, 0x118, 0xf);
    sum += DPP0(sum, 0x142, 0xa); sum += DPP0(sum, 0x143, 0xc);
    asm volatile("" : "+v"(sum));
    const u32 n = rdl(sum, 63);
    p.raw = n > 62u;
    {
        LAS unsigned short* const list = (LAS unsigned short*)(sc + 2048);
        const u32 l5 = (u32)lane << 5;
        LAS unsigned short* q = list + (sum - cnt);
        if (p.raw) bits = 0;
#pragma unroll
        for (int j = 0; j < 3; ++j) {
            u32 lb; asm("v_ffbl_b32 %0, %1" : "=v"(lb) : "v"(bits));
            (bits ? q : list + 64)[j] = (unsigned short)(l5 | lb); bits &= bits - 1u;
        }
        q += 3;
        while (bits) { *q++ = (unsigned short)(l5 | __builtin_ctz(bits)); bits &= bits - 1u; }
    }
    FENCE();
    const u32 e = ((LAS unsigned short*)(sc + 2048))[lane];
    SB();
    u32 cc[2];
#pragma unroll
    for (int k = 7; k >= 0; --k) cc[k >> 2] = (k & 3) < 3 ? bfi(0x03030303u, d[k], cc[k >> 2] << 2) : d[k];
    SB();
    const u32 pos = ((~e & 4u) << 8) | ((e >> 1) & 0x3F0u) | ((~e & 3u) << 2) | ((e >> 3) & 3u);
    const u32 val = *(LAS unsigned char*)(sc + pos);
    SB();
    u32 sg = hi[7];
#pragma unroll
    for (int k = 6; k >= 0; --k) sg = bfi(c7f, sg >> 1, hi[k]);
    p.cs = (u32x3){cc[0], cc[1], sg};
    const u32 hd = lane == 63 ? (base - s) | s << 7 : min(n, 63u);
    SB();
    p.tail = lane >= 62 ? hd : pos | val << 9;
    FENCE();
}
__device__ __forceinline__ void store_blk_cs(const Pk& p, rsrc_t rs, u32 so, int lane) {
    __builtin_amdgcn_raw_buffer_store_b96(p.cs, rs, so + 2048u + lane * 12u, 0, 17);
    __builtin_amdgcn_raw_buffer_store_b16((unsigned short)p.tail, rs, so + 2816u + lane * 2u, 0, 17);
}
__device__ __forceinline__ void store_blk_raw(const Pk& p, const u32 (&hi)[8], rsrc_t rs, u32 ov, int lane) {
    if (p.raw) {
        st16_sys((u32x4){hi[0], hi[1], hi[2], hi[3]}, rs, lane * 16u, ov);
        st16_sys((u32x4){hi[4], hi[5], hi[6], hi[7]}, rs, lane * 16u + 1024u, ov);
    }
}
struct Blk { u32x4 l0, l1; u32x3 cs; u32 tail; };
__device__ __forceinline__ void load_blk(Blk& b, rsrc_t rs, u32 so, int lane) {
    b.tail = __builtin_amdgcn_raw_buffer_load_b16(rs, 2816u + lane * 2u, so, 17);
    b.cs = __builtin_amdgcn_raw_buffer_load_b96(rs, 2048u + lane * 12u, so, 17);
    b.l0 = ld16_sys(rs, lane * 16u, so);
    b.l1 = ld16_sys(rs, lane * 16u + 1024u, so);
}
__device__ __forceinline__ void unpack_exc(u32 tail, int lane, LAS char* img, u32x4& d0, u32x4& d1) {
    const u32 cn = rdl(tail, 62);
    const bool mine = (u32)lane < (cn > 62u ? 0u : cn);
    LAS unsigned char* const q = (LAS unsigned char*)img + (tail & 0x7FFu);
    if (mine) *q = (unsigned char)((tail >> 9) & 0x7Cu);
    FENCE();
    d0 = *(LAS u32x4*)(img + lane * 16); d1 = *(LAS u32x4*)(img + 1024 + lane * 16);
    FENCE();
    if (mine) *q = 0;
    FENCE();
}
template <bool SPLIT>
__device__ __forceinline__ void unpack_hi(const Blk& b, u32x4 d0, u32x4 d1, rsrc_t rs, u32 ov, int lane, u32 (&hi)[8]) {
    const u32 hb = rdl(b.tail, 63);
    const u32 b4 = (hb & 127u) * 0x01010101u, s4 = ((hb >> 7) & 1u) * 0x01010101u;
    u32 u[8];
#pragma unroll
    for (int k = 0; k < 8; ++k) {
        const u32 lo = k < 4 ? b.l0[k & 3] : b.l1[k & 3];
        u[k] = bfi(0x03030303u, b.cs[k >> 2] >> (2 * (k & 3)), k < 4 ? d0[k & 3] : d1[k & 3]) + (SPLIT ? 0u : ~(lo >> 7) & s4) + b4;
    }
    if (SPLIT && s4) {
#pragma unroll
        for (int k = 0; k < 8; ++k) u[k] += bfi((k < 4 ? b.l0[k & 3] : b.l1[k & 3]) >> 7, 0u, s4);
    }
#pragma unroll
    for (int k = 0; k < 8; ++k) hi[k] = bfi(0x7F7F7F7Fu, u[k], b.cs[2] << k);
    if (__builtin_expect(rdl(b.tail, 62) > 62u, 0)) {
        const u32x4 h0 = ld16_sys(rs, lane * 16u, ov), h1 = ld16_sys(rs, lane * 16u + 1024u, ov);
        WAITV(0);
#pragma unroll
        for (int k = 0; k < 4; ++k) { hi[k] = h0[k]; hi[4 + k] = h1[k]; }
    }
}

__device__ __forceinline__ bool timed_out(gu32* tmo, u32 call, u64 t0) {
    if (uni(__hip_atomic_load(tmo, __ATOMIC_RELAXED, __HIP_MEMORY_SCOPE_AGENT)) == call) return true;
    if (rt() - t0 > DEADLINE) { __hip_atomic_store(tmo, call, __ATOMIC_RELAXED, __HIP_MEMORY_SCOPE_AGENT); return true; }
    return false;
}
__device__ __forceinline__ void store_frags(const u32 (&hi)[8], u32x4 l0, u32x4 l1, rsrc_t rs, u32 voff) {
#pragma unroll
    for (int f = 0; f < 4; ++f) {
        u32x4 v;
#pragma unroll
        for (int p = 0; p < 4; ++p) v[p] = __builtin_amdgcn_perm(hi[4 * (f >> 1) + p], (f >> 1) ? l1[p] : l0[p], (f & 1) ? 0x07030602u : 0x05010400u);
        st16_dev(v, rs, voff + f * 1024u, 0);
    }
}

__device__ __forceinline__ void unit_load(u32 (&in)[16], const char* x, const char* w, u32 op, u32 so, int lane) {
    if (op) {
        const rsrc_t rw = mkrs(w, 512u * 4096u * 2u);
        const u32 vo = (lane >> 4) * 8u * 8192u + (lane & 7) * 16u + (lane & 8);
#pragma unroll
        for (int r = 0; r < 8; ++r) { const u32x2 v = ld8(rw, vo, so + r * 8192u); in[2 * r] = v[0]; in[2 * r + 1] = v[1]; }
    } else {
        const rsrc_t rx = mkrs(x, 4096u * 512u * 2u);
        const u32 vo = (lane & 15) * 1024u + (lane >> 4) * 16u;
#pragma unroll
        for (int m = 0; m < 4; ++m) { const u32x4 v = ld16(rx, vo, so + m * 16384u); in[4 * m] = v[0]; in[4 * m + 1] = v[1]; in[4 * m + 2] = v[2]; in[4 * m + 3] = v[3]; }
    }
}
__device__ __forceinline__ void unit_split(const u32 (&in)[16], u32 op, u32 (&lo)[8], u32 (&hi)[8]) {
    if (op) {
#pragma unroll
        for (int k = 0; k < 8; ++k) {
            const u32 a0 = in[4 * (k & 3) + (k >> 2)], a1 = in[4 * (k & 3) + 2 + (k >> 2)];
            lo[k] = __builtin_amdgcn_perm(a1, a0, 0x06020400u); hi[k] = __builtin_amdgcn_perm(a1, a0, 0x07030501u);
        }
    } else {
#pragma unroll
        for (int k = 0; k < 8; ++k) {
            const u32 a0 = in[8 * (k >> 2) + (k & 3)], a1 = in[8 * (k >> 2) + 4 + (k & 3)];
            lo[k] = __builtin_amdgcn_perm(a1, a0, 0x06040200u); hi[k] = __builtin_amdgcn_perm(a1, a0, 0x07050301u);
        }
    }
}

__device__ __forceinline__ u32 cvt_pk_bf16(float lo, float hi) {
    typedef __bf16 bf2 __attribute__((ext_vector_type(2)));
    typedef float f2 __attribute__((ext_vector_type(2)));
    return __builtin_bit_cast(u32, __builtin_convertvector((f2){lo, hi}, bf2));
}
typedef float f32x2 __attribute__((ext_vector_type(2)));
__device__ __forceinline__ float bf_top(u32 hi, u32 lo, int e) { return __builtin_bit_cast(float, __builtin_amdgcn_perm(hi, lo, 0x04000c0cu + e * 0x01010000u)); }
__device__ __forceinline__ u32 silu2(f32x2 y) {
    constexpr float LOG2E = 1.44269504088896341f;
    const f32x2 t = y * -LOG2E;
    f32x2 e;
    e.x = __builtin_amdgcn_exp2f(__builtin_amdgcn_fmed3f(t.x, -60.0f * LOG2E, 60.0f * LOG2E));
    e.y = __builtin_amdgcn_exp2f(__builtin_amdgcn_fmed3f(t.y, -60.0f * LOG2E, 60.0f * LOG2E));
    e += 1.0f;
    e.x = __builtin_amdgcn_rcpf(e.x); e.y = __builtin_amdgcn_rcpf(e.y);
    y *= e;
    return cvt_pk_bf16(y.x, y.y);
}

#define BID (blockIdx.x ^ 256u)
__global__ void __launch_bounds__(512, 4) fused(Params P) {
    constexpr int NS = 5;
    __shared__ __attribute__((aligned(16))) char lds[NS * 16384];
    const int wid = uni(threadIdx.x >> 6);
#define LANE_VARS() int tz_ = threadIdx.x; asm volatile("" : "+v"(tz_)); const int lane = tz_ & 63; const u32 v16 = lane * 16u; \
    const u32 fvoff = lane == 0 ? 0u : 0x80000000u;   (void)v16; (void)fvoff
    const u32 pk0 = PACK.e[((blockIdx.x >> 2) & 63u) * 8u + wid][0], pk1 = PACK.e[((blockIdx.x >> 2) & 63u) * 8u + wid][1];
    const u32 sw = (blockIdx.x & 2) << 11 | (blockIdx.x & 1) << 21, sx = (blockIdx.x & 2) << 17 | (blockIdx.x & 1) << 9;
    const u32 vx = (threadIdx.x & 15) * 1024u + (threadIdx.x & 48);
    asm volatile("" :: "s"(pk0), "s"(pk1), "s"(P.x), "s"(P.w), "s"(P.my), "s"(P.rank), "s"(sw), "s"(sx), "v"(vx));
    u32 in[16];
    if (blockIdx.x < NGEMM) {
        for (u32 i = pk0 >> 20; i; --i) __builtin_amdgcn_s_sleep(1);
        const u32 e = pk0 & 255u, r16 = P.rank * 16u;
        unit_load(in, P.x, P.w, e >> 7, pk1 + (e >= 128u ? sw : sx + ((e >= 112u ? r16 : e >= r16 ? 16u : 0u) << 15)), threadIdx.x & 63);
    }
    u32 rank = P.rank;
    u64 my_ = (u64)P.my;
    asm volatile("" : "+s"(rank), "+s"(my_) :: "memory");
    const u32 ra = rank >> 2, rb = (rank >> 1) & 1, rc = rank & 1;
    char* const my = (char*)my_;
    gu32* const tmo = (gu32*)(my + OFF_TMO);
    gu32* const cnt = (gu32*)(my + OFF_CNT) + BID * 8 + wid;
#define PUT_CALL() do { if (lane == 0) __hip_atomic_store(cnt, call, __ATOMIC_RELAXED, __HIP_MEMORY_SCOPE_AGENT); } while (0)
    const rsrc_t rmy = mkrs(my, WS_USED);

    if (BID < NCOMM) {
        const u32 cb = BID;
        const bool shot = cb - 64u < 128u && wid >= 4;
        const bool puller = cb >= 128u || shot, second = cb < 192u && !shot;
        const u32 src = (cb >> 3) & 7;
        const u32 wi = shot ? (((cb >> 6) - 1u) * 4u + (wid - 4)) * 8u + (cb & 7)
                     : !puller ? ((cb >> 5) * RWPC + wid) * 8u + (cb & 7)
                     : wid * 8u + ((cb + src) & 7);
        if (shot ? (wi >= NPUSH || (src ^ rank) < 2u) : puller ? (wi >= NPW || (src ^ rank) < 2u) : (wid >= RWPC || wi >= NRS)) return;
        LANE_VARS();
        const u32 call = uni(__hip_atomic_load(cnt, __ATOMIC_RELAXED, __HIP_MEMORY_SCOPE_AGENT)) + 1u;
        PUT_CALL();
        __builtin_amdgcn_s_setprio(3);
        bool ok = true;
        LAS char* const img = (LAS char*)lds + wid * 4096;
#pragma unroll
        for (int i = 0; i < 4; ++i) *(LAS u32x4*)(img + i * 1024 + v16) = (u32x4){0, 0, 0, 0};
        FENCE();
        if (puller) {
            const rsrc_t rs = mkrs(P.ws[src], WS_USED);
            const u32 par = call & 1, sj = 14u + (src ^ rank);
            const gu32* const fla = (const gu32*)(my + OFF_IFA + src * 16384u);
            const gu32* const flb = (const gu32*)(my + OFF_IFL + src * 1024u + (lane & 7) * 128u);
            u64 ma = 0, mb = shot ? ~0ull : 0ull;
            auto ready = [&](u32 u) {
                const u32 sg = u < UB ? (u - UA) >> 6 : u < UC ? 2u + (u - UB) / 40u : (64u - G0) / 8u + (u - UC) / 48u;
                return ((u < UA ? ma >> (u + 64u - UA) : mb >> sg) & 1) != 0;
            };
            auto look = [&](u32 u) {
                if (u < UA) return uni(__hip_atomic_load(fla + u * 32u, __ATOMIC_RELAXED, __HIP_MEMORY_SCOPE_SYSTEM)) >= call;
                mb = __builtin_amdgcn_ballot_w64(__hip_atomic_load(flb, __ATOMIC_RELAXED, __HIP_MEMORY_SCOPE_SYSTEM) >= call);
                return ready(u);
            };
            auto wait = [&](u32 u) {
                const u64 t0 = rt();
                for (u32 it = 1;; ++it) {
                    if (look(u)) return true;
                    if ((it & 63u) == 0 && timed_out(tmo, call, t0)) return false;
                    __builtin_amdgcn_s_sleep(1);
                }
            };
            u32 u0 = shot ? wi : NPUSH + wi;
            const u32 ho = OFF_HND + (src * 32u + wi) * 128u, tbase = NPUSH + NPW - (call - 1u) * (NU - NPUSH);
            if (second) {
                u32 hv;
                const u64 t0 = rt();
                for (u32 it = 1;; ++it) {
                    hv = uni(__hip_atomic_load((const gu32*)(my + ho), __ATOMIC_RELAXED, __HIP_MEMORY_SCOPE_AGENT));
                    if ((hv & 0x7FFFFFFFu) >= call) break;
                    if ((it & 63u) == 0 && timed_out(tmo, call, t0)) { ok = false; break; }
                    if (~mb) mb = __builtin_amdgcn_ballot_w64(__hip_atomic_load(flb, __ATOMIC_RELAXED, __HIP_MEMORY_SCOPE_SYSTEM) >= call);
                    __builtin_amdgcn_s_sleep(2);
                }
                u0 = !ok || (hv >> 31) ? NU : uni(__builtin_amdgcn_raw_ptr_buffer_atomic_add_i32(1, rmy, fvoff, OFF_TKI + src * 128u, 0)) + tbase;
                if (u0 >= NU) {
                    st4_dev(call, rmy, fvoff, OFF_TOK + (src * 64u + wi) * 64u);
                    return;
                }
            }
            u32 e0 = entry(u0, src);
            u32 b0 = (((e0 >> 7) ? rb : ra) * 2u + rc) * 512u + u0;
            const rsrc_t rd = mkrs(shot ? my + OFF_PSH + par * PSH_PAR + (src * NPUSH + u0) * BLK - b0 * BLK : P.ws[src] + OFF_IPK + par * IPK_PAR, 2048u * BLK);
            u32 tinc = shot ? 0u : 1u;
            const u32 dummy = OFF_DUM + (cb * 8u + wid) * 64u, tko = shot ? dummy : OFF_TKI + src * 128u;
            bool last = shot;
            if (ok && !(second && ready(u0))) ok = wait(u0);
            Blk N; u32 tkt = 0, fa = 0, fb = 0;
#define IISSUE(N_, b_) do { FENCE(); tkt = __builtin_amdgcn_raw_ptr_buffer_atomic_add_i32(tinc, rmy, fvoff, tko, 0); \
                load_blk(N_, rd, (b_) * BLK, lane); FENCE(); } while (0)
#define ILOOK(b_) do { fb = __builtin_amdgcn_raw_buffer_load_b32(rmy, OFF_IFL + src * 1024u + (lane & 7) * 128u, 0, 17); \
                fa = __builtin_amdgcn_raw_buffer_load_b32(rmy, lane + (UA - 64u) > max((b_) & 511u, NPUSH + NPW - 1u) ? \
                        OFF_IFA + src * 16384u + (lane + (UA - 64u)) * 128u : fvoff | (OFF_IFA + src * 16384u + (UA - 64u) * 128u), 0, 17); FENCE(); } while (0)
#define INEXT() do { u1 = last ? NU : uni(tkt) + tbase; e1 = entry(u1, src); \
                b1 = (((e1 >> 7) ? rb : ra) * 2u + rc) * 512u + (u1 & 511u); \
                if (!second && !last && u1 >= HAND) { st4_dev(u1 < NU ? call : call | 0x80000000u, rmy, fvoff, ho); tinc = 0; last = true; } } while (0)
#define IUNPACK(B_) do { u32 hi_[8]; u32x4 d0_, d1_; unpack_exc(B_.tail, lane, img, d0_, d1_); \
                unpack_hi<false>(B_, d0_, d1_, rs, OFF_IOV + par * IOV_PAR + b0 * 2048u, lane, hi_); \
                const u32 ih_ = ((e0 >= 128u ? ID_WR - 16u + ((e0 >> 3) & 15u) * 48u : (e0 >> 3) * 64u) + sj + (e0 & 7) * 6u) * 2u + (u0 & 1); \
                store_frags(hi_, B_.l0, B_.l1, rmy, v16 + OFF_IMG + ih_ * 4096u); rp = OFF_RDY + ih_ * 128u; } while (0)
            if (ok) {
                IISSUE(N, b0);
                if (!shot) ILOOK(b0);
                WAITV(0);
                u32 rp = dummy;
                u32 u1, e1, b1;
                INEXT();
                bool go = false;
                for (;;) {
                    if (!go) {
                        ma |= __builtin_amdgcn_ballot_w64(fa >= call); mb |= __builtin_amdgcn_ballot_w64(fb >= call);
                        if (u1 >= NU || (u1 >= UA && !~mb)) break;
                        go = ready(u1) || look(u1);
                    }
                    Blk B = N;
                    asm volatile("" : "+v"(B.l0), "+v"(B.l1), "+v"(B.cs), "+v"(B.tail));
                    if (go) {
                        IISSUE(N, b1);
                        ma |= __builtin_amdgcn_ballot_w64(fa >= call); mb |= __builtin_amdgcn_ballot_w64(fb >= call);
                        ILOOK(b1); WAITV(7);
                    } else {
                        WAITV(0);
                    }
                    st4_dev(call, rmy, fvoff | rp, 0);
                    IUNPACK(B);
                    if (!go) {
                        WAITV(0);
                        st4_dev(call, rmy, fvoff | rp, 0);
                        rp = dummy;
                        if (!wait(u1)) { ok = false; break; }
                        IISSUE(N, b1); ILOOK(b1);
                        WAITV(0);
                    }
                    u0 = u1; e0 = e1; b0 = b1;
                    WAITV(11); INEXT();
                    go = u1 < NU && (u1 < UA || ~mb) && ready(u1);
                    { u32 g_ = go; asm volatile("" : "+s"(u1), "+s"(e1), "+s"(b1), "+s"(g_)); go = g_; }
                    WAITV(5);
                }
                if (ok) {
                    Blk M;
                    bool tok = !second;
#define ISTEP(C_, N_) { if (!tok && u1 + TOKE >= NU) { st4_dev(call, rmy, fvoff, OFF_TOK + (src * 64u + wi) * 64u); tok = true; } \
                        if (u1 >= NU) { WAITV(0); st4_dev(call, rmy, fvoff | rp, 0); IUNPACK(C_); break; } \
                        IISSUE(N_, b1); \
                        WAITV(5);             \
                        st4_dev(call, rmy, fvoff | rp, 0); \
                        IUNPACK(C_); \
                        u0 = u1; e0 = e1; b0 = b1; \
                        WAITV(9); INEXT(); asm volatile("" : "+s"(u1), "+s"(e1), "+s"(b1)); \
                        WAITV(5); }
                    for (;;) { ISTEP(N, M) ISTEP(M, N) }
                    WAITV(0);
                    st4_dev(call, rmy, fvoff | rp, 0);
                }
            }
        } else {
            const u32 st = src & 3;
            const rsrc_t r0 = mkrs(P.ws[st * 2u], WS_USED), r1 = mkrs(P.ws[st * 2u + 1u], WS_USED);
            const rsrc_t ro = mkrs(P.out, 512u * 4096u * 2u);
            const gu32* const fl = (const gu32*)(my + ((lane & 2) ? OFF_DFL + st * 4096u : OFF_OFL + st * 8192u + (lane & 1) * 4096u));
            const u32 ovoff = ((lane >> 4) * 4u + ((lane >> 3) & 1)) * 8192u + (lane & 7) * 16u + (st >> 1) * 2097152u + (st & 1) * 4096u;
            auto wait = [&](u32 v) {
                const u64 t0 = rt();
                for (u32 it = 1;; ++it) {
                    if (__builtin_amdgcn_ballot_w64(__hip_atomic_load(fl + (v >> 3) * 32u, __ATOMIC_RELAXED, __HIP_MEMORY_SCOPE_SYSTEM) < call) == 0) return true;
                    if ((it & 63u) == 0 && timed_out(tmo, call, t0)) return false;
                    __builtin_amdgcn_s_sleep(1);
                }
            };
            u32 v0 = wi;
            {
                const u32 ts = st * 2u + ((wi >> 3) & 1);
                const gu32* tok = (const gu32*)(my + OFF_TOK + (((ts ^ rank) < 2u ? ts ^ 2u : ts) * 64u + wi % NPW) * 64u);
                const u64 t0 = rt();
                for (u32 it = 1;; ++it) {
                    const u32 tv = __hip_atomic_load(tok, __ATOMIC_RELAXED, __HIP_MEMORY_SCOPE_AGENT);
                    if (uni(tv) >= call) break;
                    if ((it & 63u) == 0 && timed_out(tmo, call, t0)) { ok = false; break; }
                    __builtin_amdgcn_s_sleep(4);
                }
            }
            if (wi >= NPW) for (int i = 0; i < RDELAY; ++i) __builtin_amdgcn_s_sleep(8);
            bool rdy = false;
            for (;;) {
                if (ok && !rdy) ok = wait(v0);
                FENCE();
                Blk A, C;
                {
                    const u32 so = OFF_OPK + (rank * 256u + v0) * BLK;
                    A.tail = __builtin_amdgcn_raw_buffer_load_b16(r0, 2816u + lane * 2u, so, 17);
                    C.tail = __builtin_amdgcn_raw_buffer_load_b16(r1, 2816u + lane * 2u, so, 17);
                    FENCE();
                    A.cs = __builtin_amdgcn_raw_buffer_load_b96(r0, 2048u + lane * 12u, so, 17);
                    C.cs = __builtin_amdgcn_raw_buffer_load_b96(r1, 2048u + lane * 12u, so, 17);
                    FENCE();
                    A.l0 = ld16_sys(r0, v16, so); C.l0 = ld16_sys(r1, v16, so);
                    FENCE();
                    A.l1 = ld16_sys(r0, v16 + 1024u, so); C.l1 = ld16_sys(r1, v16 + 1024u, so);
                }
                FENCE();
                const u32 tkt = __builtin_amdgcn_raw_ptr_buffer_atomic_add_i32(1, rmy, fvoff, OFF_TKO + st * 128u, 0);
                const u32 tm = __hip_atomic_load(tmo, __ATOMIC_RELAXED, __HIP_MEMORY_SCOPE_AGENT);
                u32x4 dg[8];
#pragma unroll
                for (int k = 0; k < 8; ++k) dg[k] = __builtin_amdgcn_raw_buffer_load_b128(rmy, v16 + (k & 3) * 1024u, OFF_DIA + (st * 256u + v0) * 8192u + (k >> 2) * 4096u, 16);
                FENCE();
                u32 oo = ovoff + (((v0 >> 2) & 3) * 64u + (v0 & 1) * 32u) * 8192u + ((v0 >> 4) * 128u + ((v0 >> 1) & 1) * 64u) * 2u;
                u32 ov = OFF_OOV + (rank * 256u + v0) * 2048u;
                asm volatile("" : "+v"(oo)); asm volatile("" : "+s"(ov));
                SB();
                if (!ok) { A.tail = 0; C.tail = 0; }
                u32 ha[8], hc[8];
                u32x4 da0, da1, dc0, dc1;
                unpack_exc(A.tail, lane, img, da0, da1);
                unpack_exc(C.tail, lane, img + 2048, dc0, dc1);
                unpack_hi<true>(A, da0, da1, r0, ov, lane, ha);
                unpack_hi<true>(C, dc0, dc1, r1, ov, lane, hc);
#pragma unroll
                for (int k = 0; k < 8; ++k) asm volatile("" : "+v"(ha[k]), "+v"(hc[k]));
                const u32 v1 = uni(tkt) - (call - 1u) * 256u + NRS;
                FENCE();
                const u32 fv = __hip_atomic_load(fl + ((v1 >> 3) & 31u) * 32u, __ATOMIC_RELAXED, __HIP_MEMORY_SCOPE_SYSTEM);
                FENCE();
                if (!ok || uni(tm) == call) {
#pragma unroll
                    for (int k = 0; k < 8; ++k) { ha[k] = 0x7F7F7F7Fu; A.l0[k & 3] = 0xC0C0C0C0u; A.l1[k & 3] = 0xC0C0C0C0u; }
                }
#pragma unroll
                for (int k = 0; k < 8; k += 2) {
                    u32 r[4];
#pragma unroll
                    for (int j = 0; j < 2; ++j) {
                        const u32 la_ = k < 4 ? A.l0[(k + j) & 3] : A.l1[(k + j) & 3], lc_ = k < 4 ? C.l0[(k + j) & 3] : C.l1[(k + j) & 3];
                        const f32x4 yd = __builtin_bit_cast(f32x4, dg[k + j]);
#pragma unroll
                        for (int e = 0; e < 2; ++e)
                            r[2 * j + e] = silu2((f32x2){bf_top(ha[k + j], la_, 2 * e), bf_top(ha[k + j], la_, 2 * e + 1)} +
                                                 (f32x2){bf_top(hc[k + j], lc_, 2 * e), bf_top(hc[k + j], lc_, 2 * e + 1)} + (f32x2){yd[2 * e], yd[2 * e + 1]});
                    }
                    u32x4 o;
#pragma unroll
                    for (int d = 0; d < 2; ++d) {
                        o[d] = __builtin_amdgcn_update_dpp(r[d], r[2 + d], 0x128, 0xf, 0xc, false);
                        o[2 + d] = __builtin_amdgcn_update_dpp(r[2 + d], r[d], 0x128, 0xf, 0x3, false);
                    }
                    st16_dev(o, ro, oo + (16u * (k >> 2) + (k & 3)) * 8192u, 0);
                }
                if (v1 >= 256u) break;
                rdy = __builtin_amdgcn_ballot_w64(fv < call) == 0;
                v0 = v1;
            }
        }
        return;
    }

    const u32 gb = BID - NCOMM;
    u32 call;
    {
        LANE_VARS();
        const u32 pc = gb & 3, ent0 = pk0 & 255u, op = ent0 >> 7;
        u32 lo[8], hi[8];
        const u32 u = (pk0 >> 8) & 511u, sg = (pk0 >> 17) & 7u, ent = entry_fix(ent0, rank);
        const bool push = u < NPUSH, early = u < UA;
        FENCE();
        const u32 cntv = __hip_atomic_load(cnt, __ATOMIC_RELAXED, __HIP_MEMORY_SCOPE_AGENT);
        FENCE();
        const u32 g0 = (op ? (pc & 2) : (pc & 2) * 2u) + (pc & 1), g1 = g0 + (op ? 4u : 2u);
        const u32 po = OFF_PSH + (rank * NPUSH + min(u, NPUSH - 1u)) * BLK;
        u64 w0 = (u64)P.ws[g0 & 7], w1 = (u64)P.ws[g1 & 7];
        asm volatile("" : "+s"(w0), "+s"(w1));
        u64 a0 = push ? w0 + po : (u64)my + OFF_IPK + (pc * 512u + u) * BLK, a1 = w1 + po;
        gu32* const df = (gu32*)((lane ? w1 : w0) + OFF_IFA + rank * 16384u + u * 128u);
        const bool dof = early && lane < 2 && ((lane ? g1 : g0) ^ rank) >= 2u;
        const u32 tile = (ent >> 3) & 15u, kd = (pc & 1) * 8u + (ent & 7);
        u32 ih = (op ? ID_WD + ((pc >> 1) * 16u + tile) * 16u + kd : u >= NU ? ID_XD + ((pc & 2) + (tile & 1)) * 16u + kd : tile * 64u + kd) * 2u + (u & 1);
        if (!(op || u >= NU || (pc >> 1) == ra)) ih = 0xFFFFFFFFu;
        asm volatile("" : "+s"(ih) : "v"(df), "s"(__builtin_amdgcn_ballot_w64(dof)));
        u32 n0 = push && (g0 ^ rank) < 2u ? 0u : BLK, n1 = !push || (g1 ^ rank) < 2u ? 0u : BLK, ps = push ? PSH_PAR : IPK_PAR, hold = push && u > PUSH0 ? ((u - PUSH0) * PUSHD) >> 4 : 0u;
        asm volatile("" : "+s"(a0), "+s"(a1), "+s"(n0), "+s"(n1), "+s"(ps), "+s"(hold));
        SB();
        call = uni(cntv) + 1u;
        const u32 par = call & 1;
        const rsrc_t d0 = mkrs((const char*)a0 + (par ? ps : 0u), n0), d1 = mkrs((const char*)a1 + (par ? ps : 0u), n1);
        unit_split(in, op, lo, hi);
        const u32x4 l0 = {lo[0], lo[1], lo[2], lo[3]}, l1 = {lo[4], lo[5], lo[6], lo[7]};
        if (hold) { asm volatile("" :: "v"(l0), "v"(l1)); for (; hold; --hold) __builtin_amdgcn_s_sleep(1); }
        st16_sys(l0, d0, v16, 0); st16_sys(l1, d0, v16 + 1024, 0);
        st16_sys(l0, d1, v16, 0); st16_sys(l1, d1, v16 + 1024, 0);
        FENCE(); SB();
        u32 mu = in[0], mi = in[1];
#pragma unroll
        for (int i = 2; i < 16; i += 2) { mu = pkmax(mu, in[i]); mi = pkmaxi(mi, in[i + 1]); }
        mu = pkmax(mu & 0x7FFF7FFFu, mi & 0x7FFF7FFFu);
        Pk pk;
        pack_hi(lo, hi, max(mu >> 16, mu & 0xFFFFu) >> 7, lane, (LAS char*)lds + wid * 4096, pk);
        store_blk_cs(pk, d0, 0, lane);
        store_blk_cs(pk, d1, 0, lane);
        store_blk_raw(pk, hi, rmy, OFF_IOV + par * IOV_PAR + (pc * 512u + u) * 2048u, lane);
        if (early) {
            WAITV(0);
            if (dof) __hip_atomic_store(df, call, __ATOMIC_RELAXED, __HIP_MEMORY_SCOPE_SYSTEM);
        }
        if (ih != 0xFFFFFFFFu) store_frags(hi, l0, l1, rmy, v16 + OFF_IMG + ih * 4096u);
        WAITV(0);
        if (ih != 0xFFFFFFFFu) st4_dev(call, rmy, fvoff, OFF_RDY + ih * 128u);
        PUT_CALL();
        BAR();
        if (wid == 3) {
            const u32 t = __builtin_amdgcn_raw_ptr_buffer_atomic_add_i32(1, rmy, fvoff, OFF_PCN + sg * 128u, 0);
            if (uni(t) == call * 32u - 1u) {
                u64 q[NR];
#pragma unroll
                for (int i = 0; i < NR; ++i) { q[i] = (u64)P.ws[i]; asm volatile("" : "+s"(q[i])); }
                u64 dv = q[0];
#pragma unroll
                for (int i = 1; i < NR; ++i) dv = lane == i ? q[i] : dv;
                if (lane < 8) __hip_atomic_store((gu32*)(dv + OFF_IFL + rank * 1024u + sg * 128u), call, __ATOMIC_RELAXED, __HIP_MEMORY_SCOPE_SYSTEM);
            }
        }
    }
    const u32 xcd = gb & 7, idx = gb >> 3;
    const u32 rt_ = (xcd >> 1) * 4 + (idx & 3), ct = (((idx >> 2) * 2) ^ ((idx >> 2) & 4u)) + (xcd & 1);

    const u32 owner = rt_ >> 1;
    const bool noown = owner == rank;
    const int pass0 = (gb & 64u) ? 0 : 1;
    const u32 drt = idx & 3, dct = xcd * 4u + ((idx >> 2) & 1) + ((idx >> 4) & 1) * 2u;
    const int N1 = noown ? 48 : (owner ^ rank) == 1u ? 64 : 56;
    u32 pv[16];
    const u32 blk = owner * 256u + ((ct * 2u + (rt_ & 1)) * 4u + (wid & 3)) * 2u + (wid >> 2);
    u64 eblk = (u64)(my + OFF_OPK + blk * BLK), eflag = (u64)(P.ws[owner & 7] + OFF_OFL + rank * 4096u + (ct * 2u + (rt_ & 1)) * 128u);
    asm volatile("" : "+s"(eblk), "+s"(eflag));

    if (wid >= 4) {
        LANE_VARS();
        const u32 l = wid - 4;
        const rsrc_t rs = mkrs(my + OFF_IMG + (l & 1) * 4096u, NID * 8192u - 4096u);
        const rsrc_t rf = mkrs(my + OFF_RDY + (l & 1) * 128u, NID * 256u - 128u);
        __builtin_amdgcn_s_setprio(2);
        for (int pass = pass0; pass < 2; ++pass) {
            const int N = pass ? N1 : 16;
            u32 idv;
            {
                const u32 hole = (owner ^ rank) - 2u;
                const u32 i = noown ? lane + 16u : lane, j = i - 16u, ks = j / 5u, r = j - ks * 5u;
                const u32 kp = min(i < 16u || hole > 5u ? i : 16u + ks * 6u + r + (r >= hole), 63u);
                const u32 cube = (l >> 1) ? (kp < 16u ? ID_WD + (rb * 16u + ct) * 16u : ID_WR - 16u + ct * 48u) + kp : rt_ * 64u + kp;
                const u32 first = (l >> 1) ? ID_WD + dct * 16u : ID_XD + drt * 16u;
                idv = pass ? cube : first + (lane & 15);
            }
            int avail = 0;
            auto window = [&](u32 n) { return (u32)(lane - avail) < n ? idv * 256u : 0x80000000u; };
            auto seen = [&](u32 f) { if (avail < 64) { const u64 miss = __builtin_amdgcn_ballot_w64(f < call) >> avail << avail; avail = miss ? (int)__builtin_ctzll(miss) : 64; } };
            auto need = [&](int s1) {
                if (__builtin_expect(avail <= s1, 0)) {
                    const u64 t0 = rt();
                    for (u32 it = 1;; ++it) {
                        FENCE();
                        seen(__builtin_amdgcn_raw_buffer_load_b32(rf, window(LWINB), 0, 16));
                        if (avail > s1) break;
                        if ((it & 63u) == 0 && timed_out(tmo, call, t0)) { avail = 64; break; }
                        __builtin_amdgcn_s_sleep(12);
                    }
                    WAITL0();
                }
            };
            const u32 sov = idv * 8192u, dl0 = l * 4096u;
            u32 so = rdl(sov, 0), dl = dl0, f = 0, fo = 0x80000000u;
            auto issue = [&](int s1, bool look) {
                need(s1);
                LAS void* const d = (LAS void*)((LAS char*)lds + dl);
                __builtin_amdgcn_raw_ptr_buffer_load_lds(rs, d, 16, v16, so, 0, 17);
                SB();
                u32 son = rdl(sov, (s1 + 1) & 63), dln = dl + 16384u;
                dln = dln == dl0 + NS * 16384u ? dl0 : dln;
                asm volatile("" : "+s"(son), "+s"(dln));
                SB();
                __builtin_amdgcn_raw_ptr_buffer_load_lds(rs, d, 16, v16, so, 1024, 17);
                SB();
                if (look) { const u64 miss = __builtin_amdgcn_ballot_w64(f < call && lane >= avail); avail = miss ? (int)__builtin_ctzll(miss) : 64; asm volatile("" : "+s"(avail)); }
                SB();
                __builtin_amdgcn_raw_ptr_buffer_load_lds(rs, d, 16, v16, so, 2048, 17);
                SB();
                if (look) { fo = window(avail < s1 + LAHEAD ? LWIN : 0); f = 0; asm volatile("" : "+v"(fo), "+v"(f)); }
                SB();
                __builtin_amdgcn_raw_ptr_buffer_load_lds(rs, d, 16, v16, so, 3072, 17);
                FENCE(); SB();
                so = son; dl = dln;
            };
            if (pass && l == 2) {
                const u64 t0 = rt();
                for (u32 it = 1; uni(__hip_atomic_load((const gu32*)(my + OFF_IFL + owner * 1024u), __ATOMIC_RELAXED, __HIP_MEMORY_SCOPE_SYSTEM)) < call; ++it) {
                    if ((it & 63u) == 0 && timed_out(tmo, call, t0)) break;
                    __builtin_amdgcn_s_sleep(12);
                }
            }
            if (!pass && gb >= 192u && ((idx ^ rank) & 6u) != 0) {
                const gu32* const h = (const gu32*)(my + OFF_HND + ((idx & 7u) * 32u + ((gb + idx) & 7u)) * 128u);
                const u64 t0 = rt();
                for (u32 it = 1; (uni(__hip_atomic_load(h, __ATOMIC_RELAXED, __HIP_MEMORY_SCOPE_AGENT)) & 0x7FFFFFFFu) < call; ++it) {
                    if ((it & 15u) == 0 && timed_out(tmo, call, t0)) break;
                    __builtin_amdgcn_s_sleep(24);
                }
                __builtin_amdgcn_s_sleep(20);
            }
            for (int s = 0; s < 5; ++s) issue(s, false);
            asm volatile("s_waitcnt vmcnt(16)" ::: "memory"); BAR();
            asm volatile("s_waitcnt vmcnt(12)" ::: "memory"); BAR();
            int s = 5;
            for (; s < N - 3; s += 4) {
                issue(s, true);
                asm volatile("buffer_load_dword %0, %1, %2, 0 offen sc1" : "+v"(f) : "v"(fo), "s"(rf) : "memory");
                asm volatile("s_waitcnt vmcnt(12)" ::: "memory"); BAR();
                issue(s + 1, false);
                asm volatile("s_waitcnt vmcnt(12)" ::: "memory"); BAR();
                issue(s + 2, false);
                asm volatile("s_waitcnt vmcnt(12)" ::: "memory"); BAR();
                issue(s + 3, false);
                asm volatile("s_waitcnt vmcnt(12)" : "+v"(f) :: "memory"); BAR();
            }
            seen(f);
            for (; s < N; ++s) { issue(s, false); asm volatile("s_waitcnt vmcnt(12)" ::: "memory"); BAR(); }
            asm volatile("s_waitcnt vmcnt(8)" ::: "memory"); BAR();
            asm volatile("s_waitcnt vmcnt(4)" ::: "memory"); BAR();
            asm volatile("s_waitcnt vmcnt(0)" ::: "memory"); BAR();
            BAR(); BAR();
            if (!pass) BAR();
        }
        __builtin_amdgcn_s_setprio(0);
        BAR();
#pragma unroll
        for (int i = 0; i < 4; ++i) {
            const u32x4 v = *(LAS u32x4*)((LAS char*)lds + 32768 + l * 4096 + i * 1024 + v16);
            pv[4 * i] = v[0]; pv[4 * i + 1] = v[1]; pv[4 * i + 2] = v[2]; pv[4 * i + 3] = v[3];
        }
    } else {
        const u32 la = (wid >> 1) * 4096, lb = 8192 + (wid & 1) * 4096;
        u32 pla = threadIdx.x * 16u, plb;
#define V16() (pla & 1023u)
        f32x4 acc[4][4];
        bf16x8 A0, A1, C0[2], C1[2], B0[4], B1[4];
#define RD(d, q, i) d = *(const LAS bf16x8*)((LAS char*)lds + p##q + (i) * 1024)
#define M1(a, B, m, n) acc[m][n] = __builtin_amdgcn_mfma_f32_16x16x32_bf16(a, B[n], acc[m][n], 0, 0, 0)
#define NEXT() do { const u32 d_ = so == (NS - 1) * 16384u ? 0u - so : 16384u; so += d_; pla += d_; plb += d_; asm volatile("" : "+v"(pla), "+v"(plb)); } while (0)
#define STEP(Cc, Bc, Cn, Bn) do { SB(); \
            M1(A0, Bc, 0, 0); SB(); RD(Bn[0], lb, 0); SB(); M1(A0, Bc, 0, 1); SB(); RD(Bn[1], lb, 1); SB(); M1(A0, Bc, 0, 2); SB(); RD(Bn[2], lb, 2); SB(); M1(A0, Bc, 0, 3); SB(); RD(A0, la, 0); SB(); \
            M1(A1, Bc, 1, 0); SB(); RD(Bn[3], lb, 3); SB(); M1(A1, Bc, 1, 1); SB(); RD(Cn[0], la, 2); SB(); M1(A1, Bc, 1, 2); SB(); RD(Cn[1], la, 3); SB(); M1(A1, Bc, 1, 3); SB(); RD(A1, la, 1); SB(); \
            M1(Cc[0], Bc, 2, 0); SB(); M1(Cc[0], Bc, 2, 1); SB(); NEXT(); SB(); M1(Cc[0], Bc, 2, 2); SB(); M1(Cc[0], Bc, 2, 3); SB(); \
            M1(Cc[1], Bc, 3, 0); SB(); WAITL0(); SB(); M1(Cc[1], Bc, 3, 1); SB(); BAR(); SB(); M1(Cc[1], Bc, 3, 2); SB(); M1(Cc[1], Bc, 3, 3); SB(); \
        } while (0)
        for (int pass = pass0;; ++pass) {
#pragma unroll
            for (int m = 0; m < 4; ++m)
#pragma unroll
                for (int n = 0; n < 4; ++n) acc[m][n] = (f32x4){0.f, 0.f, 0.f, 0.f};
            u32 so = 0;
            asm volatile("" : "+v"(pla));
            plb = V16() + lb; pla = V16() + la;
            BAR();
            RD(A0, la, 0); RD(A1, la, 1); RD(C0[0], la, 2); RD(C0[1], la, 3);
#pragma unroll
            for (int i = 0; i < 4; ++i) RD(B0[i], lb, i);
            SB(); NEXT();
            WAITL0(); BAR();
            for (int s = pass ? N1 : 16; s > 0; s -= 2) {
                STEP(C0, B0, C1, B1);
                STEP(C1, B1, C0, B0);
            }
            if (pass) break;
            {
                const u32 vo = V16() + OFF_DIA + (((drt & 2) + (dct >> 4)) * 256u + (((dct & 15u) * 2u + (drt & 1)) * 4u + wid) * 2u) * 8192u;
#pragma unroll
                for (int m = 0; m < 4; ++m)
#pragma unroll
                    for (int j = 0; j < 4; ++j)
                        st16_dev(__builtin_bit_cast(u32x4, (f32x4){acc[m][0][j], acc[m][1][j], acc[m][2][j], acc[m][3][j]}), rmy, vo + m * 4096u + j * 1024u, 0);
            }
            WAITV(0);
            BAR();
            if (wid == 0 && V16() == 0 && __hip_atomic_load(tmo, __ATOMIC_RELAXED, __HIP_MEMORY_SCOPE_AGENT) != call)
                __hip_atomic_store((gu32*)(my + OFF_DFL + (((drt & 2) + (dct >> 4)) * 32u + (dct & 15u) * 2u + (drt & 1)) * 128u), call, __ATOMIC_RELAXED, __HIP_MEMORY_SCOPE_AGENT);
        }
#pragma unroll
        for (int i = 0; i < 4; ++i) {
            u32x4 v;
#pragma unroll
            for (int e = 0; e < 4; ++e) { const int k = 2 * i + (e >> 1), m = 2 + (k >> 2), j = k & 3; v[e] = cvt_pk_bf16(acc[m][2 * (e & 1)][j], acc[m][2 * (e & 1) + 1][j]); }
            *(LAS u32x4*)((LAS char*)lds + 32768 + wid * 4096 + i * 1024 + V16()) = v;
        }
        WAITL0(); BAR();
#pragma unroll
        for (int k = 0; k < 8; ++k) { pv[2 * k] = cvt_pk_bf16(acc[k >> 2][0][k & 3], acc[k >> 2][1][k & 3]); pv[2 * k + 1] = cvt_pk_bf16(acc[k >> 2][2][k & 3], acc[k >> 2][3][k & 3]); }
    }
    {
        int tz = threadIdx.x; asm volatile("" : "+v"(tz));
        const int ln = tz & 63;
        const rsrc_t re = mkrs((const void*)eblk, BLK);
        u32 lo[8], hi[8], mu = 0, mi = 0x80008000u;
#pragma unroll
        for (int k = 0; k < 8; ++k) {
            lo[k] = __builtin_amdgcn_perm(pv[2 * k + 1], pv[2 * k], 0x06040200u); hi[k] = __builtin_amdgcn_perm(pv[2 * k + 1], pv[2 * k], 0x07050301u);
            mu = pkmax(mu, pv[2 * k]); mi = pkmaxi(mi, pv[2 * k + 1]);
        }
        mu = pkmax(mu & 0x7FFF7FFFu, mi & 0x7FFF7FFFu);
        st16_sys((u32x4){lo[0], lo[1], lo[2], lo[3]}, re, ln * 16u, 0);
        st16_sys((u32x4){lo[4], lo[5], lo[6], lo[7]}, re, ln * 16u + 1024u, 0);
        Pk pk;
        pack_hi(lo, hi, max(mu >> 16, mu & 0xFFFFu) >> 7, ln, (LAS char*)lds + wid * 4096, pk);
        store_blk_cs(pk, re, 0, ln);
        store_blk_raw(pk, hi, rmy, OFF_OOV + blk * 2048u, ln);
        FENCE();
        const u32 tm = __hip_atomic_load(tmo, __ATOMIC_RELAXED, __HIP_MEMORY_SCOPE_AGENT);
        WAITV(0);
        BAR();
        if (tz == 0 && tm != call)
            __hip_atomic_store((gu32*)eflag, call, __ATOMIC_RELAXED, __HIP_MEMORY_SCOPE_SYSTEM);
    }
}

}

extern "C" void kernel_launch(int rank, int nranks, ncclComm_t comm,
                              hipStream_t stream,
                              void* const* d_in, const int* in_sizes, int n_in,
                              void* d_out, int out_size,
                              const int* in_dtypes, int out_dtype,
                              void* const* d_out_all,
                              void* const* d_ws_all, size_t ws_bytes) {
    if (nranks != NR || (unsigned)rank >= (unsigned)NR || n_in != 2 || ws_bytes < WS_USED ||
        in_sizes[0] != 4096 * 512 || in_sizes[1] != 512 * 4096 || out_size != 512 * 4096) {
        fprintf(stderr, "kernel_launch: unexpected problem shape\n");
        return;
    }
    Params P;
    for (int p = 0; p < NR; ++p) P.ws[p] = (char*)d_ws_all[p];
    P.x = (const char*)d_in[0];
    P.w = (const char*)d_in[1];
    P.out = (char*)d_out;
    P.my = P.ws[rank];
    P.rank = rank;
    P.pad = 0;
    hipLaunchKernelGGL(fused, dim3(NCOMM + NGEMM), dim3(512), 0, stream, P);
    HIP_CHECK(hipGetLastError());
}
```

```cpp
#include <hip/hip_runtime.h>
#include <rccl/rccl.h>
#include <cstdio>

#define HIP_CHECK(expr) do { hipError_t _e = (expr); if (_e != hipSuccess) { \
    fprintf(stderr, "HIP error %s at %s:%d\n", hipGetErrorString(_e), __FILE__, __LINE__); \
    return; } } while (0)

namespace {

typedef unsigned u32;
typedef unsigned long long u64;
typedef unsigned u32x4 __attribute__((ext_vector_type(4)));
typedef unsigned u32x3 __attribute__((ext_vector_type(3)));
typedef unsigned u32x2 __attribute__((ext_vector_type(2)));
typedef short i16x2 __attribute__((ext_vector_type(2)));
typedef unsigned short u16x2 __attribute__((ext_vector_type(2)));
typedef short bf16x8 __attribute__((ext_vector_type(8)));
typedef float f32x4 __attribute__((ext_vector_type(4)));
#define LAS __attribute__((address_space(3)))
#define GAS __attribute__((address_space(1)))
typedef GAS u32 gu32;
typedef GAS u64 gu64;
typedef __amdgpu_buffer_rsrc_t rsrc_t;

constexpr int NR = 8;
constexpr int NCOMM = 256;
constexpr int NGEMM = 256;
constexpr u32 NPW = 21;
constexpr u32 HAND = 170;
constexpr u32 TOKE = 21;
constexpr u32 NRP = 35, RPP = 2;
constexpr int RDELAY = 3, RDELAY2 = 7;
constexpr u32 NPUSH = 48;
constexpr u32 PUSH0 = 16, PUSHD = 26;
constexpr u32 PSTAG = 24, PSTEP = 1;
constexpr int RFAST = 10, RSLOW = 14;
constexpr int RBASE = 40, REARLY = 3;
constexpr u32 LWIN = 8, LWINB = 5;
constexpr int LAHEAD = 10;
constexpr u32 NU = 480;
constexpr u64 DEADLINE = 100000000ull;

constexpr u32 BLK = 2944;
constexpr u32 OFF_CNT = 0;
constexpr u32 OFF_TMO = 16384;
constexpr u32 OFF_TKI = 16384 + 1024;
constexpr u32 OFF_TKO = 16384 + 2048;
constexpr u32 OFF_PCN = 16384 + 4096;
constexpr u32 OFF_IFL = 24576;
constexpr u32 OFF_TOK = 65536;
constexpr u32 OFF_IFA = 262144;
constexpr u32 OFF_DUM = 393216;
constexpr u32 OFF_OFL = 196608;
constexpr u32 OFF_DFL = 229376;
constexpr u32 OFF_HND = 98304;
constexpr u32 ID_WR = 1024, ID_XD = 1792, ID_WD = 1856, NID = 2368;
constexpr u32 KIB = 1024;
constexpr u32 OFF_RDY = 1024 * KIB;
constexpr u32 OFF_IMG = 2048 * KIB;
constexpr u32 OFF_IPK = 25088 * KIB;
constexpr u32 IPK_PAR = 6144 * KIB;
constexpr u32 OFF_IOV = 37376 * KIB;
constexpr u32 IOV_PAR = 4096 * KIB;
constexpr u32 OFF_OPK = 45568 * KIB;
constexpr u32 OFF_OOV = 51456 * KIB;
constexpr u32 OFF_DIA = 55552 * KIB;
constexpr u32 OFF_PSH = 63744 * KIB;
constexpr u32 PSH_PAR = 8u * NPUSH * BLK;
constexpr u32 WS_USED = OFF_PSH + 2u * PSH_PAR;
static_assert(WS_USED <= 65u * 1024u * KIB && OFF_OPK + 2048u * BLK <= OFF_OOV, "layout");
static_assert(OFF_RDY + NID * 256u <= OFF_IMG && OFF_IMG + NID * 8192u <= OFF_IPK && 2048u * BLK <= IPK_PAR, "layout");

struct Params {
    const char* x;
    const char* w;
    char* my;
    char* out;
    int rank;
    int pad;
    char* ws[NR];
};

struct Order { u32 e[256]; };
constexpr Order make_order() {
    constexpr int SLOW0 = 12;
    int key[17][8] = {};
    for (int ct = 0; ct < 16; ++ct) {
        int t = 8 * ct - RBASE - (ct < REARLY ? 8 * ct : 8 * (REARLY - 1) * (16 - ct) / (17 - REARLY));
        for (int ks = 7; ks >= 0; --ks) { t -= (ct >= SLOW0 && t <= 0) ? RSLOW : RFAST; key[ct][ks] = t; }
    }
    for (int ks = 0; ks < 8; ++ks) { int m = key[0][ks]; for (int ct = 1; ct < 16; ++ct) m = key[ct][ks] < m ? key[ct][ks] : m; key[16][ks] = m; }
    Order o{};
    int n = 0, next[17] = {};
    while (n < 240) {
        int best = -1;
        for (int i = 16; i >= 0; --i) if (next[i] < 8 && (best < 0 || key[i][next[i]] < key[best][next[best]])) best = i;
        if (best == 16) for (int t = 0; t < 14; ++t) o.e[n++] = (u32)(t << 3 | next[16]);
        else o.e[n++] = (u32)(128 | best << 3 | next[best]);
        ++next[best];
    }
    for (int ks = 0; ks < 8; ++ks) { o.e[n++] = (u32)(14 << 3 | ks); o.e[n++] = (u32)(15 << 3 | ks); }
    return o;
}
__constant__ const Order ORDER = make_order();
constexpr u32 G0 = 24, UA = 144u - G0, UB = UA + 128u, UC = UB + (48u - G0) * 5u;
static_assert(NPUSH >= G0 && NPUSH <= 2 * G0 && UA - 64u <= NPUSH + NPW, "whole flag groups; one look covers the ticketed early units");
struct PackTab { u32 e[512][2]; };
constexpr PackTab make_pack() {
    const Order o = make_order();
    PackTab t{};
    for (u32 g = 0; g < 64; ++g)
        for (u32 w = 0; w < 8; ++w) {
            const u32 bg = g >= 48u ? g - 48u : g >= G0 ? g - G0 + 16u : g + 64u - G0;
            const bool early = w < (g < G0 ? 2u : g < 48u ? 3u : 0u);
            const u32 u = g < G0 ? (early ? w * G0 + g + (w * G0 + g < NPUSH ? 0u : 3u * (48u - G0)) : UC + g * 6u + (w - 2))
                        : g < 48u ? (early ? NPUSH - G0 + g + w * (48u - G0) : UB + (g - G0) * 5u + (w - 3)) : UA + bg * 8u + w;
            const u32 e = o.e[u >> 1], tile = (e >> 3) & 15u, ks = e & 7u, half = u & 1u;
            const u32 delay = early || g >= 48u ? 0u : PSTAG + (bg - 16u) * PSTEP;
            t.e[g * 8 + w][0] = e | u << 8 | (bg >> 3) << 17 | delay << 20;
            t.e[g * 8 + w][1] = e >= 128u ? 32u * ks * 8192u + (128u * tile + 64u * half) * 2u
                                          : ((tile < 14u ? 512u * (tile >> 1) : 0u) + 128u * (tile & 1u) + 64u * half) * 1024u + 64u * ks;
        }
    return t;
}
__constant__ const PackTab PACK = make_pack();
__device__ __forceinline__ u32 entry_fix(u32 e, u32 src) {
    const u32 own = e - 112u + src * 16u, oth = e + (e >= src * 16u ? 16u : 0u), x = e >= 112u ? own : oth;
    return e >= 128u ? e : x;
}
__device__ __forceinline__ u32 entry(u32 u, u32 src) { return entry_fix(ORDER.e[(u >> 1) & 255u], src); }

__device__ __forceinline__ u64 rt() { return __builtin_amdgcn_s_memrealtime(); }
__device__ __forceinline__ u32 uni(u32 v) { return __builtin_amdgcn_readfirstlane(v); }
__device__ __forceinline__ u32 rdl(u32 v, u32 l) { return __builtin_amdgcn_readlane(v, l); }
__device__ __forceinline__ rsrc_t mkrs(const void* p, u32 bytes) { return __builtin_amdgcn_make_buffer_rsrc((void*)p, 0, bytes, 0x00020000); }
__device__ __forceinline__ u32x4 ld16(rsrc_t r, u32 voff, u32 soff) { return __builtin_amdgcn_raw_buffer_load_b128(r, voff, soff, 0); }
__device__ __forceinline__ u32x2 ld8(rsrc_t r, u32 voff, u32 soff) { return __builtin_amdgcn_raw_buffer_load_b64(r, voff, soff, 0); }
__device__ __forceinline__ u32x4 ld16_sys(rsrc_t r, u32 voff, u32 soff) { return __builtin_amdgcn_raw_buffer_load_b128(r, voff, soff, 17); }
__device__ __forceinline__ void st16_sys(u32x4 v, rsrc_t r, u32 voff, u32 soff) { __builtin_amdgcn_raw_buffer_store_b128(v, r, voff + soff, 0, 17); }
__device__ __forceinline__ void st16_dev(u32x4 v, rsrc_t r, u32 voff, u32 soff) { __builtin_amdgcn_raw_buffer_store_b128(v, r, voff + soff, 0, 16); }
__device__ __forceinline__ void st4_dev(u32 v, rsrc_t r, u32 voff, u32 soff) { __builtin_amdgcn_raw_buffer_store_b32(v, r, voff, soff, 16); }
#define FENCE() asm volatile("" ::: "memory")
#define WAITV(n) do { asm volatile("s_waitcnt vmcnt(" #n ")" ::: "memory"); __builtin_amdgcn_s_waitcnt(0xF70 | (n)); } while (0)
#define WAITL0() do { asm volatile("s_waitcnt lgkmcnt(0)" ::: "memory"); __builtin_amdgcn_s_waitcnt(0xC07F); } while (0)
#define SB() __builtin_amdgcn_sched_barrier(0)
#define BAR() do { FENCE(); __builtin_amdgcn_s_barrier(); FENCE(); } while (0)


__device__ __forceinline__ u32 pkmax(u32 a, u32 b) {
    return __builtin_bit_cast(u32, __builtin_elementwise_max(__builtin_bit_cast(u16x2, a), __builtin_bit_cast(u16x2, b)));
}
__device__ __forceinline__ u32 pkmaxi(u32 a, u32 b) {
    return __builtin_bit_cast(u32, __builtin_elementwise_max(__builtin_bit_cast(i16x2, a), __builtin_bit_cast(i16x2, b)));
}
struct Pk { u32x3 cs; u32 tail; bool raw; };
__device__ __forceinline__ u32 bfi(u32 m, u32 a, u32 b) { return __builtin_amdgcn_bitop3_b32(m, a, b, 0xCA); }
#define DPP0(v, ctrl, rm) (u32)__builtin_amdgcn_update_dpp(0, (int)(v), ctrl, rm, 0xf, true)
__device__ __forceinline__ void pack_hi(const u32 (&lo)[8], const u32 (&hi)[8], u32 m, int lane, LAS char* sc, Pk& p) {
    m = max(m, DPP0(m, 0x111, 0xf)); m = max(m, DPP0(m, 0x112, 0xf)); m = max(m, DPP0(m, 0x114, 0xf)); m = max(m, DPP0(m, 0x118, 0xf));
    u32 T0 = min(rdl(m, 15), rdl(m, 31)), T1 = min(rdl(m, 47), rdl(m, 63));
    asm volatile("" : "+s"(T0), "+s"(T1));
    u32 T = max(min(T0, T1), 7u);
    asm volatile("" : "+s"(T));
    const u32 L = T - 7u, s = L & 1u, base = (L + s) >> 1;
    const u32 k4 = (128u - base) * 0x01010101u,
              s4 = s * 0x01010101u;
    u32 c7f = 0x7F7F7F7Fu, c7c = 0x7C7C7C7Cu;
    asm volatile("" : "+s"(c7f), "+s"(c7c));
    u32 d[8], x[8], bits = 0;
#pragma unroll
    for (int k = 7; k >= 0; --k) {
        d[k] = (hi[k] & c7f) + ((lo[k] >> 7) & s4) + k4;
        x[k] = d[k] & c7c;
        bits = k < 7 ? bfi(c7f, bits >> 1, x[k] + c7c) : x[k] + c7c;
    }
    *(LAS u32x4*)(sc + lane * 16) = (u32x4){x[0], x[1], x[2], x[3]};
    *(LAS u32x4*)(sc + 1024 + lane * 16) = (u32x4){x[4], x[5], x[6], x[7]};
    const u32 cnt = __builtin_popcount(bits);
    u32 sum = cnt;
    sum += DPP0(sum, 0x111, 0xf); sum += DPP0(sum, 0x112, 0xf); sum += DPP0(sum, 0x114, 0xf); sum += DPP0(sum, 0x118, 0xf);
    sum += DPP0(sum, 0x142, 0xa); sum += DPP0(sum, 0x143, 0xc);
    asm volatile("" : "+v"(sum));
    const u32 n = rdl(sum, 63);
    p.raw = n > 62u;
    {
        LAS unsigned short* const list = (LAS unsigned short*)(sc + 2048);
        const u32 l5 = (u32)lane << 5;
        LAS unsigned short* q = list + (sum - cnt);
        if (p.raw) bits = 0;
#pragma unroll
        for (int j = 0; j < 3; ++j) {
            u32 lb; asm("v_ffbl_b32 %0, %1" : "=v"(lb) : "v"(bits));
            (bits ? q : list + 64)[j] = (unsigned short)(l5 | lb); bits &= bits - 1u;
        }
        q += 3;
        while (bits) { *q++ = (unsigned short)(l5 | __builtin_ctz(bits)); bits &= bits - 1u; }
    }
    FENCE();
    const u32 e = ((LAS unsigned short*)(sc + 2048))[lane];
    SB();
    u32 cc[2];
#pragma unroll
    for (int k = 7; k >= 0; --k) cc[k >> 2] = (k & 3) < 3 ? bfi(0x03030303u, d[k], cc[k >> 2] << 2) : d[k];
    SB();
    const u32 pos = ((~e & 4u) << 8) | ((e >> 1) & 0x3F0u) | ((~e & 3u) << 2) | ((e >> 3) & 3u);
    const u32 val = *(LAS unsigned char*)(sc + pos);
    SB();
    u32 sg = hi[7];
#pragma unroll
    for (int k = 6; k >= 0; --k) sg = bfi(c7f, sg >> 1, hi[k]);
    p.cs = (u32x3){cc[0], cc[1], sg};
    const u32 hd = lane == 63 ? (base - s) | s << 7 : min(n, 63u);
    SB();
    p.tail = lane >= 62 ? hd : pos | val << 9;
    FENCE();
}
__device__ __forceinline__ void store_blk_cs(const Pk& p, rsrc_t rs, u32 so, int lane) {
    __builtin_amdgcn_raw_buffer_store_b96(p.cs, rs, so + 2048u + lane * 12u, 0, 17);
    __builtin_amdgcn_raw_buffer_store_b16((unsigned short)p.tail, rs, so + 2816u + lane * 2u, 0, 17);
}
__device__ __forceinline__ void store_blk_raw(const Pk& p, const u32 (&hi)[8], rsrc_t rs, u32 ov, int lane) {
    if (p.raw) {
        st16_sys((u32x4){hi[0], hi[1], hi[2], hi[3]}, rs, lane * 16u, ov);
        st16_sys((u32x4){hi[4], hi[5], hi[6], hi[7]}, rs, lane * 16u + 1024u, ov);
    }
}
struct Blk { u32x4 l0, l1; u32x3 cs; u32 tail; };
__device__ __forceinline__ void load_blk(Blk& b, rsrc_t rs, u32 so, int lane) {
    b.tail = __builtin_amdgcn_raw_buffer_load_b16(rs, 2816u + lane * 2u, so, 17);
    b.cs = __builtin_amdgcn_raw_buffer_load_b96(rs, 2048u + lane * 12u, so, 17);
    b.l0 = ld16_sys(rs, lane * 16u, so);
    b.l1 = ld16_sys(rs, lane * 16u + 1024u, so);
}
__device__ __forceinline__ void unpack_exc(u32 tail, int lane, LAS char* img, u32x4& d0, u32x4& d1) {
    const u32 cn = rdl(tail, 62);
    const bool mine = (u32)lane < (cn > 62u ? 0u : cn);
    LAS unsigned char* const q = (LAS unsigned char*)img + (tail & 0x7FFu);
    if (mine) *q = (unsigned char)((tail >> 9) & 0x7Cu);
    FENCE();
    d0 = *(LAS u32x4*)(img + lane * 16); d1 = *(LAS u32x4*)(img + 1024 + lane * 16);
    FENCE();
    if (mine) *q = 0;
    FENCE();
}
template <bool SPLIT>
__device__ __forceinline__ void unpack_hi(const Blk& b, u32x4 d0, u32x4 d1, rsrc_t rs, u32 ov, int lane, u32 (&hi)[8]) {
    const u32 hb = rdl(b.tail, 63);
    const u32 b4 = (hb & 127u) * 0x01010101u, s4 = ((hb >> 7) & 1u) * 0x01010101u;
    u32 u[8];
#pragma unroll
    for (int k = 0; k < 8; ++k) {
        const u32 lo = k < 4 ? b.l0[k & 3] : b.l1[k & 3];
        u[k] = bfi(0x03030303u, b.cs[k >> 2] >> (2 * (k & 3)), k < 4 ? d0[k & 3] : d1[k & 3]) + (SPLIT ? 0u : ~(lo >> 7) & s4) + b4;
    }
    if (SPLIT && s4) {
#pragma unroll
        for (int k = 0; k < 8; ++k) u[k] += bfi((k < 4 ? b.l0[k & 3] : b.l1[k & 3]) >> 7, 0u, s4);
    }
#pragma unroll
    for (int k = 0; k < 8; ++k) hi[k] = bfi(0x7F7F7F7Fu, u[k], b.cs[2] << k);
    if (__builtin_expect(rdl(b.tail, 62) > 62u, 0)) {
        const u32x4 h0 = ld16_sys(rs, lane * 16u, ov), h1 = ld16_sys(rs, lane * 16u + 1024u, ov);
        WAITV(0);
#pragma unroll
        for (int k = 0; k < 4; ++k) { hi[k] = h0[k]; hi[4 + k] = h1[k]; }
    }
}

__device__ __forceinline__ bool timed_out(gu32* tmo, u32 call, u64 t0) {
    if (uni(__hip_atomic_load(tmo, __ATOMIC_RELAXED, __HIP_MEMORY_SCOPE_AGENT)) == call) return true;
    if (rt() - t0 > DEADLINE) { __hip_atomic_store(tmo, call, __ATOMIC_RELAXED, __HIP_MEMORY_SCOPE_AGENT); return true; }
    return false;
}
__device__ __forceinline__ void store_frags(const u32 (&hi)[8], u32x4 l0, u32x4 l1, rsrc_t rs, u32 voff) {
#pragma unroll
    for (int f = 0; f < 4; ++f) {
        u32x4 v;
#pragma unroll
        for (int p = 0; p < 4; ++p) v[p] = __builtin_amdgcn_perm(hi[4 * (f >> 1) + p], (f >> 1) ? l1[p] : l0[p], (f & 1) ? 0x07030602u : 0x05010400u);
        st16_dev(v, rs, voff + f * 1024u, 0);
    }
}

__device__ __forceinline__ void unit_load(u32 (&in)[16], const char* x, const char* w, u32 op, u32 so, int lane) {
    if (op) {
        const rsrc_t rw = mkrs(w, 512u * 4096u * 2u);
        const u32 vo = (lane >> 4) * 8u * 8192u + (lane & 7) * 16u + (lane & 8);
#pragma unroll
        for (int r = 0; r < 8; ++r) { const u32x2 v = ld8(rw, vo, so + r * 8192u); in[2 * r] = v[0]; in[2 * r + 1] = v[1]; }
    } else {
        const rsrc_t rx = mkrs(x, 4096u * 512u * 2u);
        const u32 vo = (lane & 15) * 1024u + (lane >> 4) * 16u;
#pragma unroll
        for (int m = 0; m < 4; ++m) { const u32x4 v = ld16(rx, vo, so + m * 16384u); in[4 * m] = v[0]; in[4 * m + 1] = v[1]; in[4 * m + 2] = v[2]; in[4 * m + 3] = v[3]; }
    }
}
__device__ __forceinline__ void unit_split(const u32 (&in)[16], u32 op, u32 (&lo)[8], u32 (&hi)[8]) {
    if (op) {
#pragma unroll
        for (int k = 0; k < 8; ++k) {
            const u32 a0 = in[4 * (k & 3) + (k >> 2)], a1 = in[4 * (k & 3) + 2 + (k >> 2)];
            lo[k] = __builtin_amdgcn_perm(a1, a0, 0x06020400u); hi[k] = __builtin_amdgcn_perm(a1, a0, 0x07030501u);
        }
    } else {
#pragma unroll
        for (int k = 0; k < 8; ++k) {
            const u32 a0 = in[8 * (k >> 2) + (k & 3)], a1 = in[8 * (k >> 2) + 4 + (k & 3)];
            lo[k] = __builtin_amdgcn_perm(a1, a0, 0x06040200u); hi[k] = __builtin_amdgcn_perm(a1, a0, 0x07050301u);
        }
    }
}

__device__ __forceinline__ u32 cvt_pk_bf16(float lo, float hi) {
    typedef __bf16 bf2 __attribute__((ext_vector_type(2)));
    typedef float f2 __attribute__((ext_vector_type(2)));
    return __builtin_bit_cast(u32, __builtin_convertvector((f2){lo, hi}, bf2));
}
typedef float f32x2 __attribute__((ext_vector_type(2)));
__device__ __forceinline__ float bf_top(u32 hi, u32 lo, int e) { return __builtin_bit_cast(float, __builtin_amdgcn_perm(hi, lo, 0x04000c0cu + e * 0x01010000u)); }
__device__ __forceinline__ u32 silu2(f32x2 y) {
    constexpr float LOG2E = 1.44269504088896341f;
    const f32x2 t = y * -LOG2E;
    f32x2 e;
    e.x = __builtin_amdgcn_exp2f(__builtin_amdgcn_fmed3f(t.x, -60.0f * LOG2E, 60.0f * LOG2E));
    e.y = __builtin_amdgcn_exp2f(__builtin_amdgcn_fmed3f(t.y, -60.0f * LOG2E, 60.0f * LOG2E));
    e += 1.0f;
    e.x = __builtin_amdgcn_rcpf(e.x); e.y = __builtin_amdgcn_rcpf(e.y);
    y *= e;
    return cvt_pk_bf16(y.x, y.y);
}

#define BID (blockIdx.x ^ 256u)
__global__ void __launch_bounds__(512, 4) fused(Params P) {
    constexpr int NS = 5;
    __shared__ __attribute__((aligned(16))) char lds[NS * 16384];
    const int wid = uni(threadIdx.x >> 6);
#define LANE_VARS() int tz_ = threadIdx.x; asm volatile("" : "+v"(tz_)); const int lane = tz_ & 63; const u32 v16 = lane * 16u; \
    const u32 fvoff = lane == 0 ? 0u : 0x80000000u;   (void)v16; (void)fvoff
    const u32 pk0 = PACK.e[((blockIdx.x >> 2) & 63u) * 8u + wid][0], pk1 = PACK.e[((blockIdx.x >> 2) & 63u) * 8u + wid][1];
    const u32 sw = (blockIdx.x & 2) << 11 | (blockIdx.x & 1) << 21, sx = (blockIdx.x & 2) << 17 | (blockIdx.x & 1) << 9;
    const u32 vx = (threadIdx.x & 15) * 1024u + (threadIdx.x & 48);
    asm volatile("" :: "s"(pk0), "s"(pk1), "s"(P.x), "s"(P.w), "s"(P.my), "s"(P.rank), "s"(sw), "s"(sx), "v"(vx));
    u32 in[16];
    if (blockIdx.x < NGEMM) {
        for (u32 i = pk0 >> 20; i; --i) __builtin_amdgcn_s_sleep(1);
        const u32 e = pk0 & 255u, r16 = P.rank * 16u;
        unit_load(in, P.x, P.w, e >> 7, pk1 + (e >= 128u ? sw : sx + ((e >= 112u ? r16 : e >= r16 ? 16u : 0u) << 15)), threadIdx.x & 63);
    }
    u32 rank = P.rank;
    u64 my_ = (u64)P.my;
    asm volatile("" : "+s"(rank), "+s"(my_) :: "memory");
    const u32 ra = rank >> 2, rb = (rank >> 1) & 1, rc = rank & 1;
    char* const my = (char*)my_;
    gu32* const tmo = (gu32*)(my + OFF_TMO);
    gu32* const cnt = (gu32*)(my + OFF_CNT) + BID * 8 + wid;
#define PUT_CALL() do { if (lane == 0) __hip_atomic_store(cnt, call, __ATOMIC_RELAXED, __HIP_MEMORY_SCOPE_AGENT); } while (0)
    const rsrc_t rmy = mkrs(my, WS_USED);

    if (BID < NCOMM) {
        const u32 cb = BID;
        const bool shot = cb - 64u < 128u && wid >= 4;
        const bool puller = cb >= 128u || shot, second = cb < 192u && !shot;
        const u32 src = (cb >> 3) & 7;
        const u32 wi = shot ? (((cb >> 6) - 1u) * 4u + (wid - 4)) * 8u + (cb & 7)
                     : !puller ? ((cb >> 5) * RPP + (wid >> 1)) * 8u + (cb & 7)
                     : wid * 8u + ((cb + src) & 7);
        if (shot ? (wi >= NPUSH || (src ^ rank) < 2u) : puller ? (wi >= NPW || (src ^ rank) < 2u) : (wid >= 2 * RPP || wi >= NRP)) return;
        LANE_VARS();
        const u32 call = uni(__hip_atomic_load(cnt, __ATOMIC_RELAXED, __HIP_MEMORY_SCOPE_AGENT)) + 1u;
        PUT_CALL();
        __builtin_amdgcn_s_setprio(3);
        bool ok = true;
        LAS char* const img = (LAS char*)lds + wid * 4096;
#pragma unroll
        for (int i = 0; i < 4; ++i) *(LAS u32x4*)(img + i * 1024 + v16) = (u32x4){0, 0, 0, 0};
        FENCE();
        if (puller) {
            const rsrc_t rs = mkrs(P.ws[src], WS_USED);
            const u32 par = call & 1, sj = 14u + (src ^ rank);
            const gu32* const fla = (const gu32*)(my + OFF_IFA + src * 16384u);
            const gu32* const flb = (const gu32*)(my + OFF_IFL + src * 1024u + (lane & 7) * 128u);
            u64 ma = 0, mb = shot ? ~0ull : 0ull;
            auto ready = [&](u32 u) {
                const u32 sg = u < UB ? (u - UA) >> 6 : u < UC ? 2u + (u - UB) / 40u : (64u - G0) / 8u + (u - UC) / 48u;
                return ((u < UA ? ma >> (u + 64u - UA) : mb >> sg) & 1) != 0;
            };
            auto look = [&](u32 u) {
                if (u < UA) return uni(__hip_atomic_load(fla + u * 32u, __ATOMIC_RELAXED, __HIP_MEMORY_SCOPE_SYSTEM)) >= call;
                mb = __builtin_amdgcn_ballot_w64(__hip_atomic_load(flb, __ATOMIC_RELAXED, __HIP_MEMORY_SCOPE_SYSTEM) >= call);
                return ready(u);
            };
            auto wait = [&](u32 u) {
                const u64 t0 = rt();
                for (u32 it = 1;; ++it) {
                    if (look(u)) return true;
                    if ((it & 63u) == 0 && timed_out(tmo, call, t0)) return false;
                    __builtin_amdgcn_s_sleep(1);
                }
            };
            u32 u0 = shot ? wi : NPUSH + wi;
            const u32 ho = OFF_HND + (src * 32u + wi) * 128u, tbase = NPUSH + NPW - (call - 1u) * (NU - NPUSH);
            if (second) {
                u32 hv;
                const u64 t0 = rt();
                for (u32 it = 1;; ++it) {
                    hv = uni(__hip_atomic_load((const gu32*)(my + ho), __ATOMIC_RELAXED, __HIP_MEMORY_SCOPE_AGENT));
                    if ((hv & 0x7FFFFFFFu) >= call) break;
                    if ((it & 63u) == 0 && timed_out(tmo, call, t0)) { ok = false; break; }
                    if (~mb) mb = __builtin_amdgcn_ballot_w64(__hip_atomic_load(flb, __ATOMIC_RELAXED, __HIP_MEMORY_SCOPE_SYSTEM) >= call);
                    __builtin_amdgcn_s_sleep(2);
                }
                u0 = !ok || (hv >> 31) ? NU : uni(__builtin_amdgcn_raw_ptr_buffer_atomic_add_i32(1, rmy, fvoff, OFF_TKI + src * 128u, 0)) + tbase;
                if (u0 >= NU) {
                    st4_dev(call, rmy, fvoff, OFF_TOK + (src * 64u + wi) * 64u);
                    return;
                }
            }
            u32 e0 = entry(u0, src);
            u32 b0 = (((e0 >> 7) ? rb : ra) * 2u + rc) * 512u + u0;
            const rsrc_t rd = mkrs(shot ? my + OFF_PSH + par * PSH_PAR + (src * NPUSH + u0) * BLK - b0 * BLK : P.ws[src] + OFF_IPK + par * IPK_PAR, 2048u * BLK);
            u32 tinc = shot ? 0u : 1u;
            const u32 dummy = OFF_DUM + (cb * 8u + wid) * 64u, tko = shot ? dummy : OFF_TKI + src * 128u;
            bool last = shot;
            if (!second && !shot) __builtin_amdgcn_s_sleep(10);
            if (ok && !(second && ready(u0))) ok = wait(u0);
            Blk N; u32 tkt = 0, fa = 0, fb = 0;
#define IISSUE(N_, b_) do { FENCE(); tkt = __builtin_amdgcn_raw_ptr_buffer_atomic_add_i32(tinc, rmy, fvoff, tko, 0); \
                load_blk(N_, rd, (b_) * BLK, lane); FENCE(); } while (0)
#define ILOOK(b_) do { fb = __builtin_amdgcn_raw_buffer_load_b32(rmy, OFF_IFL + src * 1024u + (lane & 7) * 128u, 0, 17); \
                fa = __builtin_amdgcn_raw_buffer_load_b32(rmy, lane + (UA - 64u) > max((b_) & 511u, NPUSH + NPW - 1u) ? \
                        OFF_IFA + src * 16384u + (lane + (UA - 64u)) * 128u : fvoff | (OFF_IFA + src * 16384u + (UA - 64u) * 128u), 0, 17); FENCE(); } while (0)
#define INEXT() do { u1 = last ? NU : uni(tkt) + tbase; e1 = entry(u1, src); \
                b1 = (((e1 >> 7) ? rb : ra) * 2u + rc) * 512u + (u1 & 511u); \
                if (!second && !last && u1 >= HAND) { st4_dev(u1 < NU ? call : call | 0x80000000u, rmy, fvoff, ho); tinc = 0; last = true; } } while (0)
#define IUNPACK(B_) do { u32 hi_[8]; u32x4 d0_, d1_; unpack_exc(B_.tail, lane, img, d0_, d1_); \
                unpack_hi<false>(B_, d0_, d1_, rs, OFF_IOV + par * IOV_PAR + b0 * 2048u, lane, hi_); \
                const u32 ih_ = ((e0 >= 128u ? ID_WR - 16u + ((e0 >> 3) & 15u) * 48u : (e0 >> 3) * 64u) + sj + (e0 & 7) * 6u) * 2u + (u0 & 1); \
                store_frags(hi_, B_.l0, B_.l1, rmy, v16 + OFF_IMG + ih_ * 4096u); rp = OFF_RDY + ih_ * 128u; } while (0)
            if (ok) {
                IISSUE(N, b0);
                if (!shot) ILOOK(b0);
                WAITV(0);
                u32 rp = dummy;
                u32 u1, e1, b1;
                INEXT();
                bool go = false;
                for (;;) {
                    if (!go) {
                        ma |= __builtin_amdgcn_ballot_w64(fa >= call); mb |= __builtin_amdgcn_ballot_w64(fb >= call);
                        if (u1 >= NU || (u1 >= UA && !~mb)) break;
                        go = ready(u1) || look(u1);
                    }
                    Blk B = N;
                    asm volatile("" : "+v"(B.l0), "+v"(B.l1), "+v"(B.cs), "+v"(B.tail));
                    if (go) {
                        IISSUE(N, b1);
                        ma |= __builtin_amdgcn_ballot_w64(fa >= call); mb |= __builtin_amdgcn_ballot_w64(fb >= call);
                        ILOOK(b1); WAITV(7);
                    } else {
                        WAITV(0);
                    }
                    st4_dev(call, rmy, fvoff | rp, 0);
                    IUNPACK(B);
                    if (!go) {
                        WAITV(0);
                        st4_dev(call, rmy, fvoff | rp, 0);
                        rp = dummy;
                        if (!wait(u1)) { ok = false; break; }
                        IISSUE(N, b1); ILOOK(b1);
                        WAITV(0);
                    }
                    u0 = u1; e0 = e1; b0 = b1;
                    WAITV(11); INEXT();
                    go = u1 < NU && (u1 < UA || ~mb) && ready(u1);
                    { u32 g_ = go; asm volatile("" : "+s"(u1), "+s"(e1), "+s"(b1), "+s"(g_)); go = g_; }
                    WAITV(5);
                }
                if (ok) {
                    Blk M;
                    bool tok = !second;
#define ISTEP(C_, N_) { if (!tok && u1 + TOKE >= NU) { st4_dev(call, rmy, fvoff, OFF_TOK + (src * 64u + wi) * 64u); tok = true; } \
                        if (u1 >= NU) { WAITV(0); st4_dev(call, rmy, fvoff | rp, 0); IUNPACK(C_); break; } \
                        IISSUE(N_, b1); \
                        WAITV(5);             \
                        st4_dev(call, rmy, fvoff | rp, 0); \
                        IUNPACK(C_); \
                        u0 = u1; e0 = e1; b0 = b1; \
                        WAITV(9); INEXT(); asm volatile("" : "+s"(u1), "+s"(e1), "+s"(b1)); \
                        WAITV(5); }
                    for (;;) { ISTEP(N, M) ISTEP(M, N) }
                    WAITV(0);
                    st4_dev(call, rmy, fvoff | rp, 0);
                }
            }
        } else {
            const u32 st = src & 3, c = wid & 1, so = st * 2u + c;
            const rsrc_t rs = mkrs(P.ws[so], WS_USED);
            const rsrc_t ro = mkrs(P.out, 512u * 4096u * 2u);
            const gu32* const fl = (const gu32*)(my + ((lane & 2) ? OFF_DFL + st * 4096u : OFF_OFL + st * 8192u + (lane & 1) * 4096u));
            const u32 ovoff = (c * 16u + (lane >> 4) * 4u + ((lane >> 3) & 1)) * 8192u + (lane & 7) * 16u + (st >> 1) * 2097152u + (st & 1) * 4096u;
            LAS char* xo = (LAS char*)lds + 32768 + wid * 4096 + v16;
            LAS char* xi = (LAS char*)lds + 32768 + (wid ^ 1) * 4096 + v16;
            LAS u32* const wo = (LAS u32*)(img + 2048);
            LAS u32* const wi_ = (LAS u32*)((LAS char*)lds + (wid ^ 1) * 4096 + 2048);
            WAITL0(); BAR();
            auto wait = [&](u32 v) {
                const u64 t0 = rt();
                for (u32 it = 1;; ++it) {
                    if (__builtin_amdgcn_ballot_w64(__hip_atomic_load(fl + (v >> 3) * 32u, __ATOMIC_RELAXED, __HIP_MEMORY_SCOPE_SYSTEM) < call) == 0) return true;
                    if ((it & 63u) == 0 && timed_out(tmo, call, t0)) return false;
                    __builtin_amdgcn_s_sleep(1);
                }
            };
            u32 v0 = wi;
            {
                const u32 ts = st * 2u + ((wi >> 3) & 1);
                const gu32* tok = (const gu32*)(my + OFF_TOK + (((ts ^ rank) < 2u ? ts ^ 2u : ts) * 64u + wi % NPW) * 64u);
                const u64 t0 = rt();
                for (u32 it = 1;; ++it) {
                    const u32 tv = __hip_atomic_load(tok, __ATOMIC_RELAXED, __HIP_MEMORY_SCOPE_AGENT);
                    if (uni(tv) >= call) break;
                    if ((it & 63u) == 0 && timed_out(tmo, call, t0)) { ok = false; break; }
                    __builtin_amdgcn_s_sleep(4);
                }
            }
            if (wi >= NPW) for (int i = 0; i < (wi >= 32u ? RDELAY2 : RDELAY); ++i) __builtin_amdgcn_s_sleep(8);
            bool rdy = false;
            u32 trip = 0;
            for (;;) {
                if (ok && !rdy) ok = wait(v0);
                FENCE();
                Blk B;
                const u32 tkt = __builtin_amdgcn_raw_ptr_buffer_atomic_add_i32(c ^ 1u, rmy, fvoff, c ? OFF_DUM + (cb * 8u + wid) * 64u : OFF_TKO + st * 128u, 0);
                FENCE();
                {
                    const u32 bo = OFF_OPK + (rank * 256u + v0) * BLK;
                    B.tail = __builtin_amdgcn_raw_buffer_load_b16(rs, 2816u + lane * 2u, bo, 17);
                    B.cs = __builtin_amdgcn_raw_buffer_load_b96(rs, 2048u + lane * 12u, bo, 17);
                    FENCE();
                    B.l0 = ld16_sys(rs, v16, bo); B.l1 = ld16_sys(rs, v16 + 1024u, bo);
                }
                FENCE();
                const u32 tm = __hip_atomic_load(tmo, __ATOMIC_RELAXED, __HIP_MEMORY_SCOPE_AGENT);
                u32x4 dg[4];
#pragma unroll
                for (int k = 0; k < 4; ++k) dg[k] = __builtin_amdgcn_raw_buffer_load_b128(rmy, v16 + k * 1024u, OFF_DIA + (st * 256u + v0) * 8192u + c * 4096u, 16);
                FENCE();
                u32 oo = ovoff + (((v0 >> 2) & 3) * 64u + (v0 & 1) * 32u) * 8192u + ((v0 >> 4) * 128u + ((v0 >> 1) & 1) * 64u) * 2u;
                asm volatile("" : "+v"(oo));
                ++trip;
                u32 v1 = min(uni(tkt) - (call - 1u) * 256u + NRP, 0x7FFFu);
                if (!c) __hip_atomic_store(wo + 2 + (trip & 1), trip << 16 | v1, __ATOMIC_RELAXED, __HIP_MEMORY_SCOPE_WORKGROUP);
                FENCE();
                if (c) { u32 pw; do { pw = uni(__hip_atomic_load(wi_ + 2 + (trip & 1), __ATOMIC_RELAXED, __HIP_MEMORY_SCOPE_WORKGROUP)); } while ((pw >> 16) != trip); v1 = pw & 0x7FFFu; }
                FENCE();
                const u32 fv = __hip_atomic_load(fl + ((v1 >> 3) & 31u) * 32u, __ATOMIC_RELAXED, __HIP_MEMORY_SCOPE_SYSTEM);
                FENCE();
                if (!ok) B.tail = 0;
                u32 h[8]; u32x4 d0, d1;
                unpack_exc(B.tail, lane, img, d0, d1);
                unpack_hi<true>(B, d0, d1, rs, OFF_OOV + (rank * 256u + v0) * 2048u, lane, h);
                if (!ok || uni(tm) == call) {
#pragma unroll
                    for (int k = 0; k < 8; ++k) { h[k] = 0x7F7F7F7Fu; B.l0[k & 3] = 0xC0C0C0C0u; B.l1[k & 3] = 0xC0C0C0C0u; }
                }
                u32x4 mh, ml;
                if (c) { *(LAS u32x4*)xo = (u32x4){h[0], h[1], h[2], h[3]}; *(LAS u32x4*)(xo + 1024) = B.l0; mh = (u32x4){h[4], h[5], h[6], h[7]}; ml = B.l1; }
                else   { *(LAS u32x4*)xo = (u32x4){h[4], h[5], h[6], h[7]}; *(LAS u32x4*)(xo + 1024) = B.l1; mh = (u32x4){h[0], h[1], h[2], h[3]}; ml = B.l0; }
                FENCE();
                __hip_atomic_store(wo, trip, __ATOMIC_RELAXED, __HIP_MEMORY_SCOPE_WORKGROUP);
                FENCE();
                f32x2 y[8];
#pragma unroll
                for (int k = 0; k < 4; ++k) { const f32x4 yd = __builtin_bit_cast(f32x4, dg[k]);
                    y[2 * k] = (f32x2){bf_top(mh[k], ml[k], 0), bf_top(mh[k], ml[k], 1)} + (f32x2){yd[0], yd[1]};
                    y[2 * k + 1] = (f32x2){bf_top(mh[k], ml[k], 2), bf_top(mh[k], ml[k], 3)} + (f32x2){yd[2], yd[3]}; }
#pragma unroll
                for (int i = 0; i < 8; ++i) asm volatile("" : "+v"(y[i]));
                while (uni(__hip_atomic_load(wi_, __ATOMIC_RELAXED, __HIP_MEMORY_SCOPE_WORKGROUP)) < trip) {}
                FENCE();
                const u32x4 ph = *(LAS u32x4*)xi, pl = *(LAS u32x4*)(xi + 1024);
                { const int fl_ = (trip & 1) ? 2048 : -2048; xo += fl_; xi += fl_; }
#pragma unroll
                for (int k = 0; k < 4; k += 2) {
                    u32 r[4];
#pragma unroll
                    for (int j = 0; j < 2; ++j) {
                        r[2 * j] = silu2(y[2 * (k + j)] + (f32x2){bf_top(ph[k + j], pl[k + j], 0), bf_top(ph[k + j], pl[k + j], 1)});
                        r[2 * j + 1] = silu2(y[2 * (k + j) + 1] + (f32x2){bf_top(ph[k + j], pl[k + j], 2), bf_top(ph[k + j], pl[k + j], 3)}); }
                    u32x4 o;
#pragma unroll
                    for (int d = 0; d < 2; ++d) {
                        o[d] = __builtin_amdgcn_update_dpp(r[d], r[2 + d], 0x128, 0xf, 0xc, false);
                        o[2 + d] = __builtin_amdgcn_update_dpp(r[2 + d], r[d], 0x128, 0xf, 0x3, false);
                    }
                    st16_dev(o, ro, oo + k * 8192u, 0);
                }
                if (v1 >= 256u) break;
                rdy = __builtin_amdgcn_ballot_w64(fv < call) == 0;
                v0 = v1;
            }
        }
        return;
    }

    const u32 gb = BID - NCOMM;
    u32 call;
    {
        LANE_VARS();
        const u32 pc = gb & 3, ent0 = pk0 & 255u, op = ent0 >> 7;
        u32 lo[8], hi[8];
        const u32 u = (pk0 >> 8) & 511u, sg = (pk0 >> 17) & 7u, ent = entry_fix(ent0, rank);
        const bool push = u < NPUSH, early = u < UA;
        FENCE();
        const u32 cntv = __hip_atomic_load(cnt, __ATOMIC_RELAXED, __HIP_MEMORY_SCOPE_AGENT);
        FENCE();
        const u32 g0 = (op ? (pc & 2) : (pc & 2) * 2u) + (pc & 1), g1 = g0 + (op ? 4u : 2u);
        const u32 po = OFF_PSH + (rank * NPUSH + min(u, NPUSH - 1u)) * BLK;
        u64 w0 = (u64)P.ws[g0 & 7], w1 = (u64)P.ws[g1 & 7];
        asm volatile("" : "+s"(w0), "+s"(w1));
        u64 a0 = push ? w0 + po : (u64)my + OFF_IPK + (pc * 512u + u) * BLK, a1 = w1 + po;
        gu32* const df = (gu32*)((lane ? w1 : w0) + OFF_IFA + rank * 16384u + u * 128u);
        const bool dof = early && lane < 2 && ((lane ? g1 : g0) ^ rank) >= 2u;
        const u32 tile = (ent >> 3) & 15u, kd = (pc & 1) * 8u + (ent & 7);
        u32 ih = (op ? ID_WD + ((pc >> 1) * 16u + tile) * 16u + kd : u >= NU ? ID_XD + ((pc & 2) + (tile & 1)) * 16u + kd : tile * 64u + kd) * 2u + (u & 1);
        if (!(op || u >= NU || (pc >> 1) == ra)) ih = 0xFFFFFFFFu;
        asm volatile("" : "+s"(ih) : "v"(df), "s"(__builtin_amdgcn_ballot_w64(dof)));
        u32 n0 = push && (g0 ^ rank) < 2u ? 0u : BLK, n1 = !push || (g1 ^ rank) < 2u ? 0u : BLK, ps = push ? PSH_PAR : IPK_PAR, hold = push && u > PUSH0 ? ((u - PUSH0) * PUSHD) >> 4 : 0u;
        asm volatile("" : "+s"(a0), "+s"(a1), "+s"(n0), "+s"(n1), "+s"(ps), "+s"(hold));
        SB();
        call = uni(cntv) + 1u;
        const u32 par = call & 1;
        const rsrc_t d0 = mkrs((const char*)a0 + (par ? ps : 0u), n0), d1 = mkrs((const char*)a1 + (par ? ps : 0u), n1);
        unit_split(in, op, lo, hi);
        const u32x4 l0 = {lo[0], lo[1], lo[2], lo[3]}, l1 = {lo[4], lo[5], lo[6], lo[7]};
        if (hold) { asm volatile("" :: "v"(l0), "v"(l1)); for (; hold; --hold) __builtin_amdgcn_s_sleep(1); }
        st16_sys(l0, d0, v16, 0); st16_sys(l1, d0, v16 + 1024, 0);
        st16_sys(l0, d1, v16, 0); st16_sys(l1, d1, v16 + 1024, 0);
        FENCE(); SB();
        u32 mu = in[0], mi = in[1];
#pragma unroll
        for (int i = 2; i < 16; i += 2) { mu = pkmax(mu, in[i]); mi = pkmaxi(mi, in[i + 1]); }
        mu = pkmax(mu & 0x7FFF7FFFu, mi & 0x7FFF7FFFu);
        Pk pk;
        pack_hi(lo, hi, max(mu >> 16, mu & 0xFFFFu) >> 7, lane, (LAS char*)lds + wid * 4096, pk);
        store_blk_cs(pk, d0, 0, lane);
        store_blk_cs(pk, d1, 0, lane);
        store_blk_raw(pk, hi, rmy, OFF_IOV + par * IOV_PAR + (pc * 512u + u) * 2048u, lane);
        if (early) {
            WAITV(0);
            if (dof) __hip_atomic_store(df, call, __ATOMIC_RELAXED, __HIP_MEMORY_SCOPE_SYSTEM);
        }
        if (ih != 0xFFFFFFFFu) store_frags(hi, l0, l1, rmy, v16 + OFF_IMG + ih * 4096u);
        WAITV(0);
        if (ih != 0xFFFFFFFFu) st4_dev(call, rmy, fvoff, OFF_RDY + ih * 128u);
        PUT_CALL();
        BAR();
        if (wid == 3) {
            const u32 t = __builtin_amdgcn_raw_ptr_buffer_atomic_add_i32(1, rmy, fvoff, OFF_PCN + sg * 128u, 0);
            if (uni(t) == call * 32u - 1u) {
                u64 q[NR];
#pragma unroll
                for (int i = 0; i < NR; ++i) { q[i] = (u64)P.ws[i]; asm volatile("" : "+s"(q[i])); }
                u64 dv = q[0];
#pragma unroll
                for (int i = 1; i < NR; ++i) dv = lane == i ? q[i] : dv;
                if (lane < 8) __hip_atomic_store((gu32*)(dv + OFF_IFL + rank * 1024u + sg * 128u), call, __ATOMIC_RELAXED, __HIP_MEMORY_SCOPE_SYSTEM);
            }
        }
    }
    const u32 xcd = gb & 7, idx = gb >> 3;
    const u32 rt_ = (xcd >> 1) * 4 + (idx & 3), ct = (((idx >> 2) * 2) ^ ((idx >> 2) & 4u)) + (xcd & 1);

    const u32 owner = rt_ >> 1;
    const bool noown = owner == rank;
    const int pass0 = (gb & 64u) ? 0 : 1;
    const u32 drt = idx & 3, dct = xcd * 4u + ((idx >> 2) & 1) + ((idx >> 4) & 1) * 2u;
    const int N1 = noown ? 48 : (owner ^ rank) == 1u ? 64 : 56;
    u32 pv[16];
    const u32 blk = owner * 256u + ((ct * 2u + (rt_ & 1)) * 4u + (wid & 3)) * 2u + (wid >> 2);
    u64 eblk = (u64)(my + OFF_OPK + blk * BLK), eflag = (u64)(P.ws[owner & 7] + OFF_OFL + rank * 4096u + (ct * 2u + (rt_ & 1)) * 128u);
    asm volatile("" : "+s"(eblk), "+s"(eflag));

    if (wid >= 4) {
        LANE_VARS();
        const u32 l = wid - 4;
        const rsrc_t rs = mkrs(my + OFF_IMG + (l & 1) * 4096u, NID * 8192u - 4096u);
        const rsrc_t rf = mkrs(my + OFF_RDY + (l & 1) * 128u, NID * 256u - 128u);
        __builtin_amdgcn_s_setprio(2);
        for (int pass = pass0; pass < 2; ++pass) {
            const int N = pass ? N1 : 16;
            u32 idv;
            {
                const u32 hole = (owner ^ rank) - 2u;
                const u32 i = noown ? lane + 16u : lane, j = i - 16u, ks = j / 5u, r = j - ks * 5u;
                const u32 kp = min(i < 16u || hole > 5u ? i : 16u + ks * 6u + r + (r >= hole), 63u);
                const u32 cube = (l >> 1) ? (kp < 16u ? ID_WD + (rb * 16u + ct) * 16u : ID_WR - 16u + ct * 48u) + kp : rt_ * 64u + kp;
                const u32 first = (l >> 1) ? ID_WD + dct * 16u : ID_XD + drt * 16u;
                idv = pass ? cube : first + (lane & 15);
            }
            int avail = 0;
            auto window = [&](u32 n) { return (u32)(lane - avail) < n ? idv * 256u : 0x80000000u; };
            auto seen = [&](u32 f) { if (avail < 64) { const u64 miss = __builtin_amdgcn_ballot_w64(f < call) >> avail << avail; avail = miss ? (int)__builtin_ctzll(miss) : 64; } };
            auto need = [&](int s1) {
                if (__builtin_expect(avail <= s1, 0)) {
                    const u64 t0 = rt();
                    for (u32 it = 1;; ++it) {
                        FENCE();
                        seen(__builtin_amdgcn_raw_buffer_load_b32(rf, window(LWINB), 0, 16));
                        if (avail > s1) break;
                        if ((it & 63u) == 0 && timed_out(tmo, call, t0)) { avail = 64; break; }
                        __builtin_amdgcn_s_sleep(12);
                    }
                    WAITL0();
                }
            };
            const u32 sov = idv * 8192u, dl0 = l * 4096u;
            u32 so = rdl(sov, 0), dl = dl0, f = 0, fo = 0x80000000u;
            auto issue = [&](int s1, bool look) {
                need(s1);
                LAS void* const d = (LAS void*)((LAS char*)lds + dl);
                __builtin_amdgcn_raw_ptr_buffer_load_lds(rs, d, 16, v16, so, 0, 17);
                SB();
                u32 son = rdl(sov, (s1 + 1) & 63), dln = dl + 16384u;
                dln = dln == dl0 + NS * 16384u ? dl0 : dln;
                asm volatile("" : "+s"(son), "+s"(dln));
                SB();
                __builtin_amdgcn_raw_ptr_buffer_load_lds(rs, d, 16, v16, so, 1024, 17);
                SB();
                if (look) { const u64 miss = __builtin_amdgcn_ballot_w64(f < call && lane >= avail); avail = miss ? (int)__builtin_ctzll(miss) : 64; asm volatile("" : "+s"(avail)); }
                SB();
                __builtin_amdgcn_raw_ptr_buffer_load_lds(rs, d, 16, v16, so, 2048, 17);
                SB();
                if (look) { fo = window(avail < s1 + LAHEAD ? LWIN : 0); f = 0; asm volatile("" : "+v"(fo), "+v"(f)); }
                SB();
                __builtin_amdgcn_raw_ptr_buffer_load_lds(rs, d, 16, v16, so, 3072, 17);
                FENCE(); SB();
                so = son; dl = dln;
            };
            if (pass && l == 2) {
                const u64 t0 = rt();
                for (u32 it = 1; uni(__hip_atomic_load((const gu32*)(my + OFF_IFL + owner * 1024u), __ATOMIC_RELAXED, __HIP_MEMORY_SCOPE_SYSTEM)) < call; ++it) {
                    if ((it & 63u) == 0 && timed_out(tmo, call, t0)) break;
                    __builtin_amdgcn_s_sleep(12);
                }
            }
            if (!pass && gb >= 192u && ((idx ^ rank) & 6u) != 0) {
                const gu32* const h = (const gu32*)(my + OFF_HND + ((idx & 7u) * 32u + ((gb + idx) & 7u)) * 128u);
                const u64 t0 = rt();
                for (u32 it = 1; (uni(__hip_atomic_load(h, __ATOMIC_RELAXED, __HIP_MEMORY_SCOPE_AGENT)) & 0x7FFFFFFFu) < call; ++it) {
                    if ((it & 15u) == 0 && timed_out(tmo, call, t0)) break;
                    __builtin_amdgcn_s_sleep(24);
                }
                __builtin_amdgcn_s_sleep(20);
            }
            for (int s = 0; s < 5; ++s) issue(s, false);
            asm volatile("s_waitcnt vmcnt(16)" ::: "memory"); BAR();
            asm volatile("s_waitcnt vmcnt(12)" ::: "memory"); BAR();
            int s = 5;
            for (; s < N - 3; s += 4) {
                issue(s, true);
                asm volatile("buffer_load_dword %0, %1, %2, 0 offen sc1" : "+v"(f) : "v"(fo), "s"(rf) : "memory");
                asm volatile("s_waitcnt vmcnt(12)" ::: "memory"); BAR();
                issue(s + 1, false);
                asm volatile("s_waitcnt vmcnt(12)" ::: "memory"); BAR();
                issue(s + 2, false);
                asm volatile("s_waitcnt vmcnt(12)" ::: "memory"); BAR();
                issue(s + 3, false);
                asm volatile("s_waitcnt vmcnt(12)" : "+v"(f) :: "memory"); BAR();
            }
            seen(f);
            for (; s < N; ++s) { issue(s, false); asm volatile("s_waitcnt vmcnt(12)" ::: "memory"); BAR(); }
            asm volatile("s_waitcnt vmcnt(8)" ::: "memory"); BAR();
            asm volatile("s_waitcnt vmcnt(4)" ::: "memory"); BAR();
            asm volatile("s_waitcnt vmcnt(0)" ::: "memory"); BAR();
            BAR(); BAR();
            if (!pass) BAR();
        }
        __builtin_amdgcn_s_setprio(0);
        BAR();
#pragma unroll
        for (int i = 0; i < 4; ++i) {
            const u32x4 v = *(LAS u32x4*)((LAS char*)lds + 32768 + l * 4096 + i * 1024 + v16);
            pv[4 * i] = v[0]; pv[4 * i + 1] = v[1]; pv[4 * i + 2] = v[2]; pv[4 * i + 3] = v[3];
        }
    } else {
        const u32 la = (wid >> 1) * 4096, lb = 8192 + (wid & 1) * 4096;
        u32 pla = threadIdx.x * 16u, plb;
#define V16() (pla & 1023u)
        f32x4 acc[4][4];
        bf16x8 A0, A1, C0[2], C1[2], B0[4], B1[4];
#define RD(d, q, i) d = *(const LAS bf16x8*)((LAS char*)lds + p##q + (i) * 1024)
#define M1(a, B, m, n) acc[m][n] = __builtin_amdgcn_mfma_f32_16x16x32_bf16(a, B[n], acc[m][n], 0, 0, 0)
#define NEXT() do { const u32 d_ = so == (NS - 1) * 16384u ? 0u - so : 16384u; so += d_; pla += d_; plb += d_; asm volatile("" : "+v"(pla), "+v"(plb)); } while (0)
#define STEP(Cc, Bc, Cn, Bn) do { SB(); \
            M1(A0, Bc, 0, 0); SB(); RD(Bn[0], lb, 0); SB(); M1(A0, Bc, 0, 1); SB(); RD(Bn[1], lb, 1); SB(); M1(A0, Bc, 0, 2); SB(); RD(Bn[2], lb, 2); SB(); M1(A0, Bc, 0, 3); SB(); RD(A0, la, 0); SB(); \
            M1(A1, Bc, 1, 0); SB(); RD(Bn[3], lb, 3); SB(); M1(A1, Bc, 1, 1); SB(); RD(Cn[0], la, 2); SB(); M1(A1, Bc, 1, 2); SB(); RD(Cn[1], la, 3); SB(); M1(A1, Bc, 1, 3); SB(); RD(A1, la, 1); SB(); \
            M1(Cc[0], Bc, 2, 0); SB(); M1(Cc[0], Bc, 2, 1); SB(); NEXT(); SB(); M1(Cc[0], Bc, 2, 2); SB(); M1(Cc[0], Bc, 2, 3); SB(); \
            M1(Cc[1], Bc, 3, 0); SB(); WAITL0(); SB(); M1(Cc[1], Bc, 3, 1); SB(); BAR(); SB(); M1(Cc[1], Bc, 3, 2); SB(); M1(Cc[1], Bc, 3, 3); SB(); \
        } while (0)
        for (int pass = pass0;; ++pass) {
#pragma unroll
            for (int m = 0; m < 4; ++m)
#pragma unroll
                for (int n = 0; n < 4; ++n) acc[m][n] = (f32x4){0.f, 0.f, 0.f, 0.f};
            u32 so = 0;
            asm volatile("" : "+v"(pla));
            plb = V16() + lb; pla = V16() + la;
            BAR();
            RD(A0, la, 0); RD(A1, la, 1); RD(C0[0], la, 2); RD(C0[1], la, 3);
#pragma unroll
            for (int i = 0; i < 4; ++i) RD(B0[i], lb, i);
            SB(); NEXT();
            WAITL0(); BAR();
            for (int s = pass ? N1 : 16; s > 0; s -= 2) {
                STEP(C0, B0, C1, B1);
                STEP(C1, B1, C0, B0);
            }
            if (pass) break;
            {
                const u32 vo = V16() + OFF_DIA + (((drt & 2) + (dct >> 4)) * 256u + (((dct & 15u) * 2u + (drt & 1)) * 4u + wid) * 2u) * 8192u;
#pragma unroll
                for (int m = 0; m < 4; ++m)
#pragma unroll
                    for (int j = 0; j < 4; ++j)
                        st16_dev(__builtin_bit_cast(u32x4, (f32x4){acc[m][0][j], acc[m][1][j], acc[m][2][j], acc[m][3][j]}), rmy, vo + m * 4096u + j * 1024u, 0);
            }
            WAITV(0);
            BAR();
            if (wid == 0 && V16() == 0 && __hip_atomic_load(tmo, __ATOMIC_RELAXED, __HIP_MEMORY_SCOPE_AGENT) != call)
                __hip_atomic_store((gu32*)(my + OFF_DFL + (((drt & 2) + (dct >> 4)) * 32u + (dct & 15u) * 2u + (drt & 1)) * 128u), call, __ATOMIC_RELAXED, __HIP_MEMORY_SCOPE_AGENT);
        }
#pragma unroll
        for (int i = 0; i < 4; ++i) {
            u32x4 v;
#pragma unroll
            for (int e = 0; e < 4; ++e) { const int k = 2 * i + (e >> 1), m = 2 + (k >> 2), j = k & 3; v[e] = cvt_pk_bf16(acc[m][2 * (e & 1)][j], acc[m][2 * (e & 1) + 1][j]); }
            *(LAS u32x4*)((LAS char*)lds + 32768 + wid * 4096 + i * 1024 + V16()) = v;
        }
        WAITL0(); BAR();
#pragma unroll
        for (int k = 0; k < 8; ++k) { pv[2 * k] = cvt_pk_bf16(acc[k >> 2][0][k & 3], acc[k >> 2][1][k & 3]); pv[2 * k + 1] = cvt_pk_bf16(acc[k >> 2][2][k & 3], acc[k >> 2][3][k & 3]); }
    }
    {
        int tz = threadIdx.x; asm volatile("" : "+v"(tz));
        const int ln = tz & 63;
        const rsrc_t re = mkrs((const void*)eblk, BLK);
        u32 lo[8], hi[8], mu = 0, mi = 0x80008000u;
#pragma unroll
        for (int k = 0; k < 8; ++k) {
            lo[k] = __builtin_amdgcn_perm(pv[2 * k + 1], pv[2 * k], 0x06040200u); hi[k] = __builtin_amdgcn_perm(pv[2 * k + 1], pv[2 * k], 0x07050301u);
            mu = pkmax(mu, pv[2 * k]); mi = pkmaxi(mi, pv[2 * k + 1]);
        }
        mu = pkmax(mu & 0x7FFF7FFFu, mi & 0x7FFF7FFFu);
        st16_sys((u32x4){lo[0], lo[1], lo[2], lo[3]}, re, ln * 16u, 0);
        st16_sys((u32x4){lo[4], lo[5], lo[6], lo[7]}, re, ln * 16u + 1024u, 0);
        Pk pk;
        pack_hi(lo, hi, max(mu >> 16, mu & 0xFFFFu) >> 7, ln, (LAS char*)lds + wid * 4096, pk);
        store_blk_cs(pk, re, 0, ln);
        store_blk_raw(pk, hi, rmy, OFF_OOV + blk * 2048u, ln);
        FENCE();
        const u32 tm = __hip_atomic_load(tmo, __ATOMIC_RELAXED, __HIP_MEMORY_SCOPE_AGENT);
        WAITV(0);
        BAR();
        if (tz == 0 && tm != call)
            __hip_atomic_store((gu32*)eflag, call, __ATOMIC_RELAXED, __HIP_MEMORY_SCOPE_SYSTEM);
    }
}

}

extern "C" void kernel_launch(int rank, int nranks, ncclComm_t comm,
                              hipStream_t stream,
                              void* const* d_in, const int* in_sizes, int n_in,
                              void* d_out, int out_size,
                              const int* in_dtypes, int out_dtype,
                              void* const* d_out_all,
                              void* const* d_ws_all, size_t ws_bytes) {
    if (nranks != NR || (unsigned)rank >= (unsigned)NR || n_in != 2 || ws_bytes < WS_USED ||
        in_sizes[0] != 4096 * 512 || in_sizes[1] != 512 * 4096 || out_size != 512 * 4096) {
        fprintf(stderr, "kernel_launch: unexpected problem shape\n");
        return;
    }
    Params P;
    for (int p = 0; p < NR; ++p) P.ws[p] = (char*)d_ws_all[p];
    P.x = (const char*)d_in[0];
    P.w = (const char*)d_in[1];
    P.out = (char*)d_out;
    P.my = P.ws[rank];
    P.rank = rank;
    P.pad = 0;
    hipLaunchKernelGGL(fused, dim3(NCOMM + NGEMM), dim3(512), 0, stream, P);
    HIP_CHECK(hipGetLastError());
}
```
